```python
import numpy as np

kernel_name = "gpu_ag_gemm_m8192_k8192_n8192_f32_relu"


ABSMAX_RTOL = 0.001

COMPUTE_DTYPE = "float32"
EPILOGUE = "relu"
IN_QUANT = None
OUT_QUANT = None
IN_DTYPES = {"x": "float32", "w_mat": "float32"}
WORLD_SIZE = 8

def setup_inputs():

    x = np.random.randn(8192, 8192).astype(np.float32)
    w_mat = (np.random.randn(8192, 8192) / np.sqrt(8192)).astype(np.float32)
    return {"x": x, "w_mat": w_mat}

def _epilogue(y):
    return np.maximum(y, 0.0).astype(np.float32)

def reference(x, w_mat):
    y = (x @ w_mat).astype(np.float32)
    return _epilogue(y)
```

```cpp
#include <hip/hip_runtime.h>
#include <rccl/rccl.h>
#include <cstdio>
#include <cstdint>

typedef _Float16 f16x8 __attribute__((ext_vector_type(8)));
typedef float f32x4 __attribute__((ext_vector_type(4)));
typedef unsigned u32x4 __attribute__((ext_vector_type(4)));
#define LAS __attribute__((address_space(3)))
#define GAS __attribute__((address_space(1)))

namespace {

constexpr unsigned kBlk = 16384u;
constexpr unsigned kPanel = 2u << 20;
constexpr unsigned kAOff = 1u << 20;
constexpr unsigned kASize = 34u * kPanel;
constexpr unsigned kBOff = kAOff + 2u * kASize;
constexpr unsigned kBSize = 16u * kPanel;
constexpr unsigned kWsNeed = kBOff + 2u * kBSize;
constexpr unsigned kTicket = 0, kTmo = 128, kDone = 256, kCvt = 384, kFlags = 4096;
constexpr unsigned kInBytes = 32u << 20;
constexpr int kNPush = 48, kNComp = 208, kNWG = 256;
constexpr unsigned kItems = 1024;
constexpr unsigned kDonePerCall = 240u * 4u;
constexpr unsigned long long kDeadline = 50000000ull;
constexpr unsigned kLdsPoll = 131072u, kLdsMisc = 132096u, kLdsBytes = 132160u;
constexpr int kAuxSys = 17;
constexpr int kAuxDev = 16;

struct Params {
  const float* x;
  const float* w;
  char* ws[8];
  float* out[8];
  int rank;
  uint32_t tiles[kNComp];
};

__device__ __forceinline__ __amdgpu_buffer_rsrc_t mk_rsrc(const void* p, unsigned bytes) {
  return __builtin_amdgcn_make_buffer_rsrc((void*)p, 0, bytes, 0x00020000);
}
__device__ __forceinline__ f32x4 ld16(__amdgpu_buffer_rsrc_t r, unsigned voff, unsigned soff) {
  return __builtin_bit_cast(f32x4, __builtin_amdgcn_raw_buffer_load_b128(r, voff, soff, 0));
}
template <int AUX>
__device__ __forceinline__ void st16(f16x8 v, __amdgpu_buffer_rsrc_t r, unsigned voff, unsigned soff) {
  __builtin_amdgcn_raw_buffer_store_b128(__builtin_bit_cast(u32x4, v), r, voff, soff, AUX);
}
__device__ __forceinline__ f16x8 cvt8(f32x4 a, f32x4 b) {
  f16x8 h;
#pragma unroll
  for (int i = 0; i < 4; ++i) { h[i] = (_Float16)a[i]; h[4 + i] = (_Float16)b[i]; }
  return h;
}
__device__ __forceinline__ f16x8 cvt_col(const f32x4 (&v)[8], int i) {
  f16x8 h;
#pragma unroll
  for (int j = 0; j < 8; ++j) h[j] = (_Float16)v[j][i];
  return h;
}
#define WAIT_VM(n) asm volatile("s_waitcnt vmcnt(" #n ")" ::: "memory")
#define BARRIER() do { asm volatile("" ::: "memory"); __builtin_amdgcn_s_barrier(); asm volatile("" ::: "memory"); } while (0)

__device__ __forceinline__ void flag_store(char* p, unsigned v) {
  __hip_atomic_store((GAS unsigned*)p, v, __ATOMIC_RELAXED, __HIP_MEMORY_SCOPE_SYSTEM);
}

#define STAMP(ws, slot) do {} while (0)

__device__ __forceinline__ bool wait_flags(char* myws, unsigned voff, unsigned call, int lane) {
  const unsigned long long t0 = __builtin_amdgcn_s_memrealtime();
  for (;;) {
    const unsigned v = __hip_atomic_load((GAS unsigned*)(myws + voff), __ATOMIC_RELAXED, __HIP_MEMORY_SCOPE_SYSTEM);
    const bool ok = lane >= 32 || (int)(v - call) >= 0;
    const bool dead = lane >= 32 && v == call;
    if (__any(dead)) return false;
    if (__all(ok)) return true;
    if (__builtin_amdgcn_s_memrealtime() - t0 > kDeadline) {
      if (lane == 0) __hip_atomic_store((GAS unsigned*)(myws + kTmo), call, __ATOMIC_RELAXED, __HIP_MEMORY_SCOPE_AGENT);
      return false;
    }
    __builtin_amdgcn_s_sleep(4);
  }
}

}

__global__ void __launch_bounds__(512) __attribute__((amdgpu_waves_per_eu(2, 2)))
ag_gemm_relu_kernel(const Params p) {
  extern __shared__ __attribute__((aligned(16))) char smem_raw[];
  LAS char* const lds = (LAS char*)smem_raw;
  LAS volatile unsigned* const misc = (LAS volatile unsigned*)(lds + kLdsMisc);

  const int tid = threadIdx.x;
  const int lane = tid & 63;
  const int wid = __builtin_amdgcn_readfirstlane(tid >> 6);
  const int bid = blockIdx.x;
  const int rank = p.rank & 7;
  const int g = rank >> 2, q = rank & 3;
  char* const myws = p.ws[rank];

  const unsigned kq = (unsigned)((lane & 3) ^ ((-(lane >> 4)) & 3));
  const unsigned rho = (unsigned)lane >> 2;
  const unsigned lx = (rho * 8192u + 8u * kq) * 4u;
  const unsigned lw = (8u * kq * 1024u + 4u * (4u * (rho & 3u) + (rho >> 2))) * 4u;
  const unsigned l16 = (unsigned)lane * 16u;
  const __amdgpu_buffer_rsrc_t rx = mk_rsrc(p.x, kInBytes);
  const __amdgpu_buffer_rsrc_t rw = mk_rsrc(p.w, kInBytes);

  STAMP(myws, wid == 0 ? 0 : 31);

  if (bid < 16) {
    const unsigned ph = bid >> 3, kt = bid & 7;
    const unsigned src0 = ((128u * ph + 16u * wid) * 8192u + kt * 64u) * 4u;
    f32x4 v[2][4];
    auto loads = [&](int pc, f32x4 (&d)[4]) {
      const unsigned so = src0 + (unsigned)(pc >> 4) * (256u * 8192u * 4u) + (unsigned)(pc & 15) * 2048u;
      d[0] = ld16(rx, lx, so); d[1] = ld16(rx, lx + 16u, so);
      d[2] = ld16(rx, lx + 128u, so); d[3] = ld16(rx, lx + 144u, so);
    };
    loads(0, v[0]);
    loads(1, v[1]);
    if (tid == 0) misc[0] = __hip_atomic_fetch_add((GAS unsigned*)(myws + kTicket), 1u, __ATOMIC_RELAXED, __HIP_MEMORY_SCOPE_AGENT);
    __syncthreads();
    const unsigned call = (unsigned)__builtin_amdgcn_readfirstlane((int)misc[0]) / (unsigned)kNWG + 1u;
    const unsigned par = call & 1u;
    const int d0 = g * 4 + ((q + 1) & 3), d1 = g * 4 + ((q + 2) & 3), d2 = g * 4 + ((q + 3) & 3);
    const __amdgpu_buffer_rsrc_t r0 = mk_rsrc(p.ws[d0], kWsNeed), r1 = mk_rsrc(p.ws[d1], kWsNeed), r2 = mk_rsrc(p.ws[d2], kWsNeed);
    const unsigned dst0 = kAOff + par * kASize + (8u + ph) * kPanel + kt * kBlk + (unsigned)wid * 2048u;
    char* fl = (lane == 0 ? p.ws[d0] : lane == 1 ? p.ws[d1] : p.ws[d2]) + kFlags + (4u + 3u * (2u - (unsigned)(lane & 3))) * 2048u + (unsigned)bid * 4u;
    auto piece = [&](int pc, f32x4 (&d)[4], bool more) {
      const f16x8 h0 = cvt8(d[0], d[1]), h1 = cvt8(d[2], d[3]);
      const unsigned so = dst0 + (unsigned)(pc >> 4) * (2u * kPanel) + (unsigned)(pc & 15) * (8u * kBlk);
      st16<kAuxSys>(h0, r0, l16, so + 12u * kPanel); st16<kAuxSys>(h1, r0, l16 + 1024u, so + 12u * kPanel);
      st16<kAuxSys>(h0, r1, l16, so + 6u * kPanel);  st16<kAuxSys>(h1, r1, l16 + 1024u, so + 6u * kPanel);
      st16<kAuxSys>(h0, r2, l16, so);                st16<kAuxSys>(h1, r2, l16 + 1024u, so);
      asm volatile("" ::: "memory");
      if (more) loads(pc + 2, d);
    };
    for (int pc = 0; pc < 46; pc += 2) {
      piece(pc, v[0], true);
      WAIT_VM(14);
      BARRIER();
      if (tid < 3 && pc > 0) flag_store(fl + (pc - 1) * 128, call);
      piece(pc + 1, v[1], true);
      WAIT_VM(14);
      BARRIER();
      if (tid < 3) flag_store(fl + pc * 128, call);
    }
    piece(46, v[0], false);
    WAIT_VM(6);
    BARRIER();
    if (tid < 3) flag_store(fl + 45 * 128, call);
    piece(47, v[1], false);
    WAIT_VM(6);
    BARRIER();
    if (tid < 3) flag_store(fl + 46 * 128, call);
    WAIT_VM(0);
    BARRIER();
    if (tid < 3) flag_store(fl + 47 * 128, call);
    STAMP(myws, wid == 0 ? 1 : 31);
    if (bid == 0 && tid == 0) {
      const unsigned target = call * kDonePerCall;
      const unsigned long long t0 = __builtin_amdgcn_s_memrealtime();
      bool failed = false;
      while ((int)(__hip_atomic_load((GAS unsigned*)(myws + kDone), __ATOMIC_RELAXED, __HIP_MEMORY_SCOPE_SYSTEM) - target) < 0) {
        if (__hip_atomic_load((GAS unsigned*)(myws + kTmo), __ATOMIC_RELAXED, __HIP_MEMORY_SCOPE_AGENT) == call ||
            __builtin_amdgcn_s_memrealtime() - t0 > kDeadline) {
          __hip_atomic_store((GAS unsigned*)(myws + kTmo), call, __ATOMIC_RELAXED, __HIP_MEMORY_SCOPE_AGENT);
          failed = true;
          break;
        }
        __builtin_amdgcn_s_sleep(2);
      }
      __builtin_amdgcn_fence(__ATOMIC_ACQUIRE, "");
      if (failed) __hip_atomic_store((GAS unsigned*)(p.out[rank] + (size_t)rank * 1024u * 1024u), 0x7fc00000u, __ATOMIC_RELAXED, __HIP_MEMORY_SCOPE_SYSTEM);
      STAMP(myws, 2);
    }
    return;
  }

  if (bid < kNPush) {
    const unsigned c = bid - 16;
    const int og = (g ^ 1) * 4;
    unsigned call;
    {
      const unsigned j = c >> 3, kt = c & 7;
      const unsigned src0 = ((kt * 64u + 32u * (wid & 1)) * 1024u + 256u * j + 64u * (wid >> 1)) * 4u;
      f32x4 v[2][8];
      auto loads = [&](int kc, f32x4 (&d)[8]) {
        const unsigned so = src0 + (unsigned)kc * (512u * 4096u);
#pragma unroll
        for (int i = 0; i < 8; ++i) d[i] = ld16(rw, lw, so + i * 4096u);
      };
      loads(0, v[0]);
      loads(1, v[1]);
      if (tid == 0) misc[0] = __hip_atomic_fetch_add((GAS unsigned*)(myws + kTicket), 1u, __ATOMIC_RELAXED, __HIP_MEMORY_SCOPE_AGENT);
      __syncthreads();
      call = (unsigned)__builtin_amdgcn_readfirstlane((int)misc[0]) / (unsigned)kNWG + 1u;
      const unsigned par = call & 1u;
      char* const dws = p.ws[og + j];
      const __amdgpu_buffer_rsrc_t rd = mk_rsrc(dws, kWsNeed);
      const unsigned dst0 = kBOff + par * kBSize + (8u + 2u * q + (wid >> 2)) * kPanel + kt * kBlk + ((wid >> 1) & 1) * 8192u + (wid & 1) * 1024u;
      char* fl = dws + kFlags + (21u + q) * 2048u + kt * 4u;
      auto piece = [&](int kc, f32x4 (&d)[8], bool more) {
        const unsigned so = dst0 + (unsigned)kc * (8u * kBlk);
        f16x8 h[4];
#pragma unroll
        for (int i = 0; i < 4; ++i) h[i] = cvt_col(d, i);
        st16<kAuxSys>(h[0], rd, l16, so); st16<kAuxSys>(h[1], rd, l16 + 2048u, so);
        st16<kAuxSys>(h[2], rd, l16, so + 4096u); st16<kAuxSys>(h[3], rd, l16 + 2048u, so + 4096u);
        asm volatile("" ::: "memory");
        if (more) loads(kc + 2, d);
      };
      for (int kc = 0; kc < 14; kc += 2) {
        piece(kc, v[0], true);
        WAIT_VM(20);
        BARRIER();
        if (tid == 0 && kc > 0) flag_store(fl + (kc - 1) * 128, call);
        piece(kc + 1, v[1], true);
        WAIT_VM(20);
        BARRIER();
        if (tid == 0) flag_store(fl + kc * 128, call);
      }
      piece(14, v[0], false);
      WAIT_VM(4);
      BARRIER();
      if (tid == 0) flag_store(fl + 13 * 128, call);
      piece(15, v[1], false);
      WAIT_VM(4);
      BARRIER();
      if (tid == 0) flag_store(fl + 14 * 128, call);
      STAMP(myws, wid == 0 ? 1 : 31);
      const unsigned ph = c >> 4, kt2 = (c >> 1) & 7, half = c & 1;
      const unsigned srcx = ((768u + 128u * ph + 64u * half + 16u * (wid >> 1)) * 8192u + kt2 * 64u + 32u * (wid & 1)) * 4u;
      f32x4 y[2][2];
      auto loadsx = [&](int kc, f32x4 (&d)[2]) {
        const unsigned so = srcx + (unsigned)kc * 2048u;
        d[0] = ld16(rx, lx, so); d[1] = ld16(rx, lx + 16u, so);
      };
      loadsx(0, y[0]);
      loadsx(1, y[1]);
      const __amdgpu_buffer_rsrc_t r0 = mk_rsrc(p.ws[og], kWsNeed), r1 = mk_rsrc(p.ws[og + 1], kWsNeed),
                                   r2 = mk_rsrc(p.ws[og + 2], kWsNeed), r3 = mk_rsrc(p.ws[og + 3], kWsNeed);
      const unsigned dstx = kAOff + par * kASize + (26u + 2u * q + ph) * kPanel + kt2 * kBlk + half * 8192u + (unsigned)wid * 1024u;
      char* flx = (lane == 0 ? p.ws[og] : lane == 1 ? p.ws[og + 1] : lane == 2 ? p.ws[og + 2] : p.ws[og + 3]) + kFlags + (13u + q) * 2048u + c * 4u;
      auto piecex = [&](int kc, f32x4 (&d)[2], bool more) {
        const f16x8 h = cvt8(d[0], d[1]);
        const unsigned so = dstx + (unsigned)kc * (8u * kBlk);
        st16<kAuxSys>(h, r0, l16, so); st16<kAuxSys>(h, r1, l16, so);
        st16<kAuxSys>(h, r2, l16, so); st16<kAuxSys>(h, r3, l16, so);
        asm volatile("" ::: "memory");
        if (more) loadsx(kc + 2, d);
      };
      for (int kc = 0; kc < 14; kc += 2) {
        piecex(kc, y[0], true);
        WAIT_VM(8);
        BARRIER();
        if (kc > 0) { if (tid < 4) flag_store(flx + (kc - 1) * 128, call); }
        else if (tid == 0) flag_store(fl + 15 * 128, call);
        piecex(kc + 1, y[1], true);
        WAIT_VM(8);
        BARRIER();
        if (tid < 4) flag_store(flx + kc * 128, call);
      }
      piecex(14, y[0], false);
      WAIT_VM(4);
      BARRIER();
      if (tid < 4) flag_store(flx + 13 * 128, call);
      piecex(15, y[1], false);
      WAIT_VM(4);
      BARRIER();
      if (tid < 4) flag_store(flx + 14 * 128, call);
      WAIT_VM(0);
      BARRIER();
      if (tid < 4) flag_store(flx + 15 * 128, call);
      STAMP(myws, wid == 0 ? 2 : 31);
    }
    return;
  }

  const __amdgpu_buffer_rsrc_t rws = mk_rsrc(myws, kWsNeed);
  if (tid == 0) {
    const unsigned tk = __hip_atomic_fetch_add((GAS unsigned*)(myws + kTicket), 1u, __ATOMIC_RELAXED, __HIP_MEMORY_SCOPE_AGENT);
    const unsigned cl = __hip_atomic_fetch_add((GAS unsigned*)(myws + kCvt), 1u, __ATOMIC_RELAXED, __HIP_MEMORY_SCOPE_AGENT);
    misc[0] = tk; misc[1] = cl; misc[3] = 0u;
  }
  __syncthreads();
  const unsigned call = (unsigned)__builtin_amdgcn_readfirstlane((int)misc[0]) / (unsigned)kNWG + 1u;
  const unsigned par = call & 1u;
  const unsigned abase = kAOff + par * kASize, bbase = kBOff + par * kBSize;
  {
    const unsigned cbase = (call - 1u) * (kItems + (unsigned)kNComp);
    unsigned cur = (unsigned)__builtin_amdgcn_readfirstlane((int)misc[1]) - cbase;
    f32x4 v[8];
    auto loads = [&](unsigned it) {
      const unsigned ktg = it >> 3, sub = it & 7u;
      if (sub < 4u) {
        const unsigned so = ((256u * sub + 32u * wid) * 8192u + ktg * 64u) * 4u;
#pragma unroll
        for (int s = 0; s < 2; ++s) {
          v[4 * s + 0] = ld16(rx, lx, so + s * 524288u); v[4 * s + 1] = ld16(rx, lx + 16u, so + s * 524288u);
          v[4 * s + 2] = ld16(rx, lx + 128u, so + s * 524288u); v[4 * s + 3] = ld16(rx, lx + 144u, so + s * 524288u);
        }
      } else {
        const unsigned so = ((ktg * 64u + 32u * (wid & 1)) * 1024u + 256u * (sub - 4u) + 64u * (wid >> 1)) * 4u;
#pragma unroll
        for (int i = 0; i < 8; ++i) v[i] = ld16(rw, lw, so + i * 4096u);
      }
    };
    if (cur < kItems) loads(cur);
    int flip = 0;
    while (cur < kItems) {
      unsigned nx = 0;
      if (tid == 0) nx = __hip_atomic_fetch_add((GAS unsigned*)(myws + kCvt), 1u, __ATOMIC_RELAXED, __HIP_MEMORY_SCOPE_AGENT);
      const unsigned ktg = cur >> 3, sub = cur & 7u;
      if (sub < 4u) {
        const unsigned so = abase + (2u * sub + (wid >> 2)) * kPanel + ktg * kBlk + (wid & 3) * 4096u;
#pragma unroll
        for (int i = 0; i < 4; ++i) st16<kAuxDev>(cvt8(v[2 * i], v[2 * i + 1]), rws, l16 + i * 1024u, so);
      } else {
        const unsigned so = bbase + (2u * (sub - 4u) + (wid >> 2)) * kPanel + ktg * kBlk + ((wid >> 1) & 1) * 8192u + (wid & 1) * 1024u;
        st16<kAuxDev>(cvt_col(v, 0), rws, l16, so); st16<kAuxDev>(cvt_col(v, 1), rws, l16 + 2048u, so);
        st16<kAuxDev>(cvt_col(v, 2), rws, l16, so + 4096u); st16<kAuxDev>(cvt_col(v, 3), rws, l16 + 2048u, so + 4096u);
      }
      if (tid == 0) misc[4 + flip] = nx;
      asm volatile("s_waitcnt lgkmcnt(0)" ::: "memory");
      BARRIER();
      const unsigned nxt = (unsigned)__builtin_amdgcn_readfirstlane((int)misc[4 + flip]) - cbase;
      flip ^= 1;
      if (nxt < kItems) { loads(nxt); WAIT_VM(8); } else { WAIT_VM(0); }
      BARRIER();
      if (tid == 0) flag_store(myws + kFlags + ((sub < 4u ? sub : 13u + sub) * 16u + (ktg >> 3)) * 128u + (ktg & 7u) * 4u, call);
      cur = nxt;
    }
  }
  STAMP(myws, wid == 0 ? 1 : 31);

  const uint32_t tl = p.tiles[bid - kNPush];
  const int ntiles = (int)(tl >> 30);

  if (wid >= 4) {
    const bool isA = wid < 6;
    const unsigned half = wid & 1;
    const unsigned lbase = (isA ? 0u : 16384u) + half * 8192u;
    LAS char* const scratch = lds + kLdsPoll + (wid - 4) * 256;
    const unsigned scr_addr = kLdsPoll + (unsigned)(wid - 4) * 256u + (unsigned)lane * 4u;
    bool bad = false;
    for (int ti = 0; ti < ntiles; ++ti) {
      const unsigned code = (tl >> (10 * ti)) & 1023u;
      const unsigned ap = code & 63u, bp = code >> 6;
      unsigned sbase, grp, woff, cmask;
      if (isA) {
        sbase = abase + ap * kPanel;
        if (ap < 8u) { grp = ap >> 1; woff = 0u; cmask = 7u; }
        else if (ap < 26u) { const unsigned i = (ap - 8u) / 6u, s = (ap - 8u) - 6u * i; grp = 4u + 3u * i + (s >> 1); woff = (s & 1u) * 8u; cmask = 7u; }
        else { grp = 13u + ((ap - 26u) >> 1); woff = (ap & 1u) * 16u; cmask = 15u; }
      } else {
        sbase = bbase + bp * kPanel;
        grp = 17u + (bp >> 1); woff = 0u; cmask = 7u;
      }
      sbase += half * 8192u;
      const unsigned pvoff = lane < 32 ? kFlags + grp * 2048u + (woff + ((unsigned)lane & cmask)) * 4u : kTmo;
      const unsigned pstep = lane < 32 ? 128u : 0u;
      int u = 0, okc = bad ? 16 : 0, up = 0;
      bool pend = false;
      auto issue = [&](int uu) {
        LAS char* const dl = lds + (uu & 3) * 32768 + lbase;
        const unsigned so = sbase + (unsigned)uu * kBlk;
        __builtin_amdgcn_raw_ptr_buffer_load_lds(rws, (LAS void*)dl, 16, l16, so, 0, kAuxDev);
        __builtin_amdgcn_raw_ptr_buffer_load_lds(rws, (LAS void*)dl, 16, l16, so, 1024, kAuxDev);
        __builtin_amdgcn_raw_ptr_buffer_load_lds(rws, (LAS void*)dl, 16, l16, so, 2048, kAuxDev);
        __builtin_amdgcn_raw_ptr_buffer_load_lds(rws, (LAS void*)dl, 16, l16, so, 3072, kAuxDev);
        __builtin_amdgcn_raw_ptr_buffer_load_lds(rws, (LAS void*)(dl + 4096), 16, l16, so + 4096u, 0, kAuxDev);
        __builtin_amdgcn_raw_ptr_buffer_load_lds(rws, (LAS void*)(dl + 4096), 16, l16, so + 4096u, 1024, kAuxDev);
        __builtin_amdgcn_raw_ptr_buffer_load_lds(rws, (LAS void*)(dl + 4096), 16, l16, so + 4096u, 2048, kAuxDev);
        __builtin_amdgcn_raw_ptr_buffer_load_lds(rws, (LAS void*)(dl + 4096), 16, l16, so + 4096u, 3072, kAuxDev);
      };
      for (int t = -1; t < 127; ++t) {
        if (u <= t + 1) {
          if (okc * 8 <= u) {
            if (!wait_flags(myws, pvoff + (unsigned)okc * pstep, call, lane)) { bad = true; okc = 15; }
            ++okc;
            pend = false;
          }
          const int lim = min(min(t + 4, 128), okc * 8);
          while (u < lim) { issue(u); ++u; }
        }
        const int n = u - (t + 2);
        if (n >= 2) WAIT_VM(16); else if (n == 1) WAIT_VM(8); else WAIT_VM(0);
        if (pend && up <= t + 2) {
          unsigned v;
          asm volatile("ds_read_b32 %0, %1\n\ts_waitcnt lgkmcnt(0)" : "=v"(v) : "v"(scr_addr) : "memory");
          pend = false;
          const bool ok = lane >= 32 || (int)(v - call) >= 0;
          const bool dead = lane >= 32 && v == call;
          if (__any(dead)) { bad = true; okc = 16; }
          else if (__all(ok)) ++okc;
          const int lim = min(min(t + 4, 128), okc * 8);
          while (u < lim) { issue(u); ++u; }
        }
        BARRIER();
        if (!pend && okc < 16 && u + 3 >= okc * 8) {
          __builtin_amdgcn_raw_ptr_buffer_load_lds(rws, (LAS void*)scratch, 4, pvoff + (unsigned)okc * pstep, 0, 0, kAuxSys);
          pend = true; up = u;
        }
        const int lim = min(min(t + 5, 128), okc * 8);
        while (u < lim) { issue(u); ++u; }
      }
      if (bad) { if (lane == 0) misc[3] = 1u; asm volatile("s_waitcnt lgkmcnt(0)" ::: "memory"); }
      BARRIER();
    }
    return;
  }

  {
    const int wr = wid >> 1, wc = wid & 1;
    const int fr = lane & 15, fq = lane >> 4;
    const unsigned slot = (unsigned)(4 * fr + (fq ^ ((-(fr >> 2)) & 3))) * 16u;
    LAS char* const pa = lds + wr * 8192 + slot;
    LAS char* const pb = lds + 16384 + wc * 8192 + slot;
    unsigned hi = 65536u;
    asm volatile("" : "+v"(hi));
    LAS char* const pa2 = pa + hi;
    LAS char* const pb2 = pb + hi;
    char* sigp = nullptr;
    for (int ti = 0; ti < ntiles; ++ti) {
      const unsigned code = (tl >> (10 * ti)) & 1023u;
      const unsigned ap = code & 63u, bp = code >> 6;
      f32x4 acc[4][4];
#pragma unroll
      for (int m = 0; m < 4; ++m)
#pragma unroll
        for (int n = 0; n < 4; ++n) acc[m][n] = f32x4{0.f, 0.f, 0.f, 0.f};
      f16x8 fa[2][8], fb[2][8];
      auto rd = [&](int st, f16x8 (&a)[8], f16x8 (&b)[8]) {
#pragma unroll
        for (int i = 0; i < 8; ++i) {
          a[i] = *(LAS f16x8*)((st < 2 ? pa : pa2) + (st & 1) * 32768 + i * 1024);
          b[i] = *(LAS f16x8*)((st < 2 ? pb : pb2) + (st & 1) * 32768 + i * 1024);
        }
      };
      auto mm = [&](const f16x8 (&a)[8], const f16x8 (&b)[8]) {
#pragma unroll
        for (int kh = 0; kh < 2; ++kh)
#pragma unroll
          for (int m = 0; m < 4; ++m)
#pragma unroll
            for (int n = 0; n < 4; ++n)
              acc[m][n] = __builtin_amdgcn_mfma_f32_16x16x32_f16(b[2 * n + kh], a[2 * m + kh], acc[m][n], 0, 0, 0);
      };
      auto sched = [&]() {
#pragma unroll
        for (int i = 0; i < 16; ++i) {
          __builtin_amdgcn_sched_group_barrier(0x100, 1, 0);
          __builtin_amdgcn_sched_group_barrier(0x008, 1, 0);
        }
        __builtin_amdgcn_sched_group_barrier(0x008, 16, 0);
      };
      BARRIER();
      STAMP(myws, wid == 0 ? 2 + 3 * ti : 31);
      rd(0, fa[0], fb[0]);
      for (int t = 0; t < 128; t += 4) {
        if (t == 16 && sigp != nullptr) {
          WAIT_VM(0);
          if (lane == 0) __hip_atomic_fetch_add((GAS unsigned*)sigp, 1u, __ATOMIC_RELAXED, __HIP_MEMORY_SCOPE_SYSTEM);
          sigp = nullptr;
        }
#pragma unroll
        for (int s = 0; s < 4; ++s) {
          __builtin_amdgcn_s_waitcnt(0xC07F);
          BARRIER();
          __builtin_amdgcn_sched_barrier(0);
          rd((s + 1) & 3, fa[(s + 1) & 1], fb[(s + 1) & 1]);
          mm(fa[s & 1], fb[s & 1]);
          sched();
          __builtin_amdgcn_sched_barrier(0);
        }
      }
      STAMP(myws, wid == 0 ? 3 + 3 * ti : 31);
      const bool bad = misc[3] != 0u;
      unsigned row0, owner, col0;
      if (ap < 8u) row0 = (unsigned)rank * 1024u + ap * 128u;
      else if (ap < 26u) { const unsigned i = (ap - 8u) / 6u, s = (ap - 8u) - 6u * i; row0 = (unsigned)(g * 4 + ((q + (int)i + 1) & 3)) * 1024u + s * 128u; }
      else row0 = (unsigned)((g ^ 1) * 4 + (int)((ap - 26u) >> 1)) * 1024u + 768u + (ap & 1u) * 128u;
      if (bp < 8u) { owner = (unsigned)rank; col0 = bp * 128u; }
      else { owner = (unsigned)((g ^ 1) * 4) + ((bp - 8u) >> 1); col0 = 256u * q + (bp & 1u) * 128u; }
      owner &= 7u;
      const __amdgpu_buffer_rsrc_t ro = mk_rsrc(p.out[owner], kInBytes);
      const unsigned vo = (unsigned)fr * 4096u + (unsigned)fq * 16u;
      const unsigned so = (row0 + 64u * wr) * 4096u + (col0 + 64u * wc) * 4u;
      const int lo = bad ? 0x7fc00000 : 0;
#pragma unroll
      for (int m = 0; m < 4; ++m)
#pragma unroll
        for (int j = 0; j < 4; ++j) {
          u32x4 o;
#pragma unroll
          for (int n = 0; n < 4; ++n) { const float f = acc[m][n][j]; o[n] = (unsigned)max(__float_as_int(f), lo); }
          __builtin_amdgcn_raw_buffer_store_b128(o, ro, vo + j * 64u, so + m * 65536u, kAuxSys);
        }
      if (bp >= 8u) sigp = p.ws[owner] + kDone;
      STAMP(myws, wid == 0 ? 4 + 3 * ti : 31);
    }
    if (sigp != nullptr) {
      WAIT_VM(0);
      if (lane == 0) __hip_atomic_fetch_add((GAS unsigned*)sigp, 1u, __ATOMIC_RELAXED, __HIP_MEMORY_SCOPE_SYSTEM);
    }
    STAMP(myws, wid == 0 ? 11 : 31);
  }
}

extern "C" void kernel_launch(int rank, int nranks, ncclComm_t comm, hipStream_t stream,
                              void* const* d_in, const int* in_sizes, int n_in,
                              void* d_out, int out_size,
                              const int* in_dtypes, int out_dtype,
                              void* const* d_out_all, void* const* d_ws_all, size_t ws_bytes) {
  (void)comm; (void)in_dtypes; (void)out_dtype; (void)d_out;
  if (nranks != 8 || (unsigned)rank >= 8u || n_in < 2 || ws_bytes < (size_t)kWsNeed ||
      in_sizes[0] != 8 * 1024 * 1024 || in_sizes[1] != 8 * 1024 * 1024 || out_size != 8 * 1024 * 1024) {
    fprintf(stderr, "kernel_launch: unexpected problem geometry\n");
    return;
  }
  Params p;
  p.x = (const float*)d_in[0];
  p.w = (const float*)d_in[1];
  for (int i = 0; i < 8; ++i) { p.ws[i] = (char*)d_ws_all[i]; p.out[i] = (float*)d_out_all[i]; }
  p.rank = rank;
  static const int rowsel[6] = {8, 9, 14, 15, 20, 21};
  static const int xcol[8] = {0, 4, 2, 3, 1, 5, 6, 7};
  for (int ci = 0; ci < kNComp; ++ci) {
    const int x = ci & 7, s = ci >> 3;
    unsigned t0, t1, t2 = 0, n;
    if (s < 24) t0 = (unsigned)((x < 4 ? (s >> 2) : rowsel[s >> 2]) | ((4 * (x & 3) + (s & 3)) << 6));
    else t0 = (unsigned)((6 + (s - 24)) | (xcol[x] << 6));
    if (s < 12) {
      const int r = rowsel[3 * (x >> 2) + (s >> 2)], c = 4 * (x & 3) + (s & 3);
      t1 = (unsigned)((r + 2) | (c << 6));
      t2 = (unsigned)((r + 4) | (c << 6));
      n = 3;
    } else {
      const int e = s - 12, j = x >> 1, hf = x & 1;
      int a, b;
      if (e < 8) { a = 26 + 2 * j + (e & 1); b = 4 * hf + (e >> 1); }
      else {
        const int e2 = e - 8, idx = 3 * hf + (e2 >> 1);
        a = 26 + 2 * j + (e2 & 1);
        b = 8 + idx + (idx >= 2 * j ? 2 : 0);
      }
      t1 = (unsigned)(a | (b << 6));
      n = 2;
    }
    p.tiles[ci] = t0 | (t1 << 10) | (t2 << 20) | (n << 30);
  }
  hipError_t e = hipFuncSetAttribute((const void*)ag_gemm_relu_kernel, hipFuncAttributeMaxDynamicSharedMemorySize, (int)kLdsBytes);
  if (e != hipSuccess) { fprintf(stderr, "hipFuncSetAttribute: %s\n", hipGetErrorString(e)); return; }
  hipLaunchKernelGGL(ag_gemm_relu_kernel, dim3(kNWG), dim3(512), kLdsBytes, stream, p);
  e = hipGetLastError();
  if (e != hipSuccess) fprintf(stderr, "launch: %s\n", hipGetErrorString(e));
}
```

```cpp
#include <hip/hip_runtime.h>
#include <rccl/rccl.h>
#include <cstdio>
#include <cstdint>
#include <type_traits>

typedef _Float16 f16x8 __attribute__((ext_vector_type(8)));
typedef float f32x4 __attribute__((ext_vector_type(4)));
typedef unsigned u32x4 __attribute__((ext_vector_type(4)));
#define LAS __attribute__((address_space(3)))
#define GAS __attribute__((address_space(1)))

#define NI 16u
#define NC 24u
#define KWI 4
#define KWC 3
#define M0 64u
#define CVMOD 3
#define TAUI 460
#define TAUC 470
#define TOFF 200u
#define PMODE 1
#define CRED 100
#define PDIST 3

namespace {

constexpr unsigned kBlk = 16384u;
constexpr unsigned kPanel = 2u << 20;
constexpr unsigned kAOff = 1u << 20;
constexpr unsigned kASize = 34u * kPanel;
constexpr unsigned kBOff = kAOff + 2u * kASize;
constexpr unsigned kBSize = 16u * kPanel;
constexpr unsigned kOOff = kBOff + 2u * kBSize;
constexpr unsigned kWsNeed = kOOff + 34u * 8u * 65536u;
constexpr unsigned kTicket = 0, kTmo = 128, kDone = 256, kCvt = 384, kQueue = 512;
constexpr unsigned kOFlags = 4096, kFlags = 16384;
constexpr unsigned kInBytes = 32u << 20;
constexpr int kNPush = 48, kNComp = 208, kNWG = 256;
constexpr unsigned kItems = 1024;
constexpr unsigned kDonePerCall = 240u * 8u;
constexpr unsigned long long kDeadline = 50000000ull;
constexpr unsigned kLdsPoll = 131072u, kLdsMisc = 132096u, kLdsBytes = 132160u;
constexpr int kAuxSys = 17;
constexpr int kAuxDev = 16;

struct Params {
  const float* x;
  const float* w;
  char* ws[8];
  float* out[8];
  int rank;
  int pad;
  uint32_t tiles[2 * kNComp];
};

__device__ __forceinline__ __amdgpu_buffer_rsrc_t mk_rsrc(const void* p, unsigned bytes) {
  return __builtin_amdgcn_make_buffer_rsrc((void*)p, 0, bytes, 0x00020000);
}
template <int AUX>
__device__ __forceinline__ f32x4 ld16(__amdgpu_buffer_rsrc_t r, unsigned voff, unsigned soff) {
  return __builtin_bit_cast(f32x4, __builtin_amdgcn_raw_buffer_load_b128(r, voff, soff, AUX));
}
template <int AUX>
__device__ __forceinline__ void st16(f16x8 v, __amdgpu_buffer_rsrc_t r, unsigned voff, unsigned soff) {
  __builtin_amdgcn_raw_buffer_store_b128(__builtin_bit_cast(u32x4, v), r, voff, soff, AUX);
}
__device__ __forceinline__ f16x8 cvt8(f32x4 a, f32x4 b) {
  f16x8 h;
#pragma unroll
  for (int i = 0; i < 4; ++i) { h[i] = (_Float16)a[i]; h[4 + i] = (_Float16)b[i]; }
  return h;
}
__device__ __forceinline__ f16x8 cvt_col(const f32x4 (&v)[8], int i) {
  f16x8 h;
#pragma unroll
  for (int j = 0; j < 8; ++j) h[j] = (_Float16)v[j][i];
  return h;
}
#define WAIT_VM(n) asm volatile("s_waitcnt vmcnt(%0)" :: "n"(n) : "memory")
#define WAIT_LGKM0() asm volatile("s_waitcnt lgkmcnt(0)" ::: "memory")
#define FENCE() asm volatile("" ::: "memory")
#define BARRIER() do { FENCE(); __builtin_amdgcn_s_barrier(); FENCE(); } while (0)

__device__ __forceinline__ void flag_store(char* p, unsigned v) {
  __hip_atomic_store((GAS unsigned*)p, v, __ATOMIC_RELAXED, __HIP_MEMORY_SCOPE_SYSTEM);
}
__device__ __forceinline__ unsigned draw(char* p) {
  unsigned z = 0;
  asm volatile("" : "+v"(z));
  p += z;
  return __hip_atomic_fetch_add((GAS unsigned*)p, 1u, __ATOMIC_RELAXED, __HIP_MEMORY_SCOPE_AGENT);
}
__device__ __forceinline__ unsigned uni(unsigned v) { return (unsigned)__builtin_amdgcn_readfirstlane((int)v); }
__device__ __forceinline__ unsigned lds_min4(unsigned addr) {
  u32x4 c;
  asm volatile("ds_read_b128 %0, %1\n\ts_waitcnt lgkmcnt(0)" : "=v"(c) : "v"(addr) : "memory");
  return uni(min(min(c[0], c[1]), min(c[2], c[3])));
}
__device__ __forceinline__ unsigned lds_get(unsigned addr) {
  unsigned c;
  asm volatile("ds_read_b32 %0, %1\n\ts_waitcnt lgkmcnt(0)" : "=v"(c) : "v"(addr) : "memory");
  return uni(c);
}
__device__ __forceinline__ void lds_put(unsigned addr, unsigned v) {
  asm volatile("ds_write_b32 %0, %1" :: "v"(addr), "v"(v) : "memory");
}

#define STAMP(ws, slot) do {} while (0)
#define PSTAMP(ws, cls, n) do {} while (0)

__device__ __forceinline__ int ready_run(unsigned v, unsigned call, int lane) {
  const unsigned long long b = __ballot(lane < 32 ? (int)(v - call) >= 0 : v == call);
  if ((unsigned)(b >> 32) != 0u) return -1;
  return __builtin_ctzll(~b);
}
__device__ __forceinline__ int wait_flags(char* myws, unsigned voff, unsigned call, int lane, int need) {
  const unsigned long long t0 = __builtin_amdgcn_s_memrealtime();
  for (;;) {
    const unsigned v = __hip_atomic_load((GAS unsigned*)(myws + voff), __ATOMIC_RELAXED, __HIP_MEMORY_SCOPE_SYSTEM);
    const int run = ready_run(v, call, lane);
    if (run < 0 || run >= need) return run;
    if (__builtin_amdgcn_s_memrealtime() - t0 > kDeadline) {
      if (lane == 0) __hip_atomic_store((GAS unsigned*)(myws + kTmo), call, __ATOMIC_RELAXED, __HIP_MEMORY_SCOPE_AGENT);
      return -1;
    }
    __builtin_amdgcn_s_sleep(4);
  }
}

__device__ __forceinline__ unsigned panel_row(unsigned ap, int rank) {
  const int g = rank >> 2, q = rank & 3;
  if (ap < 8u) return (unsigned)rank * 1024u + ap * 128u;
  if (ap < 26u) { const unsigned i = (ap - 8u) / 6u, s = (ap - 8u) - 6u * i; return (unsigned)(g * 4 + ((q + (int)i + 1) & 3)) * 1024u + s * 128u; }
  return (unsigned)((g ^ 1) * 4 + (int)((ap - 26u) >> 1)) * 1024u + 768u + (ap & 1u) * 128u;
}
__device__ __forceinline__ unsigned out_panel(unsigned pi, unsigned j) {
  if (pi < 6u) return pi;
  if (pi < 18u) { const unsigned e = pi - 6u, bb = e >= 6u, r = e - 6u * bb; return 8u + 6u * (r >> 1) + (r & 1u) + 2u * bb; }
  if (pi < 24u) { const unsigned ap = 8u + pi; return ap >= 26u + 2u * j ? ap + 2u : ap; }
  return 12u + 6u * (pi - 24u);
}

}

__global__ void __launch_bounds__(512) __attribute__((amdgpu_waves_per_eu(2, 2)))
ag_gemm_relu_kernel(const Params p) {
  extern __shared__ __attribute__((aligned(16))) char smem_raw[];
  LAS char* const lds = (LAS char*)smem_raw;
  LAS volatile unsigned* const misc = (LAS volatile unsigned*)(lds + kLdsMisc);

  const int tid = threadIdx.x;
  const int lane = tid & 63;
  const int wid = __builtin_amdgcn_readfirstlane(tid >> 6);
  const int bid = blockIdx.x;
  const int rank = p.rank & 7;
  const int g = rank >> 2, q = rank & 3;
  char* const myws = p.ws[rank];

  const unsigned kq = (unsigned)((lane & 3) ^ ((-(lane >> 4)) & 3));
  const unsigned rho = (unsigned)lane >> 2;
  const unsigned lx = (rho * 8192u + 4u * kq) * 4u;
  const unsigned lw = (4u * ((unsigned)lane >> 4) * 1024u + 4u * ((unsigned)lane & 15u)) * 4u;
  const unsigned l16 = (unsigned)lane * 16u;
  const __amdgpu_buffer_rsrc_t rx = mk_rsrc(p.x, kInBytes);
  const __amdgpu_buffer_rsrc_t rw = mk_rsrc(p.w, kInBytes);
  const __amdgpu_buffer_rsrc_t rws = mk_rsrc(myws, kWsNeed);

  STAMP(myws, wid == 0 ? 0 : 31);

  if (bid < kNPush) {
    if ((unsigned)bid >= NI + NC) {
      if (tid == 0) draw(myws + kTicket);
      return;
    }
    const unsigned tstart = (unsigned)__builtin_amdgcn_s_memrealtime() + TOFF;
    const bool cross = (unsigned)bid >= NI;
    const unsigned k = cross ? (unsigned)bid - NI : (unsigned)bid;
    const unsigned total = cross ? 728u : 768u;
    if (tid < 16) misc[tid] = tid == 4 ? k : (tid == 15 && !cross) ? 0x7fffffffu : tid >= 4 && tid < 8 ? 0xffffffffu : 0u;
    WAIT_LGKM0();
    BARRIER();
    if (tid == 0) misc[0] = draw(myws + kTicket) / (unsigned)kNWG + 1u;
    auto get_call = [&]() {
      unsigned c;
      while ((c = lds_get(kLdsMisc)) == 0u) __builtin_amdgcn_s_sleep(1);
      return c;
    };
    auto get_row = [&](unsigned seq) {
      unsigned c;
      while (((c = lds_get(kLdsMisc + 16u + (seq & 3u) * 4u)) >> 12) != seq) __builtin_amdgcn_s_sleep(1);
      return c & 4095u;
    };
    auto xblock = [&](unsigned m, unsigned& sp, unsigned& kt) {
      if (cross) { sp = 6u + (m & 1u); kt = (m >> 1) & 127u; }
      else if (m < 512u) { sp = 2u * (m >> 8) + (m & 1u); kt = (m >> 1) & 127u; }
      else { sp = m >> 7; kt = m & 127u; }
    };

    if (wid >= 4) {
      const unsigned pw = (unsigned)wid - 4u;
      const unsigned pj = ((unsigned)lane >> 1) & 3u;
      const unsigned pa = kLdsMisc + 32u + pw * 4u;
      unsigned call = 0;
      for (unsigned seq = 0;; ++seq) {
        const unsigned m = get_row(seq);
        if (m >= total) break;
        LAS char* const d = lds + (seq & 1u) * 65536u;
        const bool isx = !cross || (m >= 256u && m < 512u);
        unsigned dv = 0;
        auto begin = [&]() {
          while ((int)(lds_min4(kLdsMisc + 48u) + 1u - seq) < 0) __builtin_amdgcn_s_sleep(1);
          if (tid == 256) dv = draw(myws + kQueue + (cross ? 128u : 0u));
          FENCE();
        };
        auto claimed = [&]() {
          if (pw == 0u) {
            if (call == 0u) call = get_call();
            const unsigned nx = min(uni(dv) - (call - 1u) * total + (cross ? NC : NI), 4095u);
            lds_put(kLdsMisc + 16u + ((seq + 1u) & 3u) * 4u, nx | ((seq + 1u) << 12));
          }
        };
        if (isx) {
          unsigned sp, kt;
          xblock(m, sp, kt);
          const unsigned so = ((sp * 128u + pw * 32u) * 8192u + kt * 64u) * 4u;
          f32x4 v[8];
          begin();
#pragma unroll
          for (int i = 0; i < 8; ++i) v[i] = ld16<0>(rx, lx + (i & 3) * 64u, so + (i >> 2) * 524288u);
#pragma unroll
          for (int i = 0; i < 4; ++i) *(LAS f16x8*)(d + pw * 4096u + l16 + i * 1024) = cvt8(v[2 * i], v[2 * i + 1]);
          claimed();
          WAIT_LGKM0();
          lds_put(pa, 4u * seq + 4u);
        } else if (m < M0) {
          const unsigned so = (((m >> 1) * 64u + 32u * (pw & 1u)) * 1024u + 128u * (m & 1u) + 64u * (pw >> 1)) * 4u;
          f32x4 v[4][8];
          begin();
#pragma unroll
          for (int j = 0; j < 4; ++j) {
#pragma unroll
            for (int i = 0; i < 8; ++i) v[j][i] = ld16<0>(rw, lw + j * 1024u, so + ((i & 3) + 16 * (i >> 2)) * 4096u);
            FENCE();
          }
#pragma unroll
          for (int j = 0; j < 4; ++j) {
#pragma unroll
            for (int i = 0; i < 4; ++i)
              *(LAS f16x8*)(d + j * kBlk + (pw >> 1) * 8192u + (pw & 1u) * 1024u + l16 + i * 2048) = cvt_col(v[j], i);
            if (j == 0) claimed();
            WAIT_LGKM0();
            lds_put(pa, 4u * seq + j + 1u);
          }
        } else {
          if (call == 0u) call = get_call();
          const unsigned o = m - 512u;
          auto piece = [&](unsigned j) { return (out_panel(o >> 3, j) * 8u + 2u * j + ((o >> 2) & 1u)) * 4u + (o & 3u); };
          const unsigned rdy = m < 256u ? kFlags + (34u + 2u * pj + (m & 1u)) * 512u + (m >> 1) * 4u
                                        : kOFlags + piece(pj) * 8u + ((unsigned)lane & 1u) * 4u;
          const int run = wait_flags(myws, lane < 32 ? rdy : kTmo, call, lane, 32);
          begin();
          if (run >= 0) {
#pragma unroll
            for (int j = 0; j < 4; ++j) {
              LAS char* const dj = d + j * kBlk + pw * 4096u;
              const unsigned so = (m < 256u ? kBOff + (call & 1u) * kBSize + (2u * j + (m & 1u)) * kPanel + (m >> 1) * kBlk
                                            : kOOff + piece(j) * kBlk) + pw * 4096u;
              __builtin_amdgcn_raw_ptr_buffer_load_lds(rws, (LAS void*)dj, 16, l16, so, 0, kAuxDev);
              __builtin_amdgcn_raw_ptr_buffer_load_lds(rws, (LAS void*)dj, 16, l16, so, 1024, kAuxDev);
              __builtin_amdgcn_raw_ptr_buffer_load_lds(rws, (LAS void*)dj, 16, l16, so, 2048, kAuxDev);
              __builtin_amdgcn_raw_ptr_buffer_load_lds(rws, (LAS void*)dj, 16, l16, so, 3072, kAuxDev);
            }
            WAIT_VM(12);
            claimed();
            lds_put(pa, 4u * seq + 1u);
            WAIT_VM(8); lds_put(pa, 4u * seq + 2u);
            WAIT_VM(4); lds_put(pa, 4u * seq + 3u);
            WAIT_VM(0);
          } else {
#pragma unroll
            for (int i = 0; i < 16; ++i)
              *(LAS u32x4*)(d + (i >> 2) * kBlk + pw * 4096u + l16 + (i & 3) * 1024) = u32x4{0x7fc07e00u, 0x7fc07e00u, 0x7fc07e00u, 0x7fc07e00u};
            claimed();
            WAIT_LGKM0();
          }
          lds_put(pa, 4u * seq + 4u);
        }
      }
      return;
    }

    const unsigned call = get_call();
    if (!cross && wid == 3) {
      if (bid == 0 && lane == 0) {
        const unsigned target = call * kDonePerCall;
        const unsigned long long t0 = __builtin_amdgcn_s_memrealtime();
        bool failed = false;
        while ((int)(__hip_atomic_load((GAS unsigned*)(myws + kDone), __ATOMIC_RELAXED, __HIP_MEMORY_SCOPE_SYSTEM) - target) < 0) {
          if (__hip_atomic_load((GAS unsigned*)(myws + kTmo), __ATOMIC_RELAXED, __HIP_MEMORY_SCOPE_AGENT) == call ||
              __builtin_amdgcn_s_memrealtime() - t0 > kDeadline) {
            __hip_atomic_store((GAS unsigned*)(myws + kTmo), call, __ATOMIC_RELAXED, __HIP_MEMORY_SCOPE_AGENT);
            failed = true;
            break;
          }
          __builtin_amdgcn_s_sleep(2);
        }
        __builtin_amdgcn_fence(__ATOMIC_ACQUIRE, "");
        if (failed) __hip_atomic_store((GAS unsigned*)(p.out[rank] + (size_t)rank * 1024u * 1024u), 0x7fc00000u, __ATOMIC_RELAXED, __HIP_MEMORY_SCOPE_SYSTEM);
        STAMP(myws, 10);
      }
      return;
    }
    const unsigned c = (unsigned)wid;
    const int dr = (cross ? (g ^ 1) * 4 + wid : g * 4 + ((q + 1 + wid) & 3)) & 7;
    char* const dws = p.ws[dr];
    const unsigned par = call & 1u;
    const unsigned abase = kAOff + par * kASize, bbase = kBOff + par * kBSize;
    struct Unit { unsigned own, dst, flg, m; };
    auto decode = [&](unsigned m) {
      Unit u;
      u.m = m;
      u.own = 1u;
      if (cross && m < 256u) {
        const unsigned dp = 8u + 2u * q + (m & 1u);
        u.dst = bbase + dp * kPanel + (m >> 1) * kBlk;
        u.flg = kFlags + (34u + dp) * 512u + (m >> 1) * 4u;
      } else if (cross && m >= 512u) {
        const unsigned o = m - 512u;
        u.dst = (panel_row(out_panel(o >> 3, c), rank) + ((o >> 1) & 1u) * 64u) * 4096u + (256u * q + ((o >> 2) & 1u) * 128u + (o & 1u) * 64u) * 4u;
        u.flg = 1u;
      } else {
        unsigned sp, kt;
        xblock(m, sp, kt);
        const unsigned dp = cross ? 20u + 2u * q + sp : 20u - 6u * c + sp;
        u.own = 0u;
        u.dst = abase + dp * kPanel + kt * kBlk;
        u.flg = kFlags + dp * 512u + kt * 4u;
      }
      return u;
    };
    const uint64_t pdw = (uint64_t)dws, pdo = (uint64_t)p.out[dr];
    const unsigned vo = ((unsigned)lane >> 4) * 4096u + ((unsigned)lane & 15u) * 16u;
    unsigned f1 = 0, f2 = 0, m1 = 0, m2 = 0;
    auto send = [&](unsigned& f, unsigned m) {
      if (f != 0u) {
        if (lane == 0) {
          if (f == 1u) __hip_atomic_fetch_add((GAS unsigned*)(dws + kDone), 2u, __ATOMIC_RELAXED, __HIP_MEMORY_SCOPE_SYSTEM);
          else flag_store(dws + f, call);
        }
        PSTAMP(myws, cross ? 3u + c : c, m + 1u);
        if (m < 128u) PSTAMP(myws, 8u + (cross ? 3u + c : c), (m + 1u) * 4u);
      }
      f = 0u;
    };
    u32x4 xa[8], xb[8];
    auto rd = [&](u32x4 (&x)[8], unsigned seq, const Unit& u, unsigned half) {
      LAS char* const s = lds + (seq & 1u) * 65536u + u.own * c * kBlk + half * 8192u + l16;
#pragma unroll
      for (int i = 0; i < 8; ++i) x[i] = *(LAS u32x4*)(s + i * 1024);
    };
    auto st = [&](const u32x4 (&x)[8], const Unit& u, int qi) {
      const bool fw = u.flg == 1u;
      const __amdgpu_buffer_rsrc_t rdst = mk_rsrc((const void*)(fw ? +pdo : +pdw), fw ? +kInBytes : +kWsNeed);
      const unsigned stp = fw ? 16384u : 1024u, vs = fw ? +vo : +l16;
#pragma unroll
      for (int i = 0; i < 4; ++i) __builtin_amdgcn_raw_buffer_store_b128(x[(qi & 1) * 4 + i], rdst, vs, u.dst + (qi * 4u + i) * stp, kAuxSys);
      FENCE();
    };
    auto wait_prod = [&](unsigned seq, const Unit& u) {
      const unsigned ps = 4u * seq + u.own * c;
      for (int it = 0; lds_min4(kLdsMisc + 32u) <= ps; ++it) {
        if (it == 8) { WAIT_VM(0); send(f2, m2); send(f1, m1); }
        __builtin_amdgcn_s_sleep(1);
      }
    };
    auto feed = [&](auto kw_) {
      constexpr int kw = decltype(kw_)::value;
      Unit cur = decode(k);
      unsigned last = (tstart - 1000u) << 6;
      auto gate = [&](int qi) {
        WAIT_VM(4 * (kw - 1));
        if (qi == ((kw - 1) & 3)) { if (kw <= 4) send(f1, m1); else send(f2, m2); }
        const int nw = cross ? NC : NI, tau = cross ? TAUC : TAUI;
        if (tau != 0) {
          last += (unsigned)(nw * tau);
          int late;
          while ((late = (int)(((unsigned)__builtin_amdgcn_s_memrealtime() << 6) - last)) < 0) __builtin_amdgcn_s_sleep(1);
          if (late > CRED * 64) last += (unsigned)(late - CRED * 64);
        }
      };
      wait_prod(0u, cur);
      STAMP(myws, 11 + wid);
      rd(xa, 0u, cur, 0u);
      for (unsigned seq = 0; cur.m < total; ++seq) {
        const Unit nxt = decode(get_row(seq + 1u));
        gate(0); st(xa, cur, 0);
        rd(xb, seq, cur, 1u);
        gate(1); st(xa, cur, 1);
        WAIT_LGKM0();
        lds_put(kLdsMisc + 48u + c * 4u, seq + 1u);
        gate(2); st(xb, cur, 2);
        gate(3); st(xb, cur, 3);
        f2 = f1; f1 = cur.flg; m2 = m1; m1 = cur.m;
        if (nxt.m < total) { wait_prod(seq + 1u, nxt); rd(xa, seq + 1u, nxt, 0u); }
        cur = nxt;
      }
    };
    if (KWI == KWC || cross) feed(std::integral_constant<int, KWC>{});
    else feed(std::integral_constant<int, KWI>{});
    WAIT_VM(0);
    send(f2, m2);
    send(f1, m1);
    STAMP(myws, 1 + wid);
    return;
  }

  const bool isconv = ((bid - kNPush) >> 3) % CVMOD == 0;
  if (tid == 0) {
    const unsigned tk = draw(myws + kTicket);
    const unsigned cl = isconv ? draw(myws + kCvt) : 0u;
    misc[0] = tk; misc[1] = cl; misc[3] = 0u;
  }
  __syncthreads();
  const unsigned call = uni(misc[0]) / (unsigned)kNWG + 1u;
  const unsigned par = call & 1u;
  const unsigned abase = kAOff + par * kASize, bbase = kBOff + par * kBSize;
  {
    constexpr unsigned kNConv = 8u * ((26 + CVMOD - 1) / CVMOD);
    const unsigned cbase = (call - 1u) * (kItems + kNConv);
    unsigned cur = isconv ? uni(misc[1]) - cbase : kItems;
    f32x4 v[8];
    auto loads = [&](unsigned it) {
      const unsigned ktg = it >> 3, sub = it & 7u;
      if (sub < 4u) {
        const unsigned so = ((256u * sub + 32u * wid) * 8192u + ktg * 64u) * 4u;
#pragma unroll
        for (int s = 0; s < 2; ++s) {
          v[4 * s + 0] = ld16<0>(rx, lx, so + s * 524288u); v[4 * s + 1] = ld16<0>(rx, lx + 64u, so + s * 524288u);
          v[4 * s + 2] = ld16<0>(rx, lx + 128u, so + s * 524288u); v[4 * s + 3] = ld16<0>(rx, lx + 192u, so + s * 524288u);
        }
      } else {
        const unsigned so = ((ktg * 64u + 32u * (wid & 1)) * 1024u + 256u * (sub - 4u) + 64u * (wid >> 1)) * 4u;
#pragma unroll
        for (int i = 0; i < 8; ++i) v[i] = ld16<0>(rw, lw, so + ((i & 3) + 16 * (i >> 2)) * 4096u);
      }
    };
    if (cur < kItems) loads(cur);
    int flip = 0;
    while (cur < kItems) {
      unsigned nx = 0;
      if (tid == 0) nx = draw(myws + kCvt);
      const unsigned ktg = cur >> 3, sub = cur & 7u;
      if (sub < 4u) {
        const unsigned so = abase + (2u * sub + (wid >> 2)) * kPanel + ktg * kBlk + (wid & 3) * 4096u;
#pragma unroll
        for (int i = 0; i < 4; ++i) st16<kAuxDev>(cvt8(v[2 * i], v[2 * i + 1]), rws, l16 + i * 1024u, so);
      } else {
        const unsigned so = bbase + (2u * (sub - 4u) + (wid >> 2)) * kPanel + ktg * kBlk + ((wid >> 1) & 1) * 8192u + (wid & 1) * 1024u;
        st16<kAuxDev>(cvt_col(v, 0), rws, l16, so); st16<kAuxDev>(cvt_col(v, 1), rws, l16 + 2048u, so);
        st16<kAuxDev>(cvt_col(v, 2), rws, l16, so + 4096u); st16<kAuxDev>(cvt_col(v, 3), rws, l16 + 2048u, so + 4096u);
      }
      if (tid == 0) misc[4 + flip] = nx;
      WAIT_LGKM0();
      BARRIER();
      const unsigned nxt = uni(misc[4 + flip]) - cbase;
      flip ^= 1;
      if (nxt < kItems) { loads(nxt); WAIT_VM(8); } else { WAIT_VM(0); }
      BARRIER();
      if (tid < 2) flag_store(myws + kFlags + ((sub < 4u ? 2u * sub : 26u + 2u * sub) + (unsigned)tid) * 512u + ktg * 4u, call);
      cur = nxt;
    }
  }
  STAMP(myws, wid == 0 ? 1 : 31);

  const uint32_t tl0 = p.tiles[2 * (bid - kNPush)], tl1 = p.tiles[2 * (bid - kNPush) + 1];
  const int ntiles = (int)(tl1 >> 16);

  if (wid >= 4) {
    const bool isA = wid < 6;
    const unsigned lbase = (isA ? 0u : 16384u) + (unsigned)(wid & 1) * 8192u;
    LAS char* const scratch = lds + kLdsPoll + (wid - 4) * 256;
    const unsigned scr_addr = kLdsPoll + (unsigned)(wid - 4) * 256u + (unsigned)lane * 4u;
    bool bad = false;
    for (int ti = 0; ti < ntiles; ++ti) {
      const unsigned code = (ti == 0 ? tl0 : ti == 1 ? tl0 >> 16 : tl1) & 0xffffu;
      const unsigned ap = code & 63u, bp = (code >> 6) & 15u;
      const bool halft = (code >> 10) & 1u;
      if (halft && wid == 7) {
        for (int t = -1; t < 128; ++t) BARRIER();
        continue;
      }
      const unsigned half = (halft && !isA) ? (code >> 11) & 1u : (unsigned)(wid & 1);
      const unsigned sbase = (isA ? abase + ap * kPanel : bbase + bp * kPanel) + half * 8192u;
      const unsigned grp = isA ? ap : 34u + bp;
      auto pvoff = [&](int base) { return lane < 32 ? kFlags + grp * 512u + (unsigned)min(base + lane, 127) * 4u : kTmo; };
      int u = 0, okt = bad ? 128 : 0, up = 0;
      bool pend = false;
      auto issue = [&](int uu) {
        LAS char* const dl = lds + (uu & 3) * 32768 + lbase;
        const unsigned so = sbase + (unsigned)uu * kBlk;
        __builtin_amdgcn_raw_ptr_buffer_load_lds(rws, (LAS void*)dl, 16, l16, so, 0, kAuxDev);
        __builtin_amdgcn_raw_ptr_buffer_load_lds(rws, (LAS void*)dl, 16, l16, so, 1024, kAuxDev);
        __builtin_amdgcn_raw_ptr_buffer_load_lds(rws, (LAS void*)dl, 16, l16, so, 2048, kAuxDev);
        __builtin_amdgcn_raw_ptr_buffer_load_lds(rws, (LAS void*)dl, 16, l16, so, 3072, kAuxDev);
        __builtin_amdgcn_raw_ptr_buffer_load_lds(rws, (LAS void*)(dl + 4096), 16, l16, so + 4096u, 0, kAuxDev);
        __builtin_amdgcn_raw_ptr_buffer_load_lds(rws, (LAS void*)(dl + 4096), 16, l16, so + 4096u, 1024, kAuxDev);
        __builtin_amdgcn_raw_ptr_buffer_load_lds(rws, (LAS void*)(dl + 4096), 16, l16, so + 4096u, 2048, kAuxDev);
        __builtin_amdgcn_raw_ptr_buffer_load_lds(rws, (LAS void*)(dl + 4096), 16, l16, so + 4096u, 3072, kAuxDev);
      };
      for (int t = -1; t < 127; ++t) {
        if (u <= t + 1) {
          if (okt <= u) {
            const int run = wait_flags(myws, pvoff(okt), call, lane, 1);
            if (run < 0) bad = true;
            okt = run < 0 ? 128 : min(okt + run, 128);
            pend = false;
          }
          const int lim = min(t + 4, okt);
          while (u < lim) { issue(u); ++u; }
        }
        const int n = u - (t + 2);
        if (n >= 2) WAIT_VM(16); else if (n == 1) WAIT_VM(8); else WAIT_VM(0);
        if (pend && up <= t + 2) {
          unsigned v;
          asm volatile("ds_read_b32 %0, %1\n\ts_waitcnt lgkmcnt(0)" : "=v"(v) : "v"(scr_addr) : "memory");
          pend = false;
          const int run = ready_run(v, call, lane);
          if (run < 0) bad = true;
          okt = run < 0 ? 128 : min(okt + run, 128);
          const int lim = min(t + 4, okt);
          while (u < lim) { issue(u); ++u; }
        }
        BARRIER();
        if (!pend && okt < 128 && u + PDIST >= okt) {
          __builtin_amdgcn_raw_ptr_buffer_load_lds(rws, (LAS void*)scratch, 4, pvoff(okt), 0, 0, kAuxSys);
          pend = true; up = u;
        }
        const int lim = min(t + 5, okt);
        while (u < lim) { issue(u); ++u; }
      }
      if (bad) { if (lane == 0) misc[3] = 1u; WAIT_LGKM0(); }
      BARRIER();
    }
    return;
  }

  {
    const int fr = lane & 15, fq = lane >> 4;
    const unsigned slot = (unsigned)(4 * fr + (fq ^ ((-(fr >> 2)) & 3))) * 16u;
    const unsigned slotb = (unsigned)(16 * fq + 4 * (fr & 3) + (fr >> 2)) * 16u;
    unsigned hi = 65536u;
    asm volatile("" : "+v"(hi));
    for (int ti = 0; ti < ntiles; ++ti) {
      const unsigned code = (ti == 0 ? tl0 : ti == 1 ? tl0 >> 16 : tl1) & 0xffffu;
      const unsigned ap = code & 63u, bp = (code >> 6) & 15u;
      const bool halft = (code >> 10) & 1u, staged = (code >> 12) & 1u;
      const unsigned owner = (bp < 8u ? (unsigned)rank : (unsigned)((g ^ 1) * 4) + ((bp - 8u) >> 1)) & 7u;
      const unsigned col0 = bp < 8u ? bp * 128u : 256u * q + (bp & 1u) * 128u;
      const unsigned oslot = ap * 8u + (bp & 7u);
      const __amdgpu_buffer_rsrc_t ro = mk_rsrc(staged ? (void*)myws : (void*)p.out[owner], staged ? kWsNeed : kInBytes);
      const unsigned stride = staged ? 256u : 4096u;
      const unsigned orow = halft ? 32u * wid : 64u * (wid >> 1), ocol = halft ? 64u * ((code >> 11) & 1u) : 64u * (wid & 1);
      const unsigned opiece = oslot * 4u + (orow >> 6) * 2u + (ocol >> 6);
      const unsigned obase = staged ? kOOff + opiece * kBlk + (orow & 32u) * 256u : (panel_row(ap, rank) + orow) * 4096u + (col0 + ocol) * 4u;
      const unsigned vo = (unsigned)fr * stride + (unsigned)fq * 16u;

      auto run = [&](auto mconst, const unsigned aoff, const unsigned boff) {
        constexpr int M = decltype(mconst)::value;
        LAS char* const pa = lds + aoff + slot;
        LAS char* const pb = lds + boff + slotb;
        LAS char* const pa2 = pa + hi;
        LAS char* const pb2 = pb + hi;
        f32x4 acc[M][4];
#pragma unroll
        for (int m = 0; m < M; ++m)
#pragma unroll
          for (int n = 0; n < 4; ++n) acc[m][n] = f32x4{0.f, 0.f, 0.f, 0.f};
        f16x8 fa[2][2 * M], fb[2][8];
        auto rd = [&](int st, f16x8 (&a)[2 * M], f16x8 (&b)[8]) {
#pragma unroll
          for (int i = 0; i < 8; ++i) {
            if (i < 2 * M) a[i] = *(LAS f16x8*)((st < 2 ? pa : pa2) + (st & 1) * 32768 + i * 1024);
            b[i] = *(LAS f16x8*)((st < 2 ? pb : pb2) + (st & 1) * 32768 + i * 1024);
          }
        };
        auto mm = [&](const f16x8 (&a)[2 * M], const f16x8 (&b)[8]) {
#pragma unroll
          for (int kh = 0; kh < 2; ++kh)
#pragma unroll
            for (int m = 0; m < M; ++m)
#pragma unroll
              for (int n = 0; n < 4; ++n)
                acc[m][n] = __builtin_amdgcn_mfma_f32_16x16x32_f16(b[2 * n + kh], a[2 * m + kh], acc[m][n], 0, 0, 0);
        };
        BARRIER();
        STAMP(myws, wid == 0 ? 2 + 3 * ti : 31);
        rd(0, fa[0], fb[0]);
        for (int t = 0; t < 128; t += 4) {
#pragma unroll
          for (int s = 0; s < 4; ++s) {
            __builtin_amdgcn_s_waitcnt(0xC07F);
            BARRIER();
            __builtin_amdgcn_sched_barrier(0);
            rd((s + 1) & 3, fa[(s + 1) & 1], fb[(s + 1) & 1]);
            mm(fa[s & 1], fb[s & 1]);
#pragma unroll
            for (int i = 0; i < 8 + 2 * M; ++i) {
              __builtin_amdgcn_sched_group_barrier(0x100, 1, 0);
              __builtin_amdgcn_sched_group_barrier(0x008, 1, 0);
            }
            __builtin_amdgcn_sched_group_barrier(0x008, 8 * M - 8 - 2 * M, 0);
            __builtin_amdgcn_sched_barrier(0);
          }
        }
        STAMP(myws, wid == 0 ? 3 + 3 * ti : 31);
        const int lo = misc[3] != 0u ? 0x7fc00000 : 0;
        const unsigned so = obase;
#pragma unroll
        for (int m = 0; m < M; ++m)
#pragma unroll
          for (int j = 0; j < 4; ++j) {
            u32x4 o;
#pragma unroll
            for (int n = 0; n < 4; ++n) { const float f = acc[m][n][j]; o[n] = (unsigned)max(__float_as_int(f), lo); }
            __builtin_amdgcn_raw_buffer_store_b128(o, ro, vo + j * 64u, so + m * 16u * stride, kAuxSys);
          }
      };
      if (halft) run(std::integral_constant<int, 2>{}, (unsigned)wid * 4096u, 16384u);
      else run(std::integral_constant<int, 4>{}, (unsigned)(wid >> 1) * 8192u, 16384u + (unsigned)(wid & 1) * 8192u);
      if (bp >= 8u) {
        WAIT_VM(0);
        if (staged) {
          if (halft ? lane == (wid & 1) : lane < 2) flag_store(myws + kOFlags + opiece * 8u + (unsigned)lane * 4u, call);
        } else if (lane == 0) {
          __hip_atomic_fetch_add((GAS unsigned*)(p.ws[owner] + kDone), halft ? 1u : 2u, __ATOMIC_RELAXED, __HIP_MEMORY_SCOPE_SYSTEM);
        }
      }
      STAMP(myws, wid == 0 ? 4 + 3 * ti : 31);
    }
  }
}

extern "C" void kernel_launch(int rank, int nranks, ncclComm_t comm, hipStream_t stream,
                              void* const* d_in, const int* in_sizes, int n_in,
                              void* d_out, int out_size,
                              const int* in_dtypes, int out_dtype,
                              void* const* d_out_all, void* const* d_ws_all, size_t ws_bytes) {
  (void)comm; (void)in_dtypes; (void)out_dtype; (void)d_out;
  if (nranks != 8 || (unsigned)rank >= 8u || n_in < 2 || ws_bytes < (size_t)kWsNeed ||
      in_sizes[0] != 8 * 1024 * 1024 || in_sizes[1] != 8 * 1024 * 1024 || out_size != 8 * 1024 * 1024) {
    fprintf(stderr, "kernel_launch: unexpected problem geometry\n");
    return;
  }
  Params p;
  p.x = (const float*)d_in[0];
  p.w = (const float*)d_in[1];
  for (int i = 0; i < 8; ++i) { p.ws[i] = (char*)d_ws_all[i]; p.out[i] = (float*)d_out_all[i]; }
  p.rank = rank;
  p.pad = 0;
  static const int rowsel[6] = {8, 9, 14, 15, 20, 21};
  static const int xcol[8] = {0, 4, 2, 3, 1, 5, 6, 7};
  auto full = [](int a, int b) { return (unsigned)(a | (b << 6) | (b >= 8 ? 1 << 12 : 0)); };
  for (int ci = 0; ci < kNComp; ++ci) {
    const int x = ci & 7, s = ci >> 3;
    unsigned t0, t1, t2 = 0, n = 2;
    if (s < 24) t0 = full(x < 4 ? (s >> 2) : rowsel[s >> 2], 4 * (x & 3) + (s & 3));
    else t0 = full(6 + (s - 24), xcol[x]);
    if (s < 12) {
      t1 = full(rowsel[3 * (x >> 2) + (s >> 2)] + 2, 4 * (x & 3) + (s & 3));
    } else {
      const int e = s - 12, j = x >> 1, hf = x & 1;
      if (e < 8) t1 = full(26 + 2 * j + (e & 1), 4 * hf + (e >> 1));
      else {
        const int e2 = e - 8, idx = 3 * hf + (e2 >> 1);
        t1 = full(26 + 2 * j + (e2 & 1), 8 + idx + (idx >= 2 * j ? 2 : 0));
      }
    }
    if (s < 24) {
      const int e = s % 12, last = s / 12;
      t2 = (unsigned)((8 + 6 * (e >> 2) + 4 + last) | ((2 * x + ((e >> 1) & 1)) << 6) | (1 << 10) | ((e & 1) << 11));
      if (!last && x >= 4) t2 |= 1u << 12;
      n = 3;
    }
    p.tiles[2 * ci] = t0 | (t1 << 16);
    p.tiles[2 * ci + 1] = t2 | (n << 16);
  }
  hipError_t e = hipFuncSetAttribute((const void*)ag_gemm_relu_kernel, hipFuncAttributeMaxDynamicSharedMemorySize, (int)kLdsBytes);
  if (e != hipSuccess) { fprintf(stderr, "hipFuncSetAttribute: %s\n", hipGetErrorString(e)); return; }
  hipLaunchKernelGGL(ag_gemm_relu_kernel, dim3(kNWG), dim3(512), kLdsBytes, stream, p);
  e = hipGetLastError();
  if (e != hipSuccess) fprintf(stderr, "launch: %s\n", hipGetErrorString(e));
}
```

```cpp
#include <hip/hip_runtime.h>
#include <rccl/rccl.h>
#include <cstdio>
#include <cstdint>
#include <type_traits>

typedef _Float16 f16x8 __attribute__((ext_vector_type(8)));
typedef float f32x4 __attribute__((ext_vector_type(4)));
typedef unsigned u32x4 __attribute__((ext_vector_type(4)));
typedef unsigned u32x2 __attribute__((ext_vector_type(2)));
#define LAS __attribute__((address_space(3)))
#define GAS __attribute__((address_space(1)))

#define NWS 9u
#define WPM 3u
#define CVMOD 1
#define PBURST 5u
#define CVAUX kAuxSys
#define MSLEEP 4
#define LSLEEP 1
#define PFD 0
#define PDIST (PFD / 4 + 1)

namespace {

constexpr unsigned kBlk = 16384u;
constexpr unsigned kPanel = 2u << 20;
constexpr unsigned kAOff = 1u << 20;
constexpr unsigned kASize = 36u * kPanel;
constexpr unsigned kBOff = kAOff + 2u * kASize;
constexpr unsigned kBSize = 16u * kPanel;
constexpr unsigned kPiece = 16896u;
constexpr unsigned kOOff = kBOff + 2u * kBSize;
constexpr unsigned kOSize = 4u * 224u * kPiece;
constexpr unsigned kWsNeed = kOOff + 2u * kOSize;
constexpr unsigned kTicket = 0, kTmo = 128, kDone = 256, kCvt = 384, kQueue = 1024;
constexpr unsigned kFlags = 16384;
constexpr unsigned kOFlags = 81920;
constexpr unsigned kSrc = 98304;
constexpr unsigned kInBytes = 32u << 20;
constexpr int kNMove = 48, kNComp = 208, kNWG = 256;
constexpr unsigned kItems = 1024;
constexpr unsigned kRowsI = 704u, kRowsC = 688u;
constexpr unsigned kDonePerCall = 192u;
constexpr unsigned long long kDeadline = 50000000ull;
constexpr unsigned kLdsMisc = 131072u, kLdsScr = 131136u, kLdsBytes = 132160u;
constexpr unsigned kOob = 0x80000000u;
constexpr int kAuxSys = 17;
constexpr int kAuxDev = 16;

struct Params {
  const float* x;
  const float* w;
  char* ws[8];
  float* out[8];
  int rank;
  int pad;
  uint32_t tiles[3 * kNComp];
};

__device__ __forceinline__ __amdgpu_buffer_rsrc_t mk_rsrc(const void* p, unsigned bytes) {
  return __builtin_amdgcn_make_buffer_rsrc((void*)p, 0, bytes, 0x00020000);
}
template <int AUX>
__device__ __forceinline__ f32x4 ld16(__amdgpu_buffer_rsrc_t r, unsigned voff, unsigned soff) {
  return __builtin_bit_cast(f32x4, __builtin_amdgcn_raw_buffer_load_b128(r, voff, soff, AUX));
}
template <int AUX>
__device__ __forceinline__ void st16(f16x8 v, __amdgpu_buffer_rsrc_t r, unsigned voff, unsigned soff) {
  __builtin_amdgcn_raw_buffer_store_b128(__builtin_bit_cast(u32x4, v), r, voff, soff, AUX);
}
__device__ __forceinline__ f16x8 cvt8(f32x4 a, f32x4 b) {
  f16x8 h;
#pragma unroll
  for (int i = 0; i < 4; ++i) { h[i] = (_Float16)a[i]; h[4 + i] = (_Float16)b[i]; }
  return h;
}
__device__ __forceinline__ f16x8 cvt_col(const f32x4 (&v)[8], int i) {
  f16x8 h;
#pragma unroll
  for (int j = 0; j < 8; ++j) h[j] = (_Float16)v[j][i];
  return h;
}
#define WAIT_VM(n) asm volatile("s_waitcnt vmcnt(%0)" :: "n"(n) : "memory")
#define WAIT_LGKM0() asm volatile("s_waitcnt lgkmcnt(0)" ::: "memory")
#define FENCE() asm volatile("" ::: "memory")
#define BARRIER() do { FENCE(); __builtin_amdgcn_s_barrier(); FENCE(); } while (0)

__device__ __forceinline__ void flag_store(char* p, unsigned v) {
  __hip_atomic_store((GAS unsigned*)p, v, __ATOMIC_RELAXED, __HIP_MEMORY_SCOPE_SYSTEM);
}
__device__ __forceinline__ void flag_store2(char* p, unsigned lo, unsigned hi) {
  __hip_atomic_store((GAS unsigned long long*)p, (unsigned long long)lo | ((unsigned long long)hi << 32), __ATOMIC_RELAXED, __HIP_MEMORY_SCOPE_SYSTEM);
}
__device__ __forceinline__ unsigned draw(char* p) {
  unsigned z = 0;
  asm volatile("" : "+v"(z));
  p += z;
  return __hip_atomic_fetch_add((GAS unsigned*)p, 1u, __ATOMIC_RELAXED, __HIP_MEMORY_SCOPE_AGENT);
}
__device__ __forceinline__ unsigned uni(unsigned v) { return (unsigned)__builtin_amdgcn_readfirstlane((int)v); }
__device__ __forceinline__ unsigned lds_min4(unsigned addr) {
  u32x4 c;
  asm volatile("s_mov_b64 exec, 1\n\tds_read_b128 %0, %1\n\ts_mov_b64 exec, -1\n\ts_waitcnt lgkmcnt(0)" : "=v"(c) : "v"(addr) : "memory");
  return uni(min(min(c[0], c[1]), min(c[2], c[3])));
}
__device__ __forceinline__ unsigned lds_get(unsigned addr) {
  unsigned c;
  asm volatile("s_mov_b64 exec, 1\n\tds_read_b32 %0, %1\n\ts_mov_b64 exec, -1\n\ts_waitcnt lgkmcnt(0)" : "=v"(c) : "v"(addr) : "memory");
  return uni(c);
}
__device__ __forceinline__ void lds_put(unsigned addr, unsigned v) {
  asm volatile("s_mov_b64 exec, 1\n\tds_write_b32 %0, %1\n\ts_mov_b64 exec, -1" :: "v"(addr), "v"(v) : "memory");
}
__device__ __forceinline__ unsigned lanes_below(unsigned long long b) {
  return __builtin_amdgcn_mbcnt_hi((unsigned)(b >> 32), __builtin_amdgcn_mbcnt_lo((unsigned)b, 0u));
}

#define STAMP(ws, slot) do {} while (0)
#define PSTAMP(ws, cls, n) do {} while (0)
#define CSTAMP(ws, slot) do {} while (0)

__device__ __forceinline__ void set_timeout(char* myws, unsigned call) {
  __hip_atomic_store((GAS unsigned*)(myws + kTmo), call, __ATOMIC_RELAXED, __HIP_MEMORY_SCOPE_AGENT);
}
__device__ __forceinline__ unsigned long long ready_bits(unsigned v, unsigned call, int lane) {
  return __ballot(lane < 62 ? (int)(v - call) >= 0 : v == call);
}
__device__ __forceinline__ unsigned long long wait_flags(char* myws, unsigned voff, unsigned call, int lane) {
  const unsigned long long t0 = __builtin_amdgcn_s_memrealtime();
  for (;;) {
    const unsigned v = __hip_atomic_load((GAS unsigned*)(myws + voff), __ATOMIC_RELAXED, __HIP_MEMORY_SCOPE_SYSTEM);
    const unsigned long long b = ready_bits(v, call, lane);
    if ((b & 0x300000003ull) == 0x300000003ull || (b >> 62) != 0ull) return b;
    if (__builtin_amdgcn_s_memrealtime() - t0 > kDeadline) {
      set_timeout(myws, call);
      return ~0ull;
    }
    __builtin_amdgcn_s_sleep(2);
  }
}

__device__ __forceinline__ unsigned peer_of(unsigned r, unsigned i) { return (r & 4u) + ((r + i + 1u) & 3u); }
__device__ __forceinline__ unsigned panel_row(unsigned ap, unsigned rh, unsigned rank) {
  const unsigned og = (rank & 4u) ^ 4u;
  if (ap < 8u) return rank * 1024u + ap * 128u + rh * 64u;
  if (ap < 23u) { const unsigned i = (ap - 8u) / 5u, s = (ap - 8u) - 5u * i; return peer_of(rank, i) * 1024u + s * 128u + rh * 64u; }
  if (ap < 26u) return peer_of(rank, ap - 23u) * 1024u + 640u;
  if (ap < 34u) return (og + ((ap - 26u) >> 1)) * 1024u + 768u + (ap & 1u) * 128u + rh * 64u;
  return (og + 2u * (ap - 34u) + rh) * 1024u + 704u;
}
__device__ __forceinline__ unsigned park_pos(unsigned ap, unsigned rh, unsigned j) {
  if (ap < 8u) return 2u * ap + rh < 11u ? 2u * ap + rh : 99u;
  if (ap < 23u) {
    const unsigned i = (ap - 8u) / 5u, s = (ap - 8u) - 5u * i;
    return s < 2u ? 11u + 4u * i + 2u * s + rh : s < 4u ? 40u + 4u * i + 2u * s + rh : 26u + 2u * i + rh;
  }
  const unsigned src = ap < 34u ? (ap - 26u) >> 1 : 2u * (ap - 34u) + rh;
  if (src == j) return 99u;
  const unsigned e = src - (src > j ? 1u : 0u);
  return ap < 34u ? 32u + 4u * e + 2u * (ap & 1u) + rh : 23u + e;
}
__device__ __forceinline__ unsigned park_row(unsigned hp, unsigned src, unsigned rank) {
  const unsigned q = rank & 3u;
  if (hp < 11u) return src * 1024u + hp * 64u;
  if (hp < 23u) return peer_of(src, (hp - 11u) >> 2) * 1024u + ((hp - 11u) & 3u) * 64u;
  if (hp < 26u) { const unsigned e = hp - 23u; return ((rank & 4u) + e + (e >= q ? 1u : 0u)) * 1024u + 704u; }
  if (hp < 32u) return peer_of(src, (hp - 26u) >> 1) * 1024u + 512u + (hp & 1u) * 64u;
  if (hp < 44u) { const unsigned e = (hp - 32u) >> 2; return ((rank & 4u) + e + (e >= q ? 1u : 0u)) * 1024u + 768u + (hp & 3u) * 64u; }
  return peer_of(src, (hp - 44u) >> 2) * 1024u + 256u + (hp & 3u) * 64u;
}

}

__global__ void __launch_bounds__(512) __attribute__((amdgpu_waves_per_eu(2, 2)))
ag_gemm_relu_kernel(const Params p) {
  extern __shared__ __attribute__((aligned(16))) char smem_raw[];
  LAS char* const lds = (LAS char*)smem_raw;
  LAS volatile unsigned* const misc = (LAS volatile unsigned*)(lds + kLdsMisc);

  const int tid = threadIdx.x;
  const int lane = tid & 63;
  const int wid = __builtin_amdgcn_readfirstlane(tid >> 6);
  const int bid = blockIdx.x;
  const unsigned rank = (unsigned)p.rank & 7u;
  const unsigned q = rank & 3u, og = (rank & 4u) ^ 4u;
  char* const myws = p.ws[rank];
  const unsigned l16 = (unsigned)lane * 16u;
  const __amdgpu_buffer_rsrc_t rws = mk_rsrc(myws, kWsNeed);
  const int fr = lane & 15, fq = lane >> 4;

  STAMP(myws, wid == 0 ? 0 : 31);

  const bool isconv = bid >= kNMove && ((bid - kNMove) >> 3) % CVMOD == 0;
  if (tid == 0) {
    const unsigned tk = draw(myws + kTicket);
    const unsigned cl = isconv ? draw(myws + kCvt) : 0u;
    misc[0] = tk; misc[1] = cl;
  } else if (tid >= 64 && tid < 77) {
    misc[tid - 61] = 0u;
  }
  __syncthreads();
  const unsigned call = uni(misc[0]) / (unsigned)kNWG + 1u;
  const unsigned par = call & 1u;
  const unsigned abase = kAOff + par * kASize, bbase = kBOff + par * kBSize, obuf = kOOff + par * kOSize;

  if (bid < kNMove) {
    const unsigned gw = (unsigned)bid * WPM + (unsigned)wid;
    if (wid == 7) {
      if (bid == 0 && lane == 0) {
        const unsigned target = call * kDonePerCall;
        const unsigned long long t0 = __builtin_amdgcn_s_memrealtime();
        bool failed = false;
        while ((int)(__hip_atomic_load((GAS unsigned*)(myws + kDone), __ATOMIC_RELAXED, __HIP_MEMORY_SCOPE_SYSTEM) - target) < 0) {
          if (__hip_atomic_load((GAS unsigned*)(myws + kTmo), __ATOMIC_RELAXED, __HIP_MEMORY_SCOPE_AGENT) == call ||
              __builtin_amdgcn_s_memrealtime() - t0 > kDeadline) {
            set_timeout(myws, call);
            failed = true;
            break;
          }
          __builtin_amdgcn_s_sleep(2);
        }
        __builtin_amdgcn_fence(__ATOMIC_ACQUIRE, "");
        if (failed) __hip_atomic_store((GAS unsigned*)(p.out[rank] + (size_t)rank * 1024u * 1024u), 0x7fc00000u, __ATOMIC_RELAXED, __HIP_MEMORY_SCOPE_SYSTEM);
        STAMP(myws, 10);
      }
      return;
    }
    if ((unsigned)wid >= WPM || gw >= 14u * NWS) return;
    const unsigned sid = gw % 14u, k = gw / 14u;
    const bool cross = sid >= 6u;
    const unsigned c = cross ? (sid - 6u) >> 1 : sid >> 1, h = sid & 1u;
    const unsigned sr = (cross ? og + c : peer_of(rank, c)) & 7u;
    char* const sws = p.ws[sr];
    const __amdgpu_buffer_rsrc_t rs = mk_rsrc(sws, kWsNeed);
    const __amdgpu_buffer_rsrc_t ro = mk_rsrc(p.out[rank], kInBytes);
    const unsigned total = cross ? kRowsC : kRowsI;
    const unsigned nplain = cross ? 576u : kRowsI;
    const unsigned qbase = (call - 1u) * total - NWS;
    bool failed = false;
    unsigned cgrp = 99u, okt = 0, wbase = 0x10000u;
    u32x2 fw = {0u, 0u};
    unsigned sa = 0, da = 0, fa = 0, pend = 0;
    u32x4 v[4], w[4];
    auto raise = [&]() {
      if (pend != 0u && lane == 0) flag_store(myws + pend, call);
      pend = 0u;
    };
    auto fetch = [&](u32x4 (&x)[4], unsigned so) {
#pragma unroll
      for (int i = 0; i < 4; ++i) x[i] = __builtin_amdgcn_raw_buffer_load_b128(rs, l16 + i * 1024u, so, kAuxSys);
      FENCE();
    };
    auto put = [&](const u32x4 (&x)[4], unsigned so) {
#pragma unroll
      for (int i = 0; i < 4; ++i) __builtin_amdgcn_raw_buffer_store_b128(x[i], rws, l16 + i * 1024u, so, kAuxDev);
      FENCE();
    };
    auto setup = [&](unsigned m) {
      unsigned sp, kt, so, dp, db, dh = h;
      if (!cross) {
        if (m < 256u) { sp = m & 1u; kt = m >> 1; so = h; dp = 8u + 5u * c + sp; db = kt; }
        else if (m < 384u) { sp = 4u; kt = m - 256u; so = h; dp = 12u + 5u * c; db = kt; }
        else if (m < 640u) { sp = 2u + (m & 1u); kt = (m - 384u) >> 1; so = h; dp = 8u + 5u * c + sp; db = kt; }
        else { sp = 5u; db = m - 640u; kt = 2u * db + h; so = 0u; dp = 23u + c; }
      } else {
        if (m < 256u) { sp = 36u + 2u * q + (m & 1u); kt = m >> 1; so = h; dp = 44u + 2u * c + (m & 1u); db = kt; }
        else if (m < 320u) { sp = 5u; kt = 2u * (m - 256u) + h; so = 1u; dp = 34u + (c >> 1); db = kt; dh = c & 1u; }
        else { sp = 6u + (m & 1u); kt = (m - 320u) >> 1; so = h; dp = 26u + 2u * c + (m & 1u); db = kt; }
      }
      const unsigned sub = sp < 36u ? sp >> 1 : (sp - 28u) >> 1;
      if (!failed && !(sub == cgrp && kt < okt)) {
        WAIT_VM(0);
        raise();
        char* const pa = myws + (lane < 63 ? kSrc + (sr * 8u + sub) * 512u + min(kt + (unsigned)lane, 127u) * 4u : kTmo);
        const unsigned long long t0 = __builtin_amdgcn_s_memrealtime();
        for (;;) {
          const unsigned f = __hip_atomic_load((GAS unsigned*)pa, __ATOMIC_RELAXED, __HIP_MEMORY_SCOPE_SYSTEM);
          const unsigned long long b = __ballot(lane < 63 ? (int)(f - call) >= 0 : f == call);
          if ((b >> 63) != 0ull) { failed = true; break; }
          if (b & 1ull) { cgrp = sub; okt = kt + (unsigned)__builtin_ctzll(~b); break; }
          if (__builtin_amdgcn_s_memrealtime() - t0 > kDeadline) { set_timeout(myws, call); failed = true; break; }
          __builtin_amdgcn_s_sleep(MSLEEP);
        }
      }
      sa = (sp < 36u ? abase + sp * kPanel : bbase + (sp - 36u) * kPanel) + kt * kBlk + so * 8192u;
      da = (dp < 36u ? abase + dp * kPanel : bbase + (dp - 36u) * kPanel) + db * kBlk + dh * 8192u;
      fa = kFlags + dp * 1024u + db * 8u + dh * 4u;
      fetch(v, sa);
    };
    unsigned m = k;
    setup(m);
    do {
      unsigned dv = 0;
      if (lane == 0) dv = draw(myws + kQueue + sid * 128u);
      FENCE();
      put(v, da);
      raise();
      fetch(w, sa + 4096u);
      put(w, da + 4096u);
      pend = fa;
      if (h == 0u) { PSTAMP(myws, sid >> 1, m + 1u); if (m < 128u) PSTAMP(myws, 8u + (sid >> 1), (m + 1u) * 4u); }
      m = uni(dv) - qbase;
      if (m < nplain) setup(m);
    } while (m < nplain);
    WAIT_VM(0);
    raise();
    {
      bool have = false;
      unsigned buf = (unsigned)wid * 2u * 17408u, oa = 0;
      const unsigned bsum = 2u * buf + 17408u;
      const unsigned vo = (unsigned)fr * 4096u + (unsigned)fq * 16u;
      for (;;) {
        const bool more = m < total;
        const unsigned o = m - 576u, s = 2u * o + h;
        bool ready = more;
        unsigned cnt = 2048u, dv = 0;
        if (more) {
          unsigned idx = o - wbase;
          if (idx >= 32u || (int)((unsigned)__builtin_amdgcn_readlane((int)fw[0], (int)idx) - call) < 0) {
            char* const pa = myws + (lane < 32 ? kOFlags + par * 8192u + c * 2048u + min(s + 2u * (unsigned)lane, 255u) * 8u : kTmo);
            const unsigned long long t0 = __builtin_amdgcn_s_memrealtime();
            wbase = o;
            idx = 0u;
            while (!failed) {
              fw = __builtin_bit_cast(u32x2, __hip_atomic_load((GAS unsigned long long*)pa, __ATOMIC_RELAXED, __HIP_MEMORY_SCOPE_SYSTEM));
              const unsigned long long b = __ballot(lane < 32 ? (int)(fw[0] - call) >= 0 : fw[0] == call);
              if ((b >> 32) != 0ull) { failed = true; break; }
              if (b & 1ull) break;
              if (have) { ready = false; break; }
              if (__builtin_amdgcn_s_memrealtime() - t0 > kDeadline) { set_timeout(myws, call); failed = true; }
              __builtin_amdgcn_s_sleep(MSLEEP);
            }
          }
          cnt = failed ? 0u : min((unsigned)__builtin_amdgcn_readlane((int)fw[1], (int)idx), 4096u);
        }
        const unsigned bytes = 512u + 4u * cnt, sb = obuf + (q * 224u + s) * kPiece, nbuf = bsum - buf;
        auto burst = [&](unsigned f0, unsigned f1) {
          for (unsigned f = f0; f < f1; f += 1024u)
            asm volatile("s_mov_b32 m0, %0\n\ts_nop 0\n\tbuffer_load_dwordx4 %1, %2, %3 offen sc0 sc1 lds"
                         :: "s"(nbuf + f), "v"(f + l16 < bytes ? l16 : kOob), "s"(rs), "s"(sb + f) : "memory");
        };
        LAS char* const stage = lds + buf;
        u32x2 mk = {0u, 0u};
        unsigned pos = 512u;
        const unsigned fill = failed ? 0x7fc00000u : 0u;
        auto expand = [&](int hf) {
          unsigned val[32];
#pragma unroll
          for (int i = 0; i < 32; ++i) {
            const unsigned long long b = __ballot((mk[hf] >> i) & 1u);
            val[i] = *(LAS unsigned*)(stage + pos + lanes_below(b) * 4u);
            pos += 4u * (unsigned)__builtin_popcountll(b);
          }
#pragma unroll
          for (int i = 0; i < 8; ++i) {
            u32x4 ov;
#pragma unroll
            for (int n = 0; n < 4; ++n) ov[n] = (mk[hf] >> (4 * i + n)) & 1u ? val[4 * i + n] : fill;
            __builtin_amdgcn_raw_buffer_store_b128(ov, ro, vo + (i & 3) * 64u, oa + (2 * hf + (i >> 2)) * 65536u, kAuxSys);
          }
          FENCE();
        };
        if (ready) {
          asm volatile("s_mov_b64 exec, 1\n\tglobal_atomic_add %0, %1, %2, %3 sc0\n\ts_mov_b64 exec, -1"
                       : "=v"(dv) : "v"(kQueue + sid * 128u), "v"(1u), "s"(myws) : "memory");
          burst(0u, min(bytes, PBURST * 1024u));
        }
        if (have) {
          if (!failed) mk = *(LAS u32x2*)(stage + lane * 8);
          expand(0);
          WAIT_VM(8);
        } else {
          WAIT_VM(0);
        }
        if (ready) burst(PBURST * 1024u, bytes);
        if (have) {
          expand(1);
          WAIT_VM(8);
        } else {
          WAIT_VM(0);
        }
        have = ready;
        if (ready) {
          if (h == 0u) PSTAMP(myws, sid >> 1, m + 1u);
          oa = park_row(s >> 2, sr, rank) * 4096u + (256u * c + (s & 3u) * 64u) * 4u;
          buf = nbuf;
          asm volatile("" : "+v"(dv));
          m = uni(dv) - qbase;
        } else if (!more) {
          break;
        }
      }
    }
    STAMP(myws, 1 + wid);
    return;
  }

  {
    const unsigned kq = (unsigned)((lane & 3) ^ ((-(lane >> 4)) & 3));
    const unsigned rho = (unsigned)lane >> 2;
    const unsigned lx = (rho * 8192u + 4u * kq) * 4u;
    const unsigned lw = (4u * ((unsigned)lane >> 4) * 1024u + 4u * ((unsigned)lane & 15u)) * 4u;
    const __amdgpu_buffer_rsrc_t rx = mk_rsrc(p.x, kInBytes);
    const __amdgpu_buffer_rsrc_t rw = mk_rsrc(p.w, kInBytes);
    char* const pws = p.ws[lane & 7] + kSrc + rank * 4096u;
    const unsigned mine = 15u << (rank & 4u);
    constexpr unsigned kNConv = 8u * ((26 + CVMOD - 1) / CVMOD);
    const unsigned cbase = (call - 1u) * (kItems + kNConv);
    unsigned cur = isconv ? uni(misc[1]) - cbase : kItems;
    f32x4 v[8];
    auto loads = [&](unsigned it) {
      const unsigned ktg = it >> 3, sub = it & 7u;
      if (sub < 4u) {
        const unsigned so = ((256u * sub + 32u * wid) * 8192u + ktg * 64u) * 4u;
#pragma unroll
        for (int s = 0; s < 2; ++s) {
          v[4 * s + 0] = ld16<0>(rx, lx, so + s * 524288u); v[4 * s + 1] = ld16<0>(rx, lx + 64u, so + s * 524288u);
          v[4 * s + 2] = ld16<0>(rx, lx + 128u, so + s * 524288u); v[4 * s + 3] = ld16<0>(rx, lx + 192u, so + s * 524288u);
        }
      } else {
        const unsigned so = ((ktg * 64u + 32u * (wid & 1)) * 1024u + 256u * (sub - 4u) + 64u * (wid >> 1)) * 4u;
#pragma unroll
        for (int i = 0; i < 8; ++i) v[i] = ld16<0>(rw, lw, so + ((i & 3) + 16 * (i >> 2)) * 4096u);
      }
    };
    if (cur < kItems) loads(cur);
    int flip = 0;
    while (cur < kItems) {
      unsigned nx = 0;
      if (tid == 0) nx = draw(myws + kCvt);
      const unsigned ktg = cur >> 3, sub = cur & 7u;
      if (sub < 4u) {
        const unsigned so = abase + (2u * sub + (wid >> 2)) * kPanel + ktg * kBlk + (wid & 3) * 4096u;
#pragma unroll
        for (int i = 0; i < 4; ++i) st16<CVAUX>(cvt8(v[2 * i], v[2 * i + 1]), rws, l16 + i * 1024u, so);
      } else {
        const unsigned so = bbase + (2u * (sub - 4u) + (wid >> 2)) * kPanel + ktg * kBlk + ((wid >> 1) & 1) * 8192u + (wid & 1) * 1024u;
        st16<CVAUX>(cvt_col(v, 0), rws, l16, so); st16<CVAUX>(cvt_col(v, 1), rws, l16 + 2048u, so);
        st16<CVAUX>(cvt_col(v, 2), rws, l16, so + 4096u); st16<CVAUX>(cvt_col(v, 3), rws, l16 + 2048u, so + 4096u);
      }
      if (tid == 0) misc[4 + flip] = nx;
      WAIT_LGKM0();
      BARRIER();
      const unsigned nxt = uni(misc[4 + flip]) - cbase;
      flip ^= 1;
      if (nxt < kItems) { loads(nxt); WAIT_VM(8); } else { WAIT_VM(0); }
      BARRIER();
      if (tid < 2) flag_store2(myws + kFlags + ((sub < 4u ? 2u * sub : 28u + 2u * sub) + (unsigned)tid) * 1024u + ktg * 8u, call, call);
      else if (wid == 1 && lane < 8) {
        const unsigned to = sub < 2u ? mine : sub == 2u ? 255u : sub == 3u ? mine ^ 255u : 1u << (og + sub - 4u);
        if (((to & ~(1u << rank)) >> lane) & 1u) flag_store(pws + sub * 512u + ktg * 4u, call);
      }
      cur = nxt;
    }
  }
  STAMP(myws, wid == 0 ? 1 : 31);

  const uint32_t* const tl = p.tiles + 3 * (bid - kNMove);
  const int ntiles = tl[2] != 0u ? 3 : 2;
  constexpr unsigned kRdy = kLdsMisc + 32u, kFre = kLdsMisc + 48u;

  if (wid >= 4) {
    const unsigned lw = (unsigned)wid - 4u;
    LAS char* const dl = lds + lw * 32768u;
    const unsigned rdy_addr = kRdy + lw * 4u;
    LAS char* const scr = lds + kLdsScr + lw * 256u;
    const bool lb = lane >= 32;
    bool bad = false;
    unsigned gs = 0, fre = 0;
    for (int ti = 0; ti < ntiles; ++ti) {
      const unsigned code = tl[ti];
      const unsigned ap = code & 63u, bp = (code >> 6) & 15u;
      const bool small = (code >> 10) & 1u;
      const unsigned n = small ? 64u : 128u, ch = (code >> 11) & 1u;
      const unsigned sa = abase + ap * kPanel, sb = bbase + bp * kPanel + ch * 8192u;
      const unsigned bstep = small ? 2u * kBlk : kBlk, bhalf = small ? kBlk : 8192u;
      const unsigned fbase = kFlags + (lb ? 36u + bp : ap) * 1024u + (lb && small ? ch * 4u + (lane & 1) * 8u : (lane & 1) * 4u);
      const unsigned fsh = lb && small ? 4u : 3u;
      auto pvoff = [&](unsigned base) { return lane < 62 ? fbase + (min(base + (((unsigned)lane & 31u) >> 1) * 4u, n - 4u + lw) << fsh) : kTmo; };
      const unsigned la = (code >> 12) & 3u, lb2 = (code >> 16) & 3u;
      const unsigned ia = ((code >> 14) & 3u) * (128u >> la) + (unsigned)lane, ib = ((code >> 18) & 3u) * (128u >> lb2) + (unsigned)lane;
      const unsigned pfa = la != 0u && (unsigned)lane < (128u >> la) ? ia * 128u : kOob;
      const unsigned pfb = lb2 != 0u && (unsigned)lane < (128u >> lb2) ? (ib >> 6) * bhalf + (ib & 63u) * 128u : kOob;
      unsigned okt = bad ? n : lw, oka = lw, okb = lw;
      auto advance = [&](unsigned long long b) {
        if ((b >> 62) != 0ull) { bad = true; lds_put(kLdsMisc + 12u, 1u); okt = n; return; }
        oka = okt + 4u * ((unsigned)__builtin_ctz(~(unsigned)b | 0x80000000u) >> 1);
        okb = okt + 4u * ((unsigned)__builtin_ctz(~(unsigned)(b >> 32)) >> 1);
        okt = min(oka, okb);
      };
      for (unsigned u = lw; u < n; u += 4u) {
        if (okt <= u) advance(wait_flags(myws, pvoff(okt), call, lane));
        while ((int)(fre + 3u - (gs + u)) < 0) { fre = lds_min4(kFre); if ((int)(fre + 3u - (gs + u)) < 0) __builtin_amdgcn_s_sleep(LSLEEP); }
        const unsigned uu = u;
#pragma unroll
        for (int i = 0; i < 8; ++i) {
          const unsigned so = i < 4 ? sa + uu * kBlk + i * 4096u : sb + uu * bstep + (i >= 6 ? bhalf : 0u) + (i & 1) * 4096u;
          __builtin_amdgcn_raw_ptr_buffer_load_lds(rws, (LAS void*)(dl + i * 4096), 16, l16, so, 0, kAuxDev);
          __builtin_amdgcn_raw_ptr_buffer_load_lds(rws, (LAS void*)(dl + i * 4096), 16, l16, so, 1024, kAuxDev);
          __builtin_amdgcn_raw_ptr_buffer_load_lds(rws, (LAS void*)(dl + i * 4096), 16, l16, so, 2048, kAuxDev);
          __builtin_amdgcn_raw_ptr_buffer_load_lds(rws, (LAS void*)(dl + i * 4096), 16, l16, so, 3072, kAuxDev);
        }
        const unsigned up = u + PFD;
        auto ahead = [&]() {
        };
        constexpr int NPF = PFD > 0 ? 2 : 0;
        if (okt < n && okt <= u + 4u * PDIST) {
          const unsigned v = __hip_atomic_load((GAS unsigned*)(myws + pvoff(okt)), __ATOMIC_RELAXED, __HIP_MEMORY_SCOPE_SYSTEM);
          ahead();
          WAIT_VM(NPF + 1);
          lds_put(rdy_addr, gs + u + 1u);
          advance(ready_bits(v, call, lane));
        } else {
          ahead();
          WAIT_VM(NPF);
          lds_put(rdy_addr, gs + u + 1u);
        }
      }
      gs += n;
    }
    return;
  }

  {
    const unsigned slot = (unsigned)(4 * fr + (fq ^ ((-(fr >> 2)) & 3))) * 16u;
    const unsigned slotb = (unsigned)(16 * fq + 4 * (fr & 3) + (fr >> 2)) * 16u;
    const unsigned fre_addr = kFre + (unsigned)wid * 4u;
    unsigned hi = 65536u;
    asm volatile("" : "+v"(hi));
    unsigned gs = 0;
    auto need = [&](unsigned known, int st, unsigned target) {
      while ((int)(known - target) < 0) known = lds_get(kRdy + 4u * st);
    };
    for (int ti = 0; ti < ntiles; ++ti) {
      const unsigned code = tl[ti];
      const unsigned ap = code & 63u, bp = (code >> 6) & 15u;
      const bool small = (code >> 10) & 1u;
      const unsigned jo = (bp - 8u) >> 1;
      const unsigned owner = (bp < 8u ? rank : og + jo) & 7u;
      const unsigned col0 = bp < 8u ? bp * 128u : 256u * q + (bp & 1u) * 128u;
      const unsigned rh = (unsigned)wid >> 1;
      const __amdgpu_buffer_rsrc_t ro = mk_rsrc(p.out[owner], kInBytes);
      const unsigned obase = small ? (panel_row(ap, 0u, rank) + 16u * wid) * 4096u + (col0 + 64u * ((code >> 11) & 1u)) * 4u
                                   : panel_row(ap, rh, rank) * 4096u + (col0 + 64u * (wid & 1)) * 4u;
      const unsigned vo = (unsigned)fr * 4096u + (unsigned)fq * 16u;

      auto run = [&](auto mconst, const unsigned aoff, const unsigned boff) {
        constexpr int M = decltype(mconst)::value;
        constexpr int KS = M == 4 ? 2 : 4;
        constexpr int NS = M == 4 ? 128 : 64;
        constexpr int NA = M * KS, NB = 4 * KS;
        LAS char* const pa = lds + aoff + slot;
        LAS char* const pb = lds + boff + slotb;
        LAS char* const pa2 = pa + hi;
        LAS char* const pb2 = pb + hi;
        f32x4 acc[M][4];
#pragma unroll
        for (int m = 0; m < M; ++m)
#pragma unroll
          for (int n = 0; n < 4; ++n) acc[m][n] = f32x4{0.f, 0.f, 0.f, 0.f};
        f16x8 fa[2][NA], fb[2][NB];
        unsigned chk;
        auto rd = [&](int st, f16x8 (&a)[NA], f16x8 (&b)[NB]) {
#pragma unroll
          for (int i = 0; i < NB; ++i) {
            const int o = M == 4 ? i * 1024 : ((i & 3) >> 1) * 8192 + (i >> 2) * 2048 + (i & 1) * 1024;
            if (i < NA) a[i] = *(LAS f16x8*)((st < 2 ? pa : pa2) + (st & 1) * 32768 + o);
            b[i] = *(LAS f16x8*)((st < 2 ? pb : pb2) + (st & 1) * 32768 + o);
          }
          chk = *(LAS volatile unsigned*)(lds + kRdy + 4u * ((st + 1) & 3));
        };
        auto mm = [&](const f16x8 (&a)[NA], const f16x8 (&b)[NB]) {
#pragma unroll
          for (int ks = 0; ks < KS; ++ks)
#pragma unroll
            for (int m = 0; m < M; ++m)
#pragma unroll
              for (int n = 0; n < 4; ++n)
                acc[m][n] = __builtin_amdgcn_mfma_f32_16x16x32_f16(b[KS * n + ks], a[KS * m + ks], acc[m][n], 0, 0, 0);
        };
        need(gs, 0, gs + 1u);
        STAMP(myws, wid == 0 ? 2 + 3 * ti : 31);
        CSTAMP(myws, wid == 0 ? 12 + 2 * ti : 31);
        rd(0, fa[0], fb[0]);
        for (int t = 0; t < NS; t += 4) {
#pragma unroll
          for (int s = 0; s < 4; ++s) {
            __builtin_amdgcn_s_waitcnt(0xC07F);
            lds_put(fre_addr, gs + (unsigned)(t + s + 1));
            need(uni(chk), (s + 1) & 3, gs + (unsigned)(s == 3 ? min(t + 5, NS - 3) : t + s + 2));
            __builtin_amdgcn_sched_barrier(0);
            rd((s + 1) & 3, fa[(s + 1) & 1], fb[(s + 1) & 1]);
            mm(fa[s & 1], fb[s & 1]);
            if constexpr (M == 4) {
#pragma unroll
              for (int i = 0; i < 17; ++i) {
                __builtin_amdgcn_sched_group_barrier(0x100, 1, 0);
                __builtin_amdgcn_sched_group_barrier(0x008, 1, 0);
              }
              __builtin_amdgcn_sched_group_barrier(0x008, 15, 0);
            } else {
#pragma unroll
              for (int i = 0; i < 5; ++i) {
                __builtin_amdgcn_sched_group_barrier(0x100, 2, 0);
                __builtin_amdgcn_sched_group_barrier(0x008, 1, 0);
              }
#pragma unroll
              for (int i = 0; i < 11; ++i) {
                __builtin_amdgcn_sched_group_barrier(0x100, 1, 0);
                __builtin_amdgcn_sched_group_barrier(0x008, 1, 0);
              }
            }
            __builtin_amdgcn_sched_barrier(0);
          }
        }
        gs += (unsigned)NS;
        STAMP(myws, wid == 0 ? 3 + 3 * ti : 31);
        CSTAMP(myws, wid == 0 ? 13 + 2 * ti : 31);
        const int lo = misc[3] != 0u ? 0x7fc00000 : 0;
        if (M == 4 && bp >= 8u) {
          const unsigned hp = park_pos(ap, rh, jo);
          if (hp < 56u) {
            const unsigned s = hp * 4u + (bp & 1u) * 2u + ((unsigned)wid & 1u);
            const unsigned pb0 = obuf + (jo * 224u + s) * kPiece;
            unsigned pos = pb0 + 512u;
            u32x2 mk = {0u, 0u};
#pragma unroll
            for (int m = 0; m < M; ++m)
#pragma unroll
              for (int j = 0; j < 4; ++j)
#pragma unroll
                for (int n = 0; n < 4; ++n) {
                  const float f = acc[m][n][j];
                  const int iv = max(__float_as_int(f), lo);
                  const unsigned long long b = __ballot(iv > 0);
                  __builtin_amdgcn_raw_buffer_store_b32((unsigned)iv, rws, iv > 0 ? lanes_below(b) * 4u : kOob, pos, kAuxSys);
                  pos += 4u * (unsigned)__builtin_popcountll(b);
                  mk[m >> 1] |= iv > 0 ? 1u << ((m & 1) * 16 + j * 4 + n) : 0u;
                }
            __builtin_amdgcn_raw_buffer_store_b64(mk, rws, (unsigned)lane * 8u, pb0, kAuxSys);
            WAIT_VM(0);
            if (lane == 0) flag_store2(p.ws[owner] + kOFlags + par * 8192u + q * 2048u + s * 8u, call, (pos - pb0 - 512u) >> 2);
          }
        } else {
#pragma unroll
          for (int m = 0; m < M; ++m)
#pragma unroll
            for (int j = 0; j < 4; ++j) {
              u32x4 o;
#pragma unroll
              for (int n = 0; n < 4; ++n) { const float f = acc[m][n][j]; o[n] = (unsigned)max(__float_as_int(f), lo); }
              __builtin_amdgcn_raw_buffer_store_b128(o, ro, vo + j * 64u, obase + m * 65536u, kAuxSys);
            }
          if (bp >= 8u) {
            WAIT_VM(0);
            if (lane == 0) __hip_atomic_fetch_add((GAS unsigned*)(p.ws[owner] + kDone), 1u, __ATOMIC_RELAXED, __HIP_MEMORY_SCOPE_SYSTEM);
          }
        }
      };
      if (small) run(std::integral_constant<int, 1>{}, (unsigned)wid * 2048u, 16384u);
      else run(std::integral_constant<int, 4>{}, (unsigned)(wid >> 1) * 8192u, 16384u + (unsigned)(wid & 1) * 8192u);
      STAMP(myws, wid == 0 ? 4 + 3 * ti : 31);
    }
  }
}

extern "C" void kernel_launch(int rank, int nranks, ncclComm_t comm, hipStream_t stream,
                              void* const* d_in, const int* in_sizes, int n_in,
                              void* d_out, int out_size,
                              const int* in_dtypes, int out_dtype,
                              void* const* d_out_all, void* const* d_ws_all, size_t ws_bytes) {
  (void)comm; (void)in_dtypes; (void)out_dtype; (void)d_out;
  if (nranks != 8 || (unsigned)rank >= 8u || n_in < 2 || ws_bytes < (size_t)kWsNeed ||
      in_sizes[0] != 8 * 1024 * 1024 || in_sizes[1] != 8 * 1024 * 1024 || out_size != 8 * 1024 * 1024) {
    fprintf(stderr, "kernel_launch: unexpected problem geometry\n");
    return;
  }
  Params p;
  p.x = (const float*)d_in[0];
  p.w = (const float*)d_in[1];
  for (int i = 0; i < 8; ++i) { p.ws[i] = (char*)d_ws_all[i]; p.out[i] = (float*)d_out_all[i]; }
  p.rank = rank;
  p.pad = 0;
  static const int rowsel[6] = {8, 9, 13, 14, 18, 19};
  static const int xcol[8] = {0, 4, 2, 3, 1, 5, 6, 7};
  auto tile = [](int a, int b) { return (unsigned)(a | (b << 6)); };
  auto share = [](int la, int ia, int lb, int ib) { return (unsigned)((la << 12) | (ia << 14) | (lb << 16) | (ib << 18)); };
  for (int ci = 0; ci < kNComp; ++ci) {
    const int x = ci & 7, s = ci >> 3;
    unsigned t0, t1, t2 = 0;
    if (s < 24) t0 = tile(x < 4 ? (s >> 2) : rowsel[s >> 2], 4 * (x & 3) + (s & 3)) | share(2, s & 3, 2, (s >> 2) & 3);
    else t0 = tile(6 + (s - 24), xcol[x]) | share(1, 0, 1, s & 1);
    if (s < 12) {
      const int bp = 2 * (x ^ 4) + (s & 1);
      const unsigned sh = share(1, s & 1, 2, (s >> 1) & 3);
      t1 = tile(rowsel[s >> 1] + 2, bp) | sh;
      if (s < 10) { t2 = t1; t1 = tile(s < 6 ? 12 + 5 * (s >> 1) : 31 + (s >> 1), bp) | sh; }
    } else {
      const int e = s - 12, j = x >> 1, hf = x & 1;
      if (e < 8) t1 = tile(26 + 2 * j + (e & 1), 4 * hf + (e >> 1));
      else {
        const int e2 = e - 8, idx = 3 * hf + (e2 >> 1);
        t1 = tile(26 + 2 * j + (e2 & 1), 8 + idx + (idx >= 2 * j ? 2 : 0));
      }
      if (e < 12) t2 = tile(23 + (e >> 2), 2 * x + ((e >> 1) & 1)) | (1u << 10) | ((unsigned)(e & 1) << 11) | share(2, e & 3, 1, (e >> 2) & 1);
    }
    p.tiles[3 * ci] = t0;
    p.tiles[3 * ci + 1] = t1;
    p.tiles[3 * ci + 2] = t2;
  }
  hipError_t e = hipFuncSetAttribute((const void*)ag_gemm_relu_kernel, hipFuncAttributeMaxDynamicSharedMemorySize, (int)kLdsBytes);
  if (e != hipSuccess) { fprintf(stderr, "hipFuncSetAttribute: %s\n", hipGetErrorString(e)); return; }
  hipLaunchKernelGGL(ag_gemm_relu_kernel, dim3(kNWG), dim3(512), kLdsBytes, stream, p);
  e = hipGetLastError();
  if (e != hipSuccess) fprintf(stderr, "launch: %s\n", hipGetErrorString(e));
}
```

```cpp
#include <hip/hip_runtime.h>
#include <rccl/rccl.h>
#include <cstdio>
#include <cstdint>
#include <type_traits>

typedef _Float16 f16x8 __attribute__((ext_vector_type(8)));
typedef float f32x4 __attribute__((ext_vector_type(4)));
typedef unsigned u32x4 __attribute__((ext_vector_type(4)));
typedef unsigned u32x2 __attribute__((ext_vector_type(2)));
#define LAS __attribute__((address_space(3)))
#define GAS __attribute__((address_space(1)))

#define NWS 9u
#define WPM 3u
#define XSLEEP 4
#define CVFROM 13
#define PWIN 4u
#define CVAUX kAuxSys
#define MSLEEP 4
#define LSLEEP 2
#define LTHR 24u

namespace {

constexpr unsigned kBlk = 16384u;
constexpr unsigned kPanel = 2u << 20;
constexpr unsigned kAOff = 1u << 20;
constexpr unsigned kASize = 36u * kPanel;
constexpr unsigned kBOff = kAOff + 2u * kASize;
constexpr unsigned kBSize = 16u * kPanel;
constexpr unsigned kPiece = 16896u;
constexpr unsigned kOOff = kBOff + 2u * kBSize;
constexpr unsigned kOSize = 4u * 224u * kPiece;
constexpr unsigned kWsNeed = kOOff + 2u * kOSize;
constexpr unsigned kTicket = 0, kTmo = 128, kDone = 256, kCvt = 384, kQueue = 1024;
constexpr unsigned kFlags = 16384;
constexpr unsigned kOFlags = 81920;
constexpr unsigned kSrc = 98304;
constexpr unsigned kInBytes = 32u << 20;
constexpr int kNMove = 48, kNComp = 208, kNWG = 256;
constexpr unsigned kItems = 1024;
constexpr unsigned kRowsI = 704u, kRowsC = 688u;
constexpr unsigned kDonePerCall = 192u;
constexpr unsigned long long kDeadline = 50000000ull;
constexpr unsigned kSlot = 24576u, kRing = 6u * kSlot;
constexpr unsigned kLdsMisc = kRing, kLdsBytes = kRing + 128u;
constexpr unsigned kOob = 0x80000000u;
constexpr int kAuxSys = 17;
constexpr int kAuxDev = 16;

struct Params {
  const float* x;
  const float* w;
  char* ws[8];
  float* out[8];
  int rank;
  int pad;
  uint32_t tiles[3 * kNComp];
};

__device__ __forceinline__ __amdgpu_buffer_rsrc_t mk_rsrc(const void* p, unsigned bytes) {
  return __builtin_amdgcn_make_buffer_rsrc((void*)p, 0, bytes, 0x00020000);
}
template <int AUX>
__device__ __forceinline__ f32x4 ld16(__amdgpu_buffer_rsrc_t r, unsigned voff, unsigned soff) {
  return __builtin_bit_cast(f32x4, __builtin_amdgcn_raw_buffer_load_b128(r, voff, soff, AUX));
}
template <int AUX>
__device__ __forceinline__ void st16(f16x8 v, __amdgpu_buffer_rsrc_t r, unsigned voff, unsigned soff) {
  __builtin_amdgcn_raw_buffer_store_b128(__builtin_bit_cast(u32x4, v), r, voff, soff, AUX);
}
__device__ __forceinline__ f16x8 cvt8(f32x4 a, f32x4 b) {
  f16x8 h;
#pragma unroll
  for (int i = 0; i < 4; ++i) { h[i] = (_Float16)a[i]; h[4 + i] = (_Float16)b[i]; }
  return h;
}
__device__ __forceinline__ f16x8 cvt_col(const f32x4 (&v)[8], int i) {
  f16x8 h;
#pragma unroll
  for (int j = 0; j < 8; ++j) h[j] = (_Float16)v[j][i];
  return h;
}
#define WAIT_VM(n) asm volatile("s_waitcnt vmcnt(%0)" :: "n"(n) : "memory")
#define WAIT_LGKM0() asm volatile("s_waitcnt lgkmcnt(0)" ::: "memory")
#define FENCE() asm volatile("" ::: "memory")
#define UNLIKELY(c) __builtin_expect(!!(c), 0)
#define BARRIER() do { FENCE(); __builtin_amdgcn_s_barrier(); FENCE(); } while (0)

__device__ __forceinline__ void flag_store(char* p, unsigned v) {
  __hip_atomic_store((GAS unsigned*)p, v, __ATOMIC_RELAXED, __HIP_MEMORY_SCOPE_SYSTEM);
}
__device__ __forceinline__ void flag_store2(char* p, unsigned lo, unsigned hi) {
  __hip_atomic_store((GAS unsigned long long*)p, (unsigned long long)lo | ((unsigned long long)hi << 32), __ATOMIC_RELAXED, __HIP_MEMORY_SCOPE_SYSTEM);
}
__device__ __forceinline__ unsigned draw(char* p) {
  unsigned z = 0;
  asm volatile("" : "+v"(z));
  p += z;
  return __hip_atomic_fetch_add((GAS unsigned*)p, 1u, __ATOMIC_RELAXED, __HIP_MEMORY_SCOPE_AGENT);
}
__device__ __forceinline__ unsigned uni(unsigned v) { return (unsigned)__builtin_amdgcn_readfirstlane((int)v); }
__device__ __forceinline__ unsigned lanes_below(unsigned long long b) {
  return __builtin_amdgcn_mbcnt_hi((unsigned)(b >> 32), __builtin_amdgcn_mbcnt_lo((unsigned)b, 0u));
}
__device__ __forceinline__ void advance(unsigned& pos, unsigned long long b) {
  asm("s_lshl2_add_u32 %0, %1, %0" : "+s"(pos) : "s"((unsigned)__builtin_popcountll(b)) : "scc");
}

#define VSTAMP(ws, slot, val) do {} while (0)
#define STAMP(ws, slot) do {} while (0)
#define PSTAMP(ws, cls, n) do {} while (0)
#define CSTAMP(ws, slot) do {} while (0)

__device__ __forceinline__ void set_timeout(char* myws, unsigned call) {
  __hip_atomic_store((GAS unsigned*)(myws + kTmo), call, __ATOMIC_RELAXED, __HIP_MEMORY_SCOPE_AGENT);
}
__device__ __forceinline__ unsigned peer_of(unsigned r, unsigned i) { return (r & 4u) + ((r + i + 1u) & 3u); }
__device__ __forceinline__ unsigned panel_row(unsigned ap, unsigned rh, unsigned rank) {
  const unsigned og = (rank & 4u) ^ 4u;
  if (ap < 8u) return rank * 1024u + ap * 128u + rh * 64u;
  if (ap < 23u) { const unsigned i = (ap - 8u) / 5u, s = (ap - 8u) - 5u * i; return peer_of(rank, i) * 1024u + s * 128u + rh * 64u; }
  if (ap < 26u) return peer_of(rank, ap - 23u) * 1024u + 640u;
  if (ap < 34u) return (og + ((ap - 26u) >> 1)) * 1024u + 768u + (ap & 1u) * 128u + rh * 64u;
  return (og + 2u * (ap - 34u) + rh) * 1024u + 704u;
}
__device__ __forceinline__ unsigned park_pos(unsigned ap, unsigned rh, unsigned j) {
  if (ap < 8u) return 2u * ap + rh < 11u ? 2u * ap + rh : 99u;
  if (ap < 23u) {
    const unsigned i = (ap - 8u) / 5u, s = (ap - 8u) - 5u * i;
    return s < 2u ? 11u + 4u * i + 2u * s + rh : s < 4u ? 40u + 4u * i + 2u * s + rh : 26u + 2u * i + rh;
  }
  const unsigned src = ap < 34u ? (ap - 26u) >> 1 : 2u * (ap - 34u) + rh;
  if (src == j) return 99u;
  const unsigned e = src - (src > j ? 1u : 0u);
  return ap < 34u ? 32u + 4u * e + 2u * (ap & 1u) + rh : 23u + e;
}
__device__ __forceinline__ unsigned park_row(unsigned hp, unsigned src, unsigned rank) {
  const unsigned q = rank & 3u;
  if (hp < 11u) return src * 1024u + hp * 64u;
  if (hp < 23u) return peer_of(src, (hp - 11u) >> 2) * 1024u + ((hp - 11u) & 3u) * 64u;
  if (hp < 26u) { const unsigned e = hp - 23u; return ((rank & 4u) + e + (e >= q ? 1u : 0u)) * 1024u + 704u; }
  if (hp < 32u) return peer_of(src, (hp - 26u) >> 1) * 1024u + 512u + (hp & 1u) * 64u;
  if (hp < 44u) { const unsigned e = (hp - 32u) >> 2; return ((rank & 4u) + e + (e >= q ? 1u : 0u)) * 1024u + 768u + (hp & 3u) * 64u; }
  return peer_of(src, (hp - 44u) >> 2) * 1024u + 256u + (hp & 3u) * 64u;
}

}

__global__ void __launch_bounds__(512) __attribute__((amdgpu_waves_per_eu(2, 2)))
ag_gemm_relu_kernel(const Params p) {
  extern __shared__ __attribute__((aligned(16))) char smem_raw[];
  LAS char* const lds = (LAS char*)smem_raw;
  LAS volatile unsigned* const misc = (LAS volatile unsigned*)(lds + kLdsMisc);

  const int tid = threadIdx.x;
  const int lane = tid & 63;
  const int wid = __builtin_amdgcn_readfirstlane(tid >> 6);
  const int bid = blockIdx.x;
  const unsigned rank = (unsigned)p.rank & 7u;
  const unsigned q = rank & 3u, og = (rank & 4u) ^ 4u;
  char* const myws = p.ws[rank];
  const unsigned l16 = (unsigned)lane * 16u;
  const __amdgpu_buffer_rsrc_t rws = mk_rsrc(myws, kWsNeed);
  const int fr = lane & 15, fq = lane >> 4;

  STAMP(myws, wid == 0 ? 0 : 31);

  const bool isconv = bid >= kNMove + 8 * CVFROM;
  if (tid == 0) {
    const unsigned tk = draw(myws + kTicket);
    const unsigned cl = isconv ? draw(myws + kCvt) : 0u;
    misc[0] = tk; misc[1] = cl;
    misc[3] = 0u;
  }
  if (tid >= 8 && tid < 18) misc[tid] = tid == 17 ? 0x7fc00000u : 0u;
  __syncthreads();
  const unsigned call = uni(misc[0]) / (unsigned)kNWG + 1u;
  const unsigned par = call & 1u;
  const unsigned abase = kAOff + par * kASize, bbase = kBOff + par * kBSize, obuf = kOOff + par * kOSize;

  if (bid < kNMove) {
    const unsigned pw = (unsigned)wid >= WPM ? (unsigned)wid - WPM : (unsigned)wid;
    const unsigned gw = (unsigned)bid * WPM + pw;
    if (wid == 7) {
      if (bid == 0 && lane == 0) {
        const unsigned target = call * kDonePerCall;
        const unsigned long long t0 = __builtin_amdgcn_s_memrealtime();
        bool failed = false;
        while ((int)(__hip_atomic_load((GAS unsigned*)(myws + kDone), __ATOMIC_RELAXED, __HIP_MEMORY_SCOPE_SYSTEM) - target) < 0) {
          if (__hip_atomic_load((GAS unsigned*)(myws + kTmo), __ATOMIC_RELAXED, __HIP_MEMORY_SCOPE_AGENT) == call ||
              __builtin_amdgcn_s_memrealtime() - t0 > kDeadline) {
            set_timeout(myws, call);
            failed = true;
            break;
          }
          __builtin_amdgcn_s_sleep(2);
        }
        __builtin_amdgcn_fence(__ATOMIC_ACQUIRE, "");
        if (failed) __hip_atomic_store((GAS unsigned*)(p.out[rank] + (size_t)rank * 1024u * 1024u), 0x7fc00000u, __ATOMIC_RELAXED, __HIP_MEMORY_SCOPE_SYSTEM);
        STAMP(myws, 10);
      }
      return;
    }
    if ((unsigned)wid >= 2u * WPM || gw >= 14u * NWS) return;
    const unsigned sid = gw % 14u, k = gw / 14u;
    const bool cross = sid >= 6u;
    const unsigned c = cross ? (sid - 6u) >> 1 : sid >> 1, h = sid & 1u;
    const unsigned sr = (cross ? og + c : peer_of(rank, c)) & 7u;
    char* const sws = p.ws[sr];
    const __amdgpu_buffer_rsrc_t rs = mk_rsrc(sws, kWsNeed);
    LAS volatile unsigned* const hand = misc + 8u + 2u * pw;
    const unsigned buf0 = pw * 2u * 17408u;
    if ((unsigned)wid >= WPM) {
      if (!cross) return;
      const __amdgpu_buffer_rsrc_t ro = mk_rsrc(p.out[rank], kInBytes);
      const unsigned vo = (unsigned)fr * 4096u + (unsigned)fq * 16u;
      const unsigned long long t0 = __builtin_amdgcn_s_memrealtime();
      for (unsigned b = 0;; b ^= 1u) {
        unsigned hv;
        while ((hv = uni(hand[b])) == 0u) {
          if (__builtin_amdgcn_s_memrealtime() - t0 > 2u * kDeadline) { set_timeout(myws, call); return; }
          __builtin_amdgcn_s_sleep(XSLEEP);
        }
        FENCE();
        if (hv == ~0u) break;
        const unsigned oa = park_row(hv >> 4, sr, rank) * 4096u + (256u * c + (hv & 12u) * 16u) * 4u;
        unsigned pos = (unsigned)(size_t)lds + buf0 + b * 17408u, val[64];
        u32x2 mk = {~0u, ~0u};
        if ((hv & 2u) == 0u) mk = *(LAS u32x2*)(size_t)(pos + lane * 8);
        const unsigned fill = (unsigned)(size_t)lds + kLdsMisc + 64u + (hv & 2u) * 2u;
        pos += 512u;
#pragma unroll
        for (int i = 0; i < 64; ++i) {
          const bool keep = (int)(mk[i >> 5] << (i & 31)) >= 0;
          const unsigned long long bl = __ballot(keep);
          unsigned a = lanes_below(bl) * 4u + pos;
          asm volatile("" : "+v"(a));
          val[i] = *(LAS unsigned*)(size_t)(keep ? a : fill);
          advance(pos, bl);
        }
        WAIT_LGKM0();
        if (lane == 0) hand[b] = 0u;
        FENCE();
#pragma unroll
        for (int i = 0; i < 16; ++i)
          __builtin_amdgcn_raw_buffer_store_b128(u32x4{val[4 * i], val[4 * i + 1], val[4 * i + 2], val[4 * i + 3]}, ro, vo + (i & 3) * 64u, oa + (i >> 2) * 65536u, kAuxSys);
      }
      STAMP(myws, 1 + wid);
      return;
    }
    const unsigned total = cross ? kRowsC : kRowsI;
    const unsigned nplain = cross ? 576u : kRowsI;
    const unsigned qbase = (call - 1u) * total - NWS;
    bool failed = false;
    unsigned cgrp = 99u, okt = 0, wbase = 0x10000u;
    u32x2 fw = {0u, 0u};
    unsigned sa = 0, da = 0, fa = 0, pend = 0;
    unsigned s2 = 4096u, d2 = 4096u;
    u32x4 v[4], w[4];
    auto raise = [&]() {
      if (pend != 0u && lane == 0) flag_store(myws + pend, call);
      pend = 0u;
    };
    auto fetch = [&](u32x4 (&x)[4], unsigned so) {
#pragma unroll
      for (int i = 0; i < 4; ++i) x[i] = __builtin_amdgcn_raw_buffer_load_b128(rs, l16 + i * 1024u, so, kAuxSys);
      FENCE();
    };
    auto put = [&](const u32x4 (&x)[4], unsigned so) {
#pragma unroll
      for (int i = 0; i < 4; ++i) __builtin_amdgcn_raw_buffer_store_b128(x[i], rws, l16 + i * 1024u, so, kAuxDev);
      FENCE();
    };
    auto setup = [&](unsigned m) {
      unsigned sp, kt, dp, db, dh = h;
      unsigned so = h * 8192u, dof = h * 8192u;
      s2 = 4096u; d2 = 4096u;
      if (!cross) {
        if (m < 256u) { sp = m & 1u; kt = m >> 1; dp = 8u + 5u * c + sp; db = kt; }
        else if (m < 384u) { sp = 4u; kt = m - 256u; dp = 12u + 5u * c; db = kt; }
        else if (m < 640u) { sp = 2u + (m & 1u); kt = (m - 384u) >> 1; dp = 8u + 5u * c + sp; db = kt; }
        else { sp = 5u; db = m - 640u; kt = 2u * db + h; so = 0u; s2 = 8192u; dp = 23u + c; }
      } else {
        if (m < 256u) { sp = 36u + 2u * q + (m & 1u); kt = m >> 1; dp = 44u + 2u * c + (m & 1u); db = kt; }
        else if (m < 320u) { sp = 5u; kt = 2u * (m - 256u) + h; so = 4096u; s2 = 8192u; dp = 34u + (c >> 1); db = kt; dh = c & 1u; dof = dh * 4096u; d2 = 8192u; }
        else { sp = 6u + (m & 1u); kt = (m - 320u) >> 1; dp = 26u + 2u * c + (m & 1u); db = kt; }
      }
      const unsigned sub = sp < 36u ? sp >> 1 : (sp - 28u) >> 1;
      if (!failed && !(sub == cgrp && kt < okt)) {
        WAIT_VM(0);
        raise();
        char* const pa = myws + (lane < 63 ? kSrc + (sr * 8u + sub) * 512u + min(kt + (unsigned)lane, 127u) * 4u : kTmo);
        const unsigned long long t0 = __builtin_amdgcn_s_memrealtime();
        for (;;) {
          const unsigned f = __hip_atomic_load((GAS unsigned*)pa, __ATOMIC_RELAXED, __HIP_MEMORY_SCOPE_SYSTEM);
          const unsigned long long b = __ballot(lane < 63 ? (int)(f - call) >= 0 : f == call);
          if ((b >> 63) != 0ull) { failed = true; break; }
          if (b & 1ull) { cgrp = sub; okt = kt + (unsigned)__builtin_ctzll(~b); break; }
          if (__builtin_amdgcn_s_memrealtime() - t0 > kDeadline) { set_timeout(myws, call); failed = true; break; }
          __builtin_amdgcn_s_sleep(MSLEEP);
        }
      }
      sa = (sp < 36u ? abase + sp * kPanel : bbase + (sp - 36u) * kPanel) + kt * kBlk + so;
      da = (dp < 36u ? abase + dp * kPanel : bbase + (dp - 36u) * kPanel) + db * kBlk + dof;
      fa = kFlags + dp * 1024u + db * 8u + dh * 4u;
      fetch(v, sa);
    };
    unsigned m = k;
    setup(m);
    do {
      unsigned dv = 0;
      if (lane == 0) dv = draw(myws + kQueue + sid * 128u);
      FENCE();
      put(v, da);
      raise();
      fetch(w, sa + s2);
      put(w, da + d2);
      pend = fa;
      if (h == 0u) { PSTAMP(myws, sid >> 1, m + 1u); if (m < 128u) PSTAMP(myws, 8u + (sid >> 1), (m + 1u) * 4u); }
      m = uni(dv) - qbase;
      if (m < nplain) setup(m);
    } while (m < nplain);
    WAIT_VM(0);
    raise();
    if (cross) {
      unsigned hval = 0;
      const unsigned fo = lane < 32 ? kOFlags + par * 8192u + c * 2048u + h * 8u + (unsigned)lane * 16u : kTmo;
      auto stale = [&](unsigned o) {
        return o - wbase >= 32u || (int)((unsigned)__builtin_amdgcn_readlane((int)fw[0], (int)(o - wbase)) - call) < 0;
      };
      for (unsigned b = 0;; b ^= 1u) {
        const unsigned o = m - 576u, s = 2u * o + h;
        unsigned cnt = 2048u, dv = 0;
        if (m >= total || stale(o)) {
          WAIT_VM(0);
          if (hval != 0u && lane == 0) hand[b ^ 1u] = hval;
          hval = 0u;
          const unsigned long long t0 = __builtin_amdgcn_s_memrealtime();
          wbase = o;
          while (!failed && m < total) {
            fw = __builtin_bit_cast(u32x2, __hip_atomic_load((GAS unsigned long long*)(myws + fo + (lane < 32 ? o * 16u : 0u)), __ATOMIC_RELAXED, __HIP_MEMORY_SCOPE_SYSTEM));
            const unsigned long long bl = __ballot(lane < 32 ? (int)(fw[0] - call) >= 0 : fw[0] == call);
            if ((bl >> 32) != 0ull) { failed = true; break; }
            if (bl & 1ull) break;
            if (__builtin_amdgcn_s_memrealtime() - t0 > kDeadline) { set_timeout(myws, call); failed = true; }
            __builtin_amdgcn_s_sleep(MSLEEP);
          }
        }
        cnt = failed ? 0u : min((unsigned)__builtin_amdgcn_readlane((int)fw[1], (int)((o - wbase) & 31u)), 4096u);
        if (UNLIKELY(uni(hand[b]) != 0u)) {
          const unsigned long long t0 = __builtin_amdgcn_s_memrealtime();
          while (uni(hand[b]) != 0u)
            if (__builtin_amdgcn_s_memrealtime() - t0 > kDeadline) { set_timeout(myws, call); return; }
        }
        FENCE();
        if (m >= total) {
          if (lane == 0) hand[b] = ~0u;
          break;
        }
        const unsigned bytes = 512u + 4u * cnt, sb = obuf + (q * 224u + s) * kPiece, nbuf = buf0 + b * 17408u;
        auto part = [&](unsigned f0, unsigned f1) {
          for (unsigned f = f0; f < f1; f += 1024u) {
            WAIT_VM(PWIN - 1u);
            asm volatile("s_mov_b32 m0, %0\n\ts_nop 0\n\tbuffer_load_dwordx4 %1, %2, %3 offen sc0 sc1 lds"
                         :: "s"(nbuf + f), "v"(f + l16 < bytes ? l16 : kOob), "s"(rs), "s"(sb + f) : "memory");
          }
        };
        asm volatile("s_mov_b64 exec, 1\n\tglobal_atomic_add %0, %1, %2, %3 sc0\n\ts_mov_b64 exec, -1"
                     : "=v"(dv) : "v"(kQueue + sid * 128u), "v"(1u), "s"(myws) : "memory");
        part(0u, min(bytes, (PWIN - 1u) * 1024u));
        if (bytes <= (PWIN - 2u) * 1024u) WAIT_VM(0);
        WAIT_VM(PWIN - 1u);
        if (hval != 0u && lane == 0) hand[b ^ 1u] = hval;
        hval = 4u * s + (failed ? 3u : 1u);
        if (h == 0u) PSTAMP(myws, sid >> 1, m + 1u);
        asm volatile("" : "+v"(dv));
        m = uni(dv) - qbase;
        if (m < total && stale(m - 576u)) {
          wbase = m - 576u;
          asm volatile("buffer_load_dwordx2 %0, %1, %2, 0 offen sc0 sc1" : "=v"(fw) : "v"(fo + (lane < 32 ? wbase * 16u : 0u)), "s"(rws) : "memory");
          part((PWIN - 1u) * 1024u, bytes);
          if (bytes <= (2u * PWIN - 3u) * 1024u) WAIT_VM(0);
          WAIT_VM(PWIN - 1u);
          asm volatile("" : "+v"(fw) :: "memory");
        } else
          part((PWIN - 1u) * 1024u, bytes);
      }
    }
    STAMP(myws, 1 + wid);
    return;
  }

  {
    const unsigned kq = (unsigned)((lane & 3) ^ ((-(lane >> 4)) & 3));
    const unsigned rho = (unsigned)lane >> 2;
    const unsigned lx = (rho * 8192u + 4u * kq) * 4u;
    const unsigned lw = (4u * ((unsigned)lane >> 4) * 1024u + 4u * ((unsigned)lane & 15u)) * 4u;
    const __amdgpu_buffer_rsrc_t rx = mk_rsrc(p.x, kInBytes);
    const __amdgpu_buffer_rsrc_t rw = mk_rsrc(p.w, kInBytes);
    char* const pws = p.ws[lane & 7] + kSrc + rank * 4096u;
    const unsigned mine = 15u << (rank & 4u);
    constexpr unsigned kNConv = 8u * (26 - CVFROM);
    const unsigned cbase = (call - 1u) * (kItems + kNConv);
    unsigned cur = isconv ? uni(misc[1]) - cbase : kItems;
    f32x4 v[8];
    auto loads = [&](unsigned it) {
      const unsigned ktg = it >> 3, sub = it & 7u;
      if (sub < 4u) {
        const unsigned so = ((256u * sub + 32u * wid) * 8192u + ktg * 64u) * 4u;
#pragma unroll
        for (int s = 0; s < 2; ++s) {
          v[4 * s + 0] = ld16<0>(rx, lx, so + s * 524288u); v[4 * s + 1] = ld16<0>(rx, lx + 64u, so + s * 524288u);
          v[4 * s + 2] = ld16<0>(rx, lx + 128u, so + s * 524288u); v[4 * s + 3] = ld16<0>(rx, lx + 192u, so + s * 524288u);
        }
      } else {
        const unsigned so = ((ktg * 64u + 32u * (wid & 1)) * 1024u + 256u * (sub - 4u) + 64u * (wid >> 1)) * 4u;
#pragma unroll
        for (int i = 0; i < 8; ++i) v[i] = ld16<0>(rw, lw, so + ((i & 3) + 16 * (i >> 2)) * 4096u);
      }
    };
    if (cur < kItems) loads(cur);
    int flip = 0;
    while (cur < kItems) {
      unsigned nx = 0;
      if (tid == 0) nx = draw(myws + kCvt);
      const unsigned ktg = cur >> 3, sub = cur & 7u;
      if (sub < 4u) {
        const unsigned so = abase + (2u * sub + (wid >> 2)) * kPanel + ktg * kBlk + (wid & 3) * 2048u;
#pragma unroll
        for (int i = 0; i < 4; ++i) st16<CVAUX>(cvt8(v[2 * i], v[2 * i + 1]), rws, l16 + (i >> 1) * 1024u, so + (i & 1) * 8192u);
      } else {
        const unsigned so = bbase + (2u * (sub - 4u) + (wid >> 2)) * kPanel + ktg * kBlk + ((wid >> 1) & 1) * 4096u + (wid & 1) * 8192u;
#pragma unroll
        for (int i = 0; i < 4; ++i) st16<CVAUX>(cvt_col(v, i), rws, l16 + i * 1024u, so);
      }
      if (tid == 0) misc[4 + flip] = nx;
      WAIT_LGKM0();
      BARRIER();
      const unsigned nxt = uni(misc[4 + flip]) - cbase;
      flip ^= 1;
      if (nxt < kItems) { loads(nxt); WAIT_VM(8); } else { WAIT_VM(0); }
      BARRIER();
      if (tid < 2) flag_store2(myws + kFlags + ((sub < 4u ? 2u * sub : 28u + 2u * sub) + (unsigned)tid) * 1024u + ktg * 8u, call, call);
      else if (wid == 1 && lane < 8) {
        const unsigned to = sub < 2u ? mine : sub == 2u ? 255u : sub == 3u ? mine ^ 255u : 1u << (og + sub - 4u);
        if (((to & ~(1u << rank)) >> lane) & 1u) flag_store(pws + sub * 512u + ktg * 4u, call);
      }
      cur = nxt;
    }
  }
  STAMP(myws, wid == 0 ? 1 : 31);

  const uint32_t* const tl = p.tiles + 3 * (bid - kNMove);
  const int ntiles = (int)(tl[0] >> 28);
  if (wid == 7) return;

  if (wid >= 4) {
    const unsigned lw = (unsigned)wid - 4u;
    unsigned total = 0;
    for (int ti = 0; ti < ntiles; ++ti) total += ((tl[ti] >> 10) & 3u) == 2u ? 64u : 256u;
    unsigned a = 0, aend = 0, so = lw * 8192u, src = 0, step = 0, g2 = 0, v2 = l16, idle = 0;
    unsigned ok = 0, f0 = 0, f1 = 0, fb = 0, fsh = 0, fmax = 0, fv = 0;
    unsigned rdy = 0, askat = 0, lookat = ~0u;
    int ta = 0, tf = 0;
    auto next_flags = [&]() {
      while (tf < ntiles) {
        const unsigned code = tl[tf++];
        const unsigned ap = code & 63u, bp = (code >> 6) & 15u, shape = (code >> 10) & 3u;
        f0 = f1;
        f1 += shape == 2u ? 64u : 256u;
        if (shape == 3u && lw == 2u) { ok = f1; continue; }
        const unsigned grp = lw == 0u ? ap : shape == 0u ? (lw == 1u ? ap + 1u : 36u + bp) : shape == 1u ? 35u + bp + lw : 36u + bp;
        fb = kFlags + grp * 1024u;
        fsh = shape != 2u ? 0u : lw == 0u ? 1u : 2u;
        fmax = fsh == 1u ? 127u : 255u;
        break;
      }
    };
    auto known = [&]() {
      rdy = min(ok, aend);
      unsigned from = max(ok, LTHR);
      asm volatile("" : "+s"(from));
      askat = ok >= total ? ~0u : from - LTHR;
    };
    auto ask = [&]() {
      const unsigned off = lane < 63 ? fb + min(((ok - f0) << fsh) + (unsigned)lane, fmax) * 4u : kTmo;
      asm volatile("buffer_load_dword %0, %1, %2, 0 offen sc0 sc1" : "=v"(fv) : "v"(off), "s"(rws) : "memory");
    };
    auto give_up = [&]() {
      ok = total;
      if (lane == 0) misc[3] = 1u;
      WAIT_LGKM0();
    };
    auto look = [&]() {
      asm volatile("" : "+v"(fv) :: "memory");
      const unsigned long long b = __ballot(lane < 63 ? (int)(fv - call) >= 0 : fv == call);
      if (UNLIKELY((b >> 63) != 0ull)) {
        give_up();
      } else {
        const unsigned run = (unsigned)__builtin_ctzll(~b);
        ok = min(ok + (fsh == 0u ? run & ~1u : run >> fsh), f1);
        if (UNLIKELY(ok == f1)) next_flags();
      }
      lookat = ~0u;
    };
    auto issue = [&](bool busy) {
      LAS char* const dl = lds + so;
      if (busy) {
        __builtin_amdgcn_raw_ptr_buffer_load_lds(rws, (LAS void*)dl, 16, l16, src, 0, kAuxDev);
        __builtin_amdgcn_raw_ptr_buffer_load_lds(rws, (LAS void*)dl, 16, l16, src, 1024, kAuxDev);
        __builtin_amdgcn_raw_ptr_buffer_load_lds(rws, (LAS void*)dl, 16, v2, src, 2048, kAuxDev);
        __builtin_amdgcn_raw_ptr_buffer_load_lds(rws, (LAS void*)dl, 16, v2, src, 3072, kAuxDev);
        __builtin_amdgcn_raw_ptr_buffer_load_lds(rws, (LAS void*)(dl + 4096), 16, l16, src + g2, 0, kAuxDev);
        __builtin_amdgcn_raw_ptr_buffer_load_lds(rws, (LAS void*)(dl + 4096), 16, l16, src + g2, 1024, kAuxDev);
        __builtin_amdgcn_raw_ptr_buffer_load_lds(rws, (LAS void*)(dl + 4096), 16, v2, src + g2, 2048, kAuxDev);
        __builtin_amdgcn_raw_ptr_buffer_load_lds(rws, (LAS void*)(dl + 4096), 16, v2, src + g2, 3072, kAuxDev);
      }
      FENCE();
      src += step;
      so = so >= kRing - kSlot ? so - (kRing - kSlot) : so + kSlot;
      ++a;
    };
    next_flags();
    known();
    for (unsigned j = 0; j < total; ++j) {
      if (a == j + 4u) {
        unsigned e = min(rdy + 1u, askat);
        asm volatile("" : "+s"(e));
        e = min(e, lookat);
        if (idle != 0u) {
          WAIT_VM(0);
          while (a + 1u < e) { issue(false); BARRIER(); }
        }
        while (a + 1u < e) {
          issue(true);
          WAIT_VM(32);
          BARRIER();
        }
        j = a - 4u;
      }
      unsigned lim = min(j + 5u, rdy);
      if (UNLIKELY(a >= lim)) {
        if (a == aend) {
          const unsigned code = tl[ta++];
          const unsigned ap = code & 63u, bp = (code >> 6) & 15u, shape = (code >> 10) & 3u;
          if (lw == 0u || (lw == 1u && shape == 0u)) src = abase + (ap + lw) * kPanel;
          else src = bbase + (bp + (shape == 1u ? lw - 1u : 0u)) * kPanel;
          step = 8192u; g2 = 4096u;
          idle = shape == 3u && lw == 2u ? 1u : 0u;
          if (shape == 2u) {
            g2 = 8192u;
            if (lw == 0u) { src += ((code >> 13) & 1u) * 2048u; step = kBlk; v2 = l16 + 2048u; }
            else { src += ((code >> 12) & 1u) * 4096u + (lw - 1u) * kBlk; step = 2u * kBlk; }
          }
          aend = ta == ntiles ? ~0u : aend + 256u;
          rdy = min(ok, aend);
        }
        if (a <= j && ok <= a) {
          WAIT_VM(0);
          if (lookat != ~0u) look();
          const unsigned long long t0 = __builtin_amdgcn_s_memrealtime();
          while (ok <= a) {
            ask();
            WAIT_VM(0);
            look();
            if (ok > a) break;
            if (__builtin_amdgcn_s_memrealtime() - t0 > kDeadline) { set_timeout(myws, call); give_up(); break; }
            __builtin_amdgcn_s_sleep(LSLEEP);
          }
          known();
        }
        lim = min(j + 5u, rdy);
      }
      if (a < lim) {
        issue(idle == 0u);
        if (UNLIKELY(a < lim)) issue(idle == 0u);
      }
      if (UNLIKELY(a >= askat)) { ask(); askat = ~0u; lookat = a + 4u; }
      const unsigned n = idle != 0u ? 0u : a - 1u - j;
      if (UNLIKELY(n < 4u)) {
        WAIT_VM(24);
        if (n < 3u) {
          WAIT_VM(16);
          if (n < 2u) {
            WAIT_VM(8);
            if (n < 1u) WAIT_VM(0);
          }
        }
      }
      WAIT_VM(32);
      if (UNLIKELY(j + 5u >= lookat)) { look(); known(); }
      BARRIER();
    }
    return;
  }

  {
    const unsigned slot = (unsigned)(4 * fr + (fq ^ ((-(fr >> 2)) & 3))) * 16u;
    const unsigned slotb = (unsigned)(16 * fq + 4 * (fr & 3) + (fr >> 2)) * 16u;
    unsigned so = 0;
    for (int ti = 0; ti < ntiles; ++ti) {
      const unsigned code = tl[ti];
      const unsigned shape = (code >> 10) & 3u;
      const unsigned wh = (unsigned)wid >> 1, wl = (unsigned)wid & 1u;
      const unsigned ap = (code & 63u) + (shape == 0u ? wh : 0u), bp = ((code >> 6) & 15u) + (shape == 1u ? wh : 0u);
      const unsigned jo = (bp - 8u) >> 1;
      const unsigned owner = (bp < 8u ? rank : og + jo) & 7u;
      const unsigned col0 = bp < 8u ? bp * 128u : 256u * q + (bp & 1u) * 128u;
      const __amdgpu_buffer_rsrc_t ro = mk_rsrc(p.out[owner], kInBytes);
      const unsigned vo = (unsigned)fr * 4096u + (unsigned)fq * 16u;

      auto run = [&](auto mconst, const unsigned aoff, const unsigned boff) {
        constexpr int M = decltype(mconst)::value;
        constexpr int KS = M > 1 ? 1 : 4;
        constexpr int NS = M > 1 ? 256 : 64;
        constexpr int NA = M * KS, NB = 4 * KS;
        const unsigned va = aoff + slot, vb = boff + slotb;
        f32x4 acc[M][4];
#pragma unroll
        for (int m = 0; m < M; ++m)
#pragma unroll
          for (int n = 0; n < 4; ++n) acc[m][n] = f32x4{0.f, 0.f, 0.f, 0.f};
        f16x8 fa[2][NA], fb[2][NB];
        auto rd = [&](f16x8 (&a)[NA], f16x8 (&b)[NB]) {
          LAS char* const pa = lds + (va + so);
          LAS char* const pb = lds + (vb + so);
#pragma unroll
          for (int i = 0; i < (NA > NB ? NA : NB); ++i) {
            if (i < NB) b[i] = *(LAS f16x8*)(pb + i * 1024);
            if (i < NA) a[i] = *(LAS f16x8*)(pa + i * (M > 1 ? 1024 : 2048));
          }
        };
        auto mm = [&](const f16x8 (&a)[NA], const f16x8 (&b)[NB]) {
#pragma unroll
          for (int ks = 0; ks < KS; ++ks)
#pragma unroll
            for (int m = 0; m < M; ++m)
#pragma unroll
              for (int n = 0; n < 4; ++n)
                acc[m][n] = __builtin_amdgcn_mfma_f32_16x16x32_f16(b[4 * ks + n], a[KS * m + ks], acc[m][n], 0, 0, 0);
        };
        BARRIER();
        STAMP(myws, wid == 0 ? 2 + 3 * ti : 31);
        CSTAMP(myws, wid == 0 ? 12 + 2 * ti : 31);
        rd(fa[0], fb[0]);
        __builtin_amdgcn_s_waitcnt(0xC07F);
        BARRIER();
        so = so == kRing - kSlot ? 0u : so + kSlot;
        if constexpr (M > 1) {
          unsigned ua = va + so, ub = vb + so;
          for (int t = 0; t < NS; t += 2) {
#pragma unroll
            for (int s = 0; s < 2; ++s) {
              __builtin_amdgcn_sched_barrier(0);
#pragma unroll
              for (int i = 0; i < 4 * M; ++i) {
                acc[i >> 2][i & 3] = __builtin_amdgcn_mfma_f32_16x16x32_f16(fb[s][i & 3], fa[s][i >> 2], acc[i >> 2][i & 3], 0, 0, 0);
                __builtin_amdgcn_sched_barrier(0);
                if (M == 8 && (i & 1) == 0 && i < 24) {
                  const int r = i >> 1;
                  if (r < 4) fb[s ^ 1][r] = *(LAS f16x8*)(lds + ub + r * 1024);
                  else fa[s ^ 1][r - 4] = *(LAS f16x8*)(lds + ua + (r - 4) * 1024);
                }
                if (M == 4 && i < 4) {
                  fb[s ^ 1][i] = *(LAS f16x8*)(lds + ub + i * 1024);
                  fa[s ^ 1][i] = *(LAS f16x8*)(lds + ua + i * 1024);
                }
                if (i == 4 * M - 10) { so = so == kRing - kSlot ? 0u : so + kSlot; asm volatile("" : "+s"(so)); }
                if (i == 4 * M - 6) { ua = va + so; asm volatile("" : "+v"(ua)); }
                if (i == 4 * M - 4) { ub = vb + so; asm volatile("" : "+v"(ub)); }
                if (i == 4 * M - 2) {
                  __builtin_amdgcn_s_waitcnt(0xC07F);
                  if (t < NS - 2) BARRIER();
                }
                __builtin_amdgcn_sched_barrier(0);
              }
            }
          }
          so = so == 0u ? kRing - kSlot : so - kSlot;
        } else {
          for (int t = 0; t < NS; t += 2) {
#pragma unroll
            for (int s = 0; s < 2; ++s) {
              __builtin_amdgcn_sched_barrier(0);
              rd(fa[s ^ 1], fb[s ^ 1]);
              mm(fa[s], fb[s]);
#pragma unroll
              for (int i = 0; i < 4; ++i) {
                __builtin_amdgcn_sched_group_barrier(0x100, 2, 0);
                __builtin_amdgcn_sched_group_barrier(0x008, 1, 0);
              }
#pragma unroll
              for (int i = 0; i < 12; ++i) {
                __builtin_amdgcn_sched_group_barrier(0x100, 1, 0);
                __builtin_amdgcn_sched_group_barrier(0x008, 1, 0);
              }
              __builtin_amdgcn_sched_barrier(0);
              so = so == kRing - kSlot ? 0u : so + kSlot;
              __builtin_amdgcn_s_waitcnt(0xC07F);
              if (t < NS - 2) BARRIER();
            }
          }
        }
        STAMP(myws, wid == 0 ? 3 + 3 * ti : 31);
        CSTAMP(myws, wid == 0 ? 13 + 2 * ti : 31);
        const int lo = misc[3] != 0u ? 0x7fc00000 : 0;
        if (M > 1 && bp >= 8u) {
          const __amdgpu_buffer_rsrc_t rw4 = __builtin_amdgcn_make_buffer_rsrc((void*)myws, 4, kWsNeed / 4u, 0x00020000);
          if (UNLIKELY(lo != 0)) {
#pragma unroll
            for (int m = 0; m < M; ++m)
#pragma unroll
              for (int n = 0; n < 4; ++n) acc[m][n] = __builtin_bit_cast(f32x4, u32x4{0x7fc00000u, 0x7fc00000u, 0x7fc00000u, 0x7fc00000u});
          }
#pragma unroll
          for (int rh = 0; rh < M / 4; ++rh) {
            const unsigned hp = park_pos(ap, M == 8 ? (unsigned)rh : wh, jo);
            if (hp < 56u) {
              const unsigned s = hp * 4u + (bp & 1u) * 2u + wl;
              const unsigned pb0 = obuf + (jo * 224u + s) * kPiece;
              unsigned pos = pb0 + 512u;
              u32x2 mk = {0u, 0u};
#pragma unroll
              for (int m = 0; m < 4; ++m)
#pragma unroll
                for (int j = 0; j < 4; ++j)
#pragma unroll
                  for (int n = 0; n < 4; ++n) {
                    const float f = acc[4 * rh + m][n][j];
                    const unsigned iv = __float_as_uint(f);
                    const unsigned long long b = __ballot((int)iv >= 0);
                    const unsigned idx = lanes_below(b) | (iv & kOob);
                    asm volatile("buffer_store_dword %0, %1, %2, %3 idxen sc0 sc1" :: "v"(iv), "v"(idx), "s"(rw4), "s"(pos) : "memory");
                    advance(pos, b);
                    mk[m >> 1] = __builtin_amdgcn_alignbit(mk[m >> 1], iv, 31);
                  }
              __builtin_amdgcn_raw_buffer_store_b64(mk, rws, (unsigned)lane * 8u, pb0, kAuxSys);
              WAIT_VM(0);
              if (lane == 0) flag_store2(p.ws[owner] + kOFlags + par * 8192u + q * 2048u + s * 8u, call, (pos - pb0 - 512u) >> 2);
            }
          }
        } else {
#pragma unroll
          for (int m = 0; m < M; ++m) {
            const unsigned ob = M > 1 ? (panel_row(ap, M == 8 ? (unsigned)(m >> 2) : wh, rank) + 16u * (m & 3)) * 4096u + (col0 + 64u * wl) * 4u
                                       : (panel_row(ap, 0u, rank) + 32u * ((code >> 13) & 1u) + 16u * wl) * 4096u + (col0 + 64u * ((code >> 12) & 1u)) * 4u;
#pragma unroll
            for (int j = 0; j < 4; ++j) {
              u32x4 o;
#pragma unroll
              for (int n = 0; n < 4; ++n) { const float f = acc[m][n][j]; o[n] = (unsigned)max(__float_as_int(f), lo); }
              __builtin_amdgcn_raw_buffer_store_b128(o, ro, vo + j * 64u, ob, kAuxSys);
            }
          }
          if (bp >= 8u) {
            WAIT_VM(0);
            if (lane == 0) __hip_atomic_fetch_add((GAS unsigned*)(p.ws[owner] + kDone), 1u, __ATOMIC_RELAXED, __HIP_MEMORY_SCOPE_SYSTEM);
          }
        }
      };
      if (shape == 2u) {
        if (wid >= 2) return;
        run(std::integral_constant<int, 1>{}, wl * 1024u, 8192u);
      } else if (shape == 3u) {
        run(std::integral_constant<int, 4>{}, wh * 4096u, 8192u + wl * 4096u);
      } else {
        run(std::integral_constant<int, 8>{}, shape == 0u ? wh * 8192u : 0u, (shape == 0u ? 16384u : 8192u + wh * 8192u) + wl * 4096u);
      }
      STAMP(myws, wid == 0 ? 4 + 3 * ti : 31);
    }
  }
}

extern "C" void kernel_launch(int rank, int nranks, ncclComm_t comm, hipStream_t stream,
                              void* const* d_in, const int* in_sizes, int n_in,
                              void* d_out, int out_size,
                              const int* in_dtypes, int out_dtype,
                              void* const* d_out_all, void* const* d_ws_all, size_t ws_bytes) {
  (void)comm; (void)in_dtypes; (void)out_dtype; (void)d_out;
  if (nranks != 8 || (unsigned)rank >= 8u || n_in < 2 || ws_bytes < (size_t)kWsNeed ||
      in_sizes[0] != 8 * 1024 * 1024 || in_sizes[1] != 8 * 1024 * 1024 || out_size != 8 * 1024 * 1024) {
    fprintf(stderr, "kernel_launch: unexpected problem geometry\n");
    return;
  }
  Params p;
  p.x = (const float*)d_in[0];
  p.w = (const float*)d_in[1];
  for (int i = 0; i < 8; ++i) { p.ws[i] = (char*)d_ws_all[i]; p.out[i] = (float*)d_out_all[i]; }
  p.rank = rank;
  p.pad = 0;
  static const int xcol[8] = {0, 4, 2, 3, 1, 5, 6, 7};
  auto tile = [](int a, int b, int shape) { return (unsigned)(a | (b << 6) | (shape << 10)); };
  for (int ci = 0; ci < kNComp; ++ci) {
    const int x = ci & 7, s = ci >> 3, bq = 2 * (x ^ 4);
    unsigned t[3] = {0u, 0u, 0u};
    int n = 0;
    auto far67 = [&](int y, int e) {
      const int j = y >> 1, idx = 3 * (y & 1) + (e >> 1);
      return tile(26 + 2 * j + (e & 1), 8 + idx + (idx >= 2 * j ? 2 : 0), 3);
    };
    bool last = true;
    if (s < 12) t[n++] = tile(x < 4 ? 2 * (s >> 2) : 8 + 5 * (s >> 2), 4 * (x & 3) + (s & 3), 0);
    else if (s == 12) t[n++] = tile(6, xcol[x], 0);
    else if (s < 16) t[n++] = tile(12 + 5 * (s - 13), bq, 1);
    else if (s < 18) t[n++] = tile(34 + (s - 16), bq, 1);
    else if (s < 22) t[n++] = tile(26 + 2 * (x >> 1), 4 * (x & 1) + s - 18, 0);
    else t[n++] = far67(x, s - 22);
    if (s == 12) t[n++] = far67(x, 4);
    else if (x >= 4 && s >= 10 && s < 12) t[n++] = far67(s == 10 ? x : x - 4, 5);
    else if (x >= 4 && s < 2) { t[n++] = tile(10, bq + s, 0); last = false; }
    else if (s < 12) {
      const int i = x >= 4 ? s + 2 : s;
      t[n++] = tile(10 + 5 * (i >> 2) + ((i >> 1) & 1), bq + (i & 1), 3);
      if (s < 2) last = false;
    }
    if (last) {
      const int i = s - 2;
      t[n++] = tile(23 + i / 8, 2 * x + ((i >> 1) & 1), 2) | ((unsigned)(i & 1) << 12) | ((unsigned)((i >> 2) & 1) << 13);
    }
    p.tiles[3 * ci] = t[0] | ((unsigned)n << 28);
    p.tiles[3 * ci + 1] = t[1];
    p.tiles[3 * ci + 2] = t[2];
  }
  hipError_t e = hipFuncSetAttribute((const void*)ag_gemm_relu_kernel, hipFuncAttributeMaxDynamicSharedMemorySize, (int)kLdsBytes);
  if (e != hipSuccess) { fprintf(stderr, "hipFuncSetAttribute: %s\n", hipGetErrorString(e)); return; }
  hipLaunchKernelGGL(ag_gemm_relu_kernel, dim3(kNWG), dim3(512), kLdsBytes, stream, p);
  e = hipGetLastError();
  if (e != hipSuccess) fprintf(stderr, "launch: %s\n", hipGetErrorString(e));
}
```

```cpp
#include <hip/hip_runtime.h>
#include <rccl/rccl.h>
#include <cstdio>
#include <cstdint>
#include <type_traits>

typedef _Float16 f16x8 __attribute__((ext_vector_type(8)));
typedef float f32x4 __attribute__((ext_vector_type(4)));
typedef unsigned u32x4 __attribute__((ext_vector_type(4)));
typedef unsigned u32x2 __attribute__((ext_vector_type(2)));
#define LAS __attribute__((address_space(3)))
#define GAS __attribute__((address_space(1)))

#define NWS 9u
#define WPM 3u
#define PU 1u
#define PUW 1u
#define CVLEAD 40u
#define XSLEEP 4
#define CVFROM 13
#define PWIN 4u
#define CVAUX kAuxSys
#define MSLEEP 4
#define LSLEEP 2
#define LTHR 24u

namespace {

constexpr unsigned kBlk = 16384u;
constexpr unsigned kPanel = 2u << 20;
constexpr unsigned kAOff = 1u << 20;
constexpr unsigned kASize = 36u * kPanel;
constexpr unsigned kBOff = kAOff + 2u * kASize;
constexpr unsigned kBSize = 16u * kPanel;
constexpr unsigned kPiece = 16896u;
constexpr unsigned kOOff = kBOff + 2u * kBSize;
constexpr unsigned kOSize = 4u * 224u * kPiece;
constexpr unsigned kWsNeed = kOOff + 2u * kOSize;
constexpr unsigned kTmo = 128, kDone = 256, kCvt = 384, kAll = 512, kQueue = 1024;
constexpr unsigned kTicket = 4096;
constexpr unsigned kFlags = 16384;
constexpr unsigned kOFlags = 81920;
constexpr unsigned kSrc = 98304;
constexpr unsigned kInBytes = 32u << 20;
constexpr int kNMove = 48, kNComp = 208, kNWG = 256;
constexpr unsigned kRowsI = 704u, kRowsC = 688u;
constexpr unsigned kDonePerCall = 192u;
constexpr unsigned long long kDeadline = 50000000ull;
constexpr unsigned kSlot = 24576u, kRing = 6u * kSlot;
constexpr unsigned kLdsMisc = kRing, kLdsBytes = kRing + 128u;
constexpr unsigned kOob = 0x80000000u;
constexpr int kAuxSys = 17;
constexpr int kAuxDev = 16;

struct Params {
  const float* x;
  const float* w;
  char* ws[8];
  float* out[8];
  int rank;
  int pad;
  uint32_t tiles[3 * kNComp];
};

__device__ __forceinline__ __amdgpu_buffer_rsrc_t mk_rsrc(const void* p, unsigned bytes) {
  return __builtin_amdgcn_make_buffer_rsrc((void*)p, 0, bytes, 0x00020000);
}
template <int AUX>
__device__ __forceinline__ f32x4 ld16(__amdgpu_buffer_rsrc_t r, unsigned voff, unsigned soff) {
  return __builtin_bit_cast(f32x4, __builtin_amdgcn_raw_buffer_load_b128(r, voff, soff, AUX));
}
template <int AUX>
__device__ __forceinline__ void st16(f16x8 v, __amdgpu_buffer_rsrc_t r, unsigned voff, unsigned soff) {
  __builtin_amdgcn_raw_buffer_store_b128(__builtin_bit_cast(u32x4, v), r, voff, soff, AUX);
}
__device__ __forceinline__ f16x8 cvt8(f32x4 a, f32x4 b) {
  f16x8 h;
#pragma unroll
  for (int i = 0; i < 4; ++i) { h[i] = (_Float16)a[i]; h[4 + i] = (_Float16)b[i]; }
  return h;
}
__device__ __forceinline__ f16x8 cvt_col(const f32x4 (&v)[8], int i) {
  f16x8 h;
#pragma unroll
  for (int j = 0; j < 8; ++j) h[j] = (_Float16)v[j][i];
  return h;
}
#define WAIT_VM(n) asm volatile("s_waitcnt vmcnt(%0)" :: "n"(n) : "memory")
#define WAIT_LGKM0() asm volatile("s_waitcnt lgkmcnt(0)" ::: "memory")
#define FENCE() asm volatile("" ::: "memory")
#define UNLIKELY(c) __builtin_expect(!!(c), 0)
#define BARRIER() do { FENCE(); __builtin_amdgcn_s_barrier(); FENCE(); } while (0)

__device__ __forceinline__ void flag_store(char* p, unsigned v) {
  __hip_atomic_store((GAS unsigned*)p, v, __ATOMIC_RELAXED, __HIP_MEMORY_SCOPE_SYSTEM);
}
__device__ __forceinline__ void flag_store2(char* p, unsigned lo, unsigned hi) {
  __hip_atomic_store((GAS unsigned long long*)p, (unsigned long long)lo | ((unsigned long long)hi << 32), __ATOMIC_RELAXED, __HIP_MEMORY_SCOPE_SYSTEM);
}
__device__ __forceinline__ unsigned draw(char* p) {
  unsigned z = 0;
  asm volatile("" : "+v"(z));
  p += z;
  return __hip_atomic_fetch_add((GAS unsigned*)p, 1u, __ATOMIC_RELAXED, __HIP_MEMORY_SCOPE_AGENT);
}
__device__ __forceinline__ unsigned uni(unsigned v) { return (unsigned)__builtin_amdgcn_readfirstlane((int)v); }
__device__ __forceinline__ unsigned lanes_below(unsigned long long b) {
  return __builtin_amdgcn_mbcnt_hi((unsigned)(b >> 32), __builtin_amdgcn_mbcnt_lo((unsigned)b, 0u));
}
__device__ __forceinline__ void advance(unsigned& pos, unsigned long long b) {
  asm("s_lshl2_add_u32 %0, %1, %0" : "+s"(pos) : "s"((unsigned)__builtin_popcountll(b)) : "scc");
}

#define VSTAMP(ws, slot, val) do {} while (0)
#define STAMP(ws, slot) do {} while (0)
#define PSTAMP(ws, cls, n) do {} while (0)
#define CSTAMP(ws, slot) do {} while (0)

__device__ __forceinline__ void set_timeout(char* myws, unsigned call) {
  __hip_atomic_store((GAS unsigned*)(myws + kTmo), call, __ATOMIC_RELAXED, __HIP_MEMORY_SCOPE_AGENT);
}
__device__ __forceinline__ unsigned peer_of(unsigned r, unsigned i) { return (r & 4u) + ((r + i + 1u) & 3u); }
__device__ __forceinline__ unsigned panel_row(unsigned ap, unsigned rh, unsigned rank) {
  const unsigned og = (rank & 4u) ^ 4u;
  if (ap < 8u) return rank * 1024u + ap * 128u + rh * 64u;
  if (ap < 23u) { const unsigned i = (ap - 8u) / 5u, s = (ap - 8u) - 5u * i; return peer_of(rank, i) * 1024u + s * 128u + rh * 64u; }
  if (ap < 26u) return peer_of(rank, ap - 23u) * 1024u + 640u;
  if (ap < 34u) return (og + ((ap - 26u) >> 1)) * 1024u + 768u + (ap & 1u) * 128u + rh * 64u;
  return (og + 2u * (ap - 34u) + rh) * 1024u + 704u;
}
__device__ __forceinline__ unsigned park_pos(unsigned ap, unsigned rh, unsigned j) {
  if (ap < 8u) return 2u * ap + rh < 11u ? 2u * ap + rh : 99u;
  if (ap < 23u) {
    const unsigned i = (ap - 8u) / 5u, s = (ap - 8u) - 5u * i;
    return s < 2u ? 11u + 4u * i + 2u * s + rh : s < 4u ? 40u + 4u * i + 2u * s + rh : 26u + 2u * i + rh;
  }
  const unsigned src = ap < 34u ? (ap - 26u) >> 1 : 2u * (ap - 34u) + rh;
  if (src == j) return 99u;
  const unsigned e = src - (src > j ? 1u : 0u);
  return ap < 34u ? 32u + 4u * e + 2u * (ap & 1u) + rh : 23u + e;
}
__device__ __forceinline__ unsigned park_row(unsigned hp, unsigned src, unsigned rank) {
  const unsigned q = rank & 3u;
  if (hp < 11u) return src * 1024u + hp * 64u;
  if (hp < 23u) return peer_of(src, (hp - 11u) >> 2) * 1024u + ((hp - 11u) & 3u) * 64u;
  if (hp < 26u) { const unsigned e = hp - 23u; return ((rank & 4u) + e + (e >= q ? 1u : 0u)) * 1024u + 704u; }
  if (hp < 32u) return peer_of(src, (hp - 26u) >> 1) * 1024u + 512u + (hp & 1u) * 64u;
  if (hp < 44u) { const unsigned e = (hp - 32u) >> 2; return ((rank & 4u) + e + (e >= q ? 1u : 0u)) * 1024u + 768u + (hp & 3u) * 64u; }
  return peer_of(src, (hp - 44u) >> 2) * 1024u + 256u + (hp & 3u) * 64u;
}

}

__global__ void __launch_bounds__(512) __attribute__((amdgpu_waves_per_eu(2, 2)))
ag_gemm_relu_kernel(const Params p) {
  extern __shared__ __attribute__((aligned(16))) char smem_raw[];
  LAS char* const lds = (LAS char*)smem_raw;
  LAS volatile unsigned* const misc = (LAS volatile unsigned*)(lds + kLdsMisc);

  const int tid = threadIdx.x;
  const int lane = tid & 63;
  const int wid = __builtin_amdgcn_readfirstlane(tid >> 6);
  const int bid = blockIdx.x;
  const unsigned rank = (unsigned)p.rank & 7u;
  const unsigned q = rank & 3u, og = (rank & 4u) ^ 4u;
  char* const myws = p.ws[rank];
  const unsigned l16 = (unsigned)lane * 16u;
  const __amdgpu_buffer_rsrc_t rws = mk_rsrc(myws, kWsNeed);
  const int fr = lane & 15, fq = lane >> 4;

  STAMP(myws, wid == 0 ? 0 : 31);

  const unsigned lx = (((unsigned)lane >> 2) * 8192u + 4u * (unsigned)((lane & 3) ^ ((-(lane >> 4)) & 3))) * 4u;
  const unsigned lw = (4u * ((unsigned)lane >> 4) * 1024u + 4u * ((unsigned)lane & 15u)) * 4u;
  const __amdgpu_buffer_rsrc_t rx = mk_rsrc(p.x, kInBytes);
  const __amdgpu_buffer_rsrc_t rw = mk_rsrc(p.w, kInBytes);
  const unsigned pw = (unsigned)wid >= WPM ? (unsigned)wid - WPM : (unsigned)wid;
  const unsigned gw = (unsigned)bid * WPM + pw;
  const unsigned sid = gw % 14u, k = gw / 14u;
  const bool cross = sid >= 6u;
  const unsigned c = cross ? (sid - 6u) >> 1 : sid >> 1, h = sid & 1u;
  const unsigned sr = (cross ? og + c : peer_of(rank, c)) & 7u;
  char* const sws = p.ws[sr];
  const __amdgpu_buffer_rsrc_t rs = mk_rsrc(sws, kWsNeed);
  LAS volatile unsigned* const hand = misc + 8u + 2u * pw;
  const unsigned buf0 = pw * 2u * 17408u;
  if (bid < kNMove && (unsigned)wid >= WPM && (unsigned)wid < 2u * WPM && gw < 14u * NWS) {
    unsigned call = 0;
    auto push = [&](auto wconst) {
      constexpr bool W = decltype(wconst)::value;
      f32x4 pv[PU][2][8];
#pragma unroll
      for (unsigned u = 0; u < PU; ++u)
#pragma unroll
        for (int i = 0; i < 16; ++i) {
          const unsigned m = u * NWS + k, kk = (m >> 1) * 64u + h * 32u;
          if (W) pv[u][i >> 3][i & 7] = ld16<0>(rw, lw + (i >> 3) * 256u, (kk * 1024u + 256u * c + (m & 1u) * 128u) * 4u + ((i & 3) + 16 * ((i >> 2) & 1)) * 4096u);
          else pv[u][i >> 3][i & 7] = ld16<0>(rx, lx + (i & 1) * 64u, ((m & 1u) * 128u * 8192u + kk) * 4u + (i >> 1) * 524288u);
        }
      __syncthreads();
      call = uni(misc[0]) + 1u;
      const unsigned base = W ? kBOff + (call & 1u) * kBSize + (8u + 2u * q) * kPanel : kAOff + (call & 1u) * kASize + (18u - 5u * c) * kPanel;
      char* const fl = sws + kFlags + (W ? 44u + 2u * q : 18u - 5u * c) * 1024u + h * 4u;
#pragma unroll
      for (unsigned j = 0; j < 2u * PU; ++j) {
        const unsigned u = j >> 1, r = j & 1u, m = u * NWS + k;
        f16x8 hv[4];
#pragma unroll
        for (int i = 0; i < 4; ++i) hv[i] = W ? cvt_col(pv[u][r], i) : cvt8(pv[u][r][2 * i], pv[u][r][2 * i + 1]);
        asm volatile("" :: "v"(hv[0]), "v"(hv[1]), "v"(hv[2]), "v"(hv[3]));
        if (j >= PUW) WAIT_VM(4u * (PUW - 1u));
        if (j <= 2u) STAMP(myws, 8 + wid + 3 * j);
        if (u > 0u && r == PUW - 1u && lane == 0) flag_store(fl + ((m - NWS) & 1u) * 1024u + ((m - NWS) >> 1) * 8u, call);
#pragma unroll
        for (int i = 0; i < 4; ++i) st16<kAuxSys>(hv[i], rs, l16 + i * 1024u, base + (m & 1u) * kPanel + (m >> 1) * kBlk + h * 8192u + r * 4096u);
        FENCE();
      }
      WAIT_VM(0);
      const unsigned m = (PU - 1u) * NWS + k;
      if (lane == 0) flag_store(fl + (m & 1u) * 1024u + (m >> 1) * 8u, call);
    };
    if (cross) push(std::true_type{}); else push(std::false_type{});
    STAMP(myws, 4 + wid);
    if (!cross) return;
    const __amdgpu_buffer_rsrc_t ro = mk_rsrc(p.out[rank], kInBytes);
    const unsigned vo = (unsigned)fr * 4096u + (unsigned)fq * 16u;
    const unsigned long long t0 = __builtin_amdgcn_s_memrealtime();
    for (unsigned b = 0;; b ^= 1u) {
      unsigned hv;
      while ((hv = uni(hand[b])) == 0u) {
        if (__builtin_amdgcn_s_memrealtime() - t0 > 2u * kDeadline) { set_timeout(myws, call); return; }
        __builtin_amdgcn_s_sleep(XSLEEP);
      }
      FENCE();
      if (hv == ~0u) break;
      const unsigned oa = park_row(hv >> 4, sr, rank) * 4096u + (256u * c + (hv & 12u) * 16u) * 4u;
      unsigned pos = (unsigned)(size_t)lds + buf0 + b * 17408u, val[64];
      u32x2 mk = {~0u, ~0u};
      if ((hv & 2u) == 0u) mk = *(LAS u32x2*)(size_t)(pos + lane * 8);
      const unsigned fill = (unsigned)(size_t)lds + kLdsMisc + 64u + (hv & 2u) * 2u;
      pos += 512u;
#pragma unroll
      for (int i = 0; i < 64; ++i) {
        const bool keep = (int)(mk[i >> 5] << (i & 31)) >= 0;
        const unsigned long long bl = __ballot(keep);
        unsigned a = lanes_below(bl) * 4u + pos;
        asm volatile("" : "+v"(a));
        val[i] = *(LAS unsigned*)(size_t)(keep ? a : fill);
        advance(pos, bl);
      }
      WAIT_LGKM0();
      if (lane == 0) hand[b] = 0u;
      FENCE();
#pragma unroll
      for (int i = 0; i < 16; ++i)
        __builtin_amdgcn_raw_buffer_store_b128(u32x4{val[4 * i], val[4 * i + 1], val[4 * i + 2], val[4 * i + 3]}, ro, vo + (i & 3) * 64u, oa + (i >> 2) * 65536u, kAuxSys);
    }
    STAMP(myws, 1 + wid);
    return;
  }
  const bool isconv = bid >= kNMove + 8 * CVFROM;
  unsigned tk = 0;
  if (tid == 0) tk = draw(myws + kTicket + (unsigned)bid * 32u);
  FENCE();
  f32x4 cv[8];
  auto item = [&](unsigned n, unsigned& ktg, unsigned& sub) {
    if (n < 5u * CVLEAD) { ktg = n / 5u; sub = n - 5u * ktg; sub += sub ? 3u : 0u; }
    else if (n < 8u * CVLEAD) { ktg = (n - 5u * CVLEAD) / 3u; sub = n - 5u * CVLEAD - 3u * ktg + 1u; }
    else { ktg = n >> 3; sub = n & 7u; }
  };
  auto loads = [&](unsigned n) {
    unsigned ktg, sub;
    item(n, ktg, sub);
    if (sub < 4u) {
      const unsigned so = ((256u * sub + 32u * wid) * 8192u + ktg * 64u) * 4u;
#pragma unroll
      for (int s = 0; s < 2; ++s) {
        cv[4 * s + 0] = ld16<0>(rx, lx, so + s * 524288u); cv[4 * s + 1] = ld16<0>(rx, lx + 64u, so + s * 524288u);
        cv[4 * s + 2] = ld16<0>(rx, lx + 128u, so + s * 524288u); cv[4 * s + 3] = ld16<0>(rx, lx + 192u, so + s * 524288u);
      }
    } else {
      const unsigned so = ((ktg * 64u + 32u * (wid & 1)) * 1024u + 256u * (sub - 4u) + 64u * (wid >> 1)) * 4u;
#pragma unroll
      for (int i = 0; i < 8; ++i) cv[i] = ld16<0>(rw, lw, so + ((i & 3) + 16 * (i >> 2)) * 4096u);
    }
  };
  if (isconv) loads((unsigned)(bid - kNMove - 8 * CVFROM));
  FENCE();
  if (tid == 0) {
    misc[0] = tk;
    misc[3] = 0u;
  }
  if (tid >= 8 && tid < 18) misc[tid] = tid == 17 ? 0x7fc00000u : 0u;
  __syncthreads();
  const unsigned call = uni(misc[0]) + 1u;
  const unsigned par = call & 1u;
  const unsigned abase = kAOff + par * kASize, bbase = kBOff + par * kBSize, obuf = kOOff + par * kOSize;

  if (bid < kNMove) {
    if (wid == 7) {
      if (bid == 0 && lane == 0) {
        const unsigned target = call * kDonePerCall;
        const unsigned long long t0 = __builtin_amdgcn_s_memrealtime();
        bool failed = false;
        while ((int)(__hip_atomic_load((GAS unsigned*)(myws + kDone), __ATOMIC_RELAXED, __HIP_MEMORY_SCOPE_SYSTEM) - target) < 0) {
          if (__hip_atomic_load((GAS unsigned*)(myws + kTmo), __ATOMIC_RELAXED, __HIP_MEMORY_SCOPE_AGENT) == call ||
              __builtin_amdgcn_s_memrealtime() - t0 > kDeadline) {
            set_timeout(myws, call);
            failed = true;
            break;
          }
          __builtin_amdgcn_s_sleep(2);
        }
        __builtin_amdgcn_fence(__ATOMIC_ACQUIRE, "");
        if (failed) __hip_atomic_store((GAS unsigned*)(p.out[rank] + (size_t)rank * 1024u * 1024u), 0x7fc00000u, __ATOMIC_RELAXED, __HIP_MEMORY_SCOPE_SYSTEM);
        STAMP(myws, 10);
      }
      return;
    }
    if ((unsigned)wid >= WPM || gw >= 14u * NWS) return;
    const unsigned total = cross ? kRowsC : kRowsI;
    const unsigned nplain = cross ? 576u : kRowsI;
    const unsigned qbase = (call - 1u) * (total - PU * NWS) - (PU + 1u) * NWS;
    bool failed = false, all = false;
    unsigned cgrp = 99u, okt = 0, wbase = 0x10000u;
    u32x2 fw = {0u, 0u};
    unsigned sa = 0, da = 0, fa = 0, pend = 0;
    unsigned s2 = 4096u, d2 = 4096u;
    u32x4 v[4], w[4];
    auto raise = [&]() {
      if (pend != 0u && lane == 0) flag_store(myws + pend, call);
      pend = 0u;
    };
    auto fetch = [&](u32x4 (&x)[4], unsigned so) {
#pragma unroll
      for (int i = 0; i < 4; ++i) x[i] = __builtin_amdgcn_raw_buffer_load_b128(rs, l16 + i * 1024u, so, kAuxSys);
      FENCE();
    };
    auto put = [&](const u32x4 (&x)[4], unsigned so) {
#pragma unroll
      for (int i = 0; i < 4; ++i) __builtin_amdgcn_raw_buffer_store_b128(x[i], rws, l16 + i * 1024u, so, kAuxDev);
      FENCE();
    };
    auto setup = [&](unsigned m) {
      unsigned sp, kt, dp, db, dh = h;
      unsigned so = h * 8192u, dof = h * 8192u;
      s2 = 4096u; d2 = 4096u;
      if (!cross) {
        if (m < 256u) { sp = m & 1u; kt = m >> 1; dp = 8u + 5u * c + sp; db = kt; }
        else if (m < 384u) { sp = 4u; kt = m - 256u; dp = 12u + 5u * c; db = kt; }
        else if (m < 640u) { sp = 2u + (m & 1u); kt = (m - 384u) >> 1; dp = 8u + 5u * c + sp; db = kt; }
        else { sp = 5u; db = m - 640u; kt = 2u * db + h; so = 0u; s2 = 8192u; dp = 23u + c; }
      } else {
        if (m < 256u) { sp = 36u + 2u * q + (m & 1u); kt = m >> 1; dp = 44u + 2u * c + (m & 1u); db = kt; }
        else if (m < 320u) { sp = 5u; kt = 2u * (m - 256u) + h; so = 4096u; s2 = 8192u; dp = 34u + (c >> 1); db = kt; dh = c & 1u; dof = dh * 4096u; d2 = 8192u; }
        else { sp = 6u + (m & 1u); kt = (m - 320u) >> 1; dp = 26u + 2u * c + (m & 1u); db = kt; }
      }
      const unsigned sub = sp < 36u ? sp >> 1 : (sp - 28u) >> 1;
      if (!all && !(sub == cgrp && kt < okt)) {
        WAIT_VM(0);
        raise();
        char* const pa = myws + (lane < 62 ? kSrc + (sr * 8u + sub) * 512u + min(kt + (unsigned)lane, 127u) * 4u : lane == 62 ? kAll + sr * 4u : kTmo);
        const unsigned long long t0 = __builtin_amdgcn_s_memrealtime();
        for (;;) {
          const unsigned f = __hip_atomic_load((GAS unsigned*)pa, __ATOMIC_RELAXED, __HIP_MEMORY_SCOPE_SYSTEM);
          const unsigned long long b = __ballot(lane < 63 ? (int)(f - call) >= 0 : f == call);
          if ((b >> 62) != 0ull) { failed = (b >> 63) != 0ull; all = true; break; }
          if (b & 1ull) { cgrp = sub; okt = kt + (unsigned)__builtin_ctzll(~b); break; }
          if (__builtin_amdgcn_s_memrealtime() - t0 > kDeadline) { set_timeout(myws, call); failed = true; all = true; break; }
          __builtin_amdgcn_s_sleep(MSLEEP);
        }
      }
      sa = (sp < 36u ? abase + sp * kPanel : bbase + (sp - 36u) * kPanel) + kt * kBlk + so;
      da = (dp < 36u ? abase + dp * kPanel : bbase + (dp - 36u) * kPanel) + db * kBlk + dof;
      fa = kFlags + dp * 1024u + db * 8u + dh * 4u;
      fetch(v, sa);
    };
    unsigned m = PU * NWS + k;
    setup(m);
    STAMP(myws, 20 + wid);
    do {
      unsigned dv = 0;
      if (lane == 0) dv = draw(myws + kQueue + sid * 128u);
      FENCE();
      put(v, da);
      raise();
      fetch(w, sa + s2);
      put(w, da + d2);
      pend = fa;
      if (h == 0u) { PSTAMP(myws, sid >> 1, m + 1u); if (m < 128u) PSTAMP(myws, 8u + (sid >> 1), (m + 1u) * 4u); }
      m = uni(dv) - qbase;
      if (m < nplain) setup(m);
    } while (m < nplain);
    WAIT_VM(0);
    raise();
    if (cross) {
      unsigned hval = 0;
      const unsigned fo = lane < 32 ? kOFlags + par * 8192u + c * 2048u + h * 8u + (unsigned)lane * 16u : kTmo;
      auto stale = [&](unsigned o) {
        return o - wbase >= 32u || (int)((unsigned)__builtin_amdgcn_readlane((int)fw[0], (int)(o - wbase)) - call) < 0;
      };
      for (unsigned b = 0;; b ^= 1u) {
        const unsigned o = m - 576u, s = 2u * o + h;
        unsigned cnt = 2048u, dv = 0;
        if (m >= total || stale(o)) {
          WAIT_VM(0);
          if (hval != 0u && lane == 0) hand[b ^ 1u] = hval;
          hval = 0u;
          const unsigned long long t0 = __builtin_amdgcn_s_memrealtime();
          wbase = o;
          while (!failed && m < total) {
            fw = __builtin_bit_cast(u32x2, __hip_atomic_load((GAS unsigned long long*)(myws + fo + (lane < 32 ? o * 16u : 0u)), __ATOMIC_RELAXED, __HIP_MEMORY_SCOPE_SYSTEM));
            const unsigned long long bl = __ballot(lane < 32 ? (int)(fw[0] - call) >= 0 : fw[0] == call);
            if ((bl >> 32) != 0ull) { failed = true; break; }
            if (bl & 1ull) break;
            if (__builtin_amdgcn_s_memrealtime() - t0 > kDeadline) { set_timeout(myws, call); failed = true; }
            __builtin_amdgcn_s_sleep(MSLEEP);
          }
        }
        cnt = failed ? 0u : min((unsigned)__builtin_amdgcn_readlane((int)fw[1], (int)((o - wbase) & 31u)), 4096u);
        if (UNLIKELY(uni(hand[b]) != 0u)) {
          const unsigned long long t0 = __builtin_amdgcn_s_memrealtime();
          while (uni(hand[b]) != 0u)
            if (__builtin_amdgcn_s_memrealtime() - t0 > kDeadline) { set_timeout(myws, call); return; }
        }
        FENCE();
        if (m >= total) {
          if (lane == 0) hand[b] = ~0u;
          break;
        }
        const unsigned bytes = 512u + 4u * cnt, sb = obuf + (q * 224u + s) * kPiece, nbuf = buf0 + b * 17408u;
        auto part = [&](unsigned f0, unsigned f1) {
          for (unsigned f = f0; f < f1; f += 1024u) {
            WAIT_VM(PWIN - 1u);
            asm volatile("s_mov_b32 m0, %0\n\ts_nop 0\n\tbuffer_load_dwordx4 %1, %2, %3 offen sc0 sc1 lds"
                         :: "s"(nbuf + f), "v"(f + l16 < bytes ? l16 : kOob), "s"(rs), "s"(sb + f) : "memory");
          }
        };
        asm volatile("s_mov_b64 exec, 1\n\tglobal_atomic_add %0, %1, %2, %3 sc0\n\ts_mov_b64 exec, -1"
                     : "=v"(dv) : "v"(kQueue + sid * 128u), "v"(1u), "s"(myws) : "memory");
        part(0u, min(bytes, (PWIN - 1u) * 1024u));
        if (bytes <= (PWIN - 2u) * 1024u) WAIT_VM(0);
        WAIT_VM(PWIN - 1u);
        if (hval != 0u && lane == 0) hand[b ^ 1u] = hval;
        hval = 4u * s + (failed ? 3u : 1u);
        if (h == 0u) PSTAMP(myws, sid >> 1, m + 1u);
        asm volatile("" : "+v"(dv));
        m = uni(dv) - qbase;
        if (m < total && stale(m - 576u)) {
          wbase = m - 576u;
          asm volatile("buffer_load_dwordx2 %0, %1, %2, 0 offen sc0 sc1" : "=v"(fw) : "v"(fo + (lane < 32 ? wbase * 16u : 0u)), "s"(rws) : "memory");
          part((PWIN - 1u) * 1024u, bytes);
          if (bytes <= (2u * PWIN - 3u) * 1024u) WAIT_VM(0);
          WAIT_VM(PWIN - 1u);
          asm volatile("" : "+v"(fw) :: "memory");
        } else
          part((PWIN - 1u) * 1024u, bytes);
      }
    }
    STAMP(myws, 1 + wid);
    return;
  }

  {
    constexpr unsigned kNConv = 8u * (26u - CVFROM);
    char* const pws = p.ws[lane & 7];
    const unsigned mine = 15u << (rank & 4u);
    STAMP(myws, wid == 0 ? 27 : 31);
    for (unsigned n = isconv ? (unsigned)(bid - kNMove - 8 * CVFROM) : 1024u; n < 1024u; n += kNConv) {
      unsigned ktg, sub;
      item(n, ktg, sub);
      if (sub < 4u) {
        const unsigned so = abase + (2u * sub + (wid >> 2)) * kPanel + ktg * kBlk + (wid & 3) * 2048u;
#pragma unroll
        for (int i = 0; i < 4; ++i) st16<CVAUX>(cvt8(cv[2 * i], cv[2 * i + 1]), rws, l16 + (i >> 1) * 1024u, so + (i & 1) * 8192u);
      } else {
        const unsigned so = bbase + (2u * (sub - 4u) + (wid >> 2)) * kPanel + ktg * kBlk + ((wid >> 1) & 1) * 4096u + (wid & 1) * 8192u;
#pragma unroll
        for (int i = 0; i < 4; ++i) st16<CVAUX>(cvt_col(cv, i), rws, l16 + i * 1024u, so);
      }
      FENCE();
      const bool ahead = n >= kNConv && n + kNConv < 1024u;
      if (ahead) { loads(n + kNConv); WAIT_VM(8); } else { WAIT_VM(0); }
      if (n < kNConv) STAMP(myws, wid == 0 ? 30 : 31);
      BARRIER();
      if (n < 2u * kNConv) STAMP(myws, wid == 0 ? (n < kNConv ? 28 : 29) : 31);
      const unsigned to = sub < 2u ? mine : sub == 2u ? 255u : sub == 3u ? mine ^ 255u : 1u << (og + sub - 4u);
      if (tid < 2) flag_store2(myws + kFlags + ((sub < 4u ? 2u * sub : 28u + 2u * sub) + (unsigned)tid) * 1024u + ktg * 8u, call, call);
      else if (wid == 1 && lane < 8 && (((to & ~(1u << rank)) >> lane) & 1u)) flag_store(pws + kSrc + rank * 4096u + sub * 512u + ktg * 4u, call);
      FENCE();
      if (n < kNConv) loads(n + kNConv);
    }
    if (isconv && wid == 7) {
      unsigned dv = 0;
      if (lane == 0) dv = draw(myws + kCvt);
      if (uni(dv) == call * kNConv - 1u && lane < 8) flag_store(pws + kAll + rank * 4u, call);
    }
  }
  STAMP(myws, wid == 0 ? 1 : 31);

  const uint32_t* const tl = p.tiles + 3 * (bid - kNMove);
  const int ntiles = (int)(tl[0] >> 28);
  if (wid == 7) return;

  if (wid >= 4) {
    const unsigned lw = (unsigned)wid - 4u;
    unsigned total = 0;
    for (int ti = 0; ti < ntiles; ++ti) total += ((tl[ti] >> 10) & 3u) == 2u ? 64u : 256u;
    unsigned a = 0, aend = 0, so = lw * 8192u, src = 0, step = 0, g2 = 0, v2 = l16, idle = 0;
    unsigned ok = 0, f0 = 0, f1 = 0, fb = 0, fsh = 0, fmax = 0, fv = 0;
    unsigned rdy = 0, askat = 0, lookat = ~0u;
    int ta = 0, tf = 0;
    auto next_flags = [&]() {
      while (tf < ntiles) {
        const unsigned code = tl[tf++];
        const unsigned ap = code & 63u, bp = (code >> 6) & 15u, shape = (code >> 10) & 3u;
        f0 = f1;
        f1 += shape == 2u ? 64u : 256u;
        if (shape == 3u && lw == 2u) { ok = f1; continue; }
        const unsigned grp = lw == 0u ? ap : shape == 0u ? (lw == 1u ? ap + 1u : 36u + bp) : shape == 1u ? 35u + bp + lw : 36u + bp;
        fb = kFlags + grp * 1024u;
        fsh = shape != 2u ? 0u : lw == 0u ? 1u : 2u;
        fmax = fsh == 1u ? 127u : 255u;
        break;
      }
    };
    auto known = [&]() {
      rdy = min(ok, aend);
      unsigned from = max(ok, LTHR);
      asm volatile("" : "+s"(from));
      askat = ok >= total ? ~0u : from - LTHR;
    };
    auto ask = [&]() {
      const unsigned off = lane < 63 ? fb + min(((ok - f0) << fsh) + (unsigned)lane, fmax) * 4u : kTmo;
      asm volatile("buffer_load_dword %0, %1, %2, 0 offen sc0 sc1" : "=v"(fv) : "v"(off), "s"(rws) : "memory");
    };
    auto give_up = [&]() {
      ok = total;
      if (lane == 0) misc[3] = 1u;
      WAIT_LGKM0();
    };
    auto look = [&]() {
      asm volatile("" : "+v"(fv) :: "memory");
      const unsigned long long b = __ballot(lane < 63 ? (int)(fv - call) >= 0 : fv == call);
      if (UNLIKELY((b >> 63) != 0ull)) {
        give_up();
      } else {
        const unsigned run = (unsigned)__builtin_ctzll(~b);
        ok = min(ok + (fsh == 0u ? run & ~1u : run >> fsh), f1);
        if (UNLIKELY(ok == f1)) next_flags();
      }
      lookat = ~0u;
    };
    auto issue = [&](bool busy) {
      LAS char* const dl = lds + so;
      if (busy) {
        __builtin_amdgcn_raw_ptr_buffer_load_lds(rws, (LAS void*)dl, 16, l16, src, 0, kAuxDev);
        __builtin_amdgcn_raw_ptr_buffer_load_lds(rws, (LAS void*)dl, 16, l16, src, 1024, kAuxDev);
        __builtin_amdgcn_raw_ptr_buffer_load_lds(rws, (LAS void*)dl, 16, v2, src, 2048, kAuxDev);
        __builtin_amdgcn_raw_ptr_buffer_load_lds(rws, (LAS void*)dl, 16, v2, src, 3072, kAuxDev);
        __builtin_amdgcn_raw_ptr_buffer_load_lds(rws, (LAS void*)(dl + 4096), 16, l16, src + g2, 0, kAuxDev);
        __builtin_amdgcn_raw_ptr_buffer_load_lds(rws, (LAS void*)(dl + 4096), 16, l16, src + g2, 1024, kAuxDev);
        __builtin_amdgcn_raw_ptr_buffer_load_lds(rws, (LAS void*)(dl + 4096), 16, v2, src + g2, 2048, kAuxDev);
        __builtin_amdgcn_raw_ptr_buffer_load_lds(rws, (LAS void*)(dl + 4096), 16, v2, src + g2, 3072, kAuxDev);
      }
      FENCE();
      src += step;
      so = so >= kRing - kSlot ? so - (kRing - kSlot) : so + kSlot;
      ++a;
    };
    next_flags();
    known();
    for (unsigned j = 0; j < total; ++j) {
      if (a == j + 4u) {
        unsigned e = min(rdy + 1u, askat);
        asm volatile("" : "+s"(e));
        e = min(e, lookat);
        if (idle != 0u) {
          WAIT_VM(0);
          while (a + 1u < e) { issue(false); BARRIER(); }
        }
        while (a + 1u < e) {
          issue(true);
          WAIT_VM(32);
          BARRIER();
        }
        j = a - 4u;
      }
      unsigned lim = min(j + 5u, rdy);
      if (UNLIKELY(a >= lim)) {
        if (a == aend) {
          const unsigned code = tl[ta++];
          const unsigned ap = code & 63u, bp = (code >> 6) & 15u, shape = (code >> 10) & 3u;
          if (lw == 0u || (lw == 1u && shape == 0u)) src = abase + (ap + lw) * kPanel;
          else src = bbase + (bp + (shape == 1u ? lw - 1u : 0u)) * kPanel;
          step = 8192u; g2 = 4096u;
          idle = shape == 3u && lw == 2u ? 1u : 0u;
          if (shape == 2u) {
            g2 = 8192u;
            if (lw == 0u) { src += ((code >> 13) & 1u) * 2048u; step = kBlk; v2 = l16 + 2048u; }
            else { src += ((code >> 12) & 1u) * 4096u + (lw - 1u) * kBlk; step = 2u * kBlk; }
          }
          aend = ta == ntiles ? ~0u : aend + 256u;
          rdy = min(ok, aend);
        }
        if (a <= j && ok <= a) {
          WAIT_VM(0);
          if (lookat != ~0u) look();
          const unsigned long long t0 = __builtin_amdgcn_s_memrealtime();
          while (ok <= a) {
            ask();
            WAIT_VM(0);
            look();
            if (ok > a) break;
            if (__builtin_amdgcn_s_memrealtime() - t0 > kDeadline) { set_timeout(myws, call); give_up(); break; }
            __builtin_amdgcn_s_sleep(LSLEEP);
          }
          known();
        }
        lim = min(j + 5u, rdy);
      }
      if (a < lim) {
        issue(idle == 0u);
        if (UNLIKELY(a < lim)) issue(idle == 0u);
      }
      if (UNLIKELY(a >= askat)) { ask(); askat = ~0u; lookat = a + 4u; }
      const unsigned n = idle != 0u ? 0u : a - 1u - j;
      if (UNLIKELY(n < 4u)) {
        WAIT_VM(24);
        if (n < 3u) {
          WAIT_VM(16);
          if (n < 2u) {
            WAIT_VM(8);
            if (n < 1u) WAIT_VM(0);
          }
        }
      }
      WAIT_VM(32);
      if (UNLIKELY(j + 5u >= lookat)) { look(); known(); }
      BARRIER();
    }
    return;
  }

  {
    const unsigned slot = (unsigned)(4 * fr + (fq ^ ((-(fr >> 2)) & 3))) * 16u;
    const unsigned slotb = (unsigned)(16 * fq + 4 * (fr & 3) + (fr >> 2)) * 16u;
    unsigned so = 0;
    for (int ti = 0; ti < ntiles; ++ti) {
      const unsigned code = tl[ti];
      const unsigned shape = (code >> 10) & 3u;
      const unsigned wh = (unsigned)wid >> 1, wl = (unsigned)wid & 1u;
      const unsigned ap = (code & 63u) + (shape == 0u ? wh : 0u), bp = ((code >> 6) & 15u) + (shape == 1u ? wh : 0u);
      const unsigned jo = (bp - 8u) >> 1;
      const unsigned owner = (bp < 8u ? rank : og + jo) & 7u;
      const unsigned col0 = bp < 8u ? bp * 128u : 256u * q + (bp & 1u) * 128u;
      const __amdgpu_buffer_rsrc_t ro = mk_rsrc(p.out[owner], kInBytes);
      const unsigned vo = (unsigned)fr * 4096u + (unsigned)fq * 16u;

      auto run = [&](auto mconst, const unsigned aoff, const unsigned boff) {
        constexpr int M = decltype(mconst)::value;
        constexpr int KS = M > 1 ? 1 : 4;
        constexpr int NS = M > 1 ? 256 : 64;
        constexpr int NA = M * KS, NB = 4 * KS;
        const unsigned va = aoff + slot, vb = boff + slotb;
        f32x4 acc[M][4];
#pragma unroll
        for (int m = 0; m < M; ++m)
#pragma unroll
          for (int n = 0; n < 4; ++n) acc[m][n] = f32x4{0.f, 0.f, 0.f, 0.f};
        f16x8 fa[2][NA], fb[2][NB];
        auto rd = [&](f16x8 (&a)[NA], f16x8 (&b)[NB]) {
          LAS char* const pa = lds + (va + so);
          LAS char* const pb = lds + (vb + so);
#pragma unroll
          for (int i = 0; i < (NA > NB ? NA : NB); ++i) {
            if (i < NB) b[i] = *(LAS f16x8*)(pb + i * 1024);
            if (i < NA) a[i] = *(LAS f16x8*)(pa + i * (M > 1 ? 1024 : 2048));
          }
        };
        auto mm = [&](const f16x8 (&a)[NA], const f16x8 (&b)[NB]) {
#pragma unroll
          for (int ks = 0; ks < KS; ++ks)
#pragma unroll
            for (int m = 0; m < M; ++m)
#pragma unroll
              for (int n = 0; n < 4; ++n)
                acc[m][n] = __builtin_amdgcn_mfma_f32_16x16x32_f16(b[4 * ks + n], a[KS * m + ks], acc[m][n], 0, 0, 0);
        };
        BARRIER();
        STAMP(myws, wid == 0 ? 2 + 3 * ti : 31);
        CSTAMP(myws, wid == 0 ? 12 + 2 * ti : 31);
        rd(fa[0], fb[0]);
        __builtin_amdgcn_s_waitcnt(0xC07F);
        BARRIER();
        so = so == kRing - kSlot ? 0u : so + kSlot;
        if constexpr (M > 1) {
          unsigned ua = va + so, ub = vb + so;
          for (int t = 0; t < NS; t += 2) {
#pragma unroll
            for (int s = 0; s < 2; ++s) {
              __builtin_amdgcn_sched_barrier(0);
#pragma unroll
              for (int i = 0; i < 4 * M; ++i) {
                acc[i >> 2][i & 3] = __builtin_amdgcn_mfma_f32_16x16x32_f16(fb[s][i & 3], fa[s][i >> 2], acc[i >> 2][i & 3], 0, 0, 0);
                __builtin_amdgcn_sched_barrier(0);
                if (M == 8 && (i & 1) == 0 && i < 24) {
                  const int r = i >> 1;
                  if (r < 4) fb[s ^ 1][r] = *(LAS f16x8*)(lds + ub + r * 1024);
                  else fa[s ^ 1][r - 4] = *(LAS f16x8*)(lds + ua + (r - 4) * 1024);
                }
                if (M == 4 && i < 4) {
                  fb[s ^ 1][i] = *(LAS f16x8*)(lds + ub + i * 1024);
                  fa[s ^ 1][i] = *(LAS f16x8*)(lds + ua + i * 1024);
                }
                if (i == 4 * M - 10) { so = so == kRing - kSlot ? 0u : so + kSlot; asm volatile("" : "+s"(so)); }
                if (i == 4 * M - 6) { ua = va + so; asm volatile("" : "+v"(ua)); }
                if (i == 4 * M - 4) { ub = vb + so; asm volatile("" : "+v"(ub)); }
                if (i == 4 * M - 2) {
                  __builtin_amdgcn_s_waitcnt(0xC07F);
                  if (t < NS - 2) BARRIER();
                }
                __builtin_amdgcn_sched_barrier(0);
              }
            }
          }
          so = so == 0u ? kRing - kSlot : so - kSlot;
        } else {
          for (int t = 0; t < NS; t += 2) {
#pragma unroll
            for (int s = 0; s < 2; ++s) {
              __builtin_amdgcn_sched_barrier(0);
              rd(fa[s ^ 1], fb[s ^ 1]);
              mm(fa[s], fb[s]);
#pragma unroll
              for (int i = 0; i < 4; ++i) {
                __builtin_amdgcn_sched_group_barrier(0x100, 2, 0);
                __builtin_amdgcn_sched_group_barrier(0x008, 1, 0);
              }
#pragma unroll
              for (int i = 0; i < 12; ++i) {
                __builtin_amdgcn_sched_group_barrier(0x100, 1, 0);
                __builtin_amdgcn_sched_group_barrier(0x008, 1, 0);
              }
              __builtin_amdgcn_sched_barrier(0);
              so = so == kRing - kSlot ? 0u : so + kSlot;
              __builtin_amdgcn_s_waitcnt(0xC07F);
              if (t < NS - 2) BARRIER();
            }
          }
        }
        STAMP(myws, wid == 0 ? 3 + 3 * ti : 31);
        CSTAMP(myws, wid == 0 ? 13 + 2 * ti : 31);
        const int lo = misc[3] != 0u ? 0x7fc00000 : 0;
        if (M > 1 && bp >= 8u) {
          const __amdgpu_buffer_rsrc_t rw4 = __builtin_amdgcn_make_buffer_rsrc((void*)myws, 4, kWsNeed / 4u, 0x00020000);
          if (UNLIKELY(lo != 0)) {
#pragma unroll
            for (int m = 0; m < M; ++m)
#pragma unroll
              for (int n = 0; n < 4; ++n) acc[m][n] = __builtin_bit_cast(f32x4, u32x4{0x7fc00000u, 0x7fc00000u, 0x7fc00000u, 0x7fc00000u});
          }
#pragma unroll
          for (int rh = 0; rh < M / 4; ++rh) {
            const unsigned hp = park_pos(ap, M == 8 ? (unsigned)rh : wh, jo);
            if (hp < 56u) {
              const unsigned s = hp * 4u + (bp & 1u) * 2u + wl;
              const unsigned pb0 = obuf + (jo * 224u + s) * kPiece;
              unsigned pos = pb0 + 512u;
              u32x2 mk = {0u, 0u};
#pragma unroll
              for (int m = 0; m < 4; ++m)
#pragma unroll
                for (int j = 0; j < 4; ++j)
#pragma unroll
                  for (int n = 0; n < 4; ++n) {
                    const float f = acc[4 * rh + m][n][j];
                    const unsigned iv = __float_as_uint(f);
                    const unsigned long long b = __ballot((int)iv >= 0);
                    const unsigned idx = lanes_below(b) | (iv & kOob);
                    asm volatile("buffer_store_dword %0, %1, %2, %3 idxen sc0 sc1" :: "v"(iv), "v"(idx), "s"(rw4), "s"(pos) : "memory");
                    advance(pos, b);
                    mk[m >> 1] = __builtin_amdgcn_alignbit(mk[m >> 1], iv, 31);
                  }
              __builtin_amdgcn_raw_buffer_store_b64(mk, rws, (unsigned)lane * 8u, pb0, kAuxSys);
              WAIT_VM(0);
              if (lane == 0) flag_store2(p.ws[owner] + kOFlags + par * 8192u + q * 2048u + s * 8u, call, (pos - pb0 - 512u) >> 2);
            }
          }
        } else {
#pragma unroll
          for (int m = 0; m < M; ++m) {
            const unsigned ob = M > 1 ? (panel_row(ap, M == 8 ? (unsigned)(m >> 2) : wh, rank) + 16u * (m & 3)) * 4096u + (col0 + 64u * wl) * 4u
                                       : (panel_row(ap, 0u, rank) + 32u * ((code >> 13) & 1u) + 16u * wl) * 4096u + (col0 + 64u * ((code >> 12) & 1u)) * 4u;
#pragma unroll
            for (int j = 0; j < 4; ++j) {
              u32x4 o;
#pragma unroll
              for (int n = 0; n < 4; ++n) { const float f = acc[m][n][j]; o[n] = (unsigned)max(__float_as_int(f), lo); }
              __builtin_amdgcn_raw_buffer_store_b128(o, ro, vo + j * 64u, ob, kAuxSys);
            }
          }
          if (bp >= 8u) {
            WAIT_VM(0);
            if (lane == 0) __hip_atomic_fetch_add((GAS unsigned*)(p.ws[owner] + kDone), 1u, __ATOMIC_RELAXED, __HIP_MEMORY_SCOPE_SYSTEM);
          }
        }
      };
      if (shape == 2u) {
        if (wid >= 2) return;
        run(std::integral_constant<int, 1>{}, wl * 1024u, 8192u);
      } else if (shape == 3u) {
        run(std::integral_constant<int, 4>{}, wh * 4096u, 8192u + wl * 4096u);
      } else {
        run(std::integral_constant<int, 8>{}, shape == 0u ? wh * 8192u : 0u, (shape == 0u ? 16384u : 8192u + wh * 8192u) + wl * 4096u);
      }
      STAMP(myws, wid == 0 ? 4 + 3 * ti : 31);
    }
  }
}

extern "C" void kernel_launch(int rank, int nranks, ncclComm_t comm, hipStream_t stream,
                              void* const* d_in, const int* in_sizes, int n_in,
                              void* d_out, int out_size,
                              const int* in_dtypes, int out_dtype,
                              void* const* d_out_all, void* const* d_ws_all, size_t ws_bytes) {
  (void)comm; (void)in_dtypes; (void)out_dtype; (void)d_out;
  if (nranks != 8 || (unsigned)rank >= 8u || n_in < 2 || ws_bytes < (size_t)kWsNeed ||
      in_sizes[0] != 8 * 1024 * 1024 || in_sizes[1] != 8 * 1024 * 1024 || out_size != 8 * 1024 * 1024) {
    fprintf(stderr, "kernel_launch: unexpected problem geometry\n");
    return;
  }
  Params p;
  p.x = (const float*)d_in[0];
  p.w = (const float*)d_in[1];
  for (int i = 0; i < 8; ++i) { p.ws[i] = (char*)d_ws_all[i]; p.out[i] = (float*)d_out_all[i]; }
  p.rank = rank;
  p.pad = 0;
  static const int xcol[8] = {0, 4, 2, 3, 1, 5, 6, 7};
  auto tile = [](int a, int b, int shape) { return (unsigned)(a | (b << 6) | (shape << 10)); };
  for (int ci = 0; ci < kNComp; ++ci) {
    const int x = ci & 7, s = ci >> 3, bq = 2 * (x ^ 4);
    unsigned t[3] = {0u, 0u, 0u};
    int n = 0;
    auto far67 = [&](int y, int e) {
      const int j = y >> 1, idx = 3 * (y & 1) + (e >> 1);
      return tile(26 + 2 * j + (e & 1), 8 + idx + (idx >= 2 * j ? 2 : 0), 3);
    };
    bool last = true;
    if (s < 12) t[n++] = tile(x < 4 ? 2 * (s >> 2) : 8 + 5 * (s >> 2), 4 * (x & 3) + (s & 3), 0);
    else if (s == 12) t[n++] = tile(6, xcol[x], 0);
    else if (s < 16) t[n++] = tile(12 + 5 * (s - 13), bq, 1);
    else if (s < 18) t[n++] = tile(34 + (s - 16), bq, 1);
    else if (s < 22) t[n++] = tile(26 + 2 * (x >> 1), 4 * (x & 1) + s - 18, 0);
    else t[n++] = far67(x, s - 22);
    if (s == 12) t[n++] = far67(x, 4);
    else if (x >= 4 && s >= 10 && s < 12) t[n++] = far67(s == 10 ? x : x - 4, 5);
    else if (x >= 4 && s < 2) { t[n++] = tile(10, bq + s, 0); last = false; }
    else if (s < 12) {
      const int i = x >= 4 ? s + 2 : s;
      t[n++] = tile(10 + 5 * (i >> 2) + ((i >> 1) & 1), bq + (i & 1), 3);
      if (s < 2) last = false;
    }
    if (last) {
      const int i = s - 2;
      t[n++] = tile(23 + i / 8, 2 * x + ((i >> 1) & 1), 2) | ((unsigned)(i & 1) << 12) | ((unsigned)((i >> 2) & 1) << 13);
    }
    p.tiles[3 * ci] = t[0] | ((unsigned)n << 28);
    p.tiles[3 * ci + 1] = t[1];
    p.tiles[3 * ci + 2] = t[2];
  }
  hipError_t e = hipFuncSetAttribute((const void*)ag_gemm_relu_kernel, hipFuncAttributeMaxDynamicSharedMemorySize, (int)kLdsBytes);
  if (e != hipSuccess) { fprintf(stderr, "hipFuncSetAttribute: %s\n", hipGetErrorString(e)); return; }
  hipLaunchKernelGGL(ag_gemm_relu_kernel, dim3(kNWG), dim3(512), kLdsBytes, stream, p);
  e = hipGetLastError();
  if (e != hipSuccess) fprintf(stderr, "launch: %s\n", hipGetErrorString(e));
}
```

```cpp
#include <hip/hip_runtime.h>
#include <rccl/rccl.h>
#include <cstdio>
#include <cstdint>
#include <type_traits>

typedef _Float16 f16x8 __attribute__((ext_vector_type(8)));
typedef float f32x4 __attribute__((ext_vector_type(4)));
typedef unsigned u32x4 __attribute__((ext_vector_type(4)));
typedef unsigned u32x2 __attribute__((ext_vector_type(2)));
#define LAS __attribute__((address_space(3)))
#define GAS __attribute__((address_space(1)))

#define NWS 9u
#define WPM 3u
#define PU 1u
#define PUW 1u
#define CVDELAY 24
#define CVFIRST 60
#define CVLEAD 40u
#define XSLEEP 4
#define CVFROM 13
#define PWIN 4u
#define CVAUX kAuxSys
#define MSLEEP 4
#define LSLEEP 2
#define LTHR 24u

namespace {

constexpr unsigned kBlk = 16384u;
constexpr unsigned kPanel = 2u << 20;
constexpr unsigned kAOff = 1u << 20;
constexpr unsigned kASize = 36u * kPanel;
constexpr unsigned kBOff = kAOff + 2u * kASize;
constexpr unsigned kBSize = 16u * kPanel;
constexpr unsigned kPiece = 16896u;
constexpr unsigned kOOff = kBOff + 2u * kBSize;
constexpr unsigned kOSize = 4u * 224u * kPiece;
constexpr unsigned kWsNeed = kOOff + 2u * kOSize;
constexpr unsigned kTmo = 128, kDone = 256, kCvt = 384, kAll = 512, kQueue = 1024;
constexpr unsigned kTicket = 4096;
constexpr unsigned kFlags = 16384;
constexpr unsigned kOFlags = 81920;
constexpr unsigned kSrc = 98304;
constexpr unsigned kInBytes = 32u << 20;
constexpr int kNMove = 48, kNComp = 208, kNWG = 256;
constexpr unsigned kRowsI = 704u, kRowsC = 688u;
constexpr unsigned kDonePerCall = 192u;
constexpr unsigned long long kDeadline = 50000000ull;
constexpr unsigned kSlot = 24576u, kRing = 6u * kSlot;
constexpr unsigned kLdsMisc = kRing, kLdsBytes = kRing + 128u;
constexpr unsigned kOob = 0x80000000u;
constexpr int kAuxSys = 17;
constexpr int kAuxDev = 16;

struct Params {
  const float* x;
  const float* w;
  char* ws[8];
  float* out[8];
  int rank;
  int pad;
  uint32_t tiles[3 * kNComp];
};

__device__ __forceinline__ __amdgpu_buffer_rsrc_t mk_rsrc(const void* p, unsigned bytes) {
  return __builtin_amdgcn_make_buffer_rsrc((void*)p, 0, bytes, 0x00020000);
}
template <int AUX>
__device__ __forceinline__ f32x4 ld16(__amdgpu_buffer_rsrc_t r, unsigned voff, unsigned soff) {
  return __builtin_bit_cast(f32x4, __builtin_amdgcn_raw_buffer_load_b128(r, voff, soff, AUX));
}
template <int AUX>
__device__ __forceinline__ void st16(f16x8 v, __amdgpu_buffer_rsrc_t r, unsigned voff, unsigned soff) {
  __builtin_amdgcn_raw_buffer_store_b128(__builtin_bit_cast(u32x4, v), r, voff, soff, AUX);
}
__device__ __forceinline__ f16x8 cvt8(f32x4 a, f32x4 b) {
  f16x8 h;
#pragma unroll
  for (int i = 0; i < 4; ++i) { h[i] = (_Float16)a[i]; h[4 + i] = (_Float16)b[i]; }
  return h;
}
__device__ __forceinline__ f16x8 cvt_col(const f32x4 (&v)[8], int i) {
  f16x8 h;
#pragma unroll
  for (int j = 0; j < 8; ++j) h[j] = (_Float16)v[j][i];
  return h;
}
#define WAIT_VM(n) asm volatile("s_waitcnt vmcnt(%0)" :: "n"(n) : "memory")
#define WAIT_LGKM0() asm volatile("s_waitcnt lgkmcnt(0)" ::: "memory")
#define FENCE() asm volatile("" ::: "memory")
#define UNLIKELY(c) __builtin_expect(!!(c), 0)
#define BARRIER() do { FENCE(); __builtin_amdgcn_s_barrier(); FENCE(); } while (0)

__device__ __forceinline__ void flag_store(char* p, unsigned v) {
  __hip_atomic_store((GAS unsigned*)p, v, __ATOMIC_RELAXED, __HIP_MEMORY_SCOPE_SYSTEM);
}
__device__ __forceinline__ void flag_store2(char* p, unsigned lo, unsigned hi) {
  __hip_atomic_store((GAS unsigned long long*)p, (unsigned long long)lo | ((unsigned long long)hi << 32), __ATOMIC_RELAXED, __HIP_MEMORY_SCOPE_SYSTEM);
}
__device__ __forceinline__ unsigned draw(char* p) {
  unsigned z = 0;
  asm volatile("" : "+v"(z));
  p += z;
  return __hip_atomic_fetch_add((GAS unsigned*)p, 1u, __ATOMIC_RELAXED, __HIP_MEMORY_SCOPE_AGENT);
}
__device__ __forceinline__ unsigned uni(unsigned v) { return (unsigned)__builtin_amdgcn_readfirstlane((int)v); }
__device__ __forceinline__ unsigned lanes_below(unsigned long long b) {
  return __builtin_amdgcn_mbcnt_hi((unsigned)(b >> 32), __builtin_amdgcn_mbcnt_lo((unsigned)b, 0u));
}
__device__ __forceinline__ void advance(unsigned& pos, unsigned long long b) {
  asm("s_lshl2_add_u32 %0, %1, %0" : "+s"(pos) : "s"((unsigned)__builtin_popcountll(b)) : "scc");
}

#define VSTAMP(ws, slot, val) do {} while (0)
#define STAMP(ws, slot) do {} while (0)
#define PSTAMP(ws, cls, n) do {} while (0)
#define CSTAMP(ws, slot) do {} while (0)

__device__ __forceinline__ void set_timeout(char* myws, unsigned call) {
  __hip_atomic_store((GAS unsigned*)(myws + kTmo), call, __ATOMIC_RELAXED, __HIP_MEMORY_SCOPE_AGENT);
}
__device__ __forceinline__ unsigned peer_of(unsigned r, unsigned i) { return (r & 4u) + ((r + i + 1u) & 3u); }
__device__ __forceinline__ unsigned panel_row(unsigned ap, unsigned rh, unsigned rank) {
  const unsigned og = (rank & 4u) ^ 4u;
  if (ap < 8u) return rank * 1024u + ap * 128u + rh * 64u;
  if (ap < 23u) { const unsigned i = (ap - 8u) / 5u, s = (ap - 8u) - 5u * i; return peer_of(rank, i) * 1024u + s * 128u + rh * 64u; }
  if (ap < 26u) return peer_of(rank, ap - 23u) * 1024u + 640u;
  if (ap < 34u) return (og + ((ap - 26u) >> 1)) * 1024u + 768u + (ap & 1u) * 128u + rh * 64u;
  return (og + 2u * (ap - 34u) + rh) * 1024u + 704u;
}
__device__ __forceinline__ unsigned park_pos(unsigned ap, unsigned rh, unsigned j) {
  if (ap < 8u) return 2u * ap + rh < 11u ? 2u * ap + rh : 99u;
  if (ap < 23u) {
    const unsigned i = (ap - 8u) / 5u, s = (ap - 8u) - 5u * i;
    return s < 2u ? 11u + 4u * i + 2u * s + rh : s < 4u ? 40u + 4u * i + 2u * s + rh : 26u + 2u * i + rh;
  }
  const unsigned src = ap < 34u ? (ap - 26u) >> 1 : 2u * (ap - 34u) + rh;
  if (src == j) return 99u;
  const unsigned e = src - (src > j ? 1u : 0u);
  return ap < 34u ? 32u + 4u * e + 2u * (ap & 1u) + rh : 23u + e;
}
__device__ __forceinline__ unsigned park_row(unsigned hp, unsigned src, unsigned rank) {
  const unsigned q = rank & 3u;
  if (hp < 11u) return src * 1024u + hp * 64u;
  if (hp < 23u) return peer_of(src, (hp - 11u) >> 2) * 1024u + ((hp - 11u) & 3u) * 64u;
  if (hp < 26u) { const unsigned e = hp - 23u; return ((rank & 4u) + e + (e >= q ? 1u : 0u)) * 1024u + 704u; }
  if (hp < 32u) return peer_of(src, (hp - 26u) >> 1) * 1024u + 512u + (hp & 1u) * 64u;
  if (hp < 44u) { const unsigned e = (hp - 32u) >> 2; return ((rank & 4u) + e + (e >= q ? 1u : 0u)) * 1024u + 768u + (hp & 3u) * 64u; }
  return peer_of(src, (hp - 44u) >> 2) * 1024u + 256u + (hp & 3u) * 64u;
}

}

__global__ void __launch_bounds__(512) __attribute__((amdgpu_waves_per_eu(2, 2)))
ag_gemm_relu_kernel(const Params p) {
  extern __shared__ __attribute__((aligned(16))) char smem_raw[];
  LAS char* const lds = (LAS char*)smem_raw;
  LAS volatile unsigned* const misc = (LAS volatile unsigned*)(lds + kLdsMisc);

  const int tid = threadIdx.x;
  const int lane = tid & 63;
  const int wid = __builtin_amdgcn_readfirstlane(tid >> 6);
  const int bid = blockIdx.x;
  const unsigned rank = (unsigned)p.rank & 7u;
  const unsigned q = rank & 3u, og = (rank & 4u) ^ 4u;
  char* const myws = p.ws[rank];
  const unsigned l16 = (unsigned)lane * 16u;
  const __amdgpu_buffer_rsrc_t rws = mk_rsrc(myws, kWsNeed);
  const int fr = lane & 15, fq = lane >> 4;

  STAMP(myws, wid == 0 ? 0 : 31);

  const unsigned lx = (((unsigned)lane >> 2) * 8192u + 4u * (unsigned)((lane & 3) ^ ((-(lane >> 4)) & 3))) * 4u;
  const unsigned lw = (4u * ((unsigned)lane >> 4) * 1024u + 4u * ((unsigned)lane & 15u)) * 4u;
  const __amdgpu_buffer_rsrc_t rx = mk_rsrc(p.x, kInBytes);
  const __amdgpu_buffer_rsrc_t rw = mk_rsrc(p.w, kInBytes);
  const unsigned pw = (unsigned)wid >= WPM ? (unsigned)wid - WPM : (unsigned)wid;
  const unsigned gw = (unsigned)bid * WPM + pw;
  const unsigned sid = gw % 14u, k = gw / 14u;
  const bool cross = sid >= 6u;
  const unsigned c = cross ? (sid - 6u) >> 1 : sid >> 1, h = sid & 1u;
  const unsigned sr = (cross ? og + c : peer_of(rank, c)) & 7u;
  char* const sws = p.ws[sr];
  const __amdgpu_buffer_rsrc_t rs = mk_rsrc(sws, kWsNeed);
  LAS volatile unsigned* const hand = misc + 8u + 2u * pw;
  const unsigned buf0 = pw * 2u * 17408u;
  if (bid < kNMove && (unsigned)wid >= WPM && (unsigned)wid < 2u * WPM && gw < 14u * NWS) {
    unsigned call = 0;
    auto push = [&](auto wconst) {
      constexpr bool W = decltype(wconst)::value;
      f32x4 pv[PU][2][8];
#pragma unroll
      for (unsigned u = 0; u < PU; ++u)
#pragma unroll
        for (int i = 0; i < 16; ++i) {
          const unsigned m = u * NWS + k, kk = (m >> 1) * 64u + h * 32u;
          if (W) pv[u][i >> 3][i & 7] = ld16<0>(rw, lw + (i >> 3) * 256u, (kk * 1024u + 256u * c + (m & 1u) * 128u) * 4u + ((i & 3) + 16 * ((i >> 2) & 1)) * 4096u);
          else pv[u][i >> 3][i & 7] = ld16<0>(rx, lx + (i & 1) * 64u, ((m & 1u) * 128u * 8192u + kk) * 4u + (i >> 1) * 524288u);
        }
      __syncthreads();
      call = uni(misc[0]) + 1u;
      const unsigned base = W ? kBOff + (call & 1u) * kBSize + (8u + 2u * q) * kPanel : kAOff + (call & 1u) * kASize + (18u - 5u * c) * kPanel;
      char* const fl = sws + kFlags + (W ? 44u + 2u * q : 18u - 5u * c) * 1024u + h * 4u;
#pragma unroll
      for (unsigned j = 0; j < 2u * PU; ++j) {
        const unsigned u = j >> 1, r = j & 1u, m = u * NWS + k;
        f16x8 hv[4];
#pragma unroll
        for (int i = 0; i < 4; ++i) hv[i] = W ? cvt_col(pv[u][r], i) : cvt8(pv[u][r][2 * i], pv[u][r][2 * i + 1]);
        asm volatile("" :: "v"(hv[0]), "v"(hv[1]), "v"(hv[2]), "v"(hv[3]));
        if (j >= PUW) WAIT_VM(4u * (PUW - 1u));
        if (j <= 2u) STAMP(myws, 8 + wid + 3 * j);
        if (u > 0u && r == PUW - 1u && lane == 0) flag_store(fl + ((m - NWS) & 1u) * 1024u + ((m - NWS) >> 1) * 8u, call);
#pragma unroll
        for (int i = 0; i < 4; ++i) st16<kAuxSys>(hv[i], rs, l16 + i * 1024u, base + (m & 1u) * kPanel + (m >> 1) * kBlk + h * 8192u + r * 4096u);
        FENCE();
      }
      WAIT_VM(0);
      const unsigned m = (PU - 1u) * NWS + k;
      if (lane == 0) flag_store(fl + (m & 1u) * 1024u + (m >> 1) * 8u, call);
    };
    if (cross) push(std::true_type{}); else push(std::false_type{});
    STAMP(myws, 4 + wid);
    if (!cross) return;
    const __amdgpu_buffer_rsrc_t ro = mk_rsrc(p.out[rank], kInBytes);
    const unsigned vo = (unsigned)fr * 4096u + (unsigned)fq * 16u;
    const unsigned long long t0 = __builtin_amdgcn_s_memrealtime();
    for (unsigned b = 0;; b ^= 1u) {
      unsigned hv;
      while ((hv = uni(hand[b])) == 0u) {
        if (__builtin_amdgcn_s_memrealtime() - t0 > 2u * kDeadline) { set_timeout(myws, call); return; }
        __builtin_amdgcn_s_sleep(XSLEEP);
      }
      FENCE();
      if (hv == ~0u) break;
      const unsigned oa = park_row(hv >> 4, sr, rank) * 4096u + (256u * c + (hv & 12u) * 16u) * 4u;
      unsigned pos = (unsigned)(size_t)lds + buf0 + b * 17408u, val[64];
      u32x2 mk = {~0u, ~0u};
      if ((hv & 2u) == 0u) mk = *(LAS u32x2*)(size_t)(pos + lane * 8);
      const unsigned fill = (unsigned)(size_t)lds + kLdsMisc + 64u + (hv & 2u) * 2u;
      pos += 512u;
#pragma unroll
      for (int i = 0; i < 64; ++i) {
        const bool keep = (int)(mk[i >> 5] << (i & 31)) >= 0;
        const unsigned long long bl = __ballot(keep);
        unsigned a = lanes_below(bl) * 4u + pos;
        asm volatile("" : "+v"(a));
        val[i] = *(LAS unsigned*)(size_t)(keep ? a : fill);
        advance(pos, bl);
      }
      WAIT_LGKM0();
      if (lane == 0) hand[b] = 0u;
      FENCE();
#pragma unroll
      for (int i = 0; i < 16; ++i)
        __builtin_amdgcn_raw_buffer_store_b128(u32x4{val[4 * i], val[4 * i + 1], val[4 * i + 2], val[4 * i + 3]}, ro, vo + (i & 3) * 64u, oa + (i >> 2) * 65536u, kAuxSys);
    }
    STAMP(myws, 1 + wid);
    return;
  }
  const bool isconv = bid >= kNMove + 8 * CVFROM;
  unsigned tk = 0;
  if (tid == 0) tk = draw(myws + kTicket + (unsigned)bid * 32u);
  FENCE();
  f32x4 cv[8];
  auto item = [&](unsigned n, unsigned& ktg, unsigned& sub) {
    if (n < 5u * CVLEAD) { ktg = n / 5u; sub = n - 5u * ktg; sub += sub ? 3u : 0u; }
    else if (n < 8u * CVLEAD) { ktg = (n - 5u * CVLEAD) / 3u; sub = n - 5u * CVLEAD - 3u * ktg + 1u; }
    else { ktg = n >> 3; sub = n & 7u; }
  };
  auto loads = [&](unsigned n) {
    unsigned ktg, sub;
    item(n, ktg, sub);
    if (sub < 4u) {
      const unsigned so = ((256u * sub + 32u * wid) * 8192u + ktg * 64u) * 4u;
#pragma unroll
      for (int s = 0; s < 2; ++s) {
        cv[4 * s + 0] = ld16<0>(rx, lx, so + s * 524288u); cv[4 * s + 1] = ld16<0>(rx, lx + 64u, so + s * 524288u);
        cv[4 * s + 2] = ld16<0>(rx, lx + 128u, so + s * 524288u); cv[4 * s + 3] = ld16<0>(rx, lx + 192u, so + s * 524288u);
      }
    } else {
      const unsigned so = ((ktg * 64u + 32u * (wid & 1)) * 1024u + 256u * (sub - 4u) + 64u * (wid >> 1)) * 4u;
#pragma unroll
      for (int i = 0; i < 8; ++i) cv[i] = ld16<0>(rw, lw, so + ((i & 3) + 16 * (i >> 2)) * 4096u);
    }
  };
  if (bid >= kNMove + 8 * CVFROM + CVFIRST) __builtin_amdgcn_s_sleep(CVDELAY);
  if (isconv) loads((unsigned)(bid - kNMove - 8 * CVFROM));
  FENCE();
  if (tid == 0) {
    misc[0] = tk;
    misc[3] = 0u;
  }
  if (tid >= 8 && tid < 18) misc[tid] = tid == 17 ? 0x7fc00000u : 0u;
  __syncthreads();
  const unsigned call = uni(misc[0]) + 1u;
  const unsigned par = call & 1u;
  const unsigned abase = kAOff + par * kASize, bbase = kBOff + par * kBSize, obuf = kOOff + par * kOSize;

  if (bid < kNMove) {
    if (wid == 7) {
      if (bid == 0 && lane == 0) {
        const unsigned target = call * kDonePerCall;
        const unsigned long long t0 = __builtin_amdgcn_s_memrealtime();
        bool failed = false;
        while ((int)(__hip_atomic_load((GAS unsigned*)(myws + kDone), __ATOMIC_RELAXED, __HIP_MEMORY_SCOPE_SYSTEM) - target) < 0) {
          if (__hip_atomic_load((GAS unsigned*)(myws + kTmo), __ATOMIC_RELAXED, __HIP_MEMORY_SCOPE_AGENT) == call ||
              __builtin_amdgcn_s_memrealtime() - t0 > kDeadline) {
            set_timeout(myws, call);
            failed = true;
            break;
          }
          __builtin_amdgcn_s_sleep(2);
        }
        __builtin_amdgcn_fence(__ATOMIC_ACQUIRE, "");
        if (failed) __hip_atomic_store((GAS unsigned*)(p.out[rank] + (size_t)rank * 1024u * 1024u), 0x7fc00000u, __ATOMIC_RELAXED, __HIP_MEMORY_SCOPE_SYSTEM);
        STAMP(myws, 10);
      }
      return;
    }
    if ((unsigned)wid >= WPM || gw >= 14u * NWS) return;
    const unsigned total = cross ? kRowsC : kRowsI;
    const unsigned nplain = cross ? 576u : kRowsI;
    const unsigned qbase = (call - 1u) * (total - PU * NWS) - (PU + 1u) * NWS;
    bool failed = false, all = false;
    unsigned cgrp = 99u, okf = 0, okt = 0, fv = 0, wbase = 0x10000u;
    u32x2 fw = {0u, 0u};
    unsigned sa = 0, da = 0, fa = 0, pend = 0;
    unsigned s2 = 4096u, d2 = 4096u;
    u32x4 v[4], w[4];
    auto raise = [&]() {
      if (pend != 0u && lane == 0) flag_store(myws + pend, call);
      pend = 0u;
    };
    auto fetch = [&](u32x4 (&x)[4], unsigned so) {
#pragma unroll
      for (int i = 0; i < 4; ++i) x[i] = __builtin_amdgcn_raw_buffer_load_b128(rs, l16 + i * 1024u, so, kAuxSys);
      FENCE();
    };
    auto put = [&](const u32x4 (&x)[4], unsigned so) {
#pragma unroll
      for (int i = 0; i < 4; ++i) __builtin_amdgcn_raw_buffer_store_b128(x[i], rws, l16 + i * 1024u, so, kAuxDev);
      FENCE();
    };
    auto setup = [&](unsigned m) {
      unsigned sp, kt, dp, db, dh = h;
      unsigned so = h * 8192u, dof = h * 8192u;
      s2 = 4096u; d2 = 4096u;
      if (!cross) {
        if (m < 256u) { sp = m & 1u; kt = m >> 1; dp = 8u + 5u * c + sp; db = kt; }
        else if (m < 384u) { sp = 4u; kt = m - 256u; dp = 12u + 5u * c; db = kt; }
        else if (m < 640u) { sp = 2u + (m & 1u); kt = (m - 384u) >> 1; dp = 8u + 5u * c + sp; db = kt; }
        else { sp = 5u; db = m - 640u; kt = 2u * db + h; so = 0u; s2 = 8192u; dp = 23u + c; }
      } else {
        if (m < 256u) { sp = 36u + 2u * q + (m & 1u); kt = m >> 1; dp = 44u + 2u * c + (m & 1u); db = kt; }
        else if (m < 320u) { sp = 5u; kt = 2u * (m - 256u) + h; so = 4096u; s2 = 8192u; dp = 34u + (c >> 1); db = kt; dh = c & 1u; dof = dh * 4096u; d2 = 8192u; }
        else { sp = 6u + (m & 1u); kt = (m - 320u) >> 1; dp = 26u + 2u * c + (m & 1u); db = kt; }
      }
      const unsigned sub = sp < 36u ? sp >> 1 : (sp - 28u) >> 1;
      auto ask = [&](unsigned k0) {
        const unsigned off = lane < 62 ? kSrc + (sr * 8u + sub) * 512u + min(k0 + (unsigned)lane, 127u) * 4u : lane == 62 ? kAll + sr * 4u : kTmo;
        asm volatile("buffer_load_dword %0, %1, %2, 0 offen sc0 sc1" : "=v"(fv) : "v"(off), "s"(rws) : "memory");
      };
      auto look = [&]() {
        asm volatile("" : "+v"(fv) :: "memory");
        const unsigned long long b = __ballot(lane < 63 ? (int)(fv - call) >= 0 : fv == call);
        if ((b >> 62) != 0ull) { failed = (b >> 63) != 0ull; all = true; }
        okt += (unsigned)__builtin_ctzll(~b);
      };
      if (!all) {
        if (cgrp != 99u) look();
        if (!all && !(sub == cgrp && kt >= okf && kt < okt)) {
          WAIT_VM(0);
          raise();
          cgrp = sub;
          okf = kt;
          const unsigned long long t0 = __builtin_amdgcn_s_memrealtime();
          for (;;) {
            okt = kt;
            ask(kt);
            WAIT_VM(0);
            look();
            if (all || okt != kt) break;
            if (__builtin_amdgcn_s_memrealtime() - t0 > kDeadline) { set_timeout(myws, call); failed = true; all = true; break; }
            __builtin_amdgcn_s_sleep(MSLEEP);
          }
        }
        if (!all) ask(okt);
      }
      sa = (sp < 36u ? abase + sp * kPanel : bbase + (sp - 36u) * kPanel) + kt * kBlk + so;
      da = (dp < 36u ? abase + dp * kPanel : bbase + (dp - 36u) * kPanel) + db * kBlk + dof;
      fa = kFlags + dp * 1024u + db * 8u + dh * 4u;
      fetch(v, sa);
    };
    unsigned m = PU * NWS + k;
    setup(m);
    STAMP(myws, 20 + wid);
    do {
      unsigned dv = 0;
      if (lane == 0) dv = draw(myws + kQueue + sid * 128u);
      FENCE();
      put(v, da);
      raise();
      fetch(w, sa + s2);
      put(w, da + d2);
      pend = fa;
      if (h == 0u) { PSTAMP(myws, sid >> 1, m + 1u); if (m < 128u) PSTAMP(myws, 8u + (sid >> 1), (m + 1u) * 4u); }
      m = uni(dv) - qbase;
      if (m < nplain) setup(m);
    } while (m < nplain);
    WAIT_VM(0);
    raise();
    if (cross) {
      unsigned hval = 0;
      const unsigned fo = lane < 32 ? kOFlags + par * 8192u + c * 2048u + h * 8u + (unsigned)lane * 16u : kTmo;
      auto stale = [&](unsigned o) {
        return o - wbase >= 32u || (int)((unsigned)__builtin_amdgcn_readlane((int)fw[0], (int)(o - wbase)) - call) < 0;
      };
      for (unsigned b = 0;; b ^= 1u) {
        const unsigned o = m - 576u, s = 2u * o + h;
        unsigned cnt = 2048u, dv = 0;
        if (m >= total || stale(o)) {
          WAIT_VM(0);
          if (hval != 0u && lane == 0) hand[b ^ 1u] = hval;
          hval = 0u;
          const unsigned long long t0 = __builtin_amdgcn_s_memrealtime();
          wbase = o;
          while (!failed && m < total) {
            fw = __builtin_bit_cast(u32x2, __hip_atomic_load((GAS unsigned long long*)(myws + fo + (lane < 32 ? o * 16u : 0u)), __ATOMIC_RELAXED, __HIP_MEMORY_SCOPE_SYSTEM));
            const unsigned long long bl = __ballot(lane < 32 ? (int)(fw[0] - call) >= 0 : fw[0] == call);
            if ((bl >> 32) != 0ull) { failed = true; break; }
            if (bl & 1ull) break;
            if (__builtin_amdgcn_s_memrealtime() - t0 > kDeadline) { set_timeout(myws, call); failed = true; }
            __builtin_amdgcn_s_sleep(MSLEEP);
          }
        }
        cnt = failed ? 0u : min((unsigned)__builtin_amdgcn_readlane((int)fw[1], (int)((o - wbase) & 31u)), 4096u);
        if (UNLIKELY(uni(hand[b]) != 0u)) {
          const unsigned long long t0 = __builtin_amdgcn_s_memrealtime();
          while (uni(hand[b]) != 0u)
            if (__builtin_amdgcn_s_memrealtime() - t0 > kDeadline) { set_timeout(myws, call); return; }
        }
        FENCE();
        if (m >= total) {
          if (lane == 0) hand[b] = ~0u;
          break;
        }
        const unsigned bytes = 512u + 4u * cnt, sb = obuf + (q * 224u + s) * kPiece, nbuf = buf0 + b * 17408u;
        auto part = [&](unsigned f0, unsigned f1) {
          for (unsigned f = f0; f < f1; f += 1024u) {
            WAIT_VM(PWIN - 1u);
            asm volatile("s_mov_b32 m0, %0\n\ts_nop 0\n\tbuffer_load_dwordx4 %1, %2, %3 offen sc0 sc1 lds"
                         :: "s"(nbuf + f), "v"(f + l16 < bytes ? l16 : kOob), "s"(rs), "s"(sb + f) : "memory");
          }
        };
        asm volatile("s_mov_b64 exec, 1\n\tglobal_atomic_add %0, %1, %2, %3 sc0\n\ts_mov_b64 exec, -1"
                     : "=v"(dv) : "v"(kQueue + sid * 128u), "v"(1u), "s"(myws) : "memory");
        part(0u, min(bytes, (PWIN - 1u) * 1024u));
        if (bytes <= (PWIN - 2u) * 1024u) WAIT_VM(0);
        WAIT_VM(PWIN - 1u);
        if (hval != 0u && lane == 0) hand[b ^ 1u] = hval;
        hval = 4u * s + (failed ? 3u : 1u);
        if (h == 0u) PSTAMP(myws, sid >> 1, m + 1u);
        asm volatile("" : "+v"(dv));
        m = uni(dv) - qbase;
        if (m < total && stale(m - 576u)) {
          wbase = m - 576u;
          asm volatile("buffer_load_dwordx2 %0, %1, %2, 0 offen sc0 sc1" : "=v"(fw) : "v"(fo + (lane < 32 ? wbase * 16u : 0u)), "s"(rws) : "memory");
          part((PWIN - 1u) * 1024u, bytes);
          if (bytes <= (2u * PWIN - 3u) * 1024u) WAIT_VM(0);
          WAIT_VM(PWIN - 1u);
          asm volatile("" : "+v"(fw) :: "memory");
        } else
          part((PWIN - 1u) * 1024u, bytes);
      }
    }
    STAMP(myws, 1 + wid);
    return;
  }

  {
    constexpr unsigned kNConv = 8u * (26u - CVFROM);
    char* const pws = p.ws[lane & 7];
    const unsigned mine = 15u << (rank & 4u);
    STAMP(myws, wid == 0 ? 27 : 31);
    for (unsigned n = isconv ? (unsigned)(bid - kNMove - 8 * CVFROM) : 1024u; n < 1024u; n += kNConv) {
      unsigned ktg, sub;
      item(n, ktg, sub);
      if (sub < 4u) {
        const unsigned so = abase + (2u * sub + (wid >> 2)) * kPanel + ktg * kBlk + (wid & 3) * 2048u;
#pragma unroll
        for (int i = 0; i < 4; ++i) st16<CVAUX>(cvt8(cv[2 * i], cv[2 * i + 1]), rws, l16 + (i >> 1) * 1024u, so + (i & 1) * 8192u);
      } else {
        const unsigned so = bbase + (2u * (sub - 4u) + (wid >> 2)) * kPanel + ktg * kBlk + ((wid >> 1) & 1) * 4096u + (wid & 1) * 8192u;
#pragma unroll
        for (int i = 0; i < 4; ++i) st16<CVAUX>(cvt_col(cv, i), rws, l16 + i * 1024u, so);
      }
      FENCE();
      const bool ahead = n >= kNConv && n + kNConv < 1024u;
      if (ahead) { loads(n + kNConv); WAIT_VM(8); } else { WAIT_VM(0); }
      if (n < kNConv) STAMP(myws, wid == 0 ? 30 : 31);
      BARRIER();
      if (n < 2u * kNConv) STAMP(myws, wid == 0 ? (n < kNConv ? 28 : 29) : 31);
      const unsigned to = sub < 2u ? mine : sub == 2u ? 255u : sub == 3u ? mine ^ 255u : 1u << (og + sub - 4u);
      if (tid < 2) flag_store2(myws + kFlags + ((sub < 4u ? 2u * sub : 28u + 2u * sub) + (unsigned)tid) * 1024u + ktg * 8u, call, call);
      else if (wid == 1 && lane < 8 && (((to & ~(1u << rank)) >> lane) & 1u)) flag_store(pws + kSrc + rank * 4096u + sub * 512u + ktg * 4u, call);
      FENCE();
      if (n < kNConv) loads(n + kNConv);
    }
    if (isconv && wid == 7) {
      unsigned dv = 0;
      if (lane == 0) dv = draw(myws + kCvt);
      if (uni(dv) == call * kNConv - 1u && lane < 8) flag_store(pws + kAll + rank * 4u, call);
    }
  }
  STAMP(myws, wid == 0 ? 1 : 31);

  const uint32_t* const tl = p.tiles + 3 * (bid - kNMove);
  const int ntiles = (int)(tl[0] >> 28);
  if (wid == 7) return;

  if (wid >= 4) {
    const unsigned lw = (unsigned)wid - 4u;
    unsigned total = 0;
    for (int ti = 0; ti < ntiles; ++ti) total += ((tl[ti] >> 10) & 3u) == 2u ? 64u : 256u;
    unsigned a = 0, aend = 0, so = lw * 8192u, src = 0, step = 0, g2 = 0, v2 = l16, idle = 0;
    unsigned ok = 0, f0 = 0, f1 = 0, fb = 0, fsh = 0, fmax = 0, fv = 0;
    unsigned rdy = 0, askat = 0, lookat = ~0u;
    int ta = 0, tf = 0;
    auto next_flags = [&]() {
      while (tf < ntiles) {
        const unsigned code = tl[tf++];
        const unsigned ap = code & 63u, bp = (code >> 6) & 15u, shape = (code >> 10) & 3u;
        f0 = f1;
        f1 += shape == 2u ? 64u : 256u;
        if (shape == 3u && lw == 2u) { ok = f1; continue; }
        const unsigned grp = lw == 0u ? ap : shape == 0u ? (lw == 1u ? ap + 1u : 36u + bp) : shape == 1u ? 35u + bp + lw : 36u + bp;
        fb = kFlags + grp * 1024u;
        fsh = shape != 2u ? 0u : lw == 0u ? 1u : 2u;
        fmax = fsh == 1u ? 127u : 255u;
        break;
      }
    };
    auto known = [&]() {
      rdy = min(ok, aend);
      unsigned from = max(ok, LTHR);
      asm volatile("" : "+s"(from));
      askat = ok >= total ? ~0u : from - LTHR;
    };
    auto ask = [&]() {
      const unsigned off = lane < 63 ? fb + min(((ok - f0) << fsh) + (unsigned)lane, fmax) * 4u : kTmo;
      asm volatile("buffer_load_dword %0, %1, %2, 0 offen sc0 sc1" : "=v"(fv) : "v"(off), "s"(rws) : "memory");
    };
    auto give_up = [&]() {
      ok = total;
      if (lane == 0) misc[3] = 1u;
      WAIT_LGKM0();
    };
    auto look = [&]() {
      asm volatile("" : "+v"(fv) :: "memory");
      const unsigned long long b = __ballot(lane < 63 ? (int)(fv - call) >= 0 : fv == call);
      if (UNLIKELY((b >> 63) != 0ull)) {
        give_up();
      } else {
        const unsigned run = (unsigned)__builtin_ctzll(~b);
        ok = min(ok + (fsh == 0u ? run & ~1u : run >> fsh), f1);
        if (UNLIKELY(ok == f1)) next_flags();
      }
      lookat = ~0u;
    };
    auto issue = [&](bool busy) {
      LAS char* const dl = lds + so;
      if (busy) {
        __builtin_amdgcn_raw_ptr_buffer_load_lds(rws, (LAS void*)dl, 16, l16, src, 0, kAuxDev);
        __builtin_amdgcn_raw_ptr_buffer_load_lds(rws, (LAS void*)dl, 16, l16, src, 1024, kAuxDev);
        __builtin_amdgcn_raw_ptr_buffer_load_lds(rws, (LAS void*)dl, 16, v2, src, 2048, kAuxDev);
        __builtin_amdgcn_raw_ptr_buffer_load_lds(rws, (LAS void*)dl, 16, v2, src, 3072, kAuxDev);
        __builtin_amdgcn_raw_ptr_buffer_load_lds(rws, (LAS void*)(dl + 4096), 16, l16, src + g2, 0, kAuxDev);
        __builtin_amdgcn_raw_ptr_buffer_load_lds(rws, (LAS void*)(dl + 4096), 16, l16, src + g2, 1024, kAuxDev);
        __builtin_amdgcn_raw_ptr_buffer_load_lds(rws, (LAS void*)(dl + 4096), 16, v2, src + g2, 2048, kAuxDev);
        __builtin_amdgcn_raw_ptr_buffer_load_lds(rws, (LAS void*)(dl + 4096), 16, v2, src + g2, 3072, kAuxDev);
      }
      FENCE();
      src += step;
      so = so >= kRing - kSlot ? so - (kRing - kSlot) : so + kSlot;
      ++a;
    };
    next_flags();
    known();
    for (unsigned j = 0; j < total; ++j) {
      if (a == j + 4u) {
        unsigned e = min(rdy + 1u, askat);
        asm volatile("" : "+s"(e));
        e = min(e, lookat);
        if (idle != 0u) {
          WAIT_VM(0);
          while (a + 1u < e) { issue(false); BARRIER(); }
        }
        while (a + 1u < e) {
          issue(true);
          WAIT_VM(32);
          BARRIER();
        }
        j = a - 4u;
      }
      unsigned lim = min(j + 5u, rdy);
      if (UNLIKELY(a >= lim)) {
        if (a == aend) {
          const unsigned code = tl[ta++];
          const unsigned ap = code & 63u, bp = (code >> 6) & 15u, shape = (code >> 10) & 3u;
          if (lw == 0u || (lw == 1u && shape == 0u)) src = abase + (ap + lw) * kPanel;
          else src = bbase + (bp + (shape == 1u ? lw - 1u : 0u)) * kPanel;
          step = 8192u; g2 = 4096u;
          idle = shape == 3u && lw == 2u ? 1u : 0u;
          if (shape == 2u) {
            g2 = 8192u;
            if (lw == 0u) { src += ((code >> 13) & 1u) * 2048u; step = kBlk; v2 = l16 + 2048u; }
            else { src += ((code >> 12) & 1u) * 4096u + (lw - 1u) * kBlk; step = 2u * kBlk; }
          }
          aend = ta == ntiles ? ~0u : aend + 256u;
          rdy = min(ok, aend);
        }
        if (a <= j && ok <= a) {
          WAIT_VM(0);
          if (lookat != ~0u) look();
          const unsigned long long t0 = __builtin_amdgcn_s_memrealtime();
          while (ok <= a) {
            ask();
            WAIT_VM(0);
            look();
            if (ok > a) break;
            if (__builtin_amdgcn_s_memrealtime() - t0 > kDeadline) { set_timeout(myws, call); give_up(); break; }
            __builtin_amdgcn_s_sleep(LSLEEP);
          }
          known();
        }
        lim = min(j + 5u, rdy);
      }
      if (a < lim) {
        issue(idle == 0u);
        if (UNLIKELY(a < lim)) issue(idle == 0u);
      }
      if (UNLIKELY(a >= askat)) { ask(); askat = ~0u; lookat = a + 4u; }
      const unsigned n = idle != 0u ? 0u : a - 1u - j;
      if (UNLIKELY(n < 4u)) {
        WAIT_VM(24);
        if (n < 3u) {
          WAIT_VM(16);
          if (n < 2u) {
            WAIT_VM(8);
            if (n < 1u) WAIT_VM(0);
          }
        }
      }
      WAIT_VM(32);
      if (UNLIKELY(j + 5u >= lookat)) { look(); known(); }
      BARRIER();
    }
    return;
  }

  {
    const unsigned slot = (unsigned)(4 * fr + (fq ^ ((-(fr >> 2)) & 3))) * 16u;
    const unsigned slotb = (unsigned)(16 * fq + 4 * (fr & 3) + (fr >> 2)) * 16u;
    unsigned so = 0;
    for (int ti = 0; ti < ntiles; ++ti) {
      const unsigned code = tl[ti];
      const unsigned shape = (code >> 10) & 3u;
      const unsigned wh = (unsigned)wid >> 1, wl = (unsigned)wid & 1u;
      const unsigned ap = (code & 63u) + (shape == 0u ? wh : 0u), bp = ((code >> 6) & 15u) + (shape == 1u ? wh : 0u);
      const unsigned jo = (bp - 8u) >> 1;
      const unsigned owner = (bp < 8u ? rank : og + jo) & 7u;
      const unsigned col0 = bp < 8u ? bp * 128u : 256u * q + (bp & 1u) * 128u;
      const __amdgpu_buffer_rsrc_t ro = mk_rsrc(p.out[owner], kInBytes);
      const unsigned vo = (unsigned)fr * 4096u + (unsigned)fq * 16u;

      auto run = [&](auto mconst, const unsigned aoff, const unsigned boff) {
        constexpr int M = decltype(mconst)::value;
        constexpr int KS = M > 1 ? 1 : 4;
        constexpr int NS = M > 1 ? 256 : 64;
        constexpr int NA = M * KS, NB = 4 * KS;
        const unsigned va = aoff + slot, vb = boff + slotb;
        f32x4 acc[M][4];
#pragma unroll
        for (int m = 0; m < M; ++m)
#pragma unroll
          for (int n = 0; n < 4; ++n) acc[m][n] = f32x4{0.f, 0.f, 0.f, 0.f};
        f16x8 fa[2][NA], fb[2][NB];
        auto rd = [&](f16x8 (&a)[NA], f16x8 (&b)[NB]) {
          LAS char* const pa = lds + (va + so);
          LAS char* const pb = lds + (vb + so);
#pragma unroll
          for (int i = 0; i < (NA > NB ? NA : NB); ++i) {
            if (i < NB) b[i] = *(LAS f16x8*)(pb + i * 1024);
            if (i < NA) a[i] = *(LAS f16x8*)(pa + i * (M > 1 ? 1024 : 2048));
          }
        };
        auto mm = [&](const f16x8 (&a)[NA], const f16x8 (&b)[NB]) {
#pragma unroll
          for (int ks = 0; ks < KS; ++ks)
#pragma unroll
            for (int m = 0; m < M; ++m)
#pragma unroll
              for (int n = 0; n < 4; ++n)
                acc[m][n] = __builtin_amdgcn_mfma_f32_16x16x32_f16(b[4 * ks + n], a[KS * m + ks], acc[m][n], 0, 0, 0);
        };
        BARRIER();
        STAMP(myws, wid == 0 ? 2 + 3 * ti : 31);
        CSTAMP(myws, wid == 0 ? 12 + 2 * ti : 31);
        rd(fa[0], fb[0]);
        __builtin_amdgcn_s_waitcnt(0xC07F);
        BARRIER();
        so = so == kRing - kSlot ? 0u : so + kSlot;
        if constexpr (M > 1) {
          unsigned ua = va + so, ub = vb + so;
          for (int t = 0; t < NS; t += 2) {
#pragma unroll
            for (int s = 0; s < 2; ++s) {
              __builtin_amdgcn_sched_barrier(0);
#pragma unroll
              for (int i = 0; i < 4 * M; ++i) {
                acc[i >> 2][i & 3] = __builtin_amdgcn_mfma_f32_16x16x32_f16(fb[s][i & 3], fa[s][i >> 2], acc[i >> 2][i & 3], 0, 0, 0);
                __builtin_amdgcn_sched_barrier(0);
                if (M == 8 && (i & 1) == 0 && i < 24) {
                  const int r = i >> 1;
                  if (r < 4) fb[s ^ 1][r] = *(LAS f16x8*)(lds + ub + r * 1024);
                  else fa[s ^ 1][r - 4] = *(LAS f16x8*)(lds + ua + (r - 4) * 1024);
                }
                if (M == 4 && i < 4) {
                  fb[s ^ 1][i] = *(LAS f16x8*)(lds + ub + i * 1024);
                  fa[s ^ 1][i] = *(LAS f16x8*)(lds + ua + i * 1024);
                }
                if (i == 4 * M - 10) { so = so == kRing - kSlot ? 0u : so + kSlot; asm volatile("" : "+s"(so)); }
                if (i == 4 * M - 6) { ua = va + so; asm volatile("" : "+v"(ua)); }
                if (i == 4 * M - 4) { ub = vb + so; asm volatile("" : "+v"(ub)); }
                if (i == 4 * M - 2) {
                  __builtin_amdgcn_s_waitcnt(0xC07F);
                  if (t < NS - 2) BARRIER();
                }
                __builtin_amdgcn_sched_barrier(0);
              }
            }
          }
          so = so == 0u ? kRing - kSlot : so - kSlot;
        } else {
          for (int t = 0; t < NS; t += 2) {
#pragma unroll
            for (int s = 0; s < 2; ++s) {
              __builtin_amdgcn_sched_barrier(0);
              rd(fa[s ^ 1], fb[s ^ 1]);
              mm(fa[s], fb[s]);
#pragma unroll
              for (int i = 0; i < 4; ++i) {
                __builtin_amdgcn_sched_group_barrier(0x100, 2, 0);
                __builtin_amdgcn_sched_group_barrier(0x008, 1, 0);
              }
#pragma unroll
              for (int i = 0; i < 12; ++i) {
                __builtin_amdgcn_sched_group_barrier(0x100, 1, 0);
                __builtin_amdgcn_sched_group_barrier(0x008, 1, 0);
              }
              __builtin_amdgcn_sched_barrier(0);
              so = so == kRing - kSlot ? 0u : so + kSlot;
              __builtin_amdgcn_s_waitcnt(0xC07F);
              if (t < NS - 2) BARRIER();
            }
          }
        }
        STAMP(myws, wid == 0 ? 3 + 3 * ti : 31);
        CSTAMP(myws, wid == 0 ? 13 + 2 * ti : 31);
        const int lo = misc[3] != 0u ? 0x7fc00000 : 0;
        if (M > 1 && bp >= 8u) {
          const __amdgpu_buffer_rsrc_t rw4 = __builtin_amdgcn_make_buffer_rsrc((void*)myws, 4, kWsNeed / 4u, 0x00020000);
          if (UNLIKELY(lo != 0)) {
#pragma unroll
            for (int m = 0; m < M; ++m)
#pragma unroll
              for (int n = 0; n < 4; ++n) acc[m][n] = __builtin_bit_cast(f32x4, u32x4{0x7fc00000u, 0x7fc00000u, 0x7fc00000u, 0x7fc00000u});
          }
#pragma unroll
          for (int rh = 0; rh < M / 4; ++rh) {
            const unsigned hp = park_pos(ap, M == 8 ? (unsigned)rh : wh, jo);
            if (hp < 56u) {
              const unsigned s = hp * 4u + (bp & 1u) * 2u + wl;
              const unsigned pb0 = obuf + (jo * 224u + s) * kPiece;
              unsigned pos = pb0 + 512u;
              u32x2 mk = {0u, 0u};
#pragma unroll
              for (int m = 0; m < 4; ++m)
#pragma unroll
                for (int j = 0; j < 4; ++j)
#pragma unroll
                  for (int n = 0; n < 4; ++n) {
                    const float f = acc[4 * rh + m][n][j];
                    const unsigned iv = __float_as_uint(f);
                    const unsigned long long b = __ballot((int)iv >= 0);
                    const unsigned idx = lanes_below(b) | (iv & kOob);
                    asm volatile("buffer_store_dword %0, %1, %2, %3 idxen sc0 sc1" :: "v"(iv), "v"(idx), "s"(rw4), "s"(pos) : "memory");
                    advance(pos, b);
                    mk[m >> 1] = __builtin_amdgcn_alignbit(mk[m >> 1], iv, 31);
                  }
              __builtin_amdgcn_raw_buffer_store_b64(mk, rws, (unsigned)lane * 8u, pb0, kAuxSys);
              WAIT_VM(0);
              if (lane == 0) flag_store2(p.ws[owner] + kOFlags + par * 8192u + q * 2048u + s * 8u, call, (pos - pb0 - 512u) >> 2);
            }
          }
        } else {
#pragma unroll
          for (int m = 0; m < M; ++m) {
            const unsigned ob = M > 1 ? (panel_row(ap, M == 8 ? (unsigned)(m >> 2) : wh, rank) + 16u * (m & 3)) * 4096u + (col0 + 64u * wl) * 4u
                                       : (panel_row(ap, 0u, rank) + 32u * ((code >> 13) & 1u) + 16u * wl) * 4096u + (col0 + 64u * ((code >> 12) & 1u)) * 4u;
#pragma unroll
            for (int j = 0; j < 4; ++j) {
              u32x4 o;
#pragma unroll
              for (int n = 0; n < 4; ++n) { const float f = acc[m][n][j]; o[n] = (unsigned)max(__float_as_int(f), lo); }
              __builtin_amdgcn_raw_buffer_store_b128(o, ro, vo + j * 64u, ob, kAuxSys);
            }
          }
          if (bp >= 8u) {
            WAIT_VM(0);
            if (lane == 0) __hip_atomic_fetch_add((GAS unsigned*)(p.ws[owner] + kDone), 1u, __ATOMIC_RELAXED, __HIP_MEMORY_SCOPE_SYSTEM);
          }
        }
      };
      if (shape == 2u) {
        if (wid >= 2) return;
        run(std::integral_constant<int, 1>{}, wl * 1024u, 8192u);
      } else if (shape == 3u) {
        run(std::integral_constant<int, 4>{}, wh * 4096u, 8192u + wl * 4096u);
      } else {
        run(std::integral_constant<int, 8>{}, shape == 0u ? wh * 8192u : 0u, (shape == 0u ? 16384u : 8192u + wh * 8192u) + wl * 4096u);
      }
      STAMP(myws, wid == 0 ? 4 + 3 * ti : 31);
    }
  }
}

extern "C" void kernel_launch(int rank, int nranks, ncclComm_t comm, hipStream_t stream,
                              void* const* d_in, const int* in_sizes, int n_in,
                              void* d_out, int out_size,
                              const int* in_dtypes, int out_dtype,
                              void* const* d_out_all, void* const* d_ws_all, size_t ws_bytes) {
  (void)comm; (void)in_dtypes; (void)out_dtype; (void)d_out;
  if (nranks != 8 || (unsigned)rank >= 8u || n_in < 2 || ws_bytes < (size_t)kWsNeed ||
      in_sizes[0] != 8 * 1024 * 1024 || in_sizes[1] != 8 * 1024 * 1024 || out_size != 8 * 1024 * 1024) {
    fprintf(stderr, "kernel_launch: unexpected problem geometry\n");
    return;
  }
  Params p;
  p.x = (const float*)d_in[0];
  p.w = (const float*)d_in[1];
  for (int i = 0; i < 8; ++i) { p.ws[i] = (char*)d_ws_all[i]; p.out[i] = (float*)d_out_all[i]; }
  p.rank = rank;
  p.pad = 0;
  static const int xcol[8] = {0, 4, 2, 3, 1, 5, 6, 7};
  auto tile = [](int a, int b, int shape) { return (unsigned)(a | (b << 6) | (shape << 10)); };
  for (int ci = 0; ci < kNComp; ++ci) {
    const int x = ci & 7, s = ci >> 3, bq = 2 * (x ^ 4);
    unsigned t[3] = {0u, 0u, 0u};
    int n = 0;
    auto far67 = [&](int y, int e) {
      const int j = y >> 1, idx = 3 * (y & 1) + (e >> 1);
      return tile(26 + 2 * j + (e & 1), 8 + idx + (idx >= 2 * j ? 2 : 0), 3);
    };
    bool last = true;
    if (s < 12) t[n++] = tile(x < 4 ? 2 * (s >> 2) : 8 + 5 * (s >> 2), 4 * (x & 3) + (s & 3), 0);
    else if (s == 12) t[n++] = tile(6, xcol[x], 0);
    else if (s < 16) t[n++] = tile(12 + 5 * (s - 13), bq, 1);
    else if (s < 18) t[n++] = tile(34 + (s - 16), bq, 1);
    else if (s < 22) t[n++] = tile(26 + 2 * (x >> 1), 4 * (x & 1) + s - 18, 0);
    else t[n++] = far67(x, s - 22);
    if (s == 12) t[n++] = far67(x, 4);
    else if (x >= 4 && s >= 10 && s < 12) t[n++] = far67(s == 10 ? x : x - 4, 5);
    else if (x >= 4 && s < 2) { t[n++] = tile(10, bq + s, 0); last = false; }
    else if (s < 12) {
      const int i = x >= 4 ? s + 2 : s;
      t[n++] = tile(10 + 5 * (i >> 2) + ((i >> 1) & 1), bq + (i & 1), 3);
      if (s < 2) last = false;
    }
    if (last) {
      const int i = s - 2;
      t[n++] = tile(23 + i / 8, 2 * x + ((i >> 1) & 1), 2) | ((unsigned)(i & 1) << 12) | ((unsigned)((i >> 2) & 1) << 13);
    }
    p.tiles[3 * ci] = t[0] | ((unsigned)n << 28);
    p.tiles[3 * ci + 1] = t[1];
    p.tiles[3 * ci + 2] = t[2];
  }
  hipError_t e = hipFuncSetAttribute((const void*)ag_gemm_relu_kernel, hipFuncAttributeMaxDynamicSharedMemorySize, (int)kLdsBytes);
  if (e != hipSuccess) { fprintf(stderr, "hipFuncSetAttribute: %s\n", hipGetErrorString(e)); return; }
  hipLaunchKernelGGL(ag_gemm_relu_kernel, dim3(kNWG), dim3(512), kLdsBytes, stream, p);
  e = hipGetLastError();
  if (e != hipSuccess) fprintf(stderr, "launch: %s\n", hipGetErrorString(e));
}
```

```cpp
#include <hip/hip_runtime.h>
#include <rccl/rccl.h>
#include <cstdio>
#include <cstdint>
#include <type_traits>

typedef _Float16 f16x8 __attribute__((ext_vector_type(8)));
typedef float f32x4 __attribute__((ext_vector_type(4)));
typedef unsigned u32x4 __attribute__((ext_vector_type(4)));
typedef unsigned u32x2 __attribute__((ext_vector_type(2)));
#define LAS __attribute__((address_space(3)))
#define GAS __attribute__((address_space(1)))

#define NWS 9u
#define WPM 3u
#define PU 1u
#define PUW 1u
#define CVDELAY 24
#define CVFIRST 60
#define CVLEAD 40u
#define XSLEEP 4
#define CVFROM 13
#define PWIN 4u
#define CVAUX kAuxSys
#define MSLEEP 4
#define LSLEEP 2
#define WLIM 80u
#define SKIPA 11
#define LTHR 24u

namespace {

constexpr unsigned kBlk = 16384u;
constexpr unsigned kPanel = 2u << 20;
constexpr unsigned kAOff = 1u << 20;
constexpr unsigned kASize = 36u * kPanel;
constexpr unsigned kBOff = kAOff + 2u * kASize;
constexpr unsigned kBSize = 16u * kPanel;
constexpr unsigned kPiece = 16896u;
constexpr unsigned kOOff = kBOff + 2u * kBSize;
constexpr unsigned kOSize = 4u * 224u * kPiece;
constexpr unsigned kWsNeed = kOOff + 2u * kOSize;
constexpr unsigned kTmo = 128, kDone = 256, kCvt = 384, kAll = 512, kQueue = 1024;
constexpr unsigned kTicket = 4096;
constexpr unsigned kFlags = 16384;
constexpr unsigned kOFlags = 81920;
constexpr unsigned kSrc = 98304;
constexpr unsigned kInBytes = 32u << 20;
constexpr int kNMove = 48, kNComp = 208, kNWG = 256;
constexpr unsigned kRowsI = 704u, kRowsC = 688u;
constexpr unsigned kDonePerCall = 192u;
constexpr unsigned long long kDeadline = 50000000ull;
constexpr unsigned kSlot = 24576u, kRing = 6u * kSlot;
constexpr unsigned kPBuf = 17408u;
constexpr unsigned kLdsMisc = 9u * kPBuf, kLdsBytes = kLdsMisc + 128u;
static_assert(kLdsMisc >= kRing, "");
constexpr unsigned kOob = 0x80000000u;
constexpr int kAuxSys = 17;
constexpr int kAuxDev = 16;

struct Params {
  const float* x;
  const float* w;
  char* ws[8];
  float* out[8];
  int rank;
  int pad;
  uint32_t tiles[3 * kNComp];
};

__device__ __forceinline__ __amdgpu_buffer_rsrc_t mk_rsrc(const void* p, unsigned bytes) {
  return __builtin_amdgcn_make_buffer_rsrc((void*)p, 0, bytes, 0x00020000);
}
template <int AUX>
__device__ __forceinline__ f32x4 ld16(__amdgpu_buffer_rsrc_t r, unsigned voff, unsigned soff) {
  return __builtin_bit_cast(f32x4, __builtin_amdgcn_raw_buffer_load_b128(r, voff, soff, AUX));
}
template <int AUX>
__device__ __forceinline__ void st16(f16x8 v, __amdgpu_buffer_rsrc_t r, unsigned voff, unsigned soff) {
  __builtin_amdgcn_raw_buffer_store_b128(__builtin_bit_cast(u32x4, v), r, voff, soff, AUX);
}
__device__ __forceinline__ f16x8 cvt8(f32x4 a, f32x4 b) {
  f16x8 h;
#pragma unroll
  for (int i = 0; i < 4; ++i) { h[i] = (_Float16)a[i]; h[4 + i] = (_Float16)b[i]; }
  return h;
}
__device__ __forceinline__ f16x8 cvt_col(const f32x4 (&v)[8], int i) {
  f16x8 h;
#pragma unroll
  for (int j = 0; j < 8; ++j) h[j] = (_Float16)v[j][i];
  return h;
}
#define WAIT_VM(n) asm volatile("s_waitcnt vmcnt(%0)" :: "n"(n) : "memory")
#define WAIT_LGKM0() asm volatile("s_waitcnt lgkmcnt(0)" ::: "memory")
#define FENCE() asm volatile("" ::: "memory")
#define UNLIKELY(c) __builtin_expect(!!(c), 0)
#define BARRIER() do { FENCE(); __builtin_amdgcn_s_barrier(); FENCE(); } while (0)

__device__ __forceinline__ void flag_store(char* p, unsigned v) {
  __hip_atomic_store((GAS unsigned*)p, v, __ATOMIC_RELAXED, __HIP_MEMORY_SCOPE_SYSTEM);
}
__device__ __forceinline__ void flag_store2(char* p, unsigned lo, unsigned hi) {
  __hip_atomic_store((GAS unsigned long long*)p, (unsigned long long)lo | ((unsigned long long)hi << 32), __ATOMIC_RELAXED, __HIP_MEMORY_SCOPE_SYSTEM);
}
__device__ __forceinline__ unsigned draw(char* p) {
  unsigned z = 0;
  asm volatile("" : "+v"(z));
  p += z;
  return __hip_atomic_fetch_add((GAS unsigned*)p, 1u, __ATOMIC_RELAXED, __HIP_MEMORY_SCOPE_AGENT);
}
__device__ __forceinline__ unsigned uni(unsigned v) { return (unsigned)__builtin_amdgcn_readfirstlane((int)v); }
__device__ __forceinline__ unsigned lanes_below(unsigned long long b) {
  return __builtin_amdgcn_mbcnt_hi((unsigned)(b >> 32), __builtin_amdgcn_mbcnt_lo((unsigned)b, 0u));
}
__device__ __forceinline__ void advance(unsigned& pos, unsigned long long b) {
  asm("s_lshl2_add_u32 %0, %1, %0" : "+s"(pos) : "s"((unsigned)__builtin_popcountll(b)) : "scc");
}

#define VSTAMP(ws, slot, val) do {} while (0)
#define STAMP(ws, slot) do {} while (0)
#define PSTAMP(ws, cls, n) do {} while (0)
#define CSTAMP(ws, slot) do {} while (0)

__device__ __forceinline__ void set_timeout(char* myws, unsigned call) {
  __hip_atomic_store((GAS unsigned*)(myws + kTmo), call, __ATOMIC_RELAXED, __HIP_MEMORY_SCOPE_AGENT);
}
__device__ __forceinline__ unsigned peer_of(unsigned r, unsigned i) { return (r & 4u) + ((r + i + 1u) & 3u); }
__device__ __forceinline__ unsigned panel_row(unsigned ap, unsigned rh, unsigned rank) {
  const unsigned og = (rank & 4u) ^ 4u;
  if (ap < 8u) return rank * 1024u + ap * 128u + rh * 64u;
  if (ap < 23u) { const unsigned i = (ap - 8u) / 5u, s = (ap - 8u) - 5u * i; return peer_of(rank, i) * 1024u + s * 128u + rh * 64u; }
  if (ap < 26u) return peer_of(rank, ap - 23u) * 1024u + 640u;
  if (ap < 34u) return (og + ((ap - 26u) >> 1)) * 1024u + 768u + (ap & 1u) * 128u + rh * 64u;
  return (og + 2u * (ap - 34u) + rh) * 1024u + 704u;
}
__device__ __forceinline__ unsigned park_pos(unsigned ap, unsigned rh, unsigned j) {
  if (ap < 8u) return 2u * ap + rh < 11u ? 2u * ap + rh : 99u;
  if (ap < 23u) {
    const unsigned i = (ap - 8u) / 5u, s = (ap - 8u) - 5u * i;
    return s < 2u ? 11u + 4u * i + 2u * s + rh : s < 4u ? 40u + 4u * i + 2u * s + rh : 26u + 2u * i + rh;
  }
  const unsigned src = ap < 34u ? (ap - 26u) >> 1 : 2u * (ap - 34u) + rh;
  if (src == j) return 99u;
  const unsigned e = src - (src > j ? 1u : 0u);
  return ap < 34u ? 32u + 4u * e + 2u * (ap & 1u) + rh : 23u + e;
}
__device__ __forceinline__ unsigned park_row(unsigned hp, unsigned src, unsigned rank) {
  const unsigned q = rank & 3u;
  if (hp < 11u) return src * 1024u + hp * 64u;
  if (hp < 23u) return peer_of(src, (hp - 11u) >> 2) * 1024u + ((hp - 11u) & 3u) * 64u;
  if (hp < 26u) { const unsigned e = hp - 23u; return ((rank & 4u) + e + (e >= q ? 1u : 0u)) * 1024u + 704u; }
  if (hp < 32u) return peer_of(src, (hp - 26u) >> 1) * 1024u + 512u + (hp & 1u) * 64u;
  if (hp < 44u) { const unsigned e = (hp - 32u) >> 2; return ((rank & 4u) + e + (e >= q ? 1u : 0u)) * 1024u + 768u + (hp & 3u) * 64u; }
  return peer_of(src, (hp - 44u) >> 2) * 1024u + 256u + (hp & 3u) * 64u;
}

}

__global__ void __launch_bounds__(512) __attribute__((amdgpu_waves_per_eu(2, 2)))
ag_gemm_relu_kernel(const Params p) {
  extern __shared__ __attribute__((aligned(16))) char smem_raw[];
  LAS char* const lds = (LAS char*)smem_raw;
  LAS volatile unsigned* const misc = (LAS volatile unsigned*)(lds + kLdsMisc);

  const int tid = threadIdx.x;
  const int lane = tid & 63;
  const int wid = __builtin_amdgcn_readfirstlane(tid >> 6);
  const int bid = blockIdx.x;
  const unsigned rank = (unsigned)p.rank & 7u;
  const unsigned q = rank & 3u, og = (rank & 4u) ^ 4u;
  char* const myws = p.ws[rank];
  const unsigned l16 = (unsigned)lane * 16u;
  const __amdgpu_buffer_rsrc_t rws = mk_rsrc(myws, kWsNeed);
  const int fr = lane & 15, fq = lane >> 4;

  STAMP(myws, wid == 0 ? 0 : 31);

  const unsigned lx = (((unsigned)lane >> 2) * 8192u + 4u * (unsigned)((lane & 3) ^ ((-(lane >> 4)) & 3))) * 4u;
  const unsigned lw = (4u * ((unsigned)lane >> 4) * 1024u + 4u * ((unsigned)lane & 15u)) * 4u;
  const __amdgpu_buffer_rsrc_t rx = mk_rsrc(p.x, kInBytes);
  const __amdgpu_buffer_rsrc_t rw = mk_rsrc(p.w, kInBytes);
  const unsigned pw = (unsigned)wid >= WPM ? (unsigned)wid - WPM : (unsigned)wid;
  const unsigned gw = (unsigned)bid * WPM + pw;
  const unsigned sid = gw % 14u, k = gw / 14u;
  const bool cross = sid >= 6u;
  const unsigned c = cross ? (sid - 6u) >> 1 : sid >> 1, h = sid & 1u;
  const unsigned sr = (cross ? og + c : peer_of(rank, c)) & 7u;
  char* const sws = p.ws[sr];
  const __amdgpu_buffer_rsrc_t rs = mk_rsrc(sws, kWsNeed);
  LAS volatile unsigned* const hand = misc + 4u + 4u * pw;
  const unsigned buf0 = pw * 3u * kPBuf;
  if (bid < kNMove && (unsigned)wid >= WPM && (unsigned)wid < 2u * WPM && gw < 14u * NWS) {
    unsigned call = 0;
    auto push = [&](auto wconst) {
      constexpr bool W = decltype(wconst)::value;
      f32x4 pv[PU][2][8];
#pragma unroll
      for (unsigned u = 0; u < PU; ++u)
#pragma unroll
        for (int i = 0; i < 16; ++i) {
          const unsigned m = u * NWS + k, kk = (m >> 1) * 64u + h * 32u;
          if (W) pv[u][i >> 3][i & 7] = ld16<0>(rw, lw + (i >> 3) * 256u, (kk * 1024u + 256u * c + (m & 1u) * 128u) * 4u + ((i & 3) + 16 * ((i >> 2) & 1)) * 4096u);
          else pv[u][i >> 3][i & 7] = ld16<0>(rx, lx + (i & 1) * 64u, ((m & 1u) * 128u * 8192u + kk) * 4u + (i >> 1) * 524288u);
        }
      __syncthreads();
      call = uni(misc[0]) + 1u;
      const unsigned base = W ? kBOff + (call & 1u) * kBSize + (8u + 2u * q) * kPanel : kAOff + (call & 1u) * kASize + (18u - 5u * c) * kPanel;
      char* const fl = sws + kFlags + (W ? 44u + 2u * q : 18u - 5u * c) * 1024u + h * 4u;
#pragma unroll
      for (unsigned j = 0; j < 2u * PU; ++j) {
        const unsigned u = j >> 1, r = j & 1u, m = u * NWS + k;
        f16x8 hv[4];
#pragma unroll
        for (int i = 0; i < 4; ++i) hv[i] = W ? cvt_col(pv[u][r], i) : cvt8(pv[u][r][2 * i], pv[u][r][2 * i + 1]);
        asm volatile("" :: "v"(hv[0]), "v"(hv[1]), "v"(hv[2]), "v"(hv[3]));
        if (j >= PUW) WAIT_VM(4u * (PUW - 1u));
        if (j <= 2u) STAMP(myws, 8 + wid + 3 * j);
        if (u > 0u && r == PUW - 1u && lane == 0) flag_store(fl + ((m - NWS) & 1u) * 1024u + ((m - NWS) >> 1) * 8u, call);
#pragma unroll
        for (int i = 0; i < 4; ++i) st16<kAuxSys>(hv[i], rs, l16 + i * 1024u, base + (m & 1u) * kPanel + (m >> 1) * kBlk + h * 8192u + r * 4096u);
        FENCE();
      }
      WAIT_VM(0);
      const unsigned m = (PU - 1u) * NWS + k;
      if (lane == 0) flag_store(fl + (m & 1u) * 1024u + (m >> 1) * 8u, call);
    };
    if (cross) push(std::true_type{}); else push(std::false_type{});
    STAMP(myws, 4 + wid);
    if (!cross) return;
    const __amdgpu_buffer_rsrc_t ro = mk_rsrc(p.out[rank], kInBytes);
    const unsigned vo = (unsigned)fq * 16384u + (unsigned)fr * 16u;
    const unsigned long long t0 = __builtin_amdgcn_s_memrealtime();
    for (unsigned b = 0;; b = b == 2u ? 0u : b + 1u) {
      unsigned hv;
      while ((hv = uni(hand[b])) == 0u) {
        if (__builtin_amdgcn_s_memrealtime() - t0 > 2u * kDeadline) { set_timeout(myws, call); return; }
        __builtin_amdgcn_s_sleep(XSLEEP);
      }
      FENCE();
      if (hv == ~0u) break;
      const unsigned oa = park_row(hv >> 4, sr, rank) * 4096u + (256u * c + (hv & 12u) * 16u) * 4u;
      unsigned pos = (unsigned)(size_t)lds + buf0 + b * kPBuf, val[64];
      u32x2 mk = {~0u, ~0u};
      if ((hv & 2u) == 0u) mk = *(LAS u32x2*)(size_t)(pos + lane * 8);
      const unsigned fill = (unsigned)(size_t)lds + kLdsMisc + 64u + (hv & 2u) * 2u;
      pos += 512u;
#pragma unroll
      for (int i = 0; i < 64; ++i) {
        const bool keep = (int)(mk[i >> 5] << (i & 31)) >= 0;
        const unsigned long long bl = __ballot(keep);
        unsigned a = lanes_below(bl) * 4u + pos;
        asm volatile("" : "+v"(a));
        val[i] = *(LAS unsigned*)(size_t)(keep ? a : fill);
        advance(pos, bl);
      }
      WAIT_LGKM0();
      if (lane == 0) hand[b] = 0u;
      FENCE();
#pragma unroll
      for (int i = 0; i < 16; ++i)
        __builtin_amdgcn_raw_buffer_store_b128(u32x4{val[4 * i], val[4 * i + 1], val[4 * i + 2], val[4 * i + 3]}, ro, vo, oa + (i >> 2) * 65536u + (i & 3) * 4096u, kAuxSys);
    }
    STAMP(myws, 1 + wid);
    return;
  }
  const bool isconv = bid >= kNMove + 8 * CVFROM;
  unsigned tk = 0;
  if (tid == 0) tk = draw(myws + kTicket + (unsigned)bid * 32u);
  FENCE();
  f32x4 cv[8];
  auto item = [&](unsigned n, unsigned& ktg, unsigned& sub) {
    if (n < 5u * CVLEAD) { ktg = n / 5u; sub = n - 5u * ktg; sub += sub ? 3u : 0u; }
    else if (n < 8u * CVLEAD) { ktg = (n - 5u * CVLEAD) / 3u; sub = n - 5u * CVLEAD - 3u * ktg + 1u; }
    else { ktg = n >> 3; sub = n & 7u; }
  };
  auto loads = [&](unsigned n) {
    unsigned ktg, sub;
    item(n, ktg, sub);
    if (sub < 4u) {
      const unsigned so = ((256u * sub + 32u * wid) * 8192u + ktg * 64u) * 4u;
#pragma unroll
      for (int s = 0; s < 2; ++s) {
        cv[4 * s + 0] = ld16<0>(rx, lx, so + s * 524288u); cv[4 * s + 1] = ld16<0>(rx, lx + 64u, so + s * 524288u);
        cv[4 * s + 2] = ld16<0>(rx, lx + 128u, so + s * 524288u); cv[4 * s + 3] = ld16<0>(rx, lx + 192u, so + s * 524288u);
      }
    } else {
      const unsigned so = ((ktg * 64u + 32u * (wid & 1)) * 1024u + 256u * (sub - 4u) + 64u * (wid >> 1)) * 4u;
#pragma unroll
      for (int i = 0; i < 8; ++i) cv[i] = ld16<0>(rw, lw, so + ((i & 3) + 16 * (i >> 2)) * 4096u);
    }
  };
  if (bid >= kNMove + 8 * CVFROM + CVFIRST) __builtin_amdgcn_s_sleep(CVDELAY);
  if (isconv) loads((unsigned)(bid - kNMove - 8 * CVFROM));
  FENCE();
  if (tid == 0) {
    misc[0] = tk;
    misc[3] = 0u;
  }
  if (tid >= 4 && tid < 18) misc[tid] = tid == 17 ? 0x7fc00000u : 0u;
  __syncthreads();
  const unsigned call = uni(misc[0]) + 1u;
  const unsigned par = call & 1u;
  const unsigned abase = kAOff + par * kASize, bbase = kBOff + par * kBSize, obuf = kOOff + par * kOSize;

  if (bid < kNMove) {
    if (wid == 7) {
      if (bid == 0 && lane == 0) {
        const unsigned target = call * kDonePerCall;
        const unsigned long long t0 = __builtin_amdgcn_s_memrealtime();
        bool failed = false;
        while ((int)(__hip_atomic_load((GAS unsigned*)(myws + kDone), __ATOMIC_RELAXED, __HIP_MEMORY_SCOPE_SYSTEM) - target) < 0) {
          if (__hip_atomic_load((GAS unsigned*)(myws + kTmo), __ATOMIC_RELAXED, __HIP_MEMORY_SCOPE_AGENT) == call ||
              __builtin_amdgcn_s_memrealtime() - t0 > kDeadline) {
            set_timeout(myws, call);
            failed = true;
            break;
          }
          __builtin_amdgcn_s_sleep(2);
        }
        __builtin_amdgcn_fence(__ATOMIC_ACQUIRE, "");
        if (failed) __hip_atomic_store((GAS unsigned*)(p.out[rank] + (size_t)rank * 1024u * 1024u), 0x7fc00000u, __ATOMIC_RELAXED, __HIP_MEMORY_SCOPE_SYSTEM);
        STAMP(myws, 10);
      }
      return;
    }
    if ((unsigned)wid >= WPM || gw >= 14u * NWS) return;
    const unsigned total = cross ? kRowsC : kRowsI;
    const unsigned nplain = cross ? 576u : kRowsI;
    const unsigned qbase = (call - 1u) * (total - PU * NWS) - (PU + 1u) * NWS;
    bool failed = false, all = false;
    unsigned cgrp = 99u, okf = 0, okt = 0, fv = 0, wbase = 0x10000u;
    u32x2 fw = {0u, 0u};
    unsigned sa = 0, da = 0, fa = 0, pend = 0;
    unsigned s2 = 4096u, d2 = 4096u;
    u32x4 v[4], w[4];
    auto raise = [&]() {
      if (pend != 0u && lane == 0) flag_store(myws + pend, call);
      pend = 0u;
    };
    auto fetch = [&](u32x4 (&x)[4], unsigned so) {
#pragma unroll
      for (int i = 0; i < 4; ++i) x[i] = __builtin_amdgcn_raw_buffer_load_b128(rs, l16 + i * 1024u, so, kAuxSys);
      FENCE();
    };
    auto put = [&](const u32x4 (&x)[4], unsigned so) {
#pragma unroll
      for (int i = 0; i < 4; ++i) __builtin_amdgcn_raw_buffer_store_b128(x[i], rws, l16 + i * 1024u, so, kAuxDev);
      FENCE();
    };
    auto setup = [&](unsigned m) {
      unsigned sp, kt, dp, db, dh = h;
      unsigned so = h * 8192u, dof = h * 8192u;
      s2 = 4096u; d2 = 4096u;
      if (!cross) {
        if (m < 256u) { sp = m & 1u; kt = m >> 1; dp = 8u + 5u * c + sp; db = kt; }
        else if (m < 384u) { sp = 4u; kt = m - 256u; dp = 12u + 5u * c; db = kt; }
        else if (m < 640u) { sp = 2u + (m & 1u); kt = (m - 384u) >> 1; dp = 8u + 5u * c + sp; db = kt; }
        else { sp = 5u; db = m - 640u; kt = 2u * db + h; so = 0u; s2 = 8192u; dp = 23u + c; }
      } else {
        if (m < 256u) { sp = 36u + 2u * q + (m & 1u); kt = m >> 1; dp = 44u + 2u * c + (m & 1u); db = kt; }
        else if (m < 320u) { sp = 5u; kt = 2u * (m - 256u) + h; so = 4096u; s2 = 8192u; dp = 34u + (c >> 1); db = kt; dh = c & 1u; dof = dh * 4096u; d2 = 8192u; }
        else { sp = 6u + (m & 1u); kt = (m - 320u) >> 1; dp = 26u + 2u * c + (m & 1u); db = kt; }
      }
      const unsigned sub = sp < 36u ? sp >> 1 : (sp - 28u) >> 1;
      auto ask = [&](unsigned k0) {
        const unsigned off = lane < 62 ? kSrc + (sr * 8u + sub) * 512u + min(k0 + (unsigned)lane, 127u) * 4u : lane == 62 ? kAll + sr * 4u : kTmo;
        asm volatile("buffer_load_dword %0, %1, %2, 0 offen sc0 sc1" : "=v"(fv) : "v"(off), "s"(rws) : "memory");
      };
      auto look = [&]() {
        asm volatile("" : "+v"(fv) :: "memory");
        const unsigned long long b = __ballot(lane < 63 ? (int)(fv - call) >= 0 : fv == call);
        if ((b >> 62) != 0ull) { failed = (b >> 63) != 0ull; all = true; }
        okt += (unsigned)__builtin_ctzll(~b);
      };
      if (!all) {
        if (cgrp != 99u) look();
        if (!all && !(sub == cgrp && kt >= okf && kt < okt)) {
          WAIT_VM(0);
          raise();
          cgrp = sub;
          okf = kt;
          const unsigned long long t0 = __builtin_amdgcn_s_memrealtime();
          for (;;) {
            okt = kt;
            ask(kt);
            WAIT_VM(0);
            look();
            if (all || okt != kt) break;
            if (__builtin_amdgcn_s_memrealtime() - t0 > kDeadline) { set_timeout(myws, call); failed = true; all = true; break; }
            __builtin_amdgcn_s_sleep(MSLEEP);
          }
        }
        if (!all) ask(okt);
      }
      sa = (sp < 36u ? abase + sp * kPanel : bbase + (sp - 36u) * kPanel) + kt * kBlk + so;
      da = (dp < 36u ? abase + dp * kPanel : bbase + (dp - 36u) * kPanel) + db * kBlk + dof;
      fa = kFlags + dp * 1024u + db * 8u + dh * 4u;
      fetch(v, sa);
    };
    unsigned m = PU * NWS + k;
    setup(m);
    STAMP(myws, 20 + wid);
    do {
      unsigned dv = 0;
      if (lane == 0) dv = draw(myws + kQueue + sid * 128u);
      FENCE();
      put(v, da);
      raise();
      fetch(w, sa + s2);
      put(w, da + d2);
      pend = fa;
      if (h == 0u) { PSTAMP(myws, sid >> 1, m + 1u); if (m < 128u) PSTAMP(myws, 8u + (sid >> 1), (m + 1u) * 4u); }
      m = uni(dv) - qbase;
      if (m < nplain) setup(m);
    } while (m < nplain);
    WAIT_VM(0);
    raise();
    if (cross) {
      unsigned hval = 0;
      const unsigned fo = lane < 32 ? kOFlags + par * 8192u + c * 2048u + h * 8u + (unsigned)lane * 16u : kTmo;
      auto stale = [&](unsigned o) {
        return o - wbase >= 32u || (int)((unsigned)__builtin_amdgcn_readlane((int)fw[0], (int)(o - wbase)) - call) < 0;
      };
      for (unsigned b = 0, pb = 0;; pb = b, b = b == 2u ? 0u : b + 1u) {
        const unsigned o = m - 576u, s = 2u * o + h;
        unsigned cnt = 2048u, dv = 0;
        if (m >= total || stale(o)) {
          WAIT_VM(0);
          if (hval != 0u && lane == 0) hand[pb] = hval;
          hval = 0u;
          const unsigned long long t0 = __builtin_amdgcn_s_memrealtime();
          wbase = o;
          while (!failed && m < total) {
            fw = __builtin_bit_cast(u32x2, __hip_atomic_load((GAS unsigned long long*)(myws + fo + (lane < 32 ? o * 16u : 0u)), __ATOMIC_RELAXED, __HIP_MEMORY_SCOPE_SYSTEM));
            const unsigned long long bl = __ballot(lane < 32 ? (int)(fw[0] - call) >= 0 : fw[0] == call);
            if ((bl >> 32) != 0ull) { failed = true; break; }
            if (bl & 1ull) break;
            if (__builtin_amdgcn_s_memrealtime() - t0 > kDeadline) { set_timeout(myws, call); failed = true; }
            __builtin_amdgcn_s_sleep(MSLEEP);
          }
        }
        cnt = failed ? 0u : min((unsigned)__builtin_amdgcn_readlane((int)fw[1], (int)((o - wbase) & 31u)), 4096u);
        if (UNLIKELY(uni(hand[b]) != 0u)) {
          const unsigned long long t0 = __builtin_amdgcn_s_memrealtime();
          while (uni(hand[b]) != 0u)
            if (__builtin_amdgcn_s_memrealtime() - t0 > kDeadline) { set_timeout(myws, call); return; }
        }
        FENCE();
        if (m >= total) {
          if (lane == 0) hand[b] = ~0u;
          break;
        }
        const unsigned bytes = 512u + 4u * cnt, sb = obuf + (q * 224u + s) * kPiece, nbuf = buf0 + b * kPBuf;
        auto part = [&](unsigned f0, unsigned f1) {
          for (unsigned f = f0; f < f1; f += 1024u) {
            WAIT_VM(PWIN - 1u);
            asm volatile("s_mov_b32 m0, %0\n\ts_nop 0\n\tbuffer_load_dwordx4 %1, %2, %3 offen sc0 sc1 lds"
                         :: "s"(nbuf + f), "v"(f + l16 < bytes ? l16 : kOob), "s"(rs), "s"(sb + f) : "memory");
          }
        };
        asm volatile("s_mov_b64 exec, 1\n\tglobal_atomic_add %0, %1, %2, %3 sc0\n\ts_mov_b64 exec, -1"
                     : "=v"(dv) : "v"(kQueue + sid * 128u), "v"(1u), "s"(myws) : "memory");
        part(0u, min(bytes, (PWIN - 1u) * 1024u));
        if (bytes <= (PWIN - 2u) * 1024u) WAIT_VM(0);
        WAIT_VM(PWIN - 1u);
        if (hval != 0u && lane == 0) hand[pb] = hval;
        hval = 4u * s + (failed ? 3u : 1u);
        if (h == 0u) PSTAMP(myws, sid >> 1, m + 1u);
        asm volatile("" : "+v"(dv));
        m = uni(dv) - qbase;
        if (m < total && stale(m - 576u)) {
          wbase = m - 576u;
          asm volatile("buffer_load_dwordx2 %0, %1, %2, 0 offen sc0 sc1" : "=v"(fw) : "v"(fo + (lane < 32 ? wbase * 16u : 0u)), "s"(rws) : "memory");
          part((PWIN - 1u) * 1024u, bytes);
          if (bytes <= (2u * PWIN - 3u) * 1024u) WAIT_VM(0);
          WAIT_VM(PWIN - 1u);
          asm volatile("" : "+v"(fw) :: "memory");
        } else
          part((PWIN - 1u) * 1024u, bytes);
      }
    }
    STAMP(myws, 1 + wid);
    return;
  }

  {
    constexpr unsigned kNConv = 8u * (26u - CVFROM);
    char* const pws = p.ws[lane & 7];
    const unsigned mine = 15u << (rank & 4u);
    STAMP(myws, wid == 0 ? 27 : 31);
    for (unsigned n = isconv ? (unsigned)(bid - kNMove - 8 * CVFROM) : 1024u; n < 1024u; n += kNConv) {
      unsigned ktg, sub;
      item(n, ktg, sub);
      if (sub < 4u) {
        const unsigned so = abase + (2u * sub + (wid >> 2)) * kPanel + ktg * kBlk + (wid & 3) * 2048u;
#pragma unroll
        for (int i = 0; i < 4; ++i) st16<CVAUX>(cvt8(cv[2 * i], cv[2 * i + 1]), rws, l16 + (i >> 1) * 1024u, so + (i & 1) * 8192u);
      } else {
        const unsigned so = bbase + (2u * (sub - 4u) + (wid >> 2)) * kPanel + ktg * kBlk + ((wid >> 1) & 1) * 4096u + (wid & 1) * 8192u;
#pragma unroll
        for (int i = 0; i < 4; ++i) st16<CVAUX>(cvt_col(cv, i), rws, l16 + i * 1024u, so);
      }
      FENCE();
      const bool ahead = n >= kNConv && n + kNConv < 1024u;
      if (ahead) { loads(n + kNConv); WAIT_VM(8); } else { WAIT_VM(0); }
      if (n < kNConv) STAMP(myws, wid == 0 ? 30 : 31);
      BARRIER();
      if (n < 2u * kNConv) STAMP(myws, wid == 0 ? (n < kNConv ? 28 : 29) : 31);
      const unsigned to = sub < 2u ? mine : sub == 2u ? 255u : sub == 3u ? mine ^ 255u : 1u << (og + sub - 4u);
      if (tid < 2) flag_store2(myws + kFlags + ((sub < 4u ? 2u * sub : 28u + 2u * sub) + (unsigned)tid) * 1024u + ktg * 8u, call, call);
      else if (wid == 1 && lane < 8 && (((to & ~(1u << rank)) >> lane) & 1u)) flag_store(pws + kSrc + rank * 4096u + sub * 512u + ktg * 4u, call);
      FENCE();
      if (n < kNConv) loads(n + kNConv);
    }
    if (isconv && wid == 7) {
      unsigned dv = 0;
      if (lane == 0) dv = draw(myws + kCvt);
      if (uni(dv) == call * kNConv - 1u && lane < 8) flag_store(pws + kAll + rank * 4u, call);
    }
  }
  STAMP(myws, wid == 0 ? 1 : 31);

  const uint32_t* const tl = p.tiles + 3 * (bid - kNMove);
  const int ntiles = (int)(tl[0] >> 28);
  if (wid == 7) return;

  if (wid >= 4) {
    const unsigned lw = (unsigned)wid - 4u;
    unsigned total = 0;
    for (int ti = 0; ti < ntiles; ++ti) total += ((tl[ti] >> 10) & 3u) == 2u ? 64u : 256u;
    unsigned a = 0, aend = 0, so = lw * 8192u, src = 0, step = 0, g2 = 0, v2 = l16, idle = 0;
    unsigned ok = 0, f0 = 0, f1 = 0, fb = 0, fsh = 0, fmax = 0, fv = 0;
    unsigned rdy = 0, askat = 0, lookat = ~0u;
    int ta = 0, tf = 0;
    auto next_flags = [&]() {
      while (tf < ntiles) {
        const unsigned code = tl[tf++];
        const unsigned ap = code & 63u, bp = (code >> 6) & 15u, shape = (code >> 10) & 3u;
        f0 = f1;
        f1 += shape == 2u ? 64u : 256u;
        if (shape == 3u && lw == 2u) { ok = f1; continue; }
        const unsigned grp = lw == 0u ? ap : shape == 0u ? (lw == 1u ? ap + 1u : 36u + bp) : shape == 1u ? 35u + bp + lw : 36u + bp;
        fb = kFlags + grp * 1024u;
        fsh = shape != 2u ? 0u : lw == 0u ? 1u : 2u;
        fmax = fsh == 1u ? 127u : 255u;
        break;
      }
    };
    auto known = [&]() {
      rdy = min(ok, aend);
      unsigned from = max(ok, LTHR);
      asm volatile("" : "+s"(from));
      askat = ok >= total ? ~0u : from - LTHR;
    };
    auto ask = [&]() {
      const unsigned off = lane < 63 ? fb + min(((ok - f0) << fsh) + (unsigned)lane, fmax) * 4u : kTmo;
      asm volatile("buffer_load_dword %0, %1, %2, 0 offen sc0 sc1" : "=v"(fv) : "v"(off), "s"(rws) : "memory");
    };
    auto give_up = [&]() {
      ok = total;
      if (lane == 0) misc[3] = 1u;
      WAIT_LGKM0();
    };
    auto look = [&]() {
      asm volatile("" : "+v"(fv) :: "memory");
      const unsigned long long b = __ballot(lane < 63 ? (int)(fv - call) >= 0 : fv == call);
      if (UNLIKELY((b >> 63) != 0ull)) {
        give_up();
      } else {
        const unsigned run = (unsigned)__builtin_ctzll(~b);
        ok = min(ok + (fsh == 0u ? run & ~1u : run >> fsh), f1);
        if (UNLIKELY(ok == f1)) next_flags();
      }
      lookat = ~0u;
    };
    auto issue = [&](bool busy) {
      LAS char* const dl = lds + so;
      if (busy) {
        __builtin_amdgcn_raw_ptr_buffer_load_lds(rws, (LAS void*)dl, 16, l16, src, 0, kAuxDev);
        __builtin_amdgcn_raw_ptr_buffer_load_lds(rws, (LAS void*)dl, 16, l16, src, 1024, kAuxDev);
        __builtin_amdgcn_raw_ptr_buffer_load_lds(rws, (LAS void*)dl, 16, v2, src, 2048, kAuxDev);
        __builtin_amdgcn_raw_ptr_buffer_load_lds(rws, (LAS void*)dl, 16, v2, src, 3072, kAuxDev);
        __builtin_amdgcn_raw_ptr_buffer_load_lds(rws, (LAS void*)(dl + 4096), 16, l16, src + g2, 0, kAuxDev);
        __builtin_amdgcn_raw_ptr_buffer_load_lds(rws, (LAS void*)(dl + 4096), 16, l16, src + g2, 1024, kAuxDev);
        __builtin_amdgcn_raw_ptr_buffer_load_lds(rws, (LAS void*)(dl + 4096), 16, v2, src + g2, 2048, kAuxDev);
        __builtin_amdgcn_raw_ptr_buffer_load_lds(rws, (LAS void*)(dl + 4096), 16, v2, src + g2, 3072, kAuxDev);
      }
      FENCE();
      src += step;
      so = so >= kRing - kSlot ? so - (kRing - kSlot) : so + kSlot;
      ++a;
    };
    next_flags();
    known();
    for (unsigned j = 0; j < total; ++j) {
      if (a == j + 4u) {
        unsigned e = min(rdy + 1u, askat);
        asm volatile("" : "+s"(e));
        e = min(e, lookat);
        if (idle != 0u) {
          WAIT_VM(0);
          while (a + 1u < e) { issue(false); BARRIER(); }
        }
        while (a + 1u < e) {
          issue(true);
          WAIT_VM(32);
          BARRIER();
        }
        j = a - 4u;
      }
      unsigned lim = min(j + 5u, rdy);
      if (UNLIKELY(a >= lim)) {
        if (a == aend) {
          const unsigned code = tl[ta++];
          const unsigned ap = code & 63u, bp = (code >> 6) & 15u, shape = (code >> 10) & 3u;
          if (lw == 0u || (lw == 1u && shape == 0u)) src = abase + (ap + lw) * kPanel;
          else src = bbase + (bp + (shape == 1u ? lw - 1u : 0u)) * kPanel;
          step = 8192u; g2 = 4096u;
          idle = shape == 3u && lw == 2u ? 1u : 0u;
          if (shape == 2u) {
            g2 = 8192u;
            if (lw == 0u) { src += ((code >> 13) & 1u) * 2048u; step = kBlk; v2 = l16 + 2048u; }
            else { src += ((code >> 12) & 1u) * 4096u + (lw - 1u) * kBlk; step = 2u * kBlk; }
          }
          aend = ta == ntiles ? ~0u : aend + 256u;
          rdy = min(ok, aend);
        }
        if (a <= j && ok <= a) {
          WAIT_VM(0);
          if (lookat != ~0u) look();
          const unsigned long long t0 = __builtin_amdgcn_s_memrealtime();
          while (ok <= a) {
            ask();
            WAIT_VM(0);
            look();
            if (ok > a) break;
            if (__builtin_amdgcn_s_memrealtime() - t0 > kDeadline) { set_timeout(myws, call); give_up(); break; }
            __builtin_amdgcn_s_sleep(LSLEEP);
          }
          known();
        }
        lim = min(j + 5u, rdy);
      }
      if (a < lim) {
        issue(idle == 0u);
        if (UNLIKELY(a < lim)) issue(idle == 0u);
      }
      if (UNLIKELY(a >= askat)) { ask(); askat = ~0u; lookat = a + 4u; }
      const unsigned n = idle != 0u ? 0u : a - 1u - j;
      if (UNLIKELY(n < 4u)) {
        WAIT_VM(24);
        if (n < 3u) {
          WAIT_VM(16);
          if (n < 2u) {
            WAIT_VM(8);
            if (n < 1u) WAIT_VM(0);
          }
        }
      }
      WAIT_VM(32);
      if (UNLIKELY(j + 5u >= lookat)) { look(); known(); }
      BARRIER();
    }
    return;
  }

  {
    const unsigned slot = (unsigned)(4 * fr + (fq ^ ((-(fr >> 2)) & 3))) * 16u;
    unsigned so = 0;
    for (int ti = 0; ti < ntiles; ++ti) {
      const unsigned code = tl[ti];
      const unsigned shape = (code >> 10) & 3u;
      const unsigned wh = (unsigned)wid >> 1, wl = (unsigned)wid & 1u;
      const unsigned ap = (code & 63u) + (shape == 0u ? wh : 0u), bp = ((code >> 6) & 15u) + (shape == 1u ? wh : 0u);
      const unsigned jo = (bp - 8u) >> 1;
      const unsigned owner = (bp < 8u ? rank : og + jo) & 7u;
      const unsigned col0 = bp < 8u ? bp * 128u : 256u * q + (bp & 1u) * 128u;
      const __amdgpu_buffer_rsrc_t ro = mk_rsrc(p.out[owner], kInBytes);
      const unsigned vo = (unsigned)fq * 16384u + (unsigned)fr * 16u;

      auto run = [&](auto mconst, const unsigned aoff, const unsigned boff) {
        constexpr int M = decltype(mconst)::value;
        constexpr int KS = M > 1 ? 1 : 4;
        constexpr int NS = M > 1 ? 256 : 64;
        constexpr int NA = M * KS, NB = 4 * KS;
        const unsigned va = aoff + slot, vb = boff + l16;
        f32x4 acc[M][4];
#pragma unroll
        for (int m = 0; m < M; ++m)
#pragma unroll
          for (int n = 0; n < 4; ++n) acc[m][n] = f32x4{0.f, 0.f, 0.f, 0.f};
        f16x8 fa[2][NA], fb[2][NB];
        auto rd = [&](f16x8 (&a)[NA], f16x8 (&b)[NB]) {
          LAS char* const pa = lds + (va + so);
          LAS char* const pb = lds + (vb + so);
#pragma unroll
          for (int i = 0; i < (NA > NB ? NA : NB); ++i) {
            if (i < NB) b[i] = *(LAS f16x8*)(pb + i * 1024);
            if (i < NA) a[i] = *(LAS f16x8*)(pa + i * (M > 1 ? 1024 : 2048));
          }
        };
        auto mm = [&](const f16x8 (&a)[NA], const f16x8 (&b)[NB]) {
#pragma unroll
          for (int ks = 0; ks < KS; ++ks)
#pragma unroll
            for (int m = 0; m < M; ++m)
#pragma unroll
              for (int n = 0; n < 4; ++n)
                acc[m][n] = __builtin_amdgcn_mfma_f32_16x16x32_f16(a[KS * m + ks], b[4 * ks + n], acc[m][n], 0, 0, 0);
        };
        BARRIER();
        STAMP(myws, wid == 0 ? 2 + 3 * ti : 31);
        CSTAMP(myws, wid == 0 ? 12 + 2 * ti : 31);
        rd(fa[0], fb[0]);
        __builtin_amdgcn_s_waitcnt(0xC07F);
        BARRIER();
        so = so == kRing - kSlot ? 0u : so + kSlot;
        if constexpr (M > 1) {
          unsigned ua = va + so, ub = vb + so;
          for (int t = 0; t < NS; t += 2) {
#pragma unroll
            for (int s = 0; s < 2; ++s) {
              __builtin_amdgcn_sched_barrier(0);
#pragma unroll
              for (int i = 0; i < 4 * M; ++i) {
                acc[i >> 2][i & 3] = __builtin_amdgcn_mfma_f32_16x16x32_f16(fa[s][i >> 2], fb[s][i & 3], acc[i >> 2][i & 3], 0, 0, 0);
                __builtin_amdgcn_sched_barrier(0);
                if (M == 8 && (i & 1) == 0 && i < 24) {
                  const int r = i >> 1;
                  if (r < 4) fb[s ^ 1][r] = *(LAS f16x8*)(lds + ub + r * 1024);
                  else fa[s ^ 1][r - 4] = *(LAS f16x8*)(lds + ua + (r - 4) * 1024);
                }
                if (M == 4 && i < 4) {
                  fb[s ^ 1][i] = *(LAS f16x8*)(lds + ub + i * 1024);
                  fa[s ^ 1][i] = *(LAS f16x8*)(lds + ua + i * 1024);
                }
                if (i == 4 * M - 10) { so = so == kRing - kSlot ? 0u : so + kSlot; asm volatile("" : "+s"(so)); }
                if (i == 4 * M - 6) { ua = va + so; asm volatile("" : "+v"(ua)); }
                if (i == 4 * M - 4) { ub = vb + so; asm volatile("" : "+v"(ub)); }
                if (i == 4 * M - 2) {
                  __builtin_amdgcn_s_waitcnt(0xC07F);
                  if (t < NS - 2) BARRIER();
                }
                __builtin_amdgcn_sched_barrier(0);
              }
            }
          }
          so = so == 0u ? kRing - kSlot : so - kSlot;
        } else {
          for (int t = 0; t < NS; t += 2) {
#pragma unroll
            for (int s = 0; s < 2; ++s) {
              __builtin_amdgcn_sched_barrier(0);
              rd(fa[s ^ 1], fb[s ^ 1]);
              mm(fa[s], fb[s]);
#pragma unroll
              for (int i = 0; i < 4; ++i) {
                __builtin_amdgcn_sched_group_barrier(0x100, 2, 0);
                __builtin_amdgcn_sched_group_barrier(0x008, 1, 0);
              }
#pragma unroll
              for (int i = 0; i < 12; ++i) {
                __builtin_amdgcn_sched_group_barrier(0x100, 1, 0);
                __builtin_amdgcn_sched_group_barrier(0x008, 1, 0);
              }
              __builtin_amdgcn_sched_barrier(0);
              so = so == kRing - kSlot ? 0u : so + kSlot;
              __builtin_amdgcn_s_waitcnt(0xC07F);
              if (t < NS - 2) BARRIER();
            }
          }
        }
        STAMP(myws, wid == 0 ? 3 + 3 * ti : 31);
        CSTAMP(myws, wid == 0 ? 13 + 2 * ti : 31);
        const int lo = misc[3] != 0u ? 0x7fc00000 : 0;
        if (M > 1 && bp >= 8u) {
          const __amdgpu_buffer_rsrc_t rw4 = __builtin_amdgcn_make_buffer_rsrc((void*)myws, 4, kWsNeed / 4u, 0x00020000);
          if (UNLIKELY(lo != 0)) {
#pragma unroll
            for (int m = 0; m < M; ++m)
#pragma unroll
              for (int n = 0; n < 4; ++n) acc[m][n] = __builtin_bit_cast(f32x4, u32x4{0x7fc00000u, 0x7fc00000u, 0x7fc00000u, 0x7fc00000u});
          }
#pragma unroll
          for (int rh = 0; rh < M / 4; ++rh) {
            const unsigned hp = park_pos(ap, M == 8 ? (unsigned)rh : wh, jo);
            if (hp < 56u) {
              const unsigned s = hp * 4u + (bp & 1u) * 2u + wl;
              const unsigned pb0 = obuf + (jo * 224u + s) * kPiece;
              unsigned pos = pb0 + 512u;
              u32x2 mk = {0u, 0u};
#pragma unroll
              for (int m = 0; m < 4; ++m)
#pragma unroll
                for (int j = 0; j < 4; ++j)
#pragma unroll
                  for (int n = 0; n < 4; ++n) {
                    const float f = acc[4 * rh + m][n][j];
                    const unsigned iv = __float_as_uint(f);
                    const unsigned long long b = __ballot((int)iv >= 0);
                    const unsigned idx = lanes_below(b) | (iv & kOob);
                    asm volatile("buffer_store_dword %0, %1, %2, %3 idxen sc0 sc1" :: "v"(iv), "v"(idx), "s"(rw4), "s"(pos) : "memory");
                    advance(pos, b);
                    mk[m >> 1] = __builtin_amdgcn_alignbit(mk[m >> 1], iv, 31);
                  }
              __builtin_amdgcn_raw_buffer_store_b64(mk, rws, (unsigned)lane * 8u, pb0, kAuxSys);
              WAIT_VM(0);
              if (lane == 0) flag_store2(p.ws[owner] + kOFlags + par * 8192u + q * 2048u + s * 8u, call, (pos - pb0 - 512u) >> 2);
            }
          }
        } else {
#pragma unroll
          for (int m = 0; m < M; ++m) {
            const unsigned ob = M > 1 ? (panel_row(ap, M == 8 ? (unsigned)(m >> 2) : wh, rank) + 16u * (m & 3)) * 4096u + (col0 + 64u * wl) * 4u
                                       : (panel_row(ap, 0u, rank) + 32u * ((code >> 13) & 1u) + 16u * wl) * 4096u + (col0 + 64u * ((code >> 12) & 1u)) * 4u;
#pragma unroll
            for (int j = 0; j < 4; ++j) {
              u32x4 o;
#pragma unroll
              for (int n = 0; n < 4; ++n) { const float f = acc[m][n][j]; o[n] = (unsigned)max(__float_as_int(f), lo); }
              __builtin_amdgcn_raw_buffer_store_b128(o, ro, vo, ob + j * 4096u, kAuxSys);
            }
          }
          if (bp >= 8u) {
            WAIT_VM(0);
            if (lane == 0) __hip_atomic_fetch_add((GAS unsigned*)(p.ws[owner] + kDone), 1u, __ATOMIC_RELAXED, __HIP_MEMORY_SCOPE_SYSTEM);
          }
        }
      };
      if (shape == 2u) {
        if (wid >= 2) return;
        run(std::integral_constant<int, 1>{}, wl * 1024u, 8192u);
      } else if (shape == 3u) {
        run(std::integral_constant<int, 4>{}, wh * 4096u, 8192u + wl * 4096u);
      } else {
        run(std::integral_constant<int, 8>{}, shape == 0u ? wh * 8192u : 0u, (shape == 0u ? 16384u : 8192u + wh * 8192u) + wl * 4096u);
      }
      STAMP(myws, wid == 0 ? 4 + 3 * ti : 31);
    }
  }
}

extern "C" void kernel_launch(int rank, int nranks, ncclComm_t comm, hipStream_t stream,
                              void* const* d_in, const int* in_sizes, int n_in,
                              void* d_out, int out_size,
                              const int* in_dtypes, int out_dtype,
                              void* const* d_out_all, void* const* d_ws_all, size_t ws_bytes) {
  (void)comm; (void)in_dtypes; (void)out_dtype; (void)d_out;
  if (nranks != 8 || (unsigned)rank >= 8u || n_in < 2 || ws_bytes < (size_t)kWsNeed ||
      in_sizes[0] != 8 * 1024 * 1024 || in_sizes[1] != 8 * 1024 * 1024 || out_size != 8 * 1024 * 1024) {
    fprintf(stderr, "kernel_launch: unexpected problem geometry\n");
    return;
  }
  Params p;
  p.x = (const float*)d_in[0];
  p.w = (const float*)d_in[1];
  for (int i = 0; i < 8; ++i) { p.ws[i] = (char*)d_ws_all[i]; p.out[i] = (float*)d_out_all[i]; }
  p.rank = rank;
  p.pad = 0;
  static const int xcol[8] = {0, 4, 2, 3, 1, 5, 6, 7};
  auto tile = [](int a, int b, int shape) { return (unsigned)(a | (b << 6) | (shape << 10)); };
  for (int ci = 0; ci < kNComp; ++ci) {
    const int x = ci & 7, s = ci >> 3, bq = 2 * (x ^ 4);
    unsigned t[3] = {0u, 0u, 0u};
    int n = 0;
    if (s < 12) t[n++] = tile(x < 4 ? 2 * (s >> 2) : 8 + 5 * (s >> 2), 4 * (x & 3) + (s & 3), 0);
    else if (s == 12) t[n++] = tile(6, xcol[x], 0);
    else if (s < 19) t[n++] = tile(12 + 5 * ((s - 13) >> 1), bq + ((s - 13) & 1), 3);
    else if (s < 23) t[n++] = tile(34 + ((s - 19) >> 1), bq + ((s - 19) & 1), 3);
    if (s < 13 || s == 25) {
      const int i = s < 13 ? s : 13, j = x >> 1, e = i >> 1, idx = 3 * (x & 1) + e;
      t[n++] = tile(26 + 2 * j + (i & 1), e < 3 ? 8 + idx + (idx >= 2 * j ? 2 : 0) : 4 * (x & 1) + e - 3, 3);
    } else {
      const int i = s - 13;
      t[n++] = tile(10 + 5 * (i >> 2) + ((i >> 1) & 1), bq + (i & 1), 3);
    }
    const int i = s == 25 ? SKIPA : s < 13 ? (s == SKIPA ? -1 : s) : s - 13;
    if (i >= 0 && i < 12) t[n++] = tile(23 + (i >> 2), s < 13 || s == 25 ? 8 + x : x, 2) | ((unsigned)(i & 1) << 12) | ((unsigned)((i >> 1) & 1) << 13);
    p.tiles[3 * ci] = t[0] | ((unsigned)n << 28);
    p.tiles[3 * ci + 1] = t[1];
    p.tiles[3 * ci + 2] = t[2];
  }
  hipError_t e = hipFuncSetAttribute((const void*)ag_gemm_relu_kernel, hipFuncAttributeMaxDynamicSharedMemorySize, (int)kLdsBytes);
  if (e != hipSuccess) { fprintf(stderr, "hipFuncSetAttribute: %s\n", hipGetErrorString(e)); return; }
  hipLaunchKernelGGL(ag_gemm_relu_kernel, dim3(kNWG), dim3(512), kLdsBytes, stream, p);
  e = hipGetLastError();
  if (e != hipSuccess) fprintf(stderr, "launch: %s\n", hipGetErrorString(e));
}
```

```cpp
#include <hip/hip_runtime.h>
#include <rccl/rccl.h>
#include <cstdio>
#include <cstdint>
#include <type_traits>

typedef _Float16 f16x8 __attribute__((ext_vector_type(8)));
typedef float f32x4 __attribute__((ext_vector_type(4)));
typedef unsigned u32x4 __attribute__((ext_vector_type(4)));
typedef unsigned u32x2 __attribute__((ext_vector_type(2)));
#define LAS __attribute__((address_space(3)))
#define GAS __attribute__((address_space(1)))

#define NWS 9u
#define WPM 3u
#define PU 1u
#define PUW 1u
#define CVDELAY 24
#define CVFIRST 60
#define CVLEAD 40u
#define XSLEEP 4
#define CVFROM 13
#define PWIN 4u
#define CVAUX kAuxSys
#define MSLEEP 4
#define LSLEEP 2
#define WLIM 80u
#define SKIPA 21
#define LTHR 24u

namespace {

constexpr unsigned kBlk = 16384u;
constexpr unsigned kPanel = 2u << 20;
constexpr unsigned kAOff = 1u << 20;
constexpr unsigned kASize = 36u * kPanel;
constexpr unsigned kBOff = kAOff + 2u * kASize;
constexpr unsigned kBSize = 16u * kPanel;
constexpr unsigned kPiece = 16896u;
constexpr unsigned kOOff = kBOff + 2u * kBSize;
constexpr unsigned kOSize = 4u * 224u * kPiece;
constexpr unsigned kWsNeed = kOOff + 2u * kOSize;
constexpr unsigned kTmo = 128, kDone = 256, kCvt = 384, kAll = 512, kQueue = 1024;
constexpr unsigned kTicket = 4096;
constexpr unsigned kFlags = 16384;
constexpr unsigned kOFlags = 81920;
constexpr unsigned kSrc = 98304;
constexpr unsigned kInBytes = 32u << 20;
constexpr int kNMove = 48, kNComp = 208, kNWG = 256;
constexpr unsigned kRowsI = 704u, kRowsC = 688u;
constexpr unsigned kDonePerCall = 192u;
constexpr unsigned long long kDeadline = 50000000ull;
constexpr unsigned kSlot = 24576u, kRing = 6u * kSlot;
constexpr unsigned kPBuf = 17408u;
constexpr unsigned kLdsMisc = 9u * kPBuf, kLdsBytes = kLdsMisc + 128u;
static_assert(kLdsMisc >= kRing, "");
constexpr unsigned kOob = 0x80000000u;
constexpr int kAuxSys = 17;
constexpr int kAuxDev = 16;

struct Params {
  const float* x;
  const float* w;
  char* ws[8];
  float* out[8];
  int rank;
  int pad;
  uint32_t tiles[3 * kNComp];
};

__device__ __forceinline__ __amdgpu_buffer_rsrc_t mk_rsrc(const void* p, unsigned bytes) {
  return __builtin_amdgcn_make_buffer_rsrc((void*)p, 0, bytes, 0x00020000);
}
template <int AUX>
__device__ __forceinline__ f32x4 ld16(__amdgpu_buffer_rsrc_t r, unsigned voff, unsigned soff) {
  return __builtin_bit_cast(f32x4, __builtin_amdgcn_raw_buffer_load_b128(r, voff, soff, AUX));
}
template <int AUX>
__device__ __forceinline__ void st16(f16x8 v, __amdgpu_buffer_rsrc_t r, unsigned voff, unsigned soff) {
  __builtin_amdgcn_raw_buffer_store_b128(__builtin_bit_cast(u32x4, v), r, voff, soff, AUX);
}
__device__ __forceinline__ f16x8 cvt8(f32x4 a, f32x4 b) {
  f16x8 h;
#pragma unroll
  for (int i = 0; i < 4; ++i) { h[i] = (_Float16)a[i]; h[4 + i] = (_Float16)b[i]; }
  return h;
}
__device__ __forceinline__ f16x8 cvt_col(const f32x4 (&v)[8], int i) {
  f16x8 h;
#pragma unroll
  for (int j = 0; j < 8; ++j) h[j] = (_Float16)v[j][i];
  return h;
}
#define WAIT_VM(n) asm volatile("s_waitcnt vmcnt(%0)" :: "n"(n) : "memory")
#define WAIT_LGKM0() asm volatile("s_waitcnt lgkmcnt(0)" ::: "memory")
#define FENCE() asm volatile("" ::: "memory")
#define UNLIKELY(c) __builtin_expect(!!(c), 0)
#define BARRIER() do { FENCE(); __builtin_amdgcn_s_barrier(); FENCE(); } while (0)

__device__ __forceinline__ void flag_store(char* p, unsigned v) {
  __hip_atomic_store((GAS unsigned*)p, v, __ATOMIC_RELAXED, __HIP_MEMORY_SCOPE_SYSTEM);
}
__device__ __forceinline__ void flag_store2(char* p, unsigned lo, unsigned hi) {
  __hip_atomic_store((GAS unsigned long long*)p, (unsigned long long)lo | ((unsigned long long)hi << 32), __ATOMIC_RELAXED, __HIP_MEMORY_SCOPE_SYSTEM);
}
__device__ __forceinline__ unsigned draw(char* p) {
  unsigned z = 0;
  asm volatile("" : "+v"(z));
  p += z;
  return __hip_atomic_fetch_add((GAS unsigned*)p, 1u, __ATOMIC_RELAXED, __HIP_MEMORY_SCOPE_AGENT);
}
__device__ __forceinline__ unsigned uni(unsigned v) { return (unsigned)__builtin_amdgcn_readfirstlane((int)v); }
__device__ __forceinline__ unsigned lanes_below(unsigned long long b) {
  return __builtin_amdgcn_mbcnt_hi((unsigned)(b >> 32), __builtin_amdgcn_mbcnt_lo((unsigned)b, 0u));
}
__device__ __forceinline__ void advance(unsigned& pos, unsigned long long b) {
  asm("s_lshl2_add_u32 %0, %1, %0" : "+s"(pos) : "s"((unsigned)__builtin_popcountll(b)) : "scc");
}

#define VSTAMP(ws, slot, val) do {} while (0)
#define STAMP(ws, slot) do {} while (0)
#define PSTAMP(ws, cls, n) do {} while (0)
#define CSTAMP(ws, slot) do {} while (0)

__device__ __forceinline__ void set_timeout(char* myws, unsigned call) {
  __hip_atomic_store((GAS unsigned*)(myws + kTmo), call, __ATOMIC_RELAXED, __HIP_MEMORY_SCOPE_AGENT);
}
__device__ __forceinline__ unsigned peer_of(unsigned r, unsigned i) { return (r & 4u) + ((r + i + 1u) & 3u); }
__device__ __forceinline__ unsigned panel_row(unsigned ap, unsigned rh, unsigned rank) {
  const unsigned og = (rank & 4u) ^ 4u;
  if (ap < 8u) return rank * 1024u + ap * 128u + rh * 64u;
  if (ap < 23u) { const unsigned i = (ap - 8u) / 5u, s = (ap - 8u) - 5u * i; return peer_of(rank, i) * 1024u + s * 128u + rh * 64u; }
  if (ap < 26u) return peer_of(rank, ap - 23u) * 1024u + 640u;
  if (ap < 34u) return (og + ((ap - 26u) >> 1)) * 1024u + 768u + (ap & 1u) * 128u + rh * 64u;
  return (og + 2u * (ap - 34u) + rh) * 1024u + 704u;
}
__device__ __forceinline__ unsigned park_pos(unsigned ap, unsigned rh, unsigned j) {
  if (ap < 8u) return 2u * ap + rh < 11u ? 2u * ap + rh : 99u;
  if (ap < 23u) {
    const unsigned i = (ap - 8u) / 5u, s = (ap - 8u) - 5u * i;
    return s < 2u ? 11u + 4u * i + 2u * s + rh : s < 4u ? 40u + 4u * i + 2u * s + rh : 26u + 2u * i + rh;
  }
  const unsigned src = ap < 34u ? (ap - 26u) >> 1 : 2u * (ap - 34u) + rh;
  if (src == j) return 99u;
  const unsigned e = src - (src > j ? 1u : 0u);
  return ap < 34u ? 32u + 4u * e + 2u * (ap & 1u) + rh : 23u + e;
}
__device__ __forceinline__ unsigned park_row(unsigned hp, unsigned src, unsigned rank) {
  const unsigned q = rank & 3u;
  if (hp < 11u) return src * 1024u + hp * 64u;
  if (hp < 23u) return peer_of(src, (hp - 11u) >> 2) * 1024u + ((hp - 11u) & 3u) * 64u;
  if (hp < 26u) { const unsigned e = hp - 23u; return ((rank & 4u) + e + (e >= q ? 1u : 0u)) * 1024u + 704u; }
  if (hp < 32u) return peer_of(src, (hp - 26u) >> 1) * 1024u + 512u + (hp & 1u) * 64u;
  if (hp < 44u) { const unsigned e = (hp - 32u) >> 2; return ((rank & 4u) + e + (e >= q ? 1u : 0u)) * 1024u + 768u + (hp & 3u) * 64u; }
  return peer_of(src, (hp - 44u) >> 2) * 1024u + 256u + (hp & 3u) * 64u;
}

}

__global__ void __launch_bounds__(512) __attribute__((amdgpu_waves_per_eu(2, 2)))
ag_gemm_relu_kernel(const Params p) {
  extern __shared__ __attribute__((aligned(16))) char smem_raw[];
  LAS char* const lds = (LAS char*)smem_raw;
  LAS volatile unsigned* const misc = (LAS volatile unsigned*)(lds + kLdsMisc);

  const int tid = threadIdx.x;
  const int lane = tid & 63;
  const int wid = __builtin_amdgcn_readfirstlane(tid >> 6);
  const int bid = blockIdx.x;
  const unsigned rank = (unsigned)p.rank & 7u;
  const unsigned q = rank & 3u, og = (rank & 4u) ^ 4u;
  char* const myws = p.ws[rank];
  const unsigned l16 = (unsigned)lane * 16u;
  const __amdgpu_buffer_rsrc_t rws = mk_rsrc(myws, kWsNeed);
  const int fr = lane & 15, fq = lane >> 4;

  STAMP(myws, wid == 0 ? 0 : 31);

  const unsigned lx = (((unsigned)lane >> 2) * 8192u + 4u * (unsigned)((lane & 3) ^ ((-(lane >> 4)) & 3))) * 4u;
  const unsigned lw = (4u * ((unsigned)lane >> 4) * 1024u + 4u * ((unsigned)lane & 15u)) * 4u;
  const __amdgpu_buffer_rsrc_t rx = mk_rsrc(p.x, kInBytes);
  const __amdgpu_buffer_rsrc_t rw = mk_rsrc(p.w, kInBytes);
  const unsigned pw = (unsigned)wid >= WPM ? (unsigned)wid - WPM : (unsigned)wid;
  const unsigned gw = (unsigned)bid * WPM + pw;
  const unsigned sid = gw % 14u, k = gw / 14u;
  const bool cross = sid >= 6u;
  const unsigned c = cross ? (sid - 6u) >> 1 : sid >> 1, h = sid & 1u;
  const unsigned sr = (cross ? og + c : peer_of(rank, c)) & 7u;
  char* const sws = p.ws[sr];
  const __amdgpu_buffer_rsrc_t rs = mk_rsrc(sws, kWsNeed);
  LAS volatile unsigned* const hand = misc + 4u + 4u * pw;
  const unsigned buf0 = pw * 3u * kPBuf;
  if (bid < kNMove && (unsigned)wid >= WPM && (unsigned)wid < 2u * WPM && gw < 14u * NWS) {
    unsigned call = 0;
    auto push = [&](auto wconst) {
      constexpr bool W = decltype(wconst)::value;
      f32x4 pv[PU][2][8];
#pragma unroll
      for (unsigned u = 0; u < PU; ++u)
#pragma unroll
        for (int i = 0; i < 16; ++i) {
          const unsigned m = u * NWS + k, kk = (m >> 1) * 64u + h * 32u;
          if (W) pv[u][i >> 3][i & 7] = ld16<0>(rw, lw + (i >> 3) * 256u, (kk * 1024u + 256u * c + (m & 1u) * 128u) * 4u + ((i & 3) + 16 * ((i >> 2) & 1)) * 4096u);
          else pv[u][i >> 3][i & 7] = ld16<0>(rx, lx + (i & 1) * 64u, ((m & 1u) * 128u * 8192u + kk) * 4u + (i >> 1) * 524288u);
        }
      __syncthreads();
      call = uni(misc[0]) + 1u;
      const unsigned base = W ? kBOff + (call & 1u) * kBSize + (8u + 2u * q) * kPanel : kAOff + (call & 1u) * kASize + (18u - 5u * c) * kPanel;
      char* const fl = sws + kFlags + (W ? 44u + 2u * q : 18u - 5u * c) * 1024u + h * 4u;
#pragma unroll
      for (unsigned j = 0; j < 2u * PU; ++j) {
        const unsigned u = j >> 1, r = j & 1u, m = u * NWS + k;
        f16x8 hv[4];
#pragma unroll
        for (int i = 0; i < 4; ++i) hv[i] = W ? cvt_col(pv[u][r], i) : cvt8(pv[u][r][2 * i], pv[u][r][2 * i + 1]);
        asm volatile("" :: "v"(hv[0]), "v"(hv[1]), "v"(hv[2]), "v"(hv[3]));
        if (j >= PUW) WAIT_VM(4u * (PUW - 1u));
        if (j <= 2u) STAMP(myws, 8 + wid + 3 * j);
        if (u > 0u && r == PUW - 1u && lane == 0) flag_store(fl + ((m - NWS) & 1u) * 1024u + ((m - NWS) >> 1) * 8u, call);
#pragma unroll
        for (int i = 0; i < 4; ++i) st16<kAuxSys>(hv[i], rs, l16 + i * 1024u, base + (m & 1u) * kPanel + (m >> 1) * kBlk + h * 8192u + r * 4096u);
        FENCE();
      }
      WAIT_VM(0);
      const unsigned m = (PU - 1u) * NWS + k;
      if (lane == 0) flag_store(fl + (m & 1u) * 1024u + (m >> 1) * 8u, call);
    };
    if (cross) push(std::true_type{}); else push(std::false_type{});
    STAMP(myws, 4 + wid);
    if (!cross) return;
    const __amdgpu_buffer_rsrc_t ro = mk_rsrc(p.out[rank], kInBytes);
    const unsigned vo = (unsigned)fq * 16384u + (unsigned)fr * 16u;
    const unsigned long long t0 = __builtin_amdgcn_s_memrealtime();
    for (unsigned b = 0;; b = b == 2u ? 0u : b + 1u) {
      unsigned hv;
      while ((hv = uni(hand[b])) == 0u) {
        if (__builtin_amdgcn_s_memrealtime() - t0 > 2u * kDeadline) { set_timeout(myws, call); return; }
        __builtin_amdgcn_s_sleep(XSLEEP);
      }
      FENCE();
      if (hv == ~0u) break;
      const unsigned oa = park_row(hv >> 4, sr, rank) * 4096u + (256u * c + (hv & 12u) * 16u) * 4u;
      unsigned pos = (unsigned)(size_t)lds + buf0 + b * kPBuf, val[64];
      u32x2 mk = {~0u, ~0u};
      if ((hv & 2u) == 0u) mk = *(LAS u32x2*)(size_t)(pos + lane * 8);
      const unsigned fill = (unsigned)(size_t)lds + kLdsMisc + 64u + (hv & 2u) * 2u;
      pos += 512u;
#pragma unroll
      for (int i = 0; i < 64; ++i) {
        const bool keep = (int)(mk[i >> 5] << (i & 31)) >= 0;
        const unsigned long long bl = __ballot(keep);
        unsigned a = lanes_below(bl) * 4u + pos;
        asm volatile("" : "+v"(a));
        val[i] = *(LAS unsigned*)(size_t)(keep ? a : fill);
        advance(pos, bl);
      }
      WAIT_LGKM0();
      if (lane == 0) hand[b] = 0u;
      FENCE();
#pragma unroll
      for (int i = 0; i < 16; ++i)
        __builtin_amdgcn_raw_buffer_store_b128(u32x4{val[4 * i], val[4 * i + 1], val[4 * i + 2], val[4 * i + 3]}, ro, vo, oa + (i >> 2) * 65536u + (i & 3) * 4096u, kAuxSys);
    }
    STAMP(myws, 1 + wid);
    return;
  }
  const bool isconv = bid >= kNMove + 8 * CVFROM;
  unsigned tk = 0;
  if (tid == 0) tk = draw(myws + kTicket + (unsigned)bid * 32u);
  FENCE();
  f32x4 cv[8];
  auto item = [&](unsigned n, unsigned& ktg, unsigned& sub) {
    if (n < 5u * CVLEAD) { ktg = n / 5u; sub = n - 5u * ktg; sub += sub ? 3u : 0u; }
    else if (n < 8u * CVLEAD) { ktg = (n - 5u * CVLEAD) / 3u; sub = n - 5u * CVLEAD - 3u * ktg + 1u; }
    else { ktg = n >> 3; sub = n & 7u; }
  };
  auto loads = [&](unsigned n) {
    unsigned ktg, sub;
    item(n, ktg, sub);
    if (sub < 4u) {
      const unsigned so = ((256u * sub + 32u * wid) * 8192u + ktg * 64u) * 4u;
#pragma unroll
      for (int s = 0; s < 2; ++s) {
        cv[4 * s + 0] = ld16<0>(rx, lx, so + s * 524288u); cv[4 * s + 1] = ld16<0>(rx, lx + 64u, so + s * 524288u);
        cv[4 * s + 2] = ld16<0>(rx, lx + 128u, so + s * 524288u); cv[4 * s + 3] = ld16<0>(rx, lx + 192u, so + s * 524288u);
      }
    } else {
      const unsigned so = ((ktg * 64u + 32u * (wid & 1)) * 1024u + 256u * (sub - 4u) + 64u * (wid >> 1)) * 4u;
#pragma unroll
      for (int i = 0; i < 8; ++i) cv[i] = ld16<0>(rw, lw, so + ((i & 3) + 16 * (i >> 2)) * 4096u);
    }
  };
  if (bid >= kNMove + 8 * CVFROM + CVFIRST) __builtin_amdgcn_s_sleep(CVDELAY);
  if (isconv) loads((unsigned)(bid - kNMove - 8 * CVFROM));
  FENCE();
  if (tid == 0) {
    misc[0] = tk;
    misc[3] = 0u;
  }
  if (tid >= 4 && tid < 18) misc[tid] = tid == 17 ? 0x7fc00000u : 0u;
  __syncthreads();
  const unsigned call = uni(misc[0]) + 1u;
  const unsigned par = call & 1u;
  const unsigned abase = kAOff + par * kASize, bbase = kBOff + par * kBSize, obuf = kOOff + par * kOSize;

  if (bid < kNMove) {
    if (wid == 7) {
      if (bid == 0 && lane == 0) {
        const unsigned target = call * kDonePerCall;
        const unsigned long long t0 = __builtin_amdgcn_s_memrealtime();
        bool failed = false;
        while ((int)(__hip_atomic_load((GAS unsigned*)(myws + kDone), __ATOMIC_RELAXED, __HIP_MEMORY_SCOPE_SYSTEM) - target) < 0) {
          if (__hip_atomic_load((GAS unsigned*)(myws + kTmo), __ATOMIC_RELAXED, __HIP_MEMORY_SCOPE_AGENT) == call ||
              __builtin_amdgcn_s_memrealtime() - t0 > kDeadline) {
            set_timeout(myws, call);
            failed = true;
            break;
          }
          __builtin_amdgcn_s_sleep(2);
        }
        __builtin_amdgcn_fence(__ATOMIC_ACQUIRE, "");
        if (failed) __hip_atomic_store((GAS unsigned*)(p.out[rank] + (size_t)rank * 1024u * 1024u), 0x7fc00000u, __ATOMIC_RELAXED, __HIP_MEMORY_SCOPE_SYSTEM);
        STAMP(myws, 10);
      }
      return;
    }
    if ((unsigned)wid >= WPM || gw >= 14u * NWS) return;
    const unsigned total = cross ? kRowsC : kRowsI;
    const unsigned nplain = cross ? 576u : kRowsI;
    const unsigned qbase = (call - 1u) * (total - PU * NWS) - (PU + 1u) * NWS;
    bool failed = false, all = false;
    unsigned cgrp = 99u, okf = 0, okt = 0, fv = 0, wbase = 0x10000u;
    u32x2 fw = {0u, 0u};
    unsigned sa = 0, da = 0, fa = 0, pend = 0;
    unsigned s2 = 4096u, d2 = 4096u;
    u32x4 v[4], w[4];
    auto raise = [&]() {
      if (pend != 0u && lane == 0) flag_store(myws + pend, call);
      pend = 0u;
    };
    auto fetch = [&](u32x4 (&x)[4], unsigned so) {
#pragma unroll
      for (int i = 0; i < 4; ++i) x[i] = __builtin_amdgcn_raw_buffer_load_b128(rs, l16 + i * 1024u, so, kAuxSys);
      FENCE();
    };
    auto put = [&](const u32x4 (&x)[4], unsigned so) {
#pragma unroll
      for (int i = 0; i < 4; ++i) __builtin_amdgcn_raw_buffer_store_b128(x[i], rws, l16 + i * 1024u, so, kAuxDev);
      FENCE();
    };
    auto setup = [&](unsigned m) {
      unsigned sp, kt, dp, db, dh = h;
      unsigned so = h * 8192u, dof = h * 8192u;
      s2 = 4096u; d2 = 4096u;
      if (!cross) {
        if (m < 256u) { sp = m & 1u; kt = m >> 1; dp = 8u + 5u * c + sp; db = kt; }
        else if (m < 384u) { sp = 4u; kt = m - 256u; dp = 12u + 5u * c; db = kt; }
        else if (m < 640u) { sp = 2u + (m & 1u); kt = (m - 384u) >> 1; dp = 8u + 5u * c + sp; db = kt; }
        else { sp = 5u; db = m - 640u; kt = 2u * db + h; so = 0u; s2 = 8192u; dp = 23u + c; }
      } else {
        if (m < 256u) { sp = 36u + 2u * q + (m & 1u); kt = m >> 1; dp = 44u + 2u * c + (m & 1u); db = kt; }
        else {
          const unsigned g = ((m - 256u) * 52429u) >> 18, r = m - 256u - 5u * g;
          if (r == 2u) { sp = 5u; kt = 2u * g + h; so = 4096u; s2 = 8192u; dp = 34u + (c >> 1); db = kt; dh = c & 1u; dof = dh * 4096u; d2 = 8192u; }
          else { sp = 6u + (r % 3u); kt = 2u * g + (r > 2u ? 1u : 0u); dp = 20u + 2u * c + sp; db = kt; }
        }
      }
      const unsigned sub = sp < 36u ? sp >> 1 : (sp - 28u) >> 1;
      auto ask = [&](unsigned k0) {
        const unsigned off = lane < 62 ? kSrc + (sr * 8u + sub) * 512u + min(k0 + (unsigned)lane, 127u) * 4u : lane == 62 ? kAll + sr * 4u : kTmo;
        asm volatile("buffer_load_dword %0, %1, %2, 0 offen sc0 sc1" : "=v"(fv) : "v"(off), "s"(rws) : "memory");
      };
      auto look = [&]() {
        asm volatile("" : "+v"(fv) :: "memory");
        const unsigned long long b = __ballot(lane < 63 ? (int)(fv - call) >= 0 : fv == call);
        if ((b >> 62) != 0ull) { failed = (b >> 63) != 0ull; all = true; }
        okt += (unsigned)__builtin_ctzll(~b);
      };
      if (!all) {
        if (cgrp != 99u) look();
        if (!all && !(sub == cgrp && kt >= okf && kt < okt)) {
          WAIT_VM(0);
          raise();
          cgrp = sub;
          okf = kt;
          const unsigned long long t0 = __builtin_amdgcn_s_memrealtime();
          for (;;) {
            okt = kt;
            ask(kt);
            WAIT_VM(0);
            look();
            if (all || okt != kt) break;
            if (__builtin_amdgcn_s_memrealtime() - t0 > kDeadline) { set_timeout(myws, call); failed = true; all = true; break; }
            __builtin_amdgcn_s_sleep(MSLEEP);
          }
        }
        if (!all) ask(okt);
      }
      sa = (sp < 36u ? abase + sp * kPanel : bbase + (sp - 36u) * kPanel) + kt * kBlk + so;
      da = (dp < 36u ? abase + dp * kPanel : bbase + (dp - 36u) * kPanel) + db * kBlk + dof;
      fa = kFlags + dp * 1024u + db * 8u + dh * 4u;
      fetch(v, sa);
    };
    unsigned m = PU * NWS + k;
    setup(m);
    STAMP(myws, 20 + wid);
    do {
      unsigned dv = 0;
      if (lane == 0) dv = draw(myws + kQueue + sid * 128u);
      FENCE();
      put(v, da);
      raise();
      fetch(w, sa + s2);
      put(w, da + d2);
      pend = fa;
      if (h == 0u) { PSTAMP(myws, sid >> 1, m + 1u); if (m < 128u) PSTAMP(myws, 8u + (sid >> 1), (m + 1u) * 4u); }
      m = uni(dv) - qbase;
      if (m < nplain) setup(m);
    } while (m < nplain);
    WAIT_VM(0);
    raise();
    if (cross) {
      unsigned hval = 0;
      const unsigned fo = lane < 32 ? kOFlags + par * 8192u + c * 2048u + h * 8u + (unsigned)lane * 16u : kTmo;
      auto stale = [&](unsigned o) {
        return o - wbase >= 32u || (int)((unsigned)__builtin_amdgcn_readlane((int)fw[0], (int)(o - wbase)) - call) < 0;
      };
      for (unsigned b = 0, pb = 0;; pb = b, b = b == 2u ? 0u : b + 1u) {
        const unsigned o = m - 576u, s = 2u * o + h;
        unsigned cnt = 2048u, dv = 0;
        if (m >= total || stale(o)) {
          WAIT_VM(0);
          if (hval != 0u && lane == 0) hand[pb] = hval;
          hval = 0u;
          const unsigned long long t0 = __builtin_amdgcn_s_memrealtime();
          wbase = o;
          while (!failed && m < total) {
            fw = __builtin_bit_cast(u32x2, __hip_atomic_load((GAS unsigned long long*)(myws + fo + (lane < 32 ? o * 16u : 0u)), __ATOMIC_RELAXED, __HIP_MEMORY_SCOPE_SYSTEM));
            const unsigned long long bl = __ballot(lane < 32 ? (int)(fw[0] - call) >= 0 : fw[0] == call);
            if ((bl >> 32) != 0ull) { failed = true; break; }
            if (bl & 1ull) break;
            if (__builtin_amdgcn_s_memrealtime() - t0 > kDeadline) { set_timeout(myws, call); failed = true; }
            __builtin_amdgcn_s_sleep(MSLEEP);
          }
        }
        cnt = failed ? 0u : min((unsigned)__builtin_amdgcn_readlane((int)fw[1], (int)((o - wbase) & 31u)), 4096u);
        if (UNLIKELY(uni(hand[b]) != 0u)) {
          const unsigned long long t0 = __builtin_amdgcn_s_memrealtime();
          while (uni(hand[b]) != 0u)
            if (__builtin_amdgcn_s_memrealtime() - t0 > kDeadline) { set_timeout(myws, call); return; }
        }
        FENCE();
        if (m >= total) {
          if (lane == 0) hand[b] = ~0u;
          break;
        }
        const unsigned bytes = 512u + 4u * cnt, sb = obuf + (q * 224u + s) * kPiece, nbuf = buf0 + b * kPBuf;
        auto part = [&](unsigned f0, unsigned f1) {
          for (unsigned f = f0; f < f1; f += 1024u) {
            WAIT_VM(PWIN - 1u);
            asm volatile("s_mov_b32 m0, %0\n\ts_nop 0\n\tbuffer_load_dwordx4 %1, %2, %3 offen sc0 sc1 lds"
                         :: "s"(nbuf + f), "v"(f + l16 < bytes ? l16 : kOob), "s"(rs), "s"(sb + f) : "memory");
          }
        };
        asm volatile("s_mov_b64 exec, 1\n\tglobal_atomic_add %0, %1, %2, %3 sc0\n\ts_mov_b64 exec, -1"
                     : "=v"(dv) : "v"(kQueue + sid * 128u), "v"(1u), "s"(myws) : "memory");
        part(0u, min(bytes, (PWIN - 1u) * 1024u));
        if (bytes <= (PWIN - 2u) * 1024u) WAIT_VM(0);
        WAIT_VM(PWIN - 1u);
        if (hval != 0u && lane == 0) hand[pb] = hval;
        hval = 4u * s + (failed ? 3u : 1u);
        if (h == 0u) PSTAMP(myws, sid >> 1, m + 1u);
        asm volatile("" : "+v"(dv));
        m = uni(dv) - qbase;
        if (m < total && stale(m - 576u)) {
          wbase = m - 576u;
          asm volatile("buffer_load_dwordx2 %0, %1, %2, 0 offen sc0 sc1" : "=v"(fw) : "v"(fo + (lane < 32 ? wbase * 16u : 0u)), "s"(rws) : "memory");
          part((PWIN - 1u) * 1024u, bytes);
          if (bytes <= (2u * PWIN - 3u) * 1024u) WAIT_VM(0);
          WAIT_VM(PWIN - 1u);
          asm volatile("" : "+v"(fw) :: "memory");
        } else
          part((PWIN - 1u) * 1024u, bytes);
      }
    }
    STAMP(myws, 1 + wid);
    return;
  }

  {
    constexpr unsigned kNConv = 8u * (26u - CVFROM);
    char* const pws = p.ws[lane & 7];
    const unsigned mine = 15u << (rank & 4u);
    STAMP(myws, wid == 0 ? 27 : 31);
    for (unsigned n = isconv ? (unsigned)(bid - kNMove - 8 * CVFROM) : 1024u; n < 1024u; n += kNConv) {
      unsigned ktg, sub;
      item(n, ktg, sub);
      if (sub < 4u) {
        const unsigned so = abase + (2u * sub + (wid >> 2)) * kPanel + ktg * kBlk + (wid & 3) * 2048u;
#pragma unroll
        for (int i = 0; i < 4; ++i) st16<CVAUX>(cvt8(cv[2 * i], cv[2 * i + 1]), rws, l16 + (i >> 1) * 1024u, so + (i & 1) * 8192u);
      } else {
        const unsigned so = bbase + (2u * (sub - 4u) + (wid >> 2)) * kPanel + ktg * kBlk + ((wid >> 1) & 1) * 4096u + (wid & 1) * 8192u;
#pragma unroll
        for (int i = 0; i < 4; ++i) st16<CVAUX>(cvt_col(cv, i), rws, l16 + i * 1024u, so);
      }
      FENCE();
      const bool ahead = n >= kNConv && n + kNConv < 1024u;
      if (ahead) { loads(n + kNConv); WAIT_VM(8); } else { WAIT_VM(0); }
      if (n < kNConv) STAMP(myws, wid == 0 ? 30 : 31);
      BARRIER();
      if (n < 2u * kNConv) STAMP(myws, wid == 0 ? (n < kNConv ? 28 : 29) : 31);
      const unsigned to = sub < 2u ? mine : sub == 2u ? 255u : sub == 3u ? mine ^ 255u : 1u << (og + sub - 4u);
      if (tid < 2) flag_store2(myws + kFlags + ((sub < 4u ? 2u * sub : 28u + 2u * sub) + (unsigned)tid) * 1024u + ktg * 8u, call, call);
      else if (wid == 1 && lane < 8 && (((to & ~(1u << rank)) >> lane) & 1u)) flag_store(pws + kSrc + rank * 4096u + sub * 512u + ktg * 4u, call);
      FENCE();
      if (n < kNConv) loads(n + kNConv);
    }
    if (isconv && wid == 7) {
      unsigned dv = 0;
      if (lane == 0) dv = draw(myws + kCvt);
      if (uni(dv) == call * kNConv - 1u && lane < 8) flag_store(pws + kAll + rank * 4u, call);
    }
  }
  STAMP(myws, wid == 0 ? 1 : 31);

  const uint32_t* const tl = p.tiles + 3 * (bid - kNMove);
  const int ntiles = (int)(tl[0] >> 28);
  if (wid == 7) return;

  if (wid >= 4) {
    const unsigned lw = (unsigned)wid - 4u;
    unsigned total = 0;
    for (int ti = 0; ti < ntiles; ++ti) total += ((tl[ti] >> 10) & 3u) == 2u ? 64u : 256u;
    unsigned a = 0, aend = 0, so = lw * 8192u, src = 0, step = 0, g2 = 0, v2 = l16, idle = 0;
    unsigned ok = 0, f0 = 0, f1 = 0, fb = 0, fsh = 0, fmax = 0, fv = 0;
    unsigned rdy = 0, askat = 0, lookat = ~0u;
    int ta = 0, tf = 0;
    auto next_flags = [&]() {
      while (tf < ntiles) {
        const unsigned code = tl[tf++];
        const unsigned ap = code & 63u, bp = (code >> 6) & 15u, shape = (code >> 10) & 3u;
        f0 = f1;
        f1 += shape == 2u ? 64u : 256u;
        if (shape == 3u && lw == 2u) { ok = f1; continue; }
        const unsigned grp = lw == 0u ? ap : shape == 0u ? (lw == 1u ? ap + 1u : 36u + bp) : shape == 1u ? 35u + bp + lw : 36u + bp;
        fb = kFlags + grp * 1024u;
        fsh = shape != 2u ? 0u : lw == 0u ? 1u : 2u;
        fmax = fsh == 1u ? 127u : 255u;
        break;
      }
    };
    auto known = [&]() {
      rdy = min(ok, aend);
      unsigned from = max(ok, LTHR);
      asm volatile("" : "+s"(from));
      askat = ok >= total ? ~0u : from - LTHR;
    };
    auto ask = [&]() {
      const unsigned off = lane < 63 ? fb + min(((ok - f0) << fsh) + (unsigned)lane, fmax) * 4u : kTmo;
      asm volatile("buffer_load_dword %0, %1, %2, 0 offen sc0 sc1" : "=v"(fv) : "v"(off), "s"(rws) : "memory");
    };
    auto give_up = [&]() {
      ok = total;
      if (lane == 0) misc[3] = 1u;
      WAIT_LGKM0();
    };
    auto look = [&]() {
      asm volatile("" : "+v"(fv) :: "memory");
      const unsigned long long b = __ballot(lane < 63 ? (int)(fv - call) >= 0 : fv == call);
      if (UNLIKELY((b >> 63) != 0ull)) {
        give_up();
      } else {
        const unsigned run = (unsigned)__builtin_ctzll(~b);
        ok = min(ok + (fsh == 0u ? run & ~1u : run >> fsh), f1);
        if (UNLIKELY(ok == f1)) next_flags();
      }
      lookat = ~0u;
    };
    auto issue = [&](bool busy) {
      LAS char* const dl = lds + so;
      if (busy) {
        __builtin_amdgcn_raw_ptr_buffer_load_lds(rws, (LAS void*)dl, 16, l16, src, 0, kAuxDev);
        __builtin_amdgcn_raw_ptr_buffer_load_lds(rws, (LAS void*)dl, 16, l16, src, 1024, kAuxDev);
        __builtin_amdgcn_raw_ptr_buffer_load_lds(rws, (LAS void*)dl, 16, v2, src, 2048, kAuxDev);
        __builtin_amdgcn_raw_ptr_buffer_load_lds(rws, (LAS void*)dl, 16, v2, src, 3072, kAuxDev);
        __builtin_amdgcn_raw_ptr_buffer_load_lds(rws, (LAS void*)(dl + 4096), 16, l16, src + g2, 0, kAuxDev);
        __builtin_amdgcn_raw_ptr_buffer_load_lds(rws, (LAS void*)(dl + 4096), 16, l16, src + g2, 1024, kAuxDev);
        __builtin_amdgcn_raw_ptr_buffer_load_lds(rws, (LAS void*)(dl + 4096), 16, v2, src + g2, 2048, kAuxDev);
        __builtin_amdgcn_raw_ptr_buffer_load_lds(rws, (LAS void*)(dl + 4096), 16, v2, src + g2, 3072, kAuxDev);
      }
      FENCE();
      src += step;
      so = so >= kRing - kSlot ? so - (kRing - kSlot) : so + kSlot;
      ++a;
    };
    next_flags();
    known();
    for (unsigned j = 0; j < total; ++j) {
      if (a == j + 4u) {
        unsigned e = min(rdy + 1u, askat);
        asm volatile("" : "+s"(e));
        e = min(e, lookat);
        if (idle != 0u) {
          WAIT_VM(0);
          while (a + 1u < e) { issue(false); BARRIER(); }
        }
        while (a + 1u < e) {
          issue(true);
          WAIT_VM(32);
          BARRIER();
        }
        j = a - 4u;
      }
      unsigned lim = min(j + 5u, rdy);
      if (UNLIKELY(a >= lim)) {
        if (a == aend) {
          const unsigned code = tl[ta++];
          const unsigned ap = code & 63u, bp = (code >> 6) & 15u, shape = (code >> 10) & 3u;
          if (lw == 0u || (lw == 1u && shape == 0u)) src = abase + (ap + lw) * kPanel;
          else src = bbase + (bp + (shape == 1u ? lw - 1u : 0u)) * kPanel;
          step = 8192u; g2 = 4096u;
          idle = shape == 3u && lw == 2u ? 1u : 0u;
          if (shape == 2u) {
            g2 = 8192u;
            if (lw == 0u) { src += ((code >> 13) & 1u) * 2048u; step = kBlk; v2 = l16 + 2048u; }
            else { src += ((code >> 12) & 1u) * 4096u + (lw - 1u) * kBlk; step = 2u * kBlk; }
          }
          aend = ta == ntiles ? ~0u : aend + 256u;
          rdy = min(ok, aend);
        }
        if (a <= j && ok <= a) {
          WAIT_VM(0);
          if (lookat != ~0u) look();
          const unsigned long long t0 = __builtin_amdgcn_s_memrealtime();
          while (ok <= a) {
            ask();
            WAIT_VM(0);
            look();
            if (ok > a) break;
            if (__builtin_amdgcn_s_memrealtime() - t0 > kDeadline) { set_timeout(myws, call); give_up(); break; }
            __builtin_amdgcn_s_sleep(LSLEEP);
          }
          known();
        }
        lim = min(j + 5u, rdy);
      }
      if (a < lim) {
        issue(idle == 0u);
        if (UNLIKELY(a < lim)) issue(idle == 0u);
      }
      if (UNLIKELY(a >= askat)) { ask(); askat = ~0u; lookat = a + 4u; }
      const unsigned n = idle != 0u ? 0u : a - 1u - j;
      if (UNLIKELY(n < 4u)) {
        WAIT_VM(24);
        if (n < 3u) {
          WAIT_VM(16);
          if (n < 2u) {
            WAIT_VM(8);
            if (n < 1u) WAIT_VM(0);
          }
        }
      }
      WAIT_VM(32);
      if (UNLIKELY(j + 5u >= lookat)) { look(); known(); }
      BARRIER();
    }
    return;
  }

  {
    const unsigned slot = (unsigned)(4 * fr + (fq ^ ((-(fr >> 2)) & 3))) * 16u;
    unsigned so = 0;
    for (int ti = 0; ti < ntiles; ++ti) {
      const unsigned code = tl[ti];
      const unsigned shape = (code >> 10) & 3u;
      const unsigned wh = (unsigned)wid >> 1, wl = (unsigned)wid & 1u;
      const unsigned ap = (code & 63u) + (shape == 0u ? wh : 0u), bp = ((code >> 6) & 15u) + (shape == 1u ? wh : 0u);
      const unsigned jo = (bp - 8u) >> 1;
      const unsigned owner = (bp < 8u ? rank : og + jo) & 7u;
      const unsigned col0 = bp < 8u ? bp * 128u : 256u * q + (bp & 1u) * 128u;
      const __amdgpu_buffer_rsrc_t ro = mk_rsrc(p.out[owner], kInBytes);
      const unsigned vo = (unsigned)fq * 16384u + (unsigned)fr * 16u;

      auto run = [&](auto mconst, const unsigned aoff, const unsigned boff) {
        constexpr int M = decltype(mconst)::value;
        constexpr int KS = M > 1 ? 1 : 4;
        constexpr int NS = M > 1 ? 256 : 64;
        constexpr int NA = M * KS, NB = 4 * KS;
        const unsigned va = aoff + slot, vb = boff + l16;
        f32x4 acc[M][4];
#pragma unroll
        for (int m = 0; m < M; ++m)
#pragma unroll
          for (int n = 0; n < 4; ++n) acc[m][n] = f32x4{0.f, 0.f, 0.f, 0.f};
        f16x8 fa[2][NA], fb[2][NB];
        auto rd = [&](f16x8 (&a)[NA], f16x8 (&b)[NB]) {
          LAS char* const pa = lds + (va + so);
          LAS char* const pb = lds + (vb + so);
#pragma unroll
          for (int i = 0; i < (NA > NB ? NA : NB); ++i) {
            if (i < NB) b[i] = *(LAS f16x8*)(pb + i * 1024);
            if (i < NA) a[i] = *(LAS f16x8*)(pa + i * (M > 1 ? 1024 : 2048));
          }
        };
        auto mm = [&](const f16x8 (&a)[NA], const f16x8 (&b)[NB]) {
#pragma unroll
          for (int ks = 0; ks < KS; ++ks)
#pragma unroll
            for (int m = 0; m < M; ++m)
#pragma unroll
              for (int n = 0; n < 4; ++n)
                acc[m][n] = __builtin_amdgcn_mfma_f32_16x16x32_f16(a[KS * m + ks], b[4 * ks + n], acc[m][n], 0, 0, 0);
        };
        BARRIER();
        STAMP(myws, wid == 0 ? 2 + 3 * ti : 31);
        CSTAMP(myws, wid == 0 ? 12 + 2 * ti : 31);
        rd(fa[0], fb[0]);
        __builtin_amdgcn_s_waitcnt(0xC07F);
        BARRIER();
        so = so == kRing - kSlot ? 0u : so + kSlot;
        if constexpr (M > 1) {
          unsigned ua = va + so, ub = vb + so;
          for (int t = 0; t < NS; t += 2) {
#pragma unroll
            for (int s = 0; s < 2; ++s) {
              __builtin_amdgcn_sched_barrier(0);
#pragma unroll
              for (int i = 0; i < 4 * M; ++i) {
                acc[i >> 2][i & 3] = __builtin_amdgcn_mfma_f32_16x16x32_f16(fa[s][i >> 2], fb[s][i & 3], acc[i >> 2][i & 3], 0, 0, 0);
                __builtin_amdgcn_sched_barrier(0);
                if (M == 8 && (i & 1) == 0 && i < 24) {
                  const int r = i >> 1;
                  if (r < 4) fb[s ^ 1][r] = *(LAS f16x8*)(lds + ub + r * 1024);
                  else fa[s ^ 1][r - 4] = *(LAS f16x8*)(lds + ua + (r - 4) * 1024);
                }
                if (M == 4 && i < 4) {
                  fb[s ^ 1][i] = *(LAS f16x8*)(lds + ub + i * 1024);
                  fa[s ^ 1][i] = *(LAS f16x8*)(lds + ua + i * 1024);
                }
                if (i == 4 * M - 10) { so = so == kRing - kSlot ? 0u : so + kSlot; asm volatile("" : "+s"(so)); }
                if (i == 4 * M - 6) { ua = va + so; asm volatile("" : "+v"(ua)); }
                if (i == 4 * M - 4) { ub = vb + so; asm volatile("" : "+v"(ub)); }
                if (i == 4 * M - 2) {
                  __builtin_amdgcn_s_waitcnt(0xC07F);
                  if (t < NS - 2) BARRIER();
                }
                __builtin_amdgcn_sched_barrier(0);
              }
            }
          }
          so = so == 0u ? kRing - kSlot : so - kSlot;
        } else {
          for (int t = 0; t < NS; t += 2) {
#pragma unroll
            for (int s = 0; s < 2; ++s) {
              __builtin_amdgcn_sched_barrier(0);
              rd(fa[s ^ 1], fb[s ^ 1]);
              mm(fa[s], fb[s]);
#pragma unroll
              for (int i = 0; i < 4; ++i) {
                __builtin_amdgcn_sched_group_barrier(0x100, 2, 0);
                __builtin_amdgcn_sched_group_barrier(0x008, 1, 0);
              }
#pragma unroll
              for (int i = 0; i < 12; ++i) {
                __builtin_amdgcn_sched_group_barrier(0x100, 1, 0);
                __builtin_amdgcn_sched_group_barrier(0x008, 1, 0);
              }
              __builtin_amdgcn_sched_barrier(0);
              so = so == kRing - kSlot ? 0u : so + kSlot;
              __builtin_amdgcn_s_waitcnt(0xC07F);
              if (t < NS - 2) BARRIER();
            }
          }
        }
        STAMP(myws, wid == 0 ? 3 + 3 * ti : 31);
        CSTAMP(myws, wid == 0 ? 13 + 2 * ti : 31);
        const int lo = misc[3] != 0u ? 0x7fc00000 : 0;
        if (M > 1 && bp >= 8u) {
          const __amdgpu_buffer_rsrc_t rw4 = __builtin_amdgcn_make_buffer_rsrc((void*)myws, 4, kWsNeed / 4u, 0x00020000);
          if (UNLIKELY(lo != 0)) {
#pragma unroll
            for (int m = 0; m < M; ++m)
#pragma unroll
              for (int n = 0; n < 4; ++n) acc[m][n] = __builtin_bit_cast(f32x4, u32x4{0x7fc00000u, 0x7fc00000u, 0x7fc00000u, 0x7fc00000u});
          }
#pragma unroll
          for (int rh = 0; rh < M / 4; ++rh) {
            const unsigned hp = park_pos(ap, M == 8 ? (unsigned)rh : wh, jo);
            if (hp < 56u) {
              const unsigned s = hp * 4u + (bp & 1u) * 2u + wl;
              const unsigned pb0 = obuf + (jo * 224u + s) * kPiece;
              unsigned pos = pb0 + 512u;
              u32x2 mk = {0u, 0u};
#pragma unroll
              for (int m = 0; m < 4; ++m)
#pragma unroll
                for (int j = 0; j < 4; ++j)
#pragma unroll
                  for (int n = 0; n < 4; ++n) {
                    const float f = acc[4 * rh + m][n][j];
                    const unsigned iv = __float_as_uint(f);
                    const unsigned long long b = __ballot((int)iv >= 0);
                    const unsigned idx = lanes_below(b) | (iv & kOob);
                    asm volatile("buffer_store_dword %0, %1, %2, %3 idxen sc0 sc1" :: "v"(iv), "v"(idx), "s"(rw4), "s"(pos) : "memory");
                    advance(pos, b);
                    mk[m >> 1] = __builtin_amdgcn_alignbit(mk[m >> 1], iv, 31);
                  }
              __builtin_amdgcn_raw_buffer_store_b64(mk, rws, (unsigned)lane * 8u, pb0, kAuxSys);
              WAIT_VM(0);
              if (lane == 0) flag_store2(p.ws[owner] + kOFlags + par * 8192u + q * 2048u + s * 8u, call, (pos - pb0 - 512u) >> 2);
            }
          }
        } else {
#pragma unroll
          for (int m = 0; m < M; ++m) {
            const unsigned ob = M > 1 ? (panel_row(ap, M == 8 ? (unsigned)(m >> 2) : wh, rank) + 16u * (m & 3)) * 4096u + (col0 + 64u * wl) * 4u
                                       : (panel_row(ap, 0u, rank) + 32u * ((code >> 13) & 1u) + 16u * wl) * 4096u + (col0 + 64u * ((code >> 12) & 1u)) * 4u;
#pragma unroll
            for (int j = 0; j < 4; ++j) {
              u32x4 o;
#pragma unroll
              for (int n = 0; n < 4; ++n) { const float f = acc[m][n][j]; o[n] = (unsigned)max(__float_as_int(f), lo); }
              __builtin_amdgcn_raw_buffer_store_b128(o, ro, vo, ob + j * 4096u, kAuxSys);
            }
          }
          if (bp >= 8u) {
            WAIT_VM(0);
            if (lane == 0) __hip_atomic_fetch_add((GAS unsigned*)(p.ws[owner] + kDone), 1u, __ATOMIC_RELAXED, __HIP_MEMORY_SCOPE_SYSTEM);
          }
        }
      };
      if (shape == 2u) {
        if (wid >= 2) return;
        run(std::integral_constant<int, 1>{}, wl * 1024u, 8192u);
      } else if (shape == 3u) {
        run(std::integral_constant<int, 4>{}, wh * 4096u, 8192u + wl * 4096u);
      } else {
        run(std::integral_constant<int, 8>{}, shape == 0u ? wh * 8192u : 0u, (shape == 0u ? 16384u : 8192u + wh * 8192u) + wl * 4096u);
      }
      STAMP(myws, wid == 0 ? 4 + 3 * ti : 31);
    }
  }
}

extern "C" void kernel_launch(int rank, int nranks, ncclComm_t comm, hipStream_t stream,
                              void* const* d_in, const int* in_sizes, int n_in,
                              void* d_out, int out_size,
                              const int* in_dtypes, int out_dtype,
                              void* const* d_out_all, void* const* d_ws_all, size_t ws_bytes) {
  (void)comm; (void)in_dtypes; (void)out_dtype; (void)d_out;
  if (nranks != 8 || (unsigned)rank >= 8u || n_in < 2 || ws_bytes < (size_t)kWsNeed ||
      in_sizes[0] != 8 * 1024 * 1024 || in_sizes[1] != 8 * 1024 * 1024 || out_size != 8 * 1024 * 1024) {
    fprintf(stderr, "kernel_launch: unexpected problem geometry\n");
    return;
  }
  Params p;
  p.x = (const float*)d_in[0];
  p.w = (const float*)d_in[1];
  for (int i = 0; i < 8; ++i) { p.ws[i] = (char*)d_ws_all[i]; p.out[i] = (float*)d_out_all[i]; }
  p.rank = rank;
  p.pad = 0;
  static const int xcol[8] = {0, 4, 2, 3, 1, 5, 6, 7};
  auto tile = [](int a, int b, int shape) { return (unsigned)(a | (b << 6) | (shape << 10)); };
  for (int ci = 0; ci < kNComp; ++ci) {
    const int x = ci & 7, s = ci >> 3, bq = 2 * (x ^ 4), j = x >> 1;
    unsigned t[3] = {0u, 0u, 0u};
    int n = 0;
    auto farb = [&](int e) { const int idx = 3 * (x & 1) + e; return 8 + idx + (idx >= 2 * j ? 2 : 0); };
    if (s < 12) {
      t[n++] = tile(x < 4 ? 2 * (s >> 2) : 8 + 5 * (s >> 2), 4 * (x & 3) + (s & 3), 0);
      t[n++] = tile(10 + 5 * (s >> 2) + ((s >> 1) & 1), bq + (s & 1), 3);
    } else if (s == 12) {
      t[n++] = tile(6, xcol[x], 0);
      t[n++] = tile(27 + 2 * j, farb(1), 3);
    } else if (s < 16) {
      t[n++] = tile(12 + 5 * (s - 13), bq, 1);
    } else if (s < 18) {
      t[n++] = tile(34 + (s - 16), bq, 1);
    } else if (s < 23) {
      t[n++] = tile(26 + 2 * j, s < 22 ? 4 * (x & 1) + s - 18 : farb(2), 0);
    } else {
      t[n++] = tile(26 + 2 * j + ((s - 23) & 1), farb((s - 23) >> 1), 3);
    }
    const int i = s < 12 ? s : s == 22 || s == SKIPA ? -1 : s - 12 - (s > SKIPA ? 1 : 0) - (s > 22 ? 1 : 0);
    if (i >= 0) t[n++] = tile(23 + (i >> 2), s < 12 ? x : 8 + x, 2) | ((unsigned)(i & 1) << 12) | ((unsigned)((i >> 1) & 1) << 13);
    p.tiles[3 * ci] = t[0] | ((unsigned)n << 28);
    p.tiles[3 * ci + 1] = t[1];
    p.tiles[3 * ci + 2] = t[2];
  }
  hipError_t e = hipFuncSetAttribute((const void*)ag_gemm_relu_kernel, hipFuncAttributeMaxDynamicSharedMemorySize, (int)kLdsBytes);
  if (e != hipSuccess) { fprintf(stderr, "hipFuncSetAttribute: %s\n", hipGetErrorString(e)); return; }
  hipLaunchKernelGGL(ag_gemm_relu_kernel, dim3(kNWG), dim3(512), kLdsBytes, stream, p);
  e = hipGetLastError();
  if (e != hipSuccess) fprintf(stderr, "launch: %s\n", hipGetErrorString(e));
}
```

```cpp
#include <hip/hip_runtime.h>
#include <rccl/rccl.h>
#include <cstdio>
#include <cstdint>
#include <type_traits>

typedef _Float16 f16x8 __attribute__((ext_vector_type(8)));
typedef float f32x4 __attribute__((ext_vector_type(4)));
typedef unsigned u32x4 __attribute__((ext_vector_type(4)));
typedef unsigned u32x2 __attribute__((ext_vector_type(2)));
#define LAS __attribute__((address_space(3)))
#define GAS __attribute__((address_space(1)))

#define NWS 9u
#define WPM 3u
#define PU 1u
#define PUW 1u
#define CVDELAY 24
#define CVFIRST 60
#define CVLEAD 40u
#define XSLEEP 4
#define CVFROM 13
#define PWIN 4u
#define CVAUX kAuxSys
#define MSLEEP 4
#define LSLEEP 2
#define WLIM 80u
#define SKIPA 21
#define ALANES 63
#define PTMO kOob
#define LTHR 24u

#define LDCOND(j) ((j) + 40u >= total)
namespace {

constexpr unsigned kBlk = 16384u;
constexpr unsigned kPanel = 2u << 20;
constexpr unsigned kAOff = 1u << 20;
constexpr unsigned kASize = 36u * kPanel;
constexpr unsigned kBOff = kAOff + 2u * kASize;
constexpr unsigned kBSize = 16u * kPanel;
constexpr unsigned kPiece = 16896u;
constexpr unsigned kOOff = kBOff + 2u * kBSize;
constexpr unsigned kOSize = 4u * 224u * kPiece;
constexpr unsigned kWsNeed = kOOff + 2u * kOSize;
constexpr unsigned kTmo = 128, kDone = 256, kCvt = 384, kAll = 512, kQueue = 1024;
constexpr unsigned kTicket = 4096;
constexpr unsigned kFlags = 16384;
constexpr unsigned kOFlags = 81920;
constexpr unsigned kSrc = 98304;
constexpr unsigned kLast = 131072, kRep = 4352;
constexpr unsigned kInBytes = 32u << 20;
constexpr int kNMove = 48, kNComp = 208, kNWG = 256;
constexpr unsigned kRowsI = 704u, kRowsC = 688u;
constexpr unsigned kDonePerCall = 192u;
constexpr unsigned long long kDeadline = 50000000ull;
constexpr unsigned kSlot = 24576u, kRing = 6u * kSlot;
constexpr unsigned kPBuf = 17408u;
constexpr unsigned kLdsMisc = 9u * kPBuf, kLdsBytes = kLdsMisc + 128u;
static_assert(kLdsMisc >= kRing, "");
constexpr unsigned kOob = 0x80000000u;
constexpr int kAuxSys = 17;
constexpr int kAuxDev = 16;

struct Params {
  const float* x;
  const float* w;
  char* ws[8];
  float* out[8];
  int rank;
  int pad;
  uint32_t tiles[3 * kNComp];
};

__device__ __forceinline__ __amdgpu_buffer_rsrc_t mk_rsrc(const void* p, unsigned bytes) {
  return __builtin_amdgcn_make_buffer_rsrc((void*)p, 0, bytes, 0x00020000);
}
template <int AUX>
__device__ __forceinline__ f32x4 ld16(__amdgpu_buffer_rsrc_t r, unsigned voff, unsigned soff) {
  return __builtin_bit_cast(f32x4, __builtin_amdgcn_raw_buffer_load_b128(r, voff, soff, AUX));
}
template <int AUX>
__device__ __forceinline__ void st16(f16x8 v, __amdgpu_buffer_rsrc_t r, unsigned voff, unsigned soff) {
  __builtin_amdgcn_raw_buffer_store_b128(__builtin_bit_cast(u32x4, v), r, voff, soff, AUX);
}
__device__ __forceinline__ f16x8 cvt8(f32x4 a, f32x4 b) {
  f16x8 h;
#pragma unroll
  for (int i = 0; i < 4; ++i) { h[i] = (_Float16)a[i]; h[4 + i] = (_Float16)b[i]; }
  return h;
}
__device__ __forceinline__ f16x8 cvt_col(const f32x4 (&v)[8], int i) {
  f16x8 h;
#pragma unroll
  for (int j = 0; j < 8; ++j) h[j] = (_Float16)v[j][i];
  return h;
}
#define WAIT_VM(n) asm volatile("s_waitcnt vmcnt(%0)" :: "n"(n) : "memory")
#define WAIT_LGKM0() asm volatile("s_waitcnt lgkmcnt(0)" ::: "memory")
#define FENCE() asm volatile("" ::: "memory")
#define UNLIKELY(c) __builtin_expect(!!(c), 0)
#define BARRIER() do { FENCE(); __builtin_amdgcn_s_barrier(); FENCE(); } while (0)

__device__ __forceinline__ void flag_store(char* p, unsigned v) {
  __hip_atomic_store((GAS unsigned*)p, v, __ATOMIC_RELAXED, __HIP_MEMORY_SCOPE_SYSTEM);
}
__device__ __forceinline__ void flag_store2(char* p, unsigned lo, unsigned hi) {
  __hip_atomic_store((GAS unsigned long long*)p, (unsigned long long)lo | ((unsigned long long)hi << 32), __ATOMIC_RELAXED, __HIP_MEMORY_SCOPE_SYSTEM);
}
__device__ __forceinline__ unsigned draw(char* p) {
  unsigned z = 0;
  asm volatile("" : "+v"(z));
  p += z;
  return __hip_atomic_fetch_add((GAS unsigned*)p, 1u, __ATOMIC_RELAXED, __HIP_MEMORY_SCOPE_AGENT);
}
__device__ __forceinline__ unsigned uni(unsigned v) { return (unsigned)__builtin_amdgcn_readfirstlane((int)v); }
__device__ __forceinline__ unsigned lanes_below(unsigned long long b) {
  return __builtin_amdgcn_mbcnt_hi((unsigned)(b >> 32), __builtin_amdgcn_mbcnt_lo((unsigned)b, 0u));
}
__device__ __forceinline__ void advance(unsigned& pos, unsigned long long b) {
  asm("s_lshl2_add_u32 %0, %1, %0" : "+s"(pos) : "s"((unsigned)__builtin_popcountll(b)) : "scc");
}

#define VSTAMP(ws, slot, val) do {} while (0)
#define STAMP(ws, slot) do {} while (0)
#define PSTAMP(ws, cls, n) do {} while (0)
#define CSTAMP(ws, slot) do {} while (0)

__device__ __forceinline__ void set_timeout(char* myws, unsigned call) {
  __hip_atomic_store((GAS unsigned*)(myws + kTmo), call, __ATOMIC_RELAXED, __HIP_MEMORY_SCOPE_AGENT);
}
__device__ __forceinline__ unsigned peer_of(unsigned r, unsigned i) { return (r & 4u) + ((r + i + 1u) & 3u); }
__device__ __forceinline__ unsigned panel_row(unsigned ap, unsigned rh, unsigned rank) {
  const unsigned og = (rank & 4u) ^ 4u;
  if (ap < 8u) return rank * 1024u + ap * 128u + rh * 64u;
  if (ap < 23u) { const unsigned i = (ap - 8u) / 5u, s = (ap - 8u) - 5u * i; return peer_of(rank, i) * 1024u + s * 128u + rh * 64u; }
  if (ap < 26u) return peer_of(rank, ap - 23u) * 1024u + 640u;
  if (ap < 34u) return (og + ((ap - 26u) >> 1)) * 1024u + 768u + (ap & 1u) * 128u + rh * 64u;
  return (og + 2u * (ap - 34u) + rh) * 1024u + 704u;
}
__device__ __forceinline__ unsigned park_pos(unsigned ap, unsigned rh, unsigned j) {
  if (ap < 8u) return 2u * ap + rh < 11u ? 2u * ap + rh : 99u;
  if (ap < 23u) {
    const unsigned i = (ap - 8u) / 5u, s = (ap - 8u) - 5u * i;
    return s < 2u ? 11u + 4u * i + 2u * s + rh : s < 4u ? 40u + 4u * i + 2u * s + rh : 26u + 2u * i + rh;
  }
  const unsigned src = ap < 34u ? (ap - 26u) >> 1 : 2u * (ap - 34u) + rh;
  if (src == j) return 99u;
  const unsigned e = src - (src > j ? 1u : 0u);
  return ap < 34u ? 32u + 4u * e + 2u * (ap & 1u) + rh : 23u + e;
}
__device__ __forceinline__ unsigned park_row(unsigned hp, unsigned src, unsigned rank) {
  const unsigned q = rank & 3u;
  if (hp < 11u) return src * 1024u + hp * 64u;
  if (hp < 23u) return peer_of(src, (hp - 11u) >> 2) * 1024u + ((hp - 11u) & 3u) * 64u;
  if (hp < 26u) { const unsigned e = hp - 23u; return ((rank & 4u) + e + (e >= q ? 1u : 0u)) * 1024u + 704u; }
  if (hp < 32u) return peer_of(src, (hp - 26u) >> 1) * 1024u + 512u + (hp & 1u) * 64u;
  if (hp < 44u) { const unsigned e = (hp - 32u) >> 2; return ((rank & 4u) + e + (e >= q ? 1u : 0u)) * 1024u + 768u + (hp & 3u) * 64u; }
  return peer_of(src, (hp - 44u) >> 2) * 1024u + 256u + (hp & 3u) * 64u;
}

}

__global__ void __launch_bounds__(512) __attribute__((amdgpu_waves_per_eu(2, 2)))
ag_gemm_relu_kernel(const Params p) {
  extern __shared__ __attribute__((aligned(16))) char smem_raw[];
  LAS char* const lds = (LAS char*)smem_raw;
  LAS volatile unsigned* const misc = (LAS volatile unsigned*)(lds + kLdsMisc);

  const int tid = threadIdx.x;
  const int lane = tid & 63;
  const int wid = __builtin_amdgcn_readfirstlane(tid >> 6);
  const int bid = blockIdx.x;
  const unsigned rank = (unsigned)p.rank & 7u;
  const unsigned q = rank & 3u, og = (rank & 4u) ^ 4u;
  char* const myws = p.ws[rank];
  const unsigned l16 = (unsigned)lane * 16u;
  const __amdgpu_buffer_rsrc_t rws = mk_rsrc(myws, kWsNeed);
  const int fr = lane & 15, fq = lane >> 4;

  STAMP(myws, wid == 0 ? 0 : 31);

  const unsigned lx = (((unsigned)lane >> 2) * 8192u + 4u * (unsigned)((lane & 3) ^ ((-(lane >> 4)) & 3))) * 4u;
  const unsigned lw = (4u * ((unsigned)lane >> 4) * 1024u + 4u * ((unsigned)lane & 15u)) * 4u;
  const __amdgpu_buffer_rsrc_t rx = mk_rsrc(p.x, kInBytes);
  const __amdgpu_buffer_rsrc_t rw = mk_rsrc(p.w, kInBytes);
  const unsigned pw = (unsigned)wid >= WPM ? (unsigned)wid - WPM : (unsigned)wid;
  const unsigned gw = (unsigned)bid * WPM + pw;
  const unsigned sid = gw % 14u, k = gw / 14u;
  const bool cross = sid >= 6u;
  const unsigned c = cross ? (sid - 6u) >> 1 : sid >> 1, h = sid & 1u;
  const unsigned sr = (cross ? og + c : peer_of(rank, c)) & 7u;
  char* const sws = p.ws[sr];
  const __amdgpu_buffer_rsrc_t rs = mk_rsrc(sws, kWsNeed);
  LAS volatile unsigned* const hand = misc + 4u + 4u * pw;
  const unsigned buf0 = pw * 3u * kPBuf;
  if (bid < kNMove && (unsigned)wid >= WPM && (unsigned)wid < 2u * WPM && gw < 14u * NWS) {
    unsigned call = 0;
    auto push = [&](auto wconst) {
      constexpr bool W = decltype(wconst)::value;
      f32x4 pv[PU][2][8];
#pragma unroll
      for (unsigned u = 0; u < PU; ++u)
#pragma unroll
        for (int i = 0; i < 16; ++i) {
          const unsigned m = u * NWS + k, kk = (m >> 1) * 64u + h * 32u;
          if (W) pv[u][i >> 3][i & 7] = ld16<0>(rw, lw + (i >> 3) * 256u, (kk * 1024u + 256u * c + (m & 1u) * 128u) * 4u + ((i & 3) + 16 * ((i >> 2) & 1)) * 4096u);
          else pv[u][i >> 3][i & 7] = ld16<0>(rx, lx + (i & 1) * 64u, ((m & 1u) * 128u * 8192u + kk) * 4u + (i >> 1) * 524288u);
        }
      __syncthreads();
      call = uni(misc[0]) + 1u;
      const unsigned base = W ? kBOff + (call & 1u) * kBSize + (8u + 2u * q) * kPanel : kAOff + (call & 1u) * kASize + (18u - 5u * c) * kPanel;
      char* const fl = sws + kFlags + (W ? 44u + 2u * q : 18u - 5u * c) * 1024u + h * 4u;
#pragma unroll
      for (unsigned j = 0; j < 2u * PU; ++j) {
        const unsigned u = j >> 1, r = j & 1u, m = u * NWS + k;
        f16x8 hv[4];
#pragma unroll
        for (int i = 0; i < 4; ++i) hv[i] = W ? cvt_col(pv[u][r], i) : cvt8(pv[u][r][2 * i], pv[u][r][2 * i + 1]);
        asm volatile("" :: "v"(hv[0]), "v"(hv[1]), "v"(hv[2]), "v"(hv[3]));
        if (j >= PUW) WAIT_VM(4u * (PUW - 1u));
        if (j <= 2u) STAMP(myws, 8 + wid + 3 * j);
        if (u > 0u && r == PUW - 1u && lane == 0) flag_store(fl + ((m - NWS) & 1u) * 1024u + ((m - NWS) >> 1) * 8u, call);
#pragma unroll
        for (int i = 0; i < 4; ++i) st16<kAuxSys>(hv[i], rs, l16 + i * 1024u, base + (m & 1u) * kPanel + (m >> 1) * kBlk + h * 8192u + r * 4096u);
        FENCE();
      }
      WAIT_VM(0);
      const unsigned m = (PU - 1u) * NWS + k;
      if (lane == 0) flag_store(fl + (m & 1u) * 1024u + (m >> 1) * 8u, call);
    };
    if (cross) push(std::true_type{}); else push(std::false_type{});
    STAMP(myws, 4 + wid);
    if (!cross) return;
    const __amdgpu_buffer_rsrc_t ro = mk_rsrc(p.out[rank], kInBytes);
    const unsigned vo = (unsigned)fq * 16384u + (unsigned)fr * 16u;
    const unsigned long long t0 = __builtin_amdgcn_s_memrealtime();
    for (unsigned b = 0;; b = b == 2u ? 0u : b + 1u) {
      unsigned hv;
      while ((hv = uni(hand[b])) == 0u) {
        if (__builtin_amdgcn_s_memrealtime() - t0 > 2u * kDeadline) { set_timeout(myws, call); return; }
        __builtin_amdgcn_s_sleep(XSLEEP);
      }
      FENCE();
      if (hv == ~0u) break;
      const unsigned oa = park_row(hv >> 4, sr, rank) * 4096u + (256u * c + (hv & 12u) * 16u) * 4u;
      unsigned pos = (unsigned)(size_t)lds + buf0 + b * kPBuf, val[64];
      u32x2 mk = {~0u, ~0u};
      if ((hv & 2u) == 0u) mk = *(LAS u32x2*)(size_t)(pos + lane * 8);
      const unsigned fill = (unsigned)(size_t)lds + kLdsMisc + 64u + (hv & 2u) * 2u;
      pos += 512u;
#pragma unroll
      for (int i = 0; i < 64; ++i) {
        const bool keep = (int)(mk[i >> 5] << (i & 31)) >= 0;
        const unsigned long long bl = __ballot(keep);
        unsigned a = lanes_below(bl) * 4u + pos;
        asm volatile("" : "+v"(a));
        val[i] = *(LAS unsigned*)(size_t)(keep ? a : fill);
        advance(pos, bl);
      }
      WAIT_LGKM0();
      if (lane == 0) hand[b] = 0u;
      FENCE();
#pragma unroll
      for (int i = 0; i < 16; ++i)
        __builtin_amdgcn_raw_buffer_store_b128(u32x4{val[4 * i], val[4 * i + 1], val[4 * i + 2], val[4 * i + 3]}, ro, vo, oa + (i >> 2) * 65536u + (i & 3) * 4096u, kAuxSys);
    }
    STAMP(myws, 1 + wid);
    return;
  }
  const bool isconv = bid >= kNMove + 8 * CVFROM;
  unsigned tk = 0;
  if (tid == 0) tk = draw(myws + kTicket + (unsigned)bid * 32u);
  FENCE();
  f32x4 cv[8];
  auto item = [&](unsigned n, unsigned& ktg, unsigned& sub) {
    if (n < 5u * CVLEAD) { ktg = n / 5u; sub = n - 5u * ktg; sub += sub ? 3u : 0u; }
    else if (n < 8u * CVLEAD) { ktg = (n - 5u * CVLEAD) / 3u; sub = n - 5u * CVLEAD - 3u * ktg + 1u; }
    else { ktg = n >> 3; sub = n & 7u; }
  };
  auto loads = [&](unsigned n) {
    unsigned ktg, sub;
    item(n, ktg, sub);
    if (sub < 4u) {
      const unsigned so = ((256u * sub + 32u * wid) * 8192u + ktg * 64u) * 4u;
#pragma unroll
      for (int s = 0; s < 2; ++s) {
        cv[4 * s + 0] = ld16<0>(rx, lx, so + s * 524288u); cv[4 * s + 1] = ld16<0>(rx, lx + 64u, so + s * 524288u);
        cv[4 * s + 2] = ld16<0>(rx, lx + 128u, so + s * 524288u); cv[4 * s + 3] = ld16<0>(rx, lx + 192u, so + s * 524288u);
      }
    } else {
      const unsigned so = ((ktg * 64u + 32u * (wid & 1)) * 1024u + 256u * (sub - 4u) + 64u * (wid >> 1)) * 4u;
#pragma unroll
      for (int i = 0; i < 8; ++i) cv[i] = ld16<0>(rw, lw, so + ((i & 3) + 16 * (i >> 2)) * 4096u);
    }
  };
  if (bid >= kNMove + 8 * CVFROM + CVFIRST) __builtin_amdgcn_s_sleep(CVDELAY);
  if (isconv) loads((unsigned)(bid - kNMove - 8 * CVFROM));
  FENCE();
  if (tid == 0) {
    misc[0] = tk;
    misc[3] = 0u;
  }
  if (tid >= 4 && tid < 18) misc[tid] = tid == 17 ? 0x7fc00000u : 0u;
  __syncthreads();
  const unsigned call = uni(misc[0]) + 1u;
  const unsigned par = call & 1u;
  const unsigned abase = kAOff + par * kASize, bbase = kBOff + par * kBSize, obuf = kOOff + par * kOSize;

  if (bid < kNMove) {
    if (wid == 7) {
      if (bid == 0 && lane == 0) {
        const unsigned target = call * kDonePerCall;
        const unsigned long long t0 = __builtin_amdgcn_s_memrealtime();
        bool failed = false;
        while ((int)(__hip_atomic_load((GAS unsigned*)(myws + kDone), __ATOMIC_RELAXED, __HIP_MEMORY_SCOPE_SYSTEM) - target) < 0) {
          if (__hip_atomic_load((GAS unsigned*)(myws + kTmo), __ATOMIC_RELAXED, __HIP_MEMORY_SCOPE_AGENT) == call ||
              __builtin_amdgcn_s_memrealtime() - t0 > kDeadline) {
            set_timeout(myws, call);
            failed = true;
            break;
          }
          __builtin_amdgcn_s_sleep(2);
        }
        __builtin_amdgcn_fence(__ATOMIC_ACQUIRE, "");
        if (failed) __hip_atomic_store((GAS unsigned*)(p.out[rank] + (size_t)rank * 1024u * 1024u), 0x7fc00000u, __ATOMIC_RELAXED, __HIP_MEMORY_SCOPE_SYSTEM);
        STAMP(myws, 10);
      }
      return;
    }
    if ((unsigned)wid >= WPM || gw >= 14u * NWS) return;
    const unsigned total = cross ? kRowsC : kRowsI;
    const unsigned nplain = cross ? 576u : kRowsI;
    const unsigned qbase = (call - 1u) * (total - PU * NWS) - (PU + 1u) * NWS;
    bool failed = false, all = false;
    unsigned cgrp = 99u, okf = 0, okt = 0, fv = 0, wbase = 0x10000u;
    u32x2 fw = {0u, 0u};
    unsigned sa = 0, da = 0, fa = 0, pend = 0;
    unsigned s2 = 4096u, d2 = 4096u;
    u32x4 v[4], w[4];
    auto raise = [&]() {
      if (pend != 0u && (unsigned)lane < (pend >= kLast ? 8u : 1u)) flag_store(myws + pend + (unsigned)lane * kRep, call);
      pend = 0u;
    };
    auto fetch = [&](u32x4 (&x)[4], unsigned so) {
#pragma unroll
      for (int i = 0; i < 4; ++i) x[i] = __builtin_amdgcn_raw_buffer_load_b128(rs, l16 + i * 1024u, so, kAuxSys);
      FENCE();
    };
    auto put = [&](const u32x4 (&x)[4], unsigned so) {
#pragma unroll
      for (int i = 0; i < 4; ++i) __builtin_amdgcn_raw_buffer_store_b128(x[i], rws, l16 + i * 1024u, so, kAuxDev);
      FENCE();
    };
    auto setup = [&](unsigned m) {
      unsigned sp, kt, dp, db, dh = h;
      unsigned so = h * 8192u, dof = h * 8192u;
      s2 = 4096u; d2 = 4096u;
      if (!cross) {
        if (m < 256u) { sp = m & 1u; kt = m >> 1; dp = 8u + 5u * c + sp; db = kt; }
        else if (m < 384u) { sp = 4u; kt = m - 256u; dp = 12u + 5u * c; db = kt; }
        else if (m < 640u) { sp = 2u + (m & 1u); kt = (m - 384u) >> 1; dp = 8u + 5u * c + sp; db = kt; }
        else { sp = 5u; db = m - 640u; kt = 2u * db + h; so = 0u; s2 = 8192u; dp = 23u + c; }
      } else {
        if (m < 256u) { sp = 36u + 2u * q + (m & 1u); kt = m >> 1; dp = 44u + 2u * c + (m & 1u); db = kt; }
        else {
          const unsigned g = ((m - 256u) * 52429u) >> 18, r = m - 256u - 5u * g;
          if (r == 2u) { sp = 5u; kt = 2u * g + h; so = 4096u; s2 = 8192u; dp = 34u + (c >> 1); db = kt; dh = c & 1u; dof = dh * 4096u; d2 = 8192u; }
          else { sp = 6u + (r % 3u); kt = 2u * g + (r > 2u ? 1u : 0u); dp = 20u + 2u * c + sp; db = kt; }
        }
      }
      const unsigned sub = sp < 36u ? sp >> 1 : (sp - 28u) >> 1;
      auto ask = [&](unsigned k0) {
        const unsigned off = lane < 62 ? kSrc + (sr * 8u + sub) * 512u + min(k0 + (unsigned)lane, 127u) * 4u : lane == 62 ? kAll + sr * 4u : kTmo;
        asm volatile("buffer_load_dword %0, %1, %2, 0 offen sc0 sc1" : "=v"(fv) : "v"(off), "s"(rws) : "memory");
      };
      auto look = [&]() {
        asm volatile("" : "+v"(fv) :: "memory");
        const unsigned long long b = __ballot(lane < 63 ? (int)(fv - call) >= 0 : fv == call);
        if ((b >> 62) != 0ull) { failed = (b >> 63) != 0ull; all = true; }
        okt += (unsigned)__builtin_ctzll(~b);
      };
      if (!all) {
        if (cgrp != 99u) look();
        if (!all && !(sub == cgrp && kt >= okf && kt < okt)) {
          WAIT_VM(0);
          raise();
          cgrp = sub;
          okf = kt;
          const unsigned long long t0 = __builtin_amdgcn_s_memrealtime();
          for (;;) {
            okt = kt;
            ask(kt);
            WAIT_VM(0);
            look();
            if (all || okt != kt) break;
            if (__builtin_amdgcn_s_memrealtime() - t0 > kDeadline) { set_timeout(myws, call); failed = true; all = true; break; }
            __builtin_amdgcn_s_sleep(MSLEEP);
          }
        }
        if (!all) ask(okt);
      }
      sa = (sp < 36u ? abase + sp * kPanel : bbase + (sp - 36u) * kPanel) + kt * kBlk + so;
      da = (dp < 36u ? abase + dp * kPanel : bbase + (dp - 36u) * kPanel) + db * kBlk + dof;
      fa = (dp - 23u < 3u ? kLast - 23u * 1024u : kFlags) + dp * 1024u + db * 8u + dh * 4u;
      fetch(v, sa);
    };
    unsigned m = PU * NWS + k;
    setup(m);
    STAMP(myws, 20 + wid);
    do {
      unsigned dv = 0;
      if (lane == 0) dv = draw(myws + kQueue + sid * 128u);
      FENCE();
      put(v, da);
      raise();
      fetch(w, sa + s2);
      put(w, da + d2);
      pend = fa;
      if (h == 0u) { PSTAMP(myws, sid >> 1, m + 1u); if (m < 128u) PSTAMP(myws, 8u + (sid >> 1), (m + 1u) * 4u); }
      m = uni(dv) - qbase;
      if (m < nplain) setup(m);
    } while (m < nplain);
    WAIT_VM(0);
    raise();
    if (cross) {
      unsigned hval = 0;
      const unsigned fo = lane < 32 ? kOFlags + par * 8192u + c * 2048u + h * 8u + (unsigned)lane * 16u : kTmo;
      auto stale = [&](unsigned o) {
        return o - wbase >= 32u || (int)((unsigned)__builtin_amdgcn_readlane((int)fw[0], (int)(o - wbase)) - call) < 0;
      };
      for (unsigned b = 0, pb = 0;; pb = b, b = b == 2u ? 0u : b + 1u) {
        const unsigned o = m - 576u, s = 2u * o + h;
        unsigned cnt = 2048u, dv = 0;
        if (m >= total || stale(o)) {
          WAIT_VM(0);
          if (hval != 0u && lane == 0) hand[pb] = hval;
          hval = 0u;
          const unsigned long long t0 = __builtin_amdgcn_s_memrealtime();
          wbase = o;
          while (!failed && m < total) {
            fw = __builtin_bit_cast(u32x2, __hip_atomic_load((GAS unsigned long long*)(myws + fo + (lane < 32 ? o * 16u : 0u)), __ATOMIC_RELAXED, __HIP_MEMORY_SCOPE_SYSTEM));
            const unsigned long long bl = __ballot(lane < 32 ? (int)(fw[0] - call) >= 0 : fw[0] == call);
            if ((bl >> 32) != 0ull) { failed = true; break; }
            if (bl & 1ull) break;
            if (__builtin_amdgcn_s_memrealtime() - t0 > kDeadline) { set_timeout(myws, call); failed = true; }
            __builtin_amdgcn_s_sleep(MSLEEP);
          }
        }
        cnt = failed ? 0u : min((unsigned)__builtin_amdgcn_readlane((int)fw[1], (int)((o - wbase) & 31u)), 4096u);
        if (UNLIKELY(uni(hand[b]) != 0u)) {
          const unsigned long long t0 = __builtin_amdgcn_s_memrealtime();
          while (uni(hand[b]) != 0u)
            if (__builtin_amdgcn_s_memrealtime() - t0 > kDeadline) { set_timeout(myws, call); return; }
        }
        FENCE();
        if (m >= total) {
          if (lane == 0) hand[b] = ~0u;
          break;
        }
        const unsigned bytes = 512u + 4u * cnt, sb = obuf + (q * 224u + s) * kPiece, nbuf = buf0 + b * kPBuf;
        auto part = [&](unsigned f0, unsigned f1) {
          for (unsigned f = f0; f < f1; f += 1024u) {
            WAIT_VM(PWIN - 1u);
            asm volatile("s_mov_b32 m0, %0\n\ts_nop 0\n\tbuffer_load_dwordx4 %1, %2, %3 offen sc0 sc1 lds"
                         :: "s"(nbuf + f), "v"(f + l16 < bytes ? l16 : kOob), "s"(rs), "s"(sb + f) : "memory");
          }
        };
        asm volatile("s_mov_b64 exec, 1\n\tglobal_atomic_add %0, %1, %2, %3 sc0\n\ts_mov_b64 exec, -1"
                     : "=v"(dv) : "v"(kQueue + sid * 128u), "v"(1u), "s"(myws) : "memory");
        part(0u, min(bytes, (PWIN - 1u) * 1024u));
        if (bytes <= (PWIN - 2u) * 1024u) WAIT_VM(0);
        WAIT_VM(PWIN - 1u);
        if (hval != 0u && lane == 0) hand[pb] = hval;
        hval = 4u * s + (failed ? 3u : 1u);
        if (h == 0u) PSTAMP(myws, sid >> 1, m + 1u);
        asm volatile("" : "+v"(dv));
        m = uni(dv) - qbase;
        if (m < total && stale(m - 576u)) {
          wbase = m - 576u;
          asm volatile("buffer_load_dwordx2 %0, %1, %2, 0 offen sc0 sc1" : "=v"(fw) : "v"(fo + (lane < 32 ? wbase * 16u : 0u)), "s"(rws) : "memory");
          part((PWIN - 1u) * 1024u, bytes);
          if (bytes <= (2u * PWIN - 3u) * 1024u) WAIT_VM(0);
          WAIT_VM(PWIN - 1u);
          asm volatile("" : "+v"(fw) :: "memory");
        } else
          part((PWIN - 1u) * 1024u, bytes);
      }
    }
    STAMP(myws, 1 + wid);
    return;
  }

  {
    constexpr unsigned kNConv = 8u * (26u - CVFROM);
    char* const pws = p.ws[lane & 7];
    const unsigned mine = 15u << (rank & 4u);
    STAMP(myws, wid == 0 ? 27 : 31);
    for (unsigned n = isconv ? (unsigned)(bid - kNMove - 8 * CVFROM) : 1024u; n < 1024u; n += kNConv) {
      unsigned ktg, sub;
      item(n, ktg, sub);
      if (sub < 4u) {
        const unsigned so = abase + (2u * sub + (wid >> 2)) * kPanel + ktg * kBlk + (wid & 3) * 2048u;
#pragma unroll
        for (int i = 0; i < 4; ++i) st16<CVAUX>(cvt8(cv[2 * i], cv[2 * i + 1]), rws, l16 + (i >> 1) * 1024u, so + (i & 1) * 8192u);
      } else {
        const unsigned so = bbase + (2u * (sub - 4u) + (wid >> 2)) * kPanel + ktg * kBlk + ((wid >> 1) & 1) * 4096u + (wid & 1) * 8192u;
#pragma unroll
        for (int i = 0; i < 4; ++i) st16<CVAUX>(cvt_col(cv, i), rws, l16 + i * 1024u, so);
      }
      FENCE();
      const bool ahead = n >= kNConv && n + kNConv < 1024u;
      if (ahead) { loads(n + kNConv); WAIT_VM(8); } else { WAIT_VM(0); }
      if (n < kNConv) STAMP(myws, wid == 0 ? 30 : 31);
      BARRIER();
      if (n < 2u * kNConv) STAMP(myws, wid == 0 ? (n < kNConv ? 28 : 29) : 31);
      const unsigned to = sub < 2u ? mine : sub == 2u ? 255u : sub == 3u ? mine ^ 255u : 1u << (og + sub - 4u);
      if (tid < 2) flag_store2(myws + kFlags + ((sub < 4u ? 2u * sub : 28u + 2u * sub) + (unsigned)tid) * 1024u + ktg * 8u, call, call);
      else if (wid == 1 && lane < 8 && (((to & ~(1u << rank)) >> lane) & 1u)) flag_store(pws + kSrc + rank * 4096u + sub * 512u + ktg * 4u, call);
      FENCE();
      if (n < kNConv) loads(n + kNConv);
    }
    if (isconv && wid == 7) {
      unsigned dv = 0;
      if (lane == 0) dv = draw(myws + kCvt);
      if (uni(dv) == call * kNConv - 1u && lane < 8) flag_store(pws + kAll + rank * 4u, call);
    }
  }
  STAMP(myws, wid == 0 ? 1 : 31);

  const uint32_t* const tl = p.tiles + 3 * (bid - kNMove);
  const int ntiles = (int)(tl[0] >> 28);
  if (wid == 7) return;

  if (wid >= 4) {
    const unsigned lw = (unsigned)wid - 4u;
    unsigned total = 0;
    for (int ti = 0; ti < ntiles; ++ti) total += ((tl[ti] >> 10) & 3u) == 2u ? 64u : 256u;
    unsigned a = 0, aend = 0, so = lw * 8192u, src = 0, step = 0, g2 = 0, v2 = l16, idle = 0;
    unsigned ok = 0, f0 = 0, f1 = 0, fb = 0, fsh = 0, fmax = 0, fv = 0;
    unsigned rdy = 0, askat = 0, lookat = ~0u;
    int ta = 0, tf = 0;
    auto next_flags = [&]() {
      while (tf < ntiles) {
        const unsigned code = tl[tf++];
        const unsigned ap = code & 63u, bp = (code >> 6) & 15u, shape = (code >> 10) & 3u;
        f0 = f1;
        f1 += shape == 2u ? 64u : 256u;
        if (shape == 3u && lw == 2u) { ok = f1; continue; }
        const unsigned grp = lw == 0u ? ap : shape == 0u ? (lw == 1u ? ap + 1u : 36u + bp) : shape == 1u ? 35u + bp + lw : 36u + bp;
        fb = kFlags + grp * 1024u;
        fsh = shape != 2u ? 0u : lw == 0u ? 1u : 2u;
        fmax = fsh == 1u ? 127u : 255u;
        if (fsh == 1u) fb += kLast - kFlags - 23u * 1024u + ((unsigned)bid & 7u) * kRep;
        break;
      }
    };
    auto known = [&]() {
      rdy = min(ok, aend);
      unsigned from = max(ok, LTHR);
      asm volatile("" : "+s"(from));
      askat = ok >= total ? ~0u : from - LTHR;
    };
    auto ask = [&](unsigned last) {
      const unsigned off = lane < ALANES ? fb + min(((ok - f0) << fsh) + (unsigned)lane, fmax) * 4u : lane == 63 ? last : kOob;
      asm volatile("buffer_load_dword %0, %1, %2, 0 offen sc0 sc1" : "=v"(fv) : "v"(off), "s"(rws) : "memory");
    };
    auto give_up = [&]() {
      ok = total;
      if (lane == 0) misc[3] = 1u;
      WAIT_LGKM0();
    };
    auto look = [&]() {
      asm volatile("" : "+v"(fv) :: "memory");
      const unsigned long long b = __ballot(lane < 63 ? (int)(fv - call) >= 0 : fv == call);
      if (UNLIKELY((b >> 63) != 0ull)) {
        give_up();
      } else {
        const unsigned run = (unsigned)__builtin_ctzll(~b);
        ok = min(ok + (fsh == 0u ? run & ~1u : run >> fsh), f1);
        if (UNLIKELY(ok == f1)) next_flags();
      }
      lookat = ~0u;
    };
    auto issue = [&](bool busy) {
      LAS char* const dl = lds + so;
      if (busy) {
        __builtin_amdgcn_raw_ptr_buffer_load_lds(rws, (LAS void*)dl, 16, l16, src, 0, kAuxDev);
        __builtin_amdgcn_raw_ptr_buffer_load_lds(rws, (LAS void*)dl, 16, l16, src, 1024, kAuxDev);
        __builtin_amdgcn_raw_ptr_buffer_load_lds(rws, (LAS void*)dl, 16, v2, src, 2048, kAuxDev);
        __builtin_amdgcn_raw_ptr_buffer_load_lds(rws, (LAS void*)dl, 16, v2, src, 3072, kAuxDev);
        __builtin_amdgcn_raw_ptr_buffer_load_lds(rws, (LAS void*)(dl + 4096), 16, l16, src + g2, 0, kAuxDev);
        __builtin_amdgcn_raw_ptr_buffer_load_lds(rws, (LAS void*)(dl + 4096), 16, l16, src + g2, 1024, kAuxDev);
        __builtin_amdgcn_raw_ptr_buffer_load_lds(rws, (LAS void*)(dl + 4096), 16, v2, src + g2, 2048, kAuxDev);
        __builtin_amdgcn_raw_ptr_buffer_load_lds(rws, (LAS void*)(dl + 4096), 16, v2, src + g2, 3072, kAuxDev);
      }
      FENCE();
      src += step;
      so = so >= kRing - kSlot ? so - (kRing - kSlot) : so + kSlot;
      ++a;
    };
    next_flags();
    known();
    for (unsigned j = 0; j < total; ++j) {
      if (a == j + 4u) {
        unsigned e = min(rdy + 1u, askat);
        asm volatile("" : "+s"(e));
        e = min(e, lookat);
        if (idle != 0u) {
          WAIT_VM(0);
          while (a + 1u < e) { issue(false); BARRIER(); }
        }
        while (a + 1u < e) {
          issue(true);
          WAIT_VM(32);
          BARRIER();
        }
        j = a - 4u;
      }
      unsigned lim = min(j + 5u, rdy);
      if (UNLIKELY(a >= lim)) {
        if (a == aend) {
          const unsigned code = tl[ta++];
          const unsigned ap = code & 63u, bp = (code >> 6) & 15u, shape = (code >> 10) & 3u;
          if (lw == 0u || (lw == 1u && shape == 0u)) src = abase + (ap + lw) * kPanel;
          else src = bbase + (bp + (shape == 1u ? lw - 1u : 0u)) * kPanel;
          step = 8192u; g2 = 4096u;
          idle = shape == 3u && lw == 2u ? 1u : 0u;
          if (shape == 2u) {
            g2 = 8192u;
            if (lw == 0u) { src += ((code >> 13) & 1u) * 2048u; step = kBlk; v2 = l16 + 2048u; }
            else { src += ((code >> 12) & 1u) * 4096u + (lw - 1u) * kBlk; step = 2u * kBlk; }
          }
          aend = ta == ntiles ? ~0u : aend + 256u;
          rdy = min(ok, aend);
        }
        if (a <= j && ok <= a) {
          WAIT_VM(0);
          if (lookat != ~0u) look();
          const unsigned long long t0 = __builtin_amdgcn_s_memrealtime();
          while (ok <= a) {
            ask(kTmo);
            WAIT_VM(0);
            look();
            if (ok > a) break;
            if (__builtin_amdgcn_s_memrealtime() - t0 > kDeadline) { set_timeout(myws, call); give_up(); break; }
            __builtin_amdgcn_s_sleep(LSLEEP);
          }
          known();
        }
        lim = min(j + 5u, rdy);
      }
      if (a < lim) {
        issue(idle == 0u);
        if (UNLIKELY(a < lim)) issue(idle == 0u);
      }
      if (UNLIKELY(a >= askat)) { ask(PTMO); askat = ~0u; lookat = a + 4u; }
      const unsigned n = idle != 0u ? 0u : a - 1u - j;
      if (UNLIKELY(n < 4u)) {
        WAIT_VM(24);
        if (n < 3u) {
          WAIT_VM(16);
          if (n < 2u) {
            WAIT_VM(8);
            if (n < 1u) WAIT_VM(0);
          }
        }
      }
      WAIT_VM(32);
      if (UNLIKELY(j + 5u >= lookat)) { look(); known(); }
      BARRIER();
    }
    return;
  }

  {
    const unsigned slot = (unsigned)(4 * fr + (fq ^ ((-(fr >> 2)) & 3))) * 16u;
    unsigned so = 0;
    for (int ti = 0; ti < ntiles; ++ti) {
      const unsigned code = tl[ti];
      const unsigned shape = (code >> 10) & 3u;
      const unsigned wh = (unsigned)wid >> 1, wl = (unsigned)wid & 1u;
      const unsigned ap = (code & 63u) + (shape == 0u ? wh : 0u), bp = ((code >> 6) & 15u) + (shape == 1u ? wh : 0u);
      const unsigned jo = (bp - 8u) >> 1;
      const unsigned owner = (bp < 8u ? rank : og + jo) & 7u;
      const unsigned col0 = bp < 8u ? bp * 128u : 256u * q + (bp & 1u) * 128u;
      const __amdgpu_buffer_rsrc_t ro = mk_rsrc(p.out[owner], kInBytes);
      const unsigned vo = (unsigned)fq * 16384u + (unsigned)fr * 16u;

      auto run = [&](auto mconst, const unsigned aoff, const unsigned boff) {
        constexpr int M = decltype(mconst)::value;
        constexpr int KS = M > 1 ? 1 : 4;
        constexpr int NS = M > 1 ? 256 : 64;
        constexpr int NA = M * KS, NB = 4 * KS;
        const unsigned va = aoff + slot, vb = boff + l16;
        f32x4 acc[M][4];
#pragma unroll
        for (int m = 0; m < M; ++m)
#pragma unroll
          for (int n = 0; n < 4; ++n) acc[m][n] = f32x4{0.f, 0.f, 0.f, 0.f};
        f16x8 fa[2][NA], fb[2][NB];
        auto rd = [&](f16x8 (&a)[NA], f16x8 (&b)[NB]) {
          LAS char* const pa = lds + (va + so);
          LAS char* const pb = lds + (vb + so);
#pragma unroll
          for (int i = 0; i < (NA > NB ? NA : NB); ++i) {
            if (i < NB) b[i] = *(LAS f16x8*)(pb + i * 1024);
            if (i < NA) a[i] = *(LAS f16x8*)(pa + i * (M > 1 ? 1024 : 2048));
          }
        };
        auto mm = [&](const f16x8 (&a)[NA], const f16x8 (&b)[NB]) {
#pragma unroll
          for (int ks = 0; ks < KS; ++ks)
#pragma unroll
            for (int m = 0; m < M; ++m)
#pragma unroll
              for (int n = 0; n < 4; ++n)
                acc[m][n] = __builtin_amdgcn_mfma_f32_16x16x32_f16(a[KS * m + ks], b[4 * ks + n], acc[m][n], 0, 0, 0);
        };
        BARRIER();
        STAMP(myws, wid == 0 ? 2 + 3 * ti : 31);
        CSTAMP(myws, wid == 0 ? 12 + 2 * ti : 31);
        rd(fa[0], fb[0]);
        __builtin_amdgcn_s_waitcnt(0xC07F);
        BARRIER();
        so = so == kRing - kSlot ? 0u : so + kSlot;
        if constexpr (M > 1) {
          unsigned ua = va + so, ub = vb + so;
          for (int t = 0; t < NS; t += 2) {
#pragma unroll
            for (int s = 0; s < 2; ++s) {
              __builtin_amdgcn_sched_barrier(0);
#pragma unroll
              for (int i = 0; i < 4 * M; ++i) {
                acc[i >> 2][i & 3] = __builtin_amdgcn_mfma_f32_16x16x32_f16(fa[s][i >> 2], fb[s][i & 3], acc[i >> 2][i & 3], 0, 0, 0);
                __builtin_amdgcn_sched_barrier(0);
                if (M == 8 && (i & 1) == 0 && i < 24) {
                  const int r = i >> 1;
                  if (r < 4) fb[s ^ 1][r] = *(LAS f16x8*)(lds + ub + r * 1024);
                  else fa[s ^ 1][r - 4] = *(LAS f16x8*)(lds + ua + (r - 4) * 1024);
                }
                if (M == 4 && i < 4) {
                  fb[s ^ 1][i] = *(LAS f16x8*)(lds + ub + i * 1024);
                  fa[s ^ 1][i] = *(LAS f16x8*)(lds + ua + i * 1024);
                }
                if (i == 4 * M - 10) { so = so == kRing - kSlot ? 0u : so + kSlot; asm volatile("" : "+s"(so)); }
                if (i == 4 * M - 6) { ua = va + so; asm volatile("" : "+v"(ua)); }
                if (i == 4 * M - 4) { ub = vb + so; asm volatile("" : "+v"(ub)); }
                if (i == 4 * M - 2) {
                  __builtin_amdgcn_s_waitcnt(0xC07F);
                  if (t < NS - 2) BARRIER();
                }
                __builtin_amdgcn_sched_barrier(0);
              }
            }
          }
          so = so == 0u ? kRing - kSlot : so - kSlot;
        } else {
          for (int t = 0; t < NS; t += 2) {
#pragma unroll
            for (int s = 0; s < 2; ++s) {
              __builtin_amdgcn_sched_barrier(0);
              rd(fa[s ^ 1], fb[s ^ 1]);
              mm(fa[s], fb[s]);
#pragma unroll
              for (int i = 0; i < 4; ++i) {
                __builtin_amdgcn_sched_group_barrier(0x100, 2, 0);
                __builtin_amdgcn_sched_group_barrier(0x008, 1, 0);
              }
#pragma unroll
              for (int i = 0; i < 12; ++i) {
                __builtin_amdgcn_sched_group_barrier(0x100, 1, 0);
                __builtin_amdgcn_sched_group_barrier(0x008, 1, 0);
              }
              __builtin_amdgcn_sched_barrier(0);
              so = so == kRing - kSlot ? 0u : so + kSlot;
              __builtin_amdgcn_s_waitcnt(0xC07F);
              if (t < NS - 2) BARRIER();
            }
          }
        }
        STAMP(myws, wid == 0 ? 3 + 3 * ti : 31);
        CSTAMP(myws, wid == 0 ? 13 + 2 * ti : 31);
        const int lo = misc[3] != 0u ? 0x7fc00000 : 0;
        if (M > 1 && bp >= 8u) {
          const __amdgpu_buffer_rsrc_t rw4 = __builtin_amdgcn_make_buffer_rsrc((void*)myws, 4, kWsNeed / 4u, 0x00020000);
          if (UNLIKELY(lo != 0)) {
#pragma unroll
            for (int m = 0; m < M; ++m)
#pragma unroll
              for (int n = 0; n < 4; ++n) acc[m][n] = __builtin_bit_cast(f32x4, u32x4{0x7fc00000u, 0x7fc00000u, 0x7fc00000u, 0x7fc00000u});
          }
#pragma unroll
          for (int rh = 0; rh < M / 4; ++rh) {
            const unsigned hp = park_pos(ap, M == 8 ? (unsigned)rh : wh, jo);
            if (hp < 56u) {
              const unsigned s = hp * 4u + (bp & 1u) * 2u + wl;
              const unsigned pb0 = obuf + (jo * 224u + s) * kPiece;
              unsigned pos = pb0 + 512u;
              u32x2 mk = {0u, 0u};
#pragma unroll
              for (int m = 0; m < 4; ++m)
#pragma unroll
                for (int j = 0; j < 4; ++j)
#pragma unroll
                  for (int n = 0; n < 4; ++n) {
                    const float f = acc[4 * rh + m][n][j];
                    const unsigned iv = __float_as_uint(f);
                    const unsigned long long b = __ballot((int)iv >= 0);
                    const unsigned idx = lanes_below(b) | (iv & kOob);
                    asm volatile("buffer_store_dword %0, %1, %2, %3 idxen sc0 sc1" :: "v"(iv), "v"(idx), "s"(rw4), "s"(pos) : "memory");
                    advance(pos, b);
                    mk[m >> 1] = __builtin_amdgcn_alignbit(mk[m >> 1], iv, 31);
                  }
              __builtin_amdgcn_raw_buffer_store_b64(mk, rws, (unsigned)lane * 8u, pb0, kAuxSys);
              WAIT_VM(0);
              if (lane == 0) flag_store2(p.ws[owner] + kOFlags + par * 8192u + q * 2048u + s * 8u, call, (pos - pb0 - 512u) >> 2);
            }
          }
        } else {
#pragma unroll
          for (int m = 0; m < M; ++m) {
            const unsigned ob = M > 1 ? (panel_row(ap, M == 8 ? (unsigned)(m >> 2) : wh, rank) + 16u * (m & 3)) * 4096u + (col0 + 64u * wl) * 4u
                                       : (panel_row(ap, 0u, rank) + 32u * ((code >> 13) & 1u) + 16u * wl) * 4096u + (col0 + 64u * ((code >> 12) & 1u)) * 4u;
#pragma unroll
            for (int j = 0; j < 4; ++j) {
              u32x4 o;
#pragma unroll
              for (int n = 0; n < 4; ++n) { const float f = acc[m][n][j]; o[n] = (unsigned)max(__float_as_int(f), lo); }
              __builtin_amdgcn_raw_buffer_store_b128(o, ro, vo, ob + j * 4096u, kAuxSys);
            }
          }
          if (bp >= 8u) {
            WAIT_VM(0);
            if (lane == 0) __hip_atomic_fetch_add((GAS unsigned*)(p.ws[owner] + kDone), 1u, __ATOMIC_RELAXED, __HIP_MEMORY_SCOPE_SYSTEM);
          }
        }
      };
      if (shape == 2u) {
        if (wid >= 2) return;
        run(std::integral_constant<int, 1>{}, wl * 1024u, 8192u);
      } else if (shape == 3u) {
        run(std::integral_constant<int, 4>{}, wh * 4096u, 8192u + wl * 4096u);
      } else {
        run(std::integral_constant<int, 8>{}, shape == 0u ? wh * 8192u : 0u, (shape == 0u ? 16384u : 8192u + wh * 8192u) + wl * 4096u);
      }
      STAMP(myws, wid == 0 ? 4 + 3 * ti : 31);
    }
  }
}

extern "C" void kernel_launch(int rank, int nranks, ncclComm_t comm, hipStream_t stream,
                              void* const* d_in, const int* in_sizes, int n_in,
                              void* d_out, int out_size,
                              const int* in_dtypes, int out_dtype,
                              void* const* d_out_all, void* const* d_ws_all, size_t ws_bytes) {
  (void)comm; (void)in_dtypes; (void)out_dtype; (void)d_out;
  if (nranks != 8 || (unsigned)rank >= 8u || n_in < 2 || ws_bytes < (size_t)kWsNeed ||
      in_sizes[0] != 8 * 1024 * 1024 || in_sizes[1] != 8 * 1024 * 1024 || out_size != 8 * 1024 * 1024) {
    fprintf(stderr, "kernel_launch: unexpected problem geometry\n");
    return;
  }
  Params p;
  p.x = (const float*)d_in[0];
  p.w = (const float*)d_in[1];
  for (int i = 0; i < 8; ++i) { p.ws[i] = (char*)d_ws_all[i]; p.out[i] = (float*)d_out_all[i]; }
  p.rank = rank;
  p.pad = 0;
  static const int xcol[8] = {0, 4, 2, 3, 1, 5, 6, 7};
  auto tile = [](int a, int b, int shape) { return (unsigned)(a | (b << 6) | (shape << 10)); };
  for (int ci = 0; ci < kNComp; ++ci) {
    const int x = ci & 7, s = ci >> 3, bq = 2 * (x ^ 4), j = x >> 1;
    unsigned t[3] = {0u, 0u, 0u};
    int n = 0;
    auto farb = [&](int e) { const int idx = 3 * (x & 1) + e; return 8 + idx + (idx >= 2 * j ? 2 : 0); };
    if (s < 12) {
      t[n++] = tile(x < 4 ? 2 * (s >> 2) : 8 + 5 * (s >> 2), 4 * (x & 3) + (s & 3), 0);
      t[n++] = tile(10 + 5 * (s >> 2) + ((s >> 1) & 1), bq + (s & 1), 3);
    } else if (s == 12) {
      t[n++] = tile(6, xcol[x], 0);
      t[n++] = tile(27 + 2 * j, farb(1), 3);
    } else if (s < 16) {
      t[n++] = tile(12 + 5 * (s - 13), bq, 1);
    } else if (s < 18) {
      t[n++] = tile(34 + (s - 16), bq, 1);
    } else if (s < 23) {
      t[n++] = tile(26 + 2 * j, s < 22 ? 4 * (x & 1) + s - 18 : farb(2), 0);
    } else {
      t[n++] = tile(26 + 2 * j + ((s - 23) & 1), farb((s - 23) >> 1), 3);
    }
    const int i = s < 12 ? s : s == 22 || s == SKIPA ? -1 : s - 12 - (s > SKIPA ? 1 : 0) - (s > 22 ? 1 : 0);
    if (i >= 0) t[n++] = tile(23 + (i >> 2), s < 12 ? x : 8 + x, 2) | ((unsigned)(i & 1) << 12) | ((unsigned)((i >> 1) & 1) << 13);
    p.tiles[3 * ci] = t[0] | ((unsigned)n << 28);
    p.tiles[3 * ci + 1] = t[1];
    p.tiles[3 * ci + 2] = t[2];
  }
  hipError_t e = hipFuncSetAttribute((const void*)ag_gemm_relu_kernel, hipFuncAttributeMaxDynamicSharedMemorySize, (int)kLdsBytes);
  if (e != hipSuccess) { fprintf(stderr, "hipFuncSetAttribute: %s\n", hipGetErrorString(e)); return; }
  hipLaunchKernelGGL(ag_gemm_relu_kernel, dim3(kNWG), dim3(512), kLdsBytes, stream, p);
  e = hipGetLastError();
  if (e != hipSuccess) fprintf(stderr, "launch: %s\n", hipGetErrorString(e));
}
```

```cpp
#include <hip/hip_runtime.h>
#include <rccl/rccl.h>
#include <cstdio>
#include <cstdint>
#include <type_traits>

typedef _Float16 f16x8 __attribute__((ext_vector_type(8)));
typedef float f32x4 __attribute__((ext_vector_type(4)));
typedef unsigned u32x4 __attribute__((ext_vector_type(4)));
typedef unsigned u32x2 __attribute__((ext_vector_type(2)));
#define LAS __attribute__((address_space(3)))
#define GAS __attribute__((address_space(1)))

#define NWS 9u
#define WPM 3u
#define PU 1u
#define PUW 1u
#define CVDELAY 24
#define CVFIRST 60
#define CVLEAD 40u
#define XSLEEP 4
#define CVFROM 13
#define PWIN 4u
#define CVAUX kAuxSys
#define MSLEEP 4
#define LSLEEP 2
#define WLIM 80u
#define SKIPA 22
#define ALANES 63
#define PTMO kOob
#define TMOEV 7u
#define SD 4
#define MASKL 0xffaaaaaaaaaa0000ull
#define MASKO MASKL
#define LDSPARK
#define NPF 8
#define SKIPB 12
#define PSLEEP 1
#define LTHR 24u

#define LDCOND(j) ((j) + 40u >= total)
namespace {

constexpr unsigned kBlk = 16384u;
constexpr unsigned kPanel = 2u << 20;
constexpr unsigned kAOff = 2u << 20;
constexpr unsigned kASize = 36u * kPanel;
constexpr unsigned kBOff = kAOff + 2u * kASize;
constexpr unsigned kBSize = 16u * kPanel;
constexpr unsigned kLast = 23u * kPanel + (1u << 20);
constexpr unsigned kPiece = 16896u;
constexpr unsigned kOOff = kBOff + 2u * kBSize;
constexpr unsigned kOSize = 4u * 224u * kPiece;
constexpr unsigned kPart = kOOff + 2u * kOSize;
constexpr unsigned kWsNeed = kPart + 96u * 16384u;
constexpr unsigned kPFlags = 131072;
constexpr unsigned kTmo = 128, kDone = 256, kCvt = 384, kAll = 512, kQueue = 1024;
constexpr unsigned kTicket = 4096;
constexpr unsigned kFlags = 1u << 20, kRep = 57600;
constexpr unsigned kOFlags = 81920;
constexpr unsigned kSrc = 98304;
constexpr unsigned kInBytes = 32u << 20;
constexpr int kNMove = 48, kNComp = 208, kNWG = 256;
constexpr unsigned kRowsI = 704u, kRowsC = 688u;
constexpr unsigned kDonePerCall = 192u;
constexpr unsigned long long kDeadline = 50000000ull;
constexpr unsigned kSlot = 24576u, kRing = 6u * kSlot;
constexpr unsigned kPBuf = 17408u;
constexpr unsigned kLdsMisc = 9u * kPBuf, kLdsBytes = kLdsMisc + 128u;
constexpr unsigned kPartLds = NPF * 16384u;
static_assert(kLdsMisc >= kRing && kPartLds + 16384u <= kRing, "");
constexpr unsigned kOob = 0x80000000u;
constexpr int kAuxSys = 17;
constexpr int kAuxDev = 16;

struct Params {
  const float* x;
  const float* w;
  char* ws[8];
  float* out[8];
  int rank;
  int pad;
  uint32_t tiles[3 * kNComp];
};

__device__ __forceinline__ __amdgpu_buffer_rsrc_t mk_rsrc(const void* p, unsigned bytes) {
  return __builtin_amdgcn_make_buffer_rsrc((void*)p, 0, bytes, 0x00020000);
}
template <int AUX>
__device__ __forceinline__ f32x4 ld16(__amdgpu_buffer_rsrc_t r, unsigned voff, unsigned soff) {
  return __builtin_bit_cast(f32x4, __builtin_amdgcn_raw_buffer_load_b128(r, voff, soff, AUX));
}
template <int AUX>
__device__ __forceinline__ void st16(f16x8 v, __amdgpu_buffer_rsrc_t r, unsigned voff, unsigned soff) {
  __builtin_amdgcn_raw_buffer_store_b128(__builtin_bit_cast(u32x4, v), r, voff, soff, AUX);
}
__device__ __forceinline__ f16x8 cvt8(f32x4 a, f32x4 b) {
  f16x8 h;
#pragma unroll
  for (int i = 0; i < 4; ++i) { h[i] = (_Float16)a[i]; h[4 + i] = (_Float16)b[i]; }
  return h;
}
__device__ __forceinline__ f16x8 cvt_col(const f32x4 (&v)[8], int i) {
  f16x8 h;
#pragma unroll
  for (int j = 0; j < 8; ++j) h[j] = (_Float16)v[j][i];
  return h;
}
#define WAIT_VM(n) asm volatile("s_waitcnt vmcnt(%0)" :: "n"(n) : "memory")
#define WAIT_LGKM0() asm volatile("s_waitcnt lgkmcnt(0)" ::: "memory")
#define FENCE() asm volatile("" ::: "memory")
#define UNLIKELY(c) __builtin_expect(!!(c), 0)
#define BARRIER() do { FENCE(); __builtin_amdgcn_s_barrier(); FENCE(); } while (0)

__device__ __forceinline__ void flag_store(char* p, unsigned v) {
  __hip_atomic_store((GAS unsigned*)p, v, __ATOMIC_RELAXED, __HIP_MEMORY_SCOPE_SYSTEM);
}
__device__ __forceinline__ void flag_store2(char* p, unsigned lo, unsigned hi) {
  __hip_atomic_store((GAS unsigned long long*)p, (unsigned long long)lo | ((unsigned long long)hi << 32), __ATOMIC_RELAXED, __HIP_MEMORY_SCOPE_SYSTEM);
}
__device__ __forceinline__ unsigned draw(char* p) {
  unsigned z = 0;
  asm volatile("" : "+v"(z));
  p += z;
  return __hip_atomic_fetch_add((GAS unsigned*)p, 1u, __ATOMIC_RELAXED, __HIP_MEMORY_SCOPE_AGENT);
}
__device__ __forceinline__ unsigned uni(unsigned v) { return (unsigned)__builtin_amdgcn_readfirstlane((int)v); }
__device__ __forceinline__ unsigned lanes_below(unsigned long long b) {
  return __builtin_amdgcn_mbcnt_hi((unsigned)(b >> 32), __builtin_amdgcn_mbcnt_lo((unsigned)b, 0u));
}
__device__ __forceinline__ void advance(unsigned& pos, unsigned long long b) {
  asm("s_lshl2_add_u32 %0, %1, %0" : "+s"(pos) : "s"((unsigned)__builtin_popcountll(b)) : "scc");
}

#define VSTAMP(ws, slot, val) do {} while (0)
#define STAMP(ws, slot) do {} while (0)
#define PSTAMP(ws, cls, n) do {} while (0)
#define CSTAMP(ws, slot) do {} while (0)

__device__ __forceinline__ void set_timeout(char* myws, unsigned call) {
  __hip_atomic_store((GAS unsigned*)(myws + kTmo), call, __ATOMIC_RELAXED, __HIP_MEMORY_SCOPE_AGENT);
}
__device__ __forceinline__ unsigned peer_of(unsigned r, unsigned i) { return (r & 4u) + ((r + i + 1u) & 3u); }
__device__ __forceinline__ unsigned panel_row(unsigned ap, unsigned rh, unsigned rank) {
  const unsigned og = (rank & 4u) ^ 4u;
  if (ap < 8u) return rank * 1024u + ap * 128u + rh * 64u;
  if (ap < 23u) { const unsigned i = (ap - 8u) / 5u, s = (ap - 8u) - 5u * i; return peer_of(rank, i) * 1024u + s * 128u + rh * 64u; }
  if (ap < 26u) return peer_of(rank, ap - 23u) * 1024u + 640u;
  if (ap < 34u) return (og + ((ap - 26u) >> 1)) * 1024u + 768u + (ap & 1u) * 128u + rh * 64u;
  return (og + 2u * (ap - 34u) + rh) * 1024u + 704u;
}
__device__ __forceinline__ unsigned park_pos(unsigned ap, unsigned rh, unsigned j) {
  if (ap < 8u) return 2u * ap + rh < 11u ? 2u * ap + rh : 99u;
  if (ap < 23u) {
    const unsigned i = (ap - 8u) / 5u, s = (ap - 8u) - 5u * i;
    return s < 2u ? 11u + 4u * i + 2u * s + rh : s < 4u ? 40u + 4u * i + 2u * s + rh : 26u + 2u * i + rh;
  }
  const unsigned src = ap < 34u ? (ap - 26u) >> 1 : 2u * (ap - 34u) + rh;
  if (src == j) return 99u;
  const unsigned e = src - (src > j ? 1u : 0u);
  return ap < 34u ? 32u + 4u * e + 2u * (ap & 1u) + rh : 23u + e;
}
__device__ __forceinline__ unsigned park_row(unsigned hp, unsigned src, unsigned rank) {
  const unsigned q = rank & 3u;
  if (hp < 11u) return src * 1024u + hp * 64u;
  if (hp < 23u) return peer_of(src, (hp - 11u) >> 2) * 1024u + ((hp - 11u) & 3u) * 64u;
  if (hp < 26u) { const unsigned e = hp - 23u; return ((rank & 4u) + e + (e >= q ? 1u : 0u)) * 1024u + 704u; }
  if (hp < 32u) return peer_of(src, (hp - 26u) >> 1) * 1024u + 512u + (hp & 1u) * 64u;
  if (hp < 44u) { const unsigned e = (hp - 32u) >> 2; return ((rank & 4u) + e + (e >= q ? 1u : 0u)) * 1024u + 768u + (hp & 3u) * 64u; }
  return peer_of(src, (hp - 44u) >> 2) * 1024u + 256u + (hp & 3u) * 64u;
}

}

__global__ void __launch_bounds__(512) __attribute__((amdgpu_waves_per_eu(2, 2)))
ag_gemm_relu_kernel(const Params p) {
  extern __shared__ __attribute__((aligned(16))) char smem_raw[];
  LAS char* const lds = (LAS char*)smem_raw;
  LAS volatile unsigned* const misc = (LAS volatile unsigned*)(lds + kLdsMisc);

  const int tid = threadIdx.x;
  const int lane = tid & 63;
  const int wid = __builtin_amdgcn_readfirstlane(tid >> 6);
  const int bid = blockIdx.x;
  const unsigned rank = (unsigned)p.rank & 7u;
  const unsigned q = rank & 3u, og = (rank & 4u) ^ 4u;
  char* const myws = p.ws[rank];
  const unsigned l16 = (unsigned)lane * 16u;
  const __amdgpu_buffer_rsrc_t rws = mk_rsrc(myws, kWsNeed);
  const int fr = lane & 15, fq = lane >> 4;

  STAMP(myws, wid == 0 ? 0 : 31);

  const unsigned lx = (((unsigned)lane >> 2) * 8192u + 4u * (unsigned)((lane & 3) ^ ((-(lane >> 4)) & 3))) * 4u;
  const unsigned lw = (4u * ((unsigned)lane >> 4) * 1024u + 4u * ((unsigned)lane & 15u)) * 4u;
  const __amdgpu_buffer_rsrc_t rx = mk_rsrc(p.x, kInBytes);
  const __amdgpu_buffer_rsrc_t rw = mk_rsrc(p.w, kInBytes);
  const unsigned pw = (unsigned)wid >= WPM ? (unsigned)wid - WPM : (unsigned)wid;
  const unsigned gw = (unsigned)bid * WPM + pw;
  const unsigned sid = gw % 14u, k = gw / 14u;
  const bool cross = sid >= 6u;
  const unsigned c = cross ? (sid - 6u) >> 1 : sid >> 1, h = sid & 1u;
  const unsigned sr = (cross ? og + c : peer_of(rank, c)) & 7u;
  char* const sws = p.ws[sr];
  const __amdgpu_buffer_rsrc_t rs = mk_rsrc(sws, kWsNeed);
  LAS volatile unsigned* const hand = misc + 4u + 4u * pw;
  const unsigned buf0 = pw * 3u * kPBuf;
  if (bid < kNMove && (unsigned)wid >= WPM && (unsigned)wid < 2u * WPM && gw < 14u * NWS) {
    unsigned call = 0;
    auto push = [&](auto wconst) {
      constexpr bool W = decltype(wconst)::value;
      f32x4 pv[PU][2][8];
#pragma unroll
      for (unsigned u = 0; u < PU; ++u)
#pragma unroll
        for (int i = 0; i < 16; ++i) {
          const unsigned m = u * NWS + k, kk = (m >> 1) * 64u + h * 32u;
          if (W) pv[u][i >> 3][i & 7] = ld16<0>(rw, lw + (i >> 3) * 256u, (kk * 1024u + 256u * c + (m & 1u) * 128u) * 4u + ((i & 3) + 16 * ((i >> 2) & 1)) * 4096u);
          else pv[u][i >> 3][i & 7] = ld16<0>(rx, lx + (i & 1) * 64u, ((m & 1u) * 128u * 8192u + kk) * 4u + (i >> 1) * 524288u);
        }
      __syncthreads();
      call = uni(misc[0]) + 1u;
      const unsigned base = W ? kBOff + (call & 1u) * kBSize + (8u + 2u * q) * kPanel : kAOff + (call & 1u) * kASize + (18u - 5u * c) * kPanel;
      char* const fl = sws + kFlags + (W ? 44u + 2u * q : 18u - 5u * c) * 1024u + h * 4u + (unsigned)lane * kRep;
#pragma unroll
      for (unsigned j = 0; j < 2u * PU; ++j) {
        const unsigned u = j >> 1, r = j & 1u, m = u * NWS + k;
        f16x8 hv[4];
#pragma unroll
        for (int i = 0; i < 4; ++i) hv[i] = W ? cvt_col(pv[u][r], i) : cvt8(pv[u][r][2 * i], pv[u][r][2 * i + 1]);
        asm volatile("" :: "v"(hv[0]), "v"(hv[1]), "v"(hv[2]), "v"(hv[3]));
        if (j >= PUW) WAIT_VM(4u * (PUW - 1u));
        if (j <= 2u) STAMP(myws, 8 + wid + 3 * j);
        if (u > 0u && r == PUW - 1u && lane < 8) flag_store(fl + ((m - NWS) & 1u) * 1024u + ((m - NWS) >> 1) * 8u, call);
#pragma unroll
        for (int i = 0; i < 4; ++i) st16<kAuxSys>(hv[i], rs, l16 + i * 1024u, base + (m & 1u) * kPanel + (m >> 1) * kBlk + h * 8192u + r * 4096u);
        FENCE();
      }
      WAIT_VM(0);
      const unsigned m = (PU - 1u) * NWS + k;
      if (lane < 8) flag_store(fl + (m & 1u) * 1024u + (m >> 1) * 8u, call);
    };
    if (cross) push(std::true_type{}); else push(std::false_type{});
    STAMP(myws, 4 + wid);
    if (!cross) return;
    const __amdgpu_buffer_rsrc_t ro = mk_rsrc(p.out[rank], kInBytes);
    const unsigned vo = (unsigned)fq * 16384u + (unsigned)fr * 16u;
    const unsigned long long t0 = __builtin_amdgcn_s_memrealtime();
    for (unsigned b = 0;; b = b == 2u ? 0u : b + 1u) {
      unsigned hv;
      while ((hv = uni(hand[b])) == 0u) {
        if (__builtin_amdgcn_s_memrealtime() - t0 > 2u * kDeadline) { set_timeout(myws, call); return; }
        __builtin_amdgcn_s_sleep(XSLEEP);
      }
      FENCE();
      if (hv == ~0u) break;
      const unsigned oa = park_row(hv >> 4, sr, rank) * 4096u + (256u * c + (hv & 12u) * 16u) * 4u;
      unsigned pos = (unsigned)(size_t)lds + buf0 + b * kPBuf, val[64];
      u32x2 mk = {~0u, ~0u};
      if ((hv & 2u) == 0u) mk = *(LAS u32x2*)(size_t)(pos + lane * 8);
      const unsigned fill = (unsigned)(size_t)lds + kLdsMisc + 64u + (hv & 2u) * 2u;
      pos += 512u;
#pragma unroll
      for (int i = 0; i < 64; ++i) {
        const bool keep = (int)(mk[i >> 5] << (i & 31)) >= 0;
        const unsigned long long bl = __ballot(keep);
        unsigned a = lanes_below(bl) * 4u + pos;
        asm volatile("" : "+v"(a));
        val[i] = *(LAS unsigned*)(size_t)(keep ? a : fill);
        advance(pos, bl);
      }
      WAIT_LGKM0();
      if (lane == 0) hand[b] = 0u;
      FENCE();
#pragma unroll
      for (int i = 0; i < 16; ++i)
        __builtin_amdgcn_raw_buffer_store_b128(u32x4{val[4 * i], val[4 * i + 1], val[4 * i + 2], val[4 * i + 3]}, ro, vo, oa + (i >> 2) * 65536u + (i & 3) * 4096u, kAuxSys);
    }
    STAMP(myws, 1 + wid);
    return;
  }
  const bool isconv = bid >= kNMove + 8 * CVFROM;
  unsigned tk = 0;
  if (tid == 0) tk = draw(myws + kTicket + (unsigned)bid * 32u);
  FENCE();
  f32x4 cv[8];
  auto item = [&](unsigned n, unsigned& ktg, unsigned& sub) {
    if (n < 5u * CVLEAD) {
      ktg = n / 5u; sub = n - 5u * ktg; sub += sub ? 3u : 0u;
    }
    else if (n < 8u * CVLEAD) { ktg = (n - 5u * CVLEAD) / 3u; sub = n - 5u * CVLEAD - 3u * ktg + 1u; }
    else { ktg = n >> 3; sub = n & 7u; }
  };
  auto loads = [&](unsigned n) {
    unsigned ktg, sub;
    item(n, ktg, sub);
    if (sub < 4u) {
      const unsigned so = ((256u * sub + 32u * wid) * 8192u + ktg * 64u) * 4u;
#pragma unroll
      for (int s = 0; s < 2; ++s) {
        cv[4 * s + 0] = ld16<0>(rx, lx, so + s * 524288u); cv[4 * s + 1] = ld16<0>(rx, lx + 64u, so + s * 524288u);
        cv[4 * s + 2] = ld16<0>(rx, lx + 128u, so + s * 524288u); cv[4 * s + 3] = ld16<0>(rx, lx + 192u, so + s * 524288u);
      }
    } else {
      const unsigned so = ((ktg * 64u + 32u * (wid & 1)) * 1024u + 256u * (sub - 4u) + 64u * (wid >> 1)) * 4u;
#pragma unroll
      for (int i = 0; i < 8; ++i) cv[i] = ld16<0>(rw, lw, so + ((i & 3) + 16 * (i >> 2)) * 4096u);
    }
  };
  if (bid >= kNMove + 8 * CVFROM + CVFIRST) __builtin_amdgcn_s_sleep(CVDELAY);
  if (isconv) loads((unsigned)(bid - kNMove - 8 * CVFROM));
  FENCE();
  if (tid == 0) {
    misc[0] = tk;
    misc[3] = 0u;
  }
  if (tid >= 4 && tid < 18) misc[tid] = tid == 17 ? 0x7fc00000u : 0u;
  __syncthreads();
  const unsigned call = uni(misc[0]) + 1u;
  const unsigned par = call & 1u;
  const unsigned abase = kAOff + par * kASize, bbase = kBOff + par * kBSize, obuf = kOOff + par * kOSize;

  if (bid < kNMove) {
    if (wid == 7) {
      if (bid == 0 && lane == 0) {
        const unsigned target = call * kDonePerCall;
        const unsigned long long t0 = __builtin_amdgcn_s_memrealtime();
        bool failed = false;
        while ((int)(__hip_atomic_load((GAS unsigned*)(myws + kDone), __ATOMIC_RELAXED, __HIP_MEMORY_SCOPE_SYSTEM) - target) < 0) {
          if (__hip_atomic_load((GAS unsigned*)(myws + kTmo), __ATOMIC_RELAXED, __HIP_MEMORY_SCOPE_AGENT) == call ||
              __builtin_amdgcn_s_memrealtime() - t0 > kDeadline) {
            set_timeout(myws, call);
            failed = true;
            break;
          }
          __builtin_amdgcn_s_sleep(2);
        }
        __builtin_amdgcn_fence(__ATOMIC_ACQUIRE, "");
        if (failed) __hip_atomic_store((GAS unsigned*)(p.out[rank] + (size_t)rank * 1024u * 1024u), 0x7fc00000u, __ATOMIC_RELAXED, __HIP_MEMORY_SCOPE_SYSTEM);
        STAMP(myws, 10);
      }
      return;
    }
    if ((unsigned)wid >= WPM || gw >= 14u * NWS) return;
    const unsigned total = cross ? kRowsC : kRowsI;
    const unsigned nplain = cross ? 576u : kRowsI;
    const unsigned qbase = (call - 1u) * (total - PU * NWS) - (PU + 1u) * NWS;
    bool failed = false, all = false;
    unsigned cgrp = 99u, okf = 0, okt = 0, fv = 0, wbase = 0x10000u;
    u32x2 fw = {0u, 0u};
    unsigned sa = 0, da = 0, fa = 0, pend = 0;
    unsigned s2 = 4096u, d2 = 4096u;
    u32x4 v[4], w[4];
    auto raise = [&]() {
      if (pend != 0u && lane < 8) flag_store(myws + pend + (unsigned)lane * kRep, call);
      pend = 0u;
    };
    auto fetch = [&](u32x4 (&x)[4], unsigned so) {
#pragma unroll
      for (int i = 0; i < 4; ++i) x[i] = __builtin_amdgcn_raw_buffer_load_b128(rs, l16 + i * 1024u, so, kAuxSys);
      FENCE();
    };
    auto put = [&](const u32x4 (&x)[4], unsigned so) {
#pragma unroll
      for (int i = 0; i < 4; ++i) __builtin_amdgcn_raw_buffer_store_b128(x[i], rws, l16 + i * 1024u, so, kAuxDev);
      FENCE();
    };
    auto setup = [&](unsigned m) {
      unsigned sp, kt, dp, db, dh = h;
      unsigned so = h * 8192u, dof = h * 8192u;
      s2 = 4096u; d2 = 4096u;
      if (!cross) {
        if (m < 256u) { sp = m & 1u; kt = m >> 1; dp = 8u + 5u * c + sp; db = kt; }
        else if (m < 384u) { sp = 4u; kt = m - 256u; dp = 12u + 5u * c; db = kt; }
        else if (m < 640u) { sp = 2u + (m & 1u); kt = (m - 384u) >> 1; dp = 8u + 5u * c + sp; db = kt; }
        else { sp = 5u; db = m - 640u; kt = 2u * db + h; so = kLast - 5u * kPanel - kt * 8192u; dp = 23u + c; }
      } else {
        if (m < 256u) { sp = 36u + 2u * q + (m & 1u); kt = m >> 1; dp = 44u + 2u * c + (m & 1u); db = kt; }
        else {
          const unsigned g = ((m - 256u) * 52429u) >> 18, r = m - 256u - 5u * g;
          if (r == 2u) { sp = 5u; kt = 2u * g + h; so = 4096u; s2 = 8192u; dp = 34u + (c >> 1); db = kt; dh = c & 1u; dof = dh * 4096u; d2 = 8192u; }
          else { sp = 6u + (r % 3u); kt = 2u * g + (r > 2u ? 1u : 0u); dp = 20u + 2u * c + sp; db = kt; }
        }
      }
      const unsigned sub = sp < 36u ? sp >> 1 : (sp - 28u) >> 1;
      auto ask = [&](unsigned k0) {
        const unsigned off = lane < 62 ? kSrc + (sr * 8u + sub) * 512u + min(k0 + (unsigned)lane, 127u) * 4u : lane == 62 ? kAll + sr * 4u : kTmo;
        asm volatile("buffer_load_dword %0, %1, %2, 0 offen sc0 sc1" : "=v"(fv) : "v"(off), "s"(rws) : "memory");
      };
      auto look = [&]() {
        asm volatile("" : "+v"(fv) :: "memory");
        const unsigned long long b = __ballot(lane < 63 ? (int)(fv - call) >= 0 : fv == call);
        if ((b >> 62) != 0ull) { failed = (b >> 63) != 0ull; all = true; }
        okt += (unsigned)__builtin_ctzll(~b);
      };
      if (!all) {
        if (cgrp != 99u) look();
        if (!all && !(sub == cgrp && kt >= okf && kt < okt)) {
          WAIT_VM(0);
          raise();
          cgrp = sub;
          okf = kt;
          const unsigned long long t0 = __builtin_amdgcn_s_memrealtime();
          for (;;) {
            okt = kt;
            ask(kt);
            WAIT_VM(0);
            look();
            if (all || okt != kt) break;
            if (__builtin_amdgcn_s_memrealtime() - t0 > kDeadline) { set_timeout(myws, call); failed = true; all = true; break; }
            __builtin_amdgcn_s_sleep(MSLEEP);
          }
        }
        if (!all) ask(okt);
      }
      sa = (sp < 36u ? abase + sp * kPanel : bbase + (sp - 36u) * kPanel) + kt * kBlk + so;
      da = (dp < 36u ? abase + dp * kPanel : bbase + (dp - 36u) * kPanel) + db * kBlk + dof;
      fa = kFlags + dp * 1024u + db * 8u + dh * 4u;
      fetch(v, sa);
    };
    unsigned m = PU * NWS + k;
    setup(m);
    STAMP(myws, 20 + wid);
    do {
      unsigned dv = 0;
      if (lane == 0) dv = draw(myws + kQueue + sid * 128u);
      FENCE();
      put(v, da);
      raise();
      fetch(w, sa + s2);
      put(w, da + d2);
      pend = fa;
      if (h == 0u) { PSTAMP(myws, sid >> 1, m + 1u); if (m < 128u) PSTAMP(myws, 8u + (sid >> 1), (m + 1u) * 4u); }
      m = uni(dv) - qbase;
      if (m < nplain) setup(m);
    } while (m < nplain);
    WAIT_VM(0);
    raise();
    if (cross) {
      unsigned hval = 0;
      const unsigned fo = lane < 32 ? kOFlags + par * 8192u + c * 2048u + h * 8u + (unsigned)lane * 16u : kTmo;
      auto stale = [&](unsigned o) {
        return o - wbase >= 32u || (int)((unsigned)__builtin_amdgcn_readlane((int)fw[0], (int)(o - wbase)) - call) < 0;
      };
      for (unsigned b = 0, pb = 0;; pb = b, b = b == 2u ? 0u : b + 1u) {
        const unsigned o = m - 576u, s = 2u * o + h;
        unsigned cnt = 2048u, dv = 0;
        if (m >= total || stale(o)) {
          WAIT_VM(0);
          if (hval != 0u && lane == 0) hand[pb] = hval;
          hval = 0u;
          const unsigned long long t0 = __builtin_amdgcn_s_memrealtime();
          wbase = o;
          while (!failed && m < total) {
            fw = __builtin_bit_cast(u32x2, __hip_atomic_load((GAS unsigned long long*)(myws + fo + (lane < 32 ? o * 16u : 0u)), __ATOMIC_RELAXED, __HIP_MEMORY_SCOPE_SYSTEM));
            const unsigned long long bl = __ballot(lane < 32 ? (int)(fw[0] - call) >= 0 : fw[0] == call);
            if ((bl >> 32) != 0ull) { failed = true; break; }
            if (bl & 1ull) break;
            if (__builtin_amdgcn_s_memrealtime() - t0 > kDeadline) { set_timeout(myws, call); failed = true; }
            __builtin_amdgcn_s_sleep(MSLEEP);
          }
        }
        cnt = failed ? 0u : min((unsigned)__builtin_amdgcn_readlane((int)fw[1], (int)((o - wbase) & 31u)), 4096u);
        if (UNLIKELY(uni(hand[b]) != 0u)) {
          const unsigned long long t0 = __builtin_amdgcn_s_memrealtime();
          while (uni(hand[b]) != 0u)
            if (__builtin_amdgcn_s_memrealtime() - t0 > kDeadline) { set_timeout(myws, call); return; }
        }
        FENCE();
        if (m >= total) {
          if (lane == 0) hand[b] = ~0u;
          break;
        }
        const unsigned bytes = 512u + 4u * cnt, sb = obuf + (q * 224u + s) * kPiece, nbuf = buf0 + b * kPBuf;
        auto part = [&](unsigned f0, unsigned f1) {
          for (unsigned f = f0; f < f1; f += 1024u) {
            WAIT_VM(PWIN - 1u);
            asm volatile("s_mov_b32 m0, %0\n\ts_nop 0\n\tbuffer_load_dwordx4 %1, %2, %3 offen sc0 sc1 lds"
                         :: "s"(nbuf + f), "v"(f + l16 < bytes ? l16 : kOob), "s"(rs), "s"(sb + f) : "memory");
          }
        };
        asm volatile("s_mov_b64 exec, 1\n\tglobal_atomic_add %0, %1, %2, %3 sc0\n\ts_mov_b64 exec, -1"
                     : "=v"(dv) : "v"(kQueue + sid * 128u), "v"(1u), "s"(myws) : "memory");
        part(0u, min(bytes, (PWIN - 1u) * 1024u));
        if (bytes <= (PWIN - 2u) * 1024u) WAIT_VM(0);
        WAIT_VM(PWIN - 1u);
        if (hval != 0u && lane == 0) hand[pb] = hval;
        hval = 4u * s + (failed ? 3u : 1u);
        if (h == 0u) PSTAMP(myws, sid >> 1, m + 1u);
        asm volatile("" : "+v"(dv));
        m = uni(dv) - qbase;
        if (m < total && stale(m - 576u)) {
          wbase = m - 576u;
          asm volatile("buffer_load_dwordx2 %0, %1, %2, 0 offen sc0 sc1" : "=v"(fw) : "v"(fo + (lane < 32 ? wbase * 16u : 0u)), "s"(rws) : "memory");
          part((PWIN - 1u) * 1024u, bytes);
          if (bytes <= (2u * PWIN - 3u) * 1024u) WAIT_VM(0);
          WAIT_VM(PWIN - 1u);
          asm volatile("" : "+v"(fw) :: "memory");
        } else
          part((PWIN - 1u) * 1024u, bytes);
      }
    }
    STAMP(myws, 1 + wid);
    return;
  }

  {
    constexpr unsigned kNConv = 8u * (26u - CVFROM);
    char* const pws = p.ws[lane & 7];
    const unsigned mine = 15u << (rank & 4u);
    STAMP(myws, wid == 0 ? 27 : 31);
    for (unsigned n = isconv ? (unsigned)(bid - kNMove - 8 * CVFROM) : 1024u; n < 1024u; n += kNConv) {
      unsigned ktg, sub;
      item(n, ktg, sub);
      if (sub < 4u) {
        const unsigned so = abase + (2u * sub + (wid >> 2)) * kPanel + ktg * kBlk + (wid & 3) * 2048u;
#pragma unroll
        for (int i = 0; i < 4; ++i) st16<CVAUX>(cvt8(cv[2 * i], cv[2 * i + 1]), rws, l16 + (i >> 1) * 1024u, so + (i & 1) * 8192u);
        if (sub == 2u && (wid >> 1) == 2) {
          const int from = (4 * fr + (fq ^ ((-(fr >> 2)) & 3))) * 4;
#pragma unroll
          for (int i = 0; i < 4; ++i) {
            u32x4 hv = __builtin_bit_cast(u32x4, cvt8(cv[2 * i], cv[2 * i + 1]));
#pragma unroll
            for (int e = 0; e < 4; ++e) hv[e] = (unsigned)__builtin_amdgcn_ds_bpermute(from, (int)hv[e]);
            __builtin_amdgcn_raw_buffer_store_b128(hv, rws, l16 + (i >> 1) * 1024u, abase + kLast + ktg * 8192u + (wid & 1) * 2048u + (i & 1) * 4096u, CVAUX);
          }
        }
      } else {
        const unsigned so = bbase + (2u * (sub - 4u) + (wid >> 2)) * kPanel + ktg * kBlk + ((wid >> 1) & 1) * 4096u + (wid & 1) * 8192u;
#pragma unroll
        for (int i = 0; i < 4; ++i) st16<CVAUX>(cvt_col(cv, i), rws, l16 + i * 1024u, so);
      }
      FENCE();
      const bool ahead = n >= kNConv && n + kNConv < 1024u;
      if (ahead) { loads(n + kNConv); WAIT_VM(8); } else { WAIT_VM(0); }
      if (n < kNConv) STAMP(myws, wid == 0 ? 30 : 31);
      BARRIER();
      if (n < 2u * kNConv) STAMP(myws, wid == 0 ? (n < kNConv ? 28 : 29) : 31);
      const unsigned to = sub < 2u ? mine : sub == 2u ? 255u : sub == 3u ? mine ^ 255u : 1u << (og + sub - 4u);
      if (tid < 16) flag_store2(myws + kFlags + ((sub < 4u ? 2u * sub : 28u + 2u * sub) + ((unsigned)tid & 1u)) * 1024u + ktg * 8u + ((unsigned)tid >> 1) * kRep, call, call);
      else if (wid == 1 && lane < 8 && (((to & ~(1u << rank)) >> lane) & 1u)) flag_store(pws + kSrc + rank * 4096u + sub * 512u + ktg * 4u, call);
      FENCE();
      if (n < kNConv) loads(n + kNConv);
    }
    if (isconv && wid == 7) {
      unsigned dv = 0;
      if (lane == 0) dv = draw(myws + kCvt);
      if (uni(dv) == call * kNConv - 1u && lane < 8) flag_store(pws + kAll + rank * 4u, call);
    }
  }
  STAMP(myws, wid == 0 ? 1 : 31);

  const uint32_t* const tl = p.tiles + 3 * (bid - kNMove);
  const unsigned small = tl[(tl[0] >> 28) - 1u] & 16383u;
  const bool hass = ((small >> 10) & 3u) == 2u;
  const int ntiles = (int)(tl[0] >> 28) - (hass ? 1 : 0);
  const bool late = (small >> 13) != 0u;
  const unsigned tau = ((small >> 6) & 15u) * 6u + (small & 63u) - 23u + 3u * ((small >> 12) & 1u);
  if (wid == 7) return;

  if (wid >= 4) {
    const unsigned lw = (unsigned)wid - 4u;
    const unsigned total = 256u * (unsigned)ntiles;
    unsigned a = 0, aend = 0, so = lw * 8192u, src = 0, idle = 0;
    unsigned ok = 0, f0 = 0, f1 = 0, fb = 0, fv = 0;
    unsigned rdy = 0, askat = 0, lookat = ~0u;
    int ta = 0, tf = 0;
    auto next_flags = [&]() {
      while (tf < ntiles) {
        const unsigned code = tl[tf++];
        const unsigned ap = code & 63u, bp = (code >> 6) & 15u, shape = (code >> 10) & 3u;
        f0 = f1;
        f1 += 256u;
        if (shape == 3u && lw == 2u) { ok = f1; continue; }
        const unsigned grp = lw == 0u ? ap : shape == 0u ? (lw == 1u ? ap + 1u : 36u + bp) : shape == 1u ? 35u + bp + lw : 36u + bp;
        fb = kFlags + grp * 1024u + ((unsigned)bid & 7u) * kRep;
        break;
      }
    };
    auto known = [&]() {
      rdy = min(ok, aend);
      unsigned from = max(ok, LTHR);
      asm volatile("" : "+s"(from));
      askat = ok >= total ? ~0u : from - LTHR;
    };
    auto ask = [&](unsigned last) {
      const unsigned off = lane < ALANES ? fb + min(ok - f0 + (unsigned)lane, 255u) * 4u : lane == 63 ? last : kOob;
      asm volatile("buffer_load_dword %0, %1, %2, 0 offen sc0 sc1" : "=v"(fv) : "v"(off), "s"(rws) : "memory");
    };
    auto give_up = [&]() {
      ok = total;
      if (lane == 0) misc[3] = 1u;
      WAIT_LGKM0();
    };
    auto look = [&]() {
      asm volatile("" : "+v"(fv) :: "memory");
      const unsigned long long b = __ballot(lane < 63 ? (int)(fv - call) >= 0 : fv == call);
      if (UNLIKELY((b >> 63) != 0ull)) {
        give_up();
      } else {
        const unsigned run = (unsigned)__builtin_ctzll(~b);
        ok = min(ok + (run & ~1u), f1);
        if (UNLIKELY(ok == f1)) next_flags();
      }
      lookat = ~0u;
    };
    auto issue = [&](bool busy) {
      LAS char* const dl = lds + so;
      if (busy) {
        __builtin_amdgcn_raw_ptr_buffer_load_lds(rws, (LAS void*)dl, 16, l16, src, 0, kAuxDev);
        __builtin_amdgcn_raw_ptr_buffer_load_lds(rws, (LAS void*)dl, 16, l16, src, 1024, kAuxDev);
        __builtin_amdgcn_raw_ptr_buffer_load_lds(rws, (LAS void*)dl, 16, l16, src, 2048, kAuxDev);
        __builtin_amdgcn_raw_ptr_buffer_load_lds(rws, (LAS void*)dl, 16, l16, src, 3072, kAuxDev);
        __builtin_amdgcn_raw_ptr_buffer_load_lds(rws, (LAS void*)(dl + 4096), 16, l16, src + 4096u, 0, kAuxDev);
        __builtin_amdgcn_raw_ptr_buffer_load_lds(rws, (LAS void*)(dl + 4096), 16, l16, src + 4096u, 1024, kAuxDev);
        __builtin_amdgcn_raw_ptr_buffer_load_lds(rws, (LAS void*)(dl + 4096), 16, l16, src + 4096u, 2048, kAuxDev);
        __builtin_amdgcn_raw_ptr_buffer_load_lds(rws, (LAS void*)(dl + 4096), 16, l16, src + 4096u, 3072, kAuxDev);
      }
      FENCE();
      src += 8192u;
      so = so >= kRing - kSlot ? so - (kRing - kSlot) : so + kSlot;
      ++a;
    };
    next_flags();
    known();
    for (unsigned j = 0; j < total; ++j) {
      if (a == j + 4u) {
        unsigned e = min(rdy + 1u, askat);
        asm volatile("" : "+s"(e));
        e = min(e, lookat);
        if (idle != 0u) {
          WAIT_VM(0);
          while (a + 1u < e) { issue(false); BARRIER(); }
        }
        while (a + 1u < e) {
          issue(true);
          WAIT_VM(32);
          BARRIER();
        }
        j = a - 4u;
      }
      unsigned lim = min(j + 5u, rdy);
      if (UNLIKELY(a >= lim)) {
        if (a == aend) {
          const unsigned code = tl[ta++];
          const unsigned ap = code & 63u, bp = (code >> 6) & 15u, shape = (code >> 10) & 3u;
          if (lw == 0u || (lw == 1u && shape == 0u)) src = abase + (ap + lw) * kPanel;
          else src = bbase + (bp + (shape == 1u ? lw - 1u : 0u)) * kPanel;
          idle = shape == 3u && lw == 2u ? 1u : 0u;
          aend = ta == ntiles ? ~0u : aend + 256u;
          rdy = min(ok, aend);
        }
        if (a <= j && ok <= a) {
          WAIT_VM(0);
          if (lookat != ~0u) look();
          const unsigned long long t0 = __builtin_amdgcn_s_memrealtime();
          for (unsigned it = 0; ok <= a; ++it) {
            ask((it & TMOEV) == TMOEV ? kTmo : kOob);
            WAIT_VM(0);
            look();
            if (ok > a) break;
            if (__builtin_amdgcn_s_memrealtime() - t0 > kDeadline) { set_timeout(myws, call); give_up(); break; }
            __builtin_amdgcn_s_sleep(LSLEEP);
          }
          known();
        }
        lim = min(j + 5u, rdy);
      }
      if (a < lim) {
        issue(idle == 0u);
        if (UNLIKELY(a < lim)) issue(idle == 0u);
      }
      if (UNLIKELY(a >= askat)) { ask(PTMO); askat = ~0u; lookat = a + 4u; }
      const unsigned n = idle != 0u ? 0u : a - 1u - j;
      if (UNLIKELY(n < 4u)) {
        WAIT_VM(24);
        if (n < 3u) {
          WAIT_VM(16);
          if (n < 2u) {
            WAIT_VM(8);
            if (n < 1u) WAIT_VM(0);
          }
        }
      }
      WAIT_VM(32);
      if (UNLIKELY(j + 5u >= lookat)) { look(); known(); }
      BARRIER();
    }
    if (hass && lw < 2u) {
      const unsigned rep = kFlags + ((unsigned)bid & 7u) * kRep;
      const unsigned long long t0 = __builtin_amdgcn_s_memrealtime();
      unsigned long long mine = (small & 512u ? MASKL : MASKO) ^ (late ? 0ull : ~0ull);
      auto expired = [&](unsigned it) {
        bool bad = __builtin_amdgcn_s_memrealtime() - t0 > kDeadline;
        if (bad) set_timeout(myws, call);
        else if ((it & TMOEV) == TMOEV) bad = uni(__hip_atomic_load((GAS unsigned*)(myws + kTmo), __ATOMIC_RELAXED, __HIP_MEMORY_SCOPE_AGENT)) == call;
        if (UNLIKELY(bad)) {
          if (lane == 0) misc[3] = 1u;
          WAIT_LGKM0();
        }
        return bad;
      };
      BARRIER();
      if (lw == 1u) {
        const unsigned bp = (small >> 6) & 15u;
        for (unsigned it = 0;; ++it) {
          const u32x4 f = __builtin_amdgcn_raw_buffer_load_b128(rws, l16, rep + (36u + bp) * 1024u, kAuxSys);
          if (__ballot((int)((f[0] - call) | (f[1] - call) | (f[2] - call) | (f[3] - call)) < 0) == 0ull || expired(it)) break;
          __builtin_amdgcn_s_sleep(PSLEEP);
        }
        if (lane == 0) misc[5] = 1u;
        for (unsigned i = NPF; i-- > 0u;) {
          const unsigned pr = 63u - (unsigned)__builtin_clzll(mine);
          mine ^= 1ull << pr;
          const unsigned sb = bbase + bp * kPanel + ((small >> 12) & 1u) * 4096u + pr * 32768u;
          WAIT_VM(32);
#pragma unroll
          for (int w = 0; w < 4; ++w) {
            LAS char* const dl = lds + (w * (NPF * 4096) + i * 4096u);
            __builtin_amdgcn_raw_ptr_buffer_load_lds(rws, (LAS void*)dl, 16, l16, sb + w * 8192u, 0, kAuxDev);
            __builtin_amdgcn_raw_ptr_buffer_load_lds(rws, (LAS void*)dl, 16, l16, sb + w * 8192u, 1024, kAuxDev);
            __builtin_amdgcn_raw_ptr_buffer_load_lds(rws, (LAS void*)dl, 16, l16, sb + w * 8192u, 2048, kAuxDev);
            __builtin_amdgcn_raw_ptr_buffer_load_lds(rws, (LAS void*)dl, 16, l16, sb + w * 8192u, 3072, kAuxDev);
          }
          FENCE();
        }
        WAIT_VM(0);
        if (lane == 0) misc[6] = 1u;
        return;
      }
      const unsigned upto = 128u - 2u * (unsigned)__builtin_clzll(mine);
      unsigned have = 0;
      bool part = late;
      for (unsigned it = 0; have < upto || part; ++it) {
        {
          const u32x2 f = __builtin_amdgcn_raw_buffer_load_b64(rws, have < upto ? (unsigned)lane * 8u : kOob, rep + (small & 63u) * 1024u, kAuxSys);
          const unsigned g = __builtin_amdgcn_raw_buffer_load_b32(rws, part ? (unsigned)(lane & 3) * 4u : kOob, kPFlags + tau * 128u, kAuxSys);
          const unsigned long long b0 = __ballot((int)(f[0] - call) < 0), b1 = __ballot((int)(f[1] - call) < 0);
          const unsigned n = max(have, min(b0 ? 2u * (unsigned)__builtin_ctzll(b0) : 128u, b1 ? 2u * (unsigned)__builtin_ctzll(b1) + 1u : 128u));
          if (n != have && lane == 0) misc[4] = n;
          if (part && __ballot((int)(g - call) < 0) == 0ull) {
#pragma unroll
            for (int i = 0; i < 16; ++i)
              __builtin_amdgcn_raw_ptr_buffer_load_lds(rws, (LAS void*)(lds + kPartLds + i * 1024), 16, l16, kPart + tau * 16384u + i * 1024u, 0, kAuxDev);
            part = false;
          }
          if (n != have) { have = n; continue; }
        }
        if (expired(it)) {
          if (lane == 0) misc[4] = 128u;
          break;
        }
        __builtin_amdgcn_s_sleep(PSLEEP);
      }
      WAIT_VM(0);
    }
    return;
  }

  {
    const unsigned slot = (unsigned)(4 * fr + (fq ^ ((-(fr >> 2)) & 3))) * 16u;
    unsigned so = 0;
    for (int ti = 0; ti < ntiles; ++ti) {
      const unsigned code = tl[ti];
      const unsigned shape = (code >> 10) & 3u;
      const unsigned wh = (unsigned)wid >> 1, wl = (unsigned)wid & 1u;
      const unsigned ap = (code & 63u) + (shape == 0u ? wh : 0u), bp = ((code >> 6) & 15u) + (shape == 1u ? wh : 0u);
      const unsigned jo = (bp - 8u) >> 1;
      const unsigned owner = (bp < 8u ? rank : og + jo) & 7u;
      const unsigned col0 = bp < 8u ? bp * 128u : 256u * q + (bp & 1u) * 128u;
      const __amdgpu_buffer_rsrc_t ro = mk_rsrc(p.out[owner], kInBytes);
      const unsigned vo = (unsigned)fq * 16384u + (unsigned)fr * 16u;

      auto run = [&](auto mconst, const unsigned aoff, const unsigned boff) {
        constexpr int M = decltype(mconst)::value;
        constexpr int NS = 256;
        const unsigned va = aoff + slot, vb = boff + l16;
        f32x4 acc[M][4];
#pragma unroll
        for (int m = 0; m < M; ++m)
#pragma unroll
          for (int n = 0; n < 4; ++n) acc[m][n] = f32x4{0.f, 0.f, 0.f, 0.f};
        f16x8 fa[2][M], fb[2][4];
        BARRIER();
        STAMP(myws, wid == 0 ? 2 + 3 * ti : 31);
        CSTAMP(myws, wid == 0 ? 12 + 2 * ti : 31);
#pragma unroll
        for (int i = 0; i < M; ++i) {
          if (i < 4) fb[0][i] = *(LAS f16x8*)(lds + (vb + so) + i * 1024);
          fa[0][i] = *(LAS f16x8*)(lds + (va + so) + i * 1024);
        }
        __builtin_amdgcn_s_waitcnt(0xC07F);
        BARRIER();
        so = so == kRing - kSlot ? 0u : so + kSlot;
        {
          unsigned ua = va + so, ub = vb + so;
          for (int t = 0; t < NS; t += 2) {
#pragma unroll
            for (int s = 0; s < 2; ++s) {
              __builtin_amdgcn_sched_barrier(0);
#pragma unroll
              for (int i = 0; i < 4 * M; ++i) {
                acc[i >> 2][i & 3] = __builtin_amdgcn_mfma_f32_16x16x32_f16(fa[s][i >> 2], fb[s][i & 3], acc[i >> 2][i & 3], 0, 0, 0);
                __builtin_amdgcn_sched_barrier(0);
                if (M == 8 && (i & 1) == 0 && i < 24) {
                  const int r = i >> 1;
                  if (r < 4) fb[s ^ 1][r] = *(LAS f16x8*)(lds + ub + r * 1024);
                  else fa[s ^ 1][r - 4] = *(LAS f16x8*)(lds + ua + (r - 4) * 1024);
                }
                if (M == 4 && i < 4) {
                  fb[s ^ 1][i] = *(LAS f16x8*)(lds + ub + i * 1024);
                  fa[s ^ 1][i] = *(LAS f16x8*)(lds + ua + i * 1024);
                }
                if (i == 4 * M - 10) { so = so == kRing - kSlot ? 0u : so + kSlot; asm volatile("" : "+s"(so)); }
                if (i == 4 * M - 6) { ua = va + so; asm volatile("" : "+v"(ua)); }
                if (i == 4 * M - 4) { ub = vb + so; asm volatile("" : "+v"(ub)); }
                if (i == 4 * M - 2) {
                  __builtin_amdgcn_s_waitcnt(0xC07F);
                  if (t < NS - 2) BARRIER();
                }
                __builtin_amdgcn_sched_barrier(0);
              }
            }
          }
          so = so == 0u ? kRing - kSlot : so - kSlot;
        }
        const bool hand_over = hass && ti == ntiles - 1;
        if (M == 4 && bp < 8u && hand_over) BARRIER();
        STAMP(myws, wid == 0 ? 3 + 3 * ti : 31);
        CSTAMP(myws, wid == 0 ? 13 + 2 * ti : 31);
        const int lo = misc[3] != 0u ? 0x7fc00000 : 0;
        if (bp >= 8u) {
          const __amdgpu_buffer_rsrc_t rw4 = __builtin_amdgcn_make_buffer_rsrc((void*)myws, 4, kWsNeed / 4u, 0x00020000);
          if (UNLIKELY(lo != 0)) {
#pragma unroll
            for (int m = 0; m < M; ++m)
#pragma unroll
              for (int n = 0; n < 4; ++n) acc[m][n] = __builtin_bit_cast(f32x4, u32x4{0x7fc00000u, 0x7fc00000u, 0x7fc00000u, 0x7fc00000u});
          }
          if constexpr (M == 4) {
            const unsigned hp = park_pos(ap, wh, jo);
            const unsigned s = hp * 4u + (bp & 1u) * 2u + wl;
            const unsigned pb0 = obuf + (jo * 224u + s) * kPiece;
            unsigned l0 = so + (unsigned)wid * kSlot;
            l0 = (unsigned)(size_t)lds + (l0 >= kRing ? l0 - kRing : l0);
            unsigned pos = l0 + 512u;
            const unsigned trash = l0 + kPBuf + (unsigned)lane * 4u;
            u32x2 mk = {0u, 0u};
#pragma unroll
            for (int m = 0; m < 4; ++m)
#pragma unroll
              for (int j = 0; j < 4; ++j)
#pragma unroll
                for (int n = 0; n < 4; ++n) {
                  const float f = acc[m][n][j];
                  const unsigned iv = __float_as_uint(f);
                  const bool keep = (int)iv >= 0;
                  const unsigned long long b = __ballot(keep);
                  unsigned a = lanes_below(b) * 4u + pos;
                  asm volatile("" : "+v"(a));
                  *(LAS unsigned*)(size_t)(keep ? a : trash) = iv;
                  advance(pos, b);
                  mk[m >> 1] = __builtin_amdgcn_alignbit(mk[m >> 1], iv, 31);
                }
            *(LAS u32x2*)(size_t)(l0 + (unsigned)lane * 8u) = mk;
            const unsigned bytes = hp < 56u ? pos - l0 : 0u;
            u32x4 v[17];
#pragma unroll
            for (int i = 0; i < 17; ++i) v[i] = *(LAS u32x4*)(size_t)(l0 + l16 + i * 1024);
            WAIT_LGKM0();
            if (hand_over) BARRIER();
#pragma unroll
            for (int i = 0; i < 17; ++i)
              if (i * 1024u < bytes) __builtin_amdgcn_raw_buffer_store_b128(v[i], rws, i * 1024u + l16 < bytes ? l16 : kOob, pb0 + i * 1024u, kAuxSys);
            WAIT_VM(0);
            if (lane == 0 && hp < 56u) flag_store2(p.ws[owner] + kOFlags + par * 8192u + q * 2048u + s * 8u, call, (bytes - 512u) >> 2);
          } else
#pragma unroll
          for (int rh = 0; rh < M / 4; ++rh) {
            const unsigned hp = park_pos(ap, M == 8 ? (unsigned)rh : wh, jo);
            const unsigned s = hp * 4u + (bp & 1u) * 2u + wl;
            const unsigned pb0 = obuf + (jo * 224u + s) * kPiece;
            unsigned pos = pb0 + 512u;
            if (hp < 56u) {
              u32x2 mk = {0u, 0u};
#pragma unroll
              for (int m = 0; m < 4; ++m)
#pragma unroll
                for (int j = 0; j < 4; ++j)
#pragma unroll
                  for (int n = 0; n < 4; ++n) {
                    const float f = acc[4 * rh + m][n][j];
                    const unsigned iv = __float_as_uint(f);
                    const unsigned long long b = __ballot((int)iv >= 0);
                    const unsigned idx = lanes_below(b) | (iv & kOob);
                    asm volatile("buffer_store_dword %0, %1, %2, %3 idxen sc0 sc1" :: "v"(iv), "v"(idx), "s"(rw4), "s"(pos) : "memory");
                    advance(pos, b);
                    mk[m >> 1] = __builtin_amdgcn_alignbit(mk[m >> 1], iv, 31);
                  }
              __builtin_amdgcn_raw_buffer_store_b64(mk, rws, (unsigned)lane * 8u, pb0, kAuxSys);
            }
            if (M == 4 && hand_over) BARRIER();
            if (hp < 56u) {
              WAIT_VM(0);
              if (lane == 0) flag_store2(p.ws[owner] + kOFlags + par * 8192u + q * 2048u + s * 8u, call, (pos - pb0 - 512u) >> 2);
            }
          }
        } else {
#pragma unroll
          for (int m = 0; m < M; ++m) {
            const unsigned ob = (panel_row(ap, M == 8 ? (unsigned)(m >> 2) : wh, rank) + 16u * (m & 3)) * 4096u + (col0 + 64u * wl) * 4u;
#pragma unroll
            for (int j = 0; j < 4; ++j) {
              u32x4 o;
#pragma unroll
              for (int n = 0; n < 4; ++n) { const float f = acc[m][n][j]; o[n] = (unsigned)max(__float_as_int(f), lo); }
              __builtin_amdgcn_raw_buffer_store_b128(o, ro, vo, ob + j * 4096u, kAuxSys);
            }
          }
        }
        if (M == 8 && hand_over) BARRIER();
      };
      if (shape == 3u) {
        run(std::integral_constant<int, 4>{}, wh * 4096u, 8192u + wl * 4096u);
      } else {
        run(std::integral_constant<int, 8>{}, shape == 0u ? wh * 8192u : 0u, (shape == 0u ? 16384u : 8192u + wh * 8192u) + wl * 4096u);
      }
      STAMP(myws, wid == 0 ? 4 + 3 * ti : 31);
    }
    if (!hass) return;
    const unsigned ap = small & 63u, bp = (small >> 6) & 15u, ch = (small >> 12) & 1u;
    const unsigned owner = (bp < 8u ? rank : og + ((bp - 8u) >> 1)) & 7u;
    const unsigned sa = abase + ap * kPanel + (unsigned)wid * 4096u, sb = bbase + bp * kPanel + ch * 4096u + (unsigned)wid * 8192u;
    unsigned long long mine = (small & 512u ? MASKL : MASKO) ^ (late ? 0ull : ~0ull);
    unsigned have = 0;
    constexpr int kNL = __builtin_popcountll(MASKL), kNE = 64 - kNL, kNO = __builtin_popcountll(MASKO);
    static_assert(kNO % SD == 0 && kNO >= NPF + SD && 64 - kNO >= NPF + SD, "");
    const int rounds = (__builtin_popcountll(mine) - NPF) / SD - 1;
    static_assert(kNE % SD == 0 && kNL % SD == 0 && NPF % SD == 0 && kNE >= NPF + SD && kNL >= NPF + SD && NPF >= 4, "");
    f32x4 acc[4][4];
#pragma unroll
    for (int m = 0; m < 4; ++m)
#pragma unroll
      for (int n = 0; n < 4; ++n) acc[m][n] = f32x4{0.f, 0.f, 0.f, 0.f};
    f32x4 ra[SD][4], rb[SD][4];
    auto await = [&](int word, unsigned need) {
      unsigned v;
      const unsigned long long t0 = __builtin_amdgcn_s_memrealtime();
      while ((v = uni(misc[word])) <= need)
        if (__builtin_amdgcn_s_memrealtime() - t0 > 2u * kDeadline) { misc[3] = 1u; v = 128u; break; }
      FENCE();
      return v;
    };
    auto fetch = [&](int u, bool cols) {
      const unsigned pr = (unsigned)__builtin_ctzll(mine), need = 2u * pr + ((unsigned)wid >> 1);
      mine &= mine - 1ull;
      if (cols) {
#pragma unroll
        for (int i = 0; i < 4; ++i) rb[u][i] = ld16<kAuxDev>(rws, l16 + i * 1024u, sb + pr * 32768u);
      }
      FENCE();
      if (have <= need) have = await(4, need);
#pragma unroll
      for (int i = 0; i < 4; ++i) ra[u][i] = ld16<kAuxDev>(rws, l16 + i * 1024u, sa + pr * 16384u);
      __builtin_amdgcn_sched_barrier(0);
    };
    auto mm = [&](int u) {
#pragma unroll
      for (int m = 0; m < 4; ++m)
#pragma unroll
        for (int n = 0; n < 4; ++n)
          acc[m][n] = __builtin_amdgcn_mfma_f32_16x16x32_f16(__builtin_bit_cast(f16x8, ra[u][m]), __builtin_bit_cast(f16x8, rb[u][n]), acc[m][n], 0, 0, 0);
    };
    LAS char* const mylds = lds + ((unsigned)wid * (NPF * 4096u) + l16);
    STAMP(myws, wid == 0 ? 2 + 3 * ntiles : 31);
    await(5, 0u);
#pragma unroll
    for (int i = 0; i < SD - 1; ++i) fetch(i, true);
    for (int i = rounds; i > 0; --i) {
#pragma unroll
      for (int u = 0; u < SD; ++u) {
        fetch((u + SD - 1) % SD, true);
        mm(u);
      }
    }
#pragma unroll
    for (int u = 0; u < SD; ++u) {
      fetch((u + SD - 1) % SD, u == 0);
      mm(u);
    }
    await(6, 0u);
#pragma unroll
    for (int i = 0; i < NPF; ++i) {
#pragma unroll
      for (int n = 0; n < 4; ++n) rb[i % SD][n] = *(LAS f32x4*)(mylds + (i * 4096 + n * 1024));
      if (i + SD - 1 < NPF) fetch((i + SD - 1) % SD, false);
      mm(i % SD);
    }
    STAMP(myws, wid == 0 ? 3 + 3 * ntiles : 31);
#pragma unroll
    for (int m = 0; m < 4; ++m)
#pragma unroll
      for (int n = 0; n < 4; ++n) *(LAS f32x4*)(mylds + (4 * m + n) * 1024) = acc[m][n];
    WAIT_LGKM0();
    BARRIER();
    f32x4 sum[4];
#pragma unroll
    for (int n = 0; n < 4; ++n) {
      LAS char* const ps = lds + ((unsigned)wid * 4096u + n * 1024 + l16);
      sum[n] = (*(LAS f32x4*)ps + *(LAS f32x4*)(ps + NPF * 4096)) + (*(LAS f32x4*)(ps + 2 * NPF * 4096) + *(LAS f32x4*)(ps + 3 * NPF * 4096));
    }
    const int lo = misc[3] != 0u ? 0x7fc00000 : 0;
    if (!late) {
#pragma unroll
      for (int n = 0; n < 4; ++n) {
        if (UNLIKELY(lo != 0)) sum[n] = __builtin_bit_cast(f32x4, u32x4{0x7fc00000u, 0x7fc00000u, 0x7fc00000u, 0x7fc00000u});
        __builtin_amdgcn_raw_buffer_store_b128(__builtin_bit_cast(u32x4, sum[n]), rws, l16 + n * 1024u, kPart + tau * 16384u + (unsigned)wid * 4096u, kAuxDev);
      }
      WAIT_VM(0);
      if (lane == 0) flag_store(myws + kPFlags + tau * 128u + (unsigned)wid * 4u, call);
      STAMP(myws, wid == 0 ? 4 + 3 * ntiles : 31);
      return;
    }
#pragma unroll
    for (int n = 0; n < 4; ++n) sum[n] += *(LAS f32x4*)(lds + (kPartLds + (unsigned)wid * 4096u + n * 1024 + l16));
    const __amdgpu_buffer_rsrc_t ro = mk_rsrc(p.out[owner], kInBytes);
    const unsigned ob = (panel_row(ap, 0u, rank) + 16u * (unsigned)wid) * 4096u + ((bp < 8u ? bp * 128u : 256u * q + (bp & 1u) * 128u) + 64u * ch) * 4u;
#pragma unroll
    for (int j = 0; j < 4; ++j) {
      u32x4 o;
#pragma unroll
      for (int n = 0; n < 4; ++n) { const float f = sum[n][j]; o[n] = (unsigned)max(__float_as_int(f), lo); }
      __builtin_amdgcn_raw_buffer_store_b128(o, ro, (unsigned)fq * 16384u + (unsigned)fr * 16u, ob + j * 4096u, kAuxSys);
    }
    if (bp >= 8u) {
      WAIT_VM(0);
      if (lane == 0) __hip_atomic_fetch_add((GAS unsigned*)(p.ws[owner] + kDone), 1u, __ATOMIC_RELAXED, __HIP_MEMORY_SCOPE_SYSTEM);
    }
    STAMP(myws, wid == 0 ? 4 + 3 * ntiles : 31);
  }
}

extern "C" void kernel_launch(int rank, int nranks, ncclComm_t comm, hipStream_t stream,
                              void* const* d_in, const int* in_sizes, int n_in,
                              void* d_out, int out_size,
                              const int* in_dtypes, int out_dtype,
                              void* const* d_out_all, void* const* d_ws_all, size_t ws_bytes) {
  (void)comm; (void)in_dtypes; (void)out_dtype; (void)d_out;
  if (nranks != 8 || (unsigned)rank >= 8u || n_in < 2 || ws_bytes < (size_t)kWsNeed ||
      in_sizes[0] != 8 * 1024 * 1024 || in_sizes[1] != 8 * 1024 * 1024 || out_size != 8 * 1024 * 1024) {
    fprintf(stderr, "kernel_launch: unexpected problem geometry\n");
    return;
  }
  Params p;
  p.x = (const float*)d_in[0];
  p.w = (const float*)d_in[1];
  for (int i = 0; i < 8; ++i) { p.ws[i] = (char*)d_ws_all[i]; p.out[i] = (float*)d_out_all[i]; }
  p.rank = rank;
  p.pad = 0;
  static const int xcol[8] = {0, 4, 2, 3, 1, 5, 6, 7};
  auto tile = [](int a, int b, int shape) { return (unsigned)(a | (b << 6) | (shape << 10)); };
  for (int ci = 0; ci < kNComp; ++ci) {
    const int x = ci & 7, s = ci >> 3, bq = 2 * (x ^ 4), j = x >> 1;
    unsigned t[3] = {0u, 0u, 0u};
    int n = 0;
    auto farb = [&](int e) { const int idx = 3 * (x & 1) + e; return 8 + idx + (idx >= 2 * j ? 2 : 0); };
    if (s < 12) {
      t[n++] = tile(x < 4 ? 2 * (s >> 2) : 8 + 5 * (s >> 2), 4 * (x & 3) + (s & 3), 0);
      t[n++] = tile(10 + 5 * (s >> 2) + ((s >> 1) & 1), bq + (s & 1), 3);
    } else if (s == 12) {
      t[n++] = tile(6, xcol[x], 0);
      t[n++] = tile(27 + 2 * j, farb(1), 3);
    } else if (s < 16) {
      t[n++] = tile(12 + 5 * (s - 13), bq, 1);
    } else if (s < 18) {
      t[n++] = tile(34 + (s - 16), bq, 1);
    } else if (s < 23) {
      t[n++] = tile(26 + 2 * j, s < 22 ? 4 * (x & 1) + s - 18 : farb(2), 0);
    } else {
      t[n++] = tile(26 + 2 * j + ((s - 23) & 1), farb((s - 23) >> 1), 3);
    }
    const int i = s < 12 ? s : s == SKIPB || s == SKIPA ? -1 : s - 12 - (s > SKIPA ? 1 : 0) - (s > SKIPB ? 1 : 0);
    if (i >= 0) t[n++] = tile(23 + i % 3, i < 6 ? 8 + x : x, 2) | ((unsigned)((i / 3) & 1) << 12) | (s < 12 ? 8192u : 0u);
    p.tiles[3 * ci] = t[0] | ((unsigned)n << 28);
    p.tiles[3 * ci + 1] = t[1];
    p.tiles[3 * ci + 2] = t[2];
  }
  hipError_t e = hipFuncSetAttribute((const void*)ag_gemm_relu_kernel, hipFuncAttributeMaxDynamicSharedMemorySize, (int)kLdsBytes);
  if (e != hipSuccess) { fprintf(stderr, "hipFuncSetAttribute: %s\n", hipGetErrorString(e)); return; }
  hipLaunchKernelGGL(ag_gemm_relu_kernel, dim3(kNWG), dim3(512), kLdsBytes, stream, p);
  e = hipGetLastError();
  if (e != hipSuccess) fprintf(stderr, "launch: %s\n", hipGetErrorString(e));
}
```

```cpp
#include <hip/hip_runtime.h>
#include <rccl/rccl.h>
#include <cstdio>
#include <cstdint>
#include <type_traits>

typedef _Float16 f16x8 __attribute__((ext_vector_type(8)));
typedef float f32x4 __attribute__((ext_vector_type(4)));
typedef unsigned u32x4 __attribute__((ext_vector_type(4)));
typedef unsigned u32x2 __attribute__((ext_vector_type(2)));
#define LAS __attribute__((address_space(3)))
#define GAS __attribute__((address_space(1)))

#define NWS 9u
#define WPM 3u
#define PU 1u
#define PUW 1u
#define CVDELAY 32
#define CVFIRST 50
#define CVLEAD 40u
#define XSLEEP 4
#define CVFROM 13
#define PWIN 4u
#define CVAUX kAuxSys
#define MSLEEP 4
#define LSLEEP 2
#define WLIM 160u
#define SKIPA 22
#define ALANES 63
#define PTMO kOob
#define TMOEV 7u
#define SD 4
#define MASKL 0xffaaaaaaaaaa0000ull
#define MASKO MASKL
#define LDSPARK
#define FPOLL
#define NARROW 32u
#define ERAISE 1200u
#define FEWF 80u
#define CVROT 10u
#define PSL 6
#define PSL0 16
#define PGATE 32u
#define CVSKIP 4u
#define FPS 8
#define NPF 8
#define SKIPB 12
#define PSLEEP 1
#define LTHR 24u

#define LDCOND(j) ((j) + 40u >= total)
namespace {

constexpr unsigned kBlk = 16384u;
constexpr unsigned kPanel = 2u << 20;
constexpr unsigned kAOff = 2u << 20;
constexpr unsigned kASize = 36u * kPanel;
constexpr unsigned kBOff = kAOff + 2u * kASize;
constexpr unsigned kBSize = 16u * kPanel;
constexpr unsigned kLast = 23u * kPanel + (1u << 20);
constexpr unsigned kPiece = 16896u;
constexpr unsigned kOOff = kBOff + 2u * kBSize;
constexpr unsigned kOSize = 4u * 224u * kPiece;
constexpr unsigned kPart = kOOff + 2u * kOSize;
constexpr unsigned kWsNeed = kPart + 96u * 16384u;
constexpr unsigned kPFlags = 131072;
constexpr unsigned kTmo = 128, kDone = 256, kCvt = 384, kAll = 512, kQueue = 1024;
constexpr unsigned kTicket = 4096;
constexpr unsigned kFlags = 1u << 20, kRep = 57600;
constexpr unsigned kOFlags = 81920;
constexpr unsigned kSrc = 98304;
constexpr unsigned kInBytes = 32u << 20;
constexpr int kNMove = 48, kNComp = 208, kNWG = 256;
constexpr unsigned kRowsI = 704u, kRowsC = 688u;
constexpr unsigned kDonePerCall = 192u;
constexpr unsigned long long kDeadline = 50000000ull;
constexpr unsigned kSlot = 24576u, kRing = 6u * kSlot;
constexpr unsigned kPBuf = 17408u;
constexpr unsigned kLdsMisc = 9u * kPBuf, kLdsBytes = kLdsMisc + 128u;
constexpr unsigned kPartLds = NPF * 16384u;
static_assert(kLdsMisc >= kRing && kPartLds + 16384u <= kRing, "");
constexpr unsigned kOob = 0x80000000u;
constexpr int kAuxSys = 17;
constexpr int kAuxDev = 16;

struct Params {
  const float* x;
  const float* w;
  char* ws[8];
  float* out[8];
  int rank;
  int pad;
  uint32_t tiles[3 * kNComp];
};

__device__ __forceinline__ __amdgpu_buffer_rsrc_t mk_rsrc(const void* p, unsigned bytes) {
  return __builtin_amdgcn_make_buffer_rsrc((void*)p, 0, bytes, 0x00020000);
}
template <int AUX>
__device__ __forceinline__ f32x4 ld16(__amdgpu_buffer_rsrc_t r, unsigned voff, unsigned soff) {
  return __builtin_bit_cast(f32x4, __builtin_amdgcn_raw_buffer_load_b128(r, voff, soff, AUX));
}
template <int AUX>
__device__ __forceinline__ void st16(f16x8 v, __amdgpu_buffer_rsrc_t r, unsigned voff, unsigned soff) {
  __builtin_amdgcn_raw_buffer_store_b128(__builtin_bit_cast(u32x4, v), r, voff, soff, AUX);
}
__device__ __forceinline__ f16x8 cvt8(f32x4 a, f32x4 b) {
  f16x8 h;
#pragma unroll
  for (int i = 0; i < 4; ++i) { h[i] = (_Float16)a[i]; h[4 + i] = (_Float16)b[i]; }
  return h;
}
__device__ __forceinline__ f16x8 cvt_col(const f32x4 (&v)[8], int i) {
  f16x8 h;
#pragma unroll
  for (int j = 0; j < 8; ++j) h[j] = (_Float16)v[j][i];
  return h;
}
#define WAIT_VM(n) asm volatile("s_waitcnt vmcnt(%0)" :: "n"(n) : "memory")
#define WAIT_LGKM0() asm volatile("s_waitcnt lgkmcnt(0)" ::: "memory")
#define FENCE() asm volatile("" ::: "memory")
#define UNLIKELY(c) __builtin_expect(!!(c), 0)
#define BARRIER() do { FENCE(); __builtin_amdgcn_s_barrier(); FENCE(); } while (0)

__device__ __forceinline__ void flag_store(char* p, unsigned v) {
  __hip_atomic_store((GAS unsigned*)p, v, __ATOMIC_RELAXED, __HIP_MEMORY_SCOPE_SYSTEM);
}
__device__ __forceinline__ void flag_store2(char* p, unsigned lo, unsigned hi) {
  __hip_atomic_store((GAS unsigned long long*)p, (unsigned long long)lo | ((unsigned long long)hi << 32), __ATOMIC_RELAXED, __HIP_MEMORY_SCOPE_SYSTEM);
}
__device__ __forceinline__ unsigned draw(char* p) {
  unsigned z = 0;
  asm volatile("" : "+v"(z));
  p += z;
  return __hip_atomic_fetch_add((GAS unsigned*)p, 1u, __ATOMIC_RELAXED, __HIP_MEMORY_SCOPE_AGENT);
}
__device__ __forceinline__ unsigned uni(unsigned v) { return (unsigned)__builtin_amdgcn_readfirstlane((int)v); }
__device__ __forceinline__ unsigned lanes_below(unsigned long long b) {
  return __builtin_amdgcn_mbcnt_hi((unsigned)(b >> 32), __builtin_amdgcn_mbcnt_lo((unsigned)b, 0u));
}
__device__ __forceinline__ void advance(unsigned& pos, unsigned long long b) {
  asm("s_lshl2_add_u32 %0, %1, %0" : "+s"(pos) : "s"((unsigned)__builtin_popcountll(b)) : "scc");
}

#define VSTAMP(ws, slot, val) do {} while (0)
#define STAMP(ws, slot) do {} while (0)
#define PSTAMP(ws, cls, n) do {} while (0)
#define CSTAMP(ws, slot) do {} while (0)

__device__ __forceinline__ void set_timeout(char* myws, unsigned call) {
  __hip_atomic_store((GAS unsigned*)(myws + kTmo), call, __ATOMIC_RELAXED, __HIP_MEMORY_SCOPE_AGENT);
}
__device__ __forceinline__ unsigned peer_of(unsigned r, unsigned i) { return (r & 4u) + ((r + i + 1u) & 3u); }
__device__ __forceinline__ unsigned panel_row(unsigned ap, unsigned rh, unsigned rank) {
  const unsigned og = (rank & 4u) ^ 4u;
  if (ap < 8u) return rank * 1024u + ap * 128u + rh * 64u;
  if (ap < 23u) { const unsigned i = (ap - 8u) / 5u, s = (ap - 8u) - 5u * i; return peer_of(rank, i) * 1024u + s * 128u + rh * 64u; }
  if (ap < 26u) return peer_of(rank, ap - 23u) * 1024u + 640u;
  if (ap < 34u) return (og + ((ap - 26u) >> 1)) * 1024u + 768u + (ap & 1u) * 128u + rh * 64u;
  return (og + 2u * (ap - 34u) + rh) * 1024u + 704u;
}
__device__ __forceinline__ unsigned park_pos(unsigned ap, unsigned rh, unsigned j) {
  if (ap < 8u) return 2u * ap + rh < 11u ? 2u * ap + rh : 99u;
  if (ap < 23u) {
    const unsigned i = (ap - 8u) / 5u, s = (ap - 8u) - 5u * i;
    return s < 2u ? 11u + 4u * i + 2u * s + rh : s < 4u ? 40u + 4u * i + 2u * s + rh : 26u + 2u * i + rh;
  }
  const unsigned src = ap < 34u ? (ap - 26u) >> 1 : 2u * (ap - 34u) + rh;
  if (src == j) return 99u;
  const unsigned e = src - (src > j ? 1u : 0u);
  return ap < 34u ? 32u + 4u * e + 2u * (ap & 1u) + rh : 23u + e;
}
__device__ __forceinline__ unsigned park_row(unsigned hp, unsigned src, unsigned rank) {
  const unsigned q = rank & 3u;
  if (hp < 11u) return src * 1024u + hp * 64u;
  if (hp < 23u) return peer_of(src, (hp - 11u) >> 2) * 1024u + ((hp - 11u) & 3u) * 64u;
  if (hp < 26u) { const unsigned e = hp - 23u; return ((rank & 4u) + e + (e >= q ? 1u : 0u)) * 1024u + 704u; }
  if (hp < 32u) return peer_of(src, (hp - 26u) >> 1) * 1024u + 512u + (hp & 1u) * 64u;
  if (hp < 44u) { const unsigned e = (hp - 32u) >> 2; return ((rank & 4u) + e + (e >= q ? 1u : 0u)) * 1024u + 768u + (hp & 3u) * 64u; }
  return peer_of(src, (hp - 44u) >> 2) * 1024u + 256u + (hp & 3u) * 64u;
}

}

__global__ void __launch_bounds__(512) __attribute__((amdgpu_waves_per_eu(2, 2)))
ag_gemm_relu_kernel(const Params p) {
  extern __shared__ __attribute__((aligned(16))) char smem_raw[];
  LAS char* const lds = (LAS char*)smem_raw;
  LAS volatile unsigned* const misc = (LAS volatile unsigned*)(lds + kLdsMisc);

  const int tid = threadIdx.x;
  const int lane = tid & 63;
  const int wid = __builtin_amdgcn_readfirstlane(tid >> 6);
  const int bid = blockIdx.x;
  const unsigned rank = (unsigned)p.rank & 7u;
  const unsigned q = rank & 3u, og = (rank & 4u) ^ 4u;
  char* const myws = p.ws[rank];
  const unsigned l16 = (unsigned)lane * 16u;
  const __amdgpu_buffer_rsrc_t rws = mk_rsrc(myws, kWsNeed);
  const int fr = lane & 15, fq = lane >> 4;

  STAMP(myws, wid == 0 ? 0 : 31);

  const unsigned lx = (((unsigned)lane >> 2) * 8192u + 4u * (unsigned)((lane & 3) ^ ((-(lane >> 4)) & 3))) * 4u;
  const unsigned lw = (4u * ((unsigned)lane >> 4) * 1024u + 4u * ((unsigned)lane & 15u)) * 4u;
  const __amdgpu_buffer_rsrc_t rx = mk_rsrc(p.x, kInBytes);
  const __amdgpu_buffer_rsrc_t rw = mk_rsrc(p.w, kInBytes);
  const unsigned pw = (unsigned)wid >= WPM ? (unsigned)wid - WPM : (unsigned)wid;
  const unsigned gw = (unsigned)bid * WPM + pw;
  const unsigned sid = gw % 14u, k = gw / 14u;
  const bool cross = sid >= 6u;
  const unsigned c = cross ? (sid - 6u) >> 1 : sid >> 1, h = sid & 1u;
  const unsigned sr = (cross ? og + c : peer_of(rank, c)) & 7u;
  char* const sws = p.ws[sr];
  const __amdgpu_buffer_rsrc_t rs = mk_rsrc(sws, kWsNeed);
  LAS volatile unsigned* const hand = misc + 4u + 4u * pw;
  const unsigned buf0 = pw * 3u * kPBuf;
  if (bid < kNMove && (unsigned)wid >= WPM && (unsigned)wid < 2u * WPM && gw < 14u * NWS) {
    unsigned call = 0;
    auto push = [&](auto wconst) {
      constexpr bool W = decltype(wconst)::value;
      f32x4 pv[PU][2][8];
#pragma unroll
      for (unsigned u = 0; u < PU; ++u)
#pragma unroll
        for (int i = 0; i < 16; ++i) {
          const unsigned m = u * NWS + k, kk = (m >> 1) * 64u + h * 32u;
          if (W) pv[u][i >> 3][i & 7] = ld16<0>(rw, lw + (i >> 3) * 256u, (kk * 1024u + 256u * c + (m & 1u) * 128u) * 4u + ((i & 3) + 16 * ((i >> 2) & 1)) * 4096u);
          else pv[u][i >> 3][i & 7] = ld16<0>(rx, lx + (i & 1) * 64u, ((m & 1u) * 128u * 8192u + kk) * 4u + (i >> 1) * 524288u);
        }
      __syncthreads();
      call = uni(misc[0]) + 1u;
      const unsigned base = W ? kBOff + (call & 1u) * kBSize + (8u + 2u * q) * kPanel : kAOff + (call & 1u) * kASize + (18u - 5u * c) * kPanel;
      char* const fl = sws + kFlags + (W ? 44u + 2u * q : 18u - 5u * c) * 1024u + h * 4u + (unsigned)lane * kRep;
#pragma unroll
      for (unsigned j = 0; j < 2u * PU; ++j) {
        const unsigned u = j >> 1, r = j & 1u, m = u * NWS + k;
        f16x8 hv[4];
#pragma unroll
        for (int i = 0; i < 4; ++i) hv[i] = W ? cvt_col(pv[u][r], i) : cvt8(pv[u][r][2 * i], pv[u][r][2 * i + 1]);
        asm volatile("" :: "v"(hv[0]), "v"(hv[1]), "v"(hv[2]), "v"(hv[3]));
        if (j >= PUW) WAIT_VM(4u * (PUW - 1u));
        if (j <= 2u) STAMP(myws, 8 + wid + 3 * j);
        if (u > 0u && r == PUW - 1u && lane < 8) flag_store(fl + ((m - NWS) & 1u) * 1024u + ((m - NWS) >> 1) * 8u, call);
#pragma unroll
        for (int i = 0; i < 4; ++i) st16<kAuxSys>(hv[i], rs, l16 + i * 1024u, base + (m & 1u) * kPanel + (m >> 1) * kBlk + h * 8192u + r * 4096u);
        FENCE();
      }
      WAIT_VM(0);
      const unsigned m = (PU - 1u) * NWS + k;
      if (lane < 8) flag_store(fl + (m & 1u) * 1024u + (m >> 1) * 8u, call);
    };
    if (cross) push(std::true_type{}); else push(std::false_type{});
    STAMP(myws, 4 + wid);
    if (!cross) return;
    const __amdgpu_buffer_rsrc_t ro = mk_rsrc(p.out[rank], kInBytes);
    const unsigned vo = (unsigned)fq * 16384u + (unsigned)fr * 16u;
    const unsigned long long t0 = __builtin_amdgcn_s_memrealtime();
    for (unsigned b = 0;; b = b == 2u ? 0u : b + 1u) {
      unsigned hv;
      while ((hv = uni(hand[b])) == 0u) {
        if (__builtin_amdgcn_s_memrealtime() - t0 > 2u * kDeadline) { set_timeout(myws, call); return; }
        __builtin_amdgcn_s_sleep(XSLEEP);
      }
      FENCE();
      if (hv == ~0u) break;
      const unsigned oa = park_row(hv >> 4, sr, rank) * 4096u + (256u * c + (hv & 12u) * 16u) * 4u;
      unsigned pos = (unsigned)(size_t)lds + buf0 + b * kPBuf, val[64];
      u32x2 mk = {~0u, ~0u};
      if ((hv & 2u) == 0u) mk = *(LAS u32x2*)(size_t)(pos + lane * 8);
      const unsigned fill = (unsigned)(size_t)lds + kLdsMisc + 64u + (hv & 2u) * 2u;
      pos += 512u;
#pragma unroll
      for (int i = 0; i < 64; ++i) {
        const bool keep = (int)(mk[i >> 5] << (i & 31)) >= 0;
        const unsigned long long bl = __ballot(keep);
        unsigned a = lanes_below(bl) * 4u + pos;
        asm volatile("" : "+v"(a));
        val[i] = *(LAS unsigned*)(size_t)(keep ? a : fill);
        advance(pos, bl);
      }
      WAIT_LGKM0();
      if (lane == 0) hand[b] = 0u;
      FENCE();
#pragma unroll
      for (int i = 0; i < 16; ++i)
        __builtin_amdgcn_raw_buffer_store_b128(u32x4{val[4 * i], val[4 * i + 1], val[4 * i + 2], val[4 * i + 3]}, ro, vo, oa + (i >> 2) * 65536u + (i & 3) * 4096u, kAuxSys);
    }
    STAMP(myws, 1 + wid);
    return;
  }
  const bool isconv = bid >= kNMove + 8 * CVFROM;
  unsigned tk = 0;
  if (tid == 0) tk = draw(myws + kTicket + (unsigned)bid * 32u);
  FENCE();
  f32x4 cv[8];
  auto item = [&](unsigned n, unsigned& ktg, unsigned& sub) {
    if (n < 5u * CVLEAD) {
      ktg = n / 5u; sub = n - 5u * ktg; sub += sub ? 3u : 0u;
      if (ktg < CVROT + CVSKIP) ktg = ktg < CVROT ? ktg + CVSKIP : ktg - CVROT;
    }
    else if (n < 8u * CVLEAD) { ktg = (n - 5u * CVLEAD) / 3u; sub = n - 5u * CVLEAD - 3u * ktg + 1u; }
    else { ktg = n >> 3; sub = n & 7u; }
  };
  auto loads = [&](unsigned n) {
    unsigned ktg, sub;
    item(n, ktg, sub);
    if (sub < 4u) {
      const unsigned so = ((256u * sub + 32u * wid) * 8192u + ktg * 64u) * 4u;
#pragma unroll
      for (int s = 0; s < 2; ++s) {
        cv[4 * s + 0] = ld16<0>(rx, lx, so + s * 524288u); cv[4 * s + 1] = ld16<0>(rx, lx + 64u, so + s * 524288u);
        cv[4 * s + 2] = ld16<0>(rx, lx + 128u, so + s * 524288u); cv[4 * s + 3] = ld16<0>(rx, lx + 192u, so + s * 524288u);
      }
    } else {
      const unsigned so = ((ktg * 64u + 32u * (wid & 1)) * 1024u + 256u * (sub - 4u) + 64u * (wid >> 1)) * 4u;
#pragma unroll
      for (int i = 0; i < 8; ++i) cv[i] = ld16<0>(rw, lw, so + ((i & 3) + 16 * (i >> 2)) * 4096u);
    }
  };
  if (bid >= kNMove + 8 * CVFROM + CVFIRST) __builtin_amdgcn_s_sleep(CVDELAY);
  if (isconv) loads((unsigned)(bid - kNMove - 8 * CVFROM));
  FENCE();
  if (tid == 0) {
    misc[0] = tk;
    misc[3] = 0u;
  }
  if (tid >= 4 && tid < 18) misc[tid] = tid == 17 ? 0x7fc00000u : 0u;
  __syncthreads();
  const unsigned call = uni(misc[0]) + 1u;
  const unsigned par = call & 1u;
  const unsigned abase = kAOff + par * kASize, bbase = kBOff + par * kBSize, obuf = kOOff + par * kOSize;

  if (bid < kNMove) {
    if (wid == 7) {
      if (bid == 0) {
        const unsigned target = call * kDonePerCall, off = lane == 0 ? kDone : lane == 1 ? kTmo : kOob;
        const unsigned long long t0 = __builtin_amdgcn_s_memrealtime();
        unsigned res = 0;
        while (res == 0u) {
          const unsigned v = __builtin_amdgcn_raw_buffer_load_b32(rws, off, 0, kAuxSys);
          FENCE();
          res = (unsigned)__ballot(lane == 0 ? (int)(v - target) >= 0 : v == call) & 3u;
          if (res == 0u && __builtin_amdgcn_s_memrealtime() - t0 > kDeadline) res = 2u;
        }
        if ((res & 1u) == 0u && lane == 0) {
          set_timeout(myws, call);
          __hip_atomic_store((GAS unsigned*)(p.out[rank] + (size_t)rank * 1024u * 1024u), 0x7fc00000u, __ATOMIC_RELAXED, __HIP_MEMORY_SCOPE_SYSTEM);
        }
        STAMP(myws, 10);
      }
      return;
    }
    if ((unsigned)wid >= WPM || gw >= 14u * NWS) return;
    const unsigned total = cross ? 2u * kRowsC : 2u * kRowsI;
    const unsigned nplain = cross ? 1152u : 2u * kRowsI;
    const unsigned qbase = (call - 1u) * (total - PU * 2u * NWS) - (PU + 1u) * 2u * NWS;
    const unsigned queue = kQueue + (sid >> 1) * 128u;
    bool failed = false, all = false;
    unsigned cgrp = 99u, okf = 0, okt = 0, fv = 0, wbase = 0x10000u;
    u32x2 fw = {0u, 0u};
    const unsigned fo = lane < 63 ? kOFlags + par * 8192u + c * 2048u + (unsigned)lane * 8u : kTmo;
    unsigned sa = 0, da = 0, fa = 0, pend = 0;
    unsigned s2 = 4096u, d2 = 4096u;
    u32x4 v[4], w[4];
    auto raise = [&]() {
      if (pend != 0u && lane < 8) flag_store(myws + pend + (unsigned)lane * kRep, call);
      pend = 0u;
    };
    auto fetch = [&](u32x4 (&x)[4], unsigned so) {
#pragma unroll
      for (int i = 0; i < 4; ++i) x[i] = __builtin_amdgcn_raw_buffer_load_b128(rs, l16 + i * 1024u, so, kAuxSys);
      FENCE();
    };
    auto put = [&](const u32x4 (&x)[4], unsigned so) {
#pragma unroll
      for (int i = 0; i < 4; ++i) __builtin_amdgcn_raw_buffer_store_b128(x[i], rws, l16 + i * 1024u, so, kAuxDev);
      FENCE();
    };
    auto setup = [&](unsigned mh) {
      const unsigned m = mh >> 1, h = mh & 1u;
      unsigned sp, kt, dp, db, dh = h;
      unsigned so = h * 8192u, dof = h * 8192u;
      s2 = 4096u; d2 = 4096u;
      if (!cross) {
        if (m < 256u) { sp = m & 1u; kt = m >> 1; dp = 8u + 5u * c + sp; db = kt; }
        else if (m < 384u) { sp = 4u; kt = m - 256u; dp = 12u + 5u * c; db = kt; }
        else if (m < 640u) { sp = 2u + (m & 1u); kt = (m - 384u) >> 1; dp = 8u + 5u * c + sp; db = kt; }
        else { sp = 5u; db = m - 640u; kt = 2u * db + h; so = kLast - 5u * kPanel - kt * 8192u; dp = 23u + c; }
      } else {
        if (m < 256u) { sp = 36u + 2u * q + (m & 1u); kt = m >> 1; dp = 44u + 2u * c + (m & 1u); db = kt; }
        else {
          const unsigned g = ((m - 256u) * 52429u) >> 18, r = m - 256u - 5u * g;
          if (r == 2u) { sp = 5u; kt = 2u * g + h; so = 4096u; s2 = 8192u; dp = 34u + (c >> 1); db = kt; dh = c & 1u; dof = dh * 4096u; d2 = 8192u; }
          else { sp = 6u + (r % 3u); kt = 2u * g + (r > 2u ? 1u : 0u); dp = 20u + 2u * c + sp; db = kt; }
        }
      }
      const unsigned sub = sp < 36u ? sp >> 1 : (sp - 28u) >> 1;
      auto ask = [&](unsigned k0) {
        const unsigned off = lane < 62 ? kSrc + (sr * 8u + sub) * 512u + min(k0 + (unsigned)lane, 127u) * 4u : lane == 62 ? kAll + sr * 4u : kTmo;
        asm volatile("buffer_load_dword %0, %1, %2, 0 offen sc0 sc1" : "=v"(fv) : "v"(off), "s"(rws) : "memory");
      };
      auto look = [&]() {
        asm volatile("" : "+v"(fv) :: "memory");
        const unsigned long long b = __ballot(lane < 63 ? (int)(fv - call) >= 0 : fv == call);
        if ((b >> 62) != 0ull) { failed = (b >> 63) != 0ull; all = true; }
        okt += (unsigned)__builtin_ctzll(~b);
      };
      if (!all) {
        if (cgrp != 99u) look();
        if (!all && !(sub == cgrp && kt >= okf && kt < okt)) {
          WAIT_VM(0);
          raise();
          cgrp = sub;
          okf = kt;
          const unsigned long long t0 = __builtin_amdgcn_s_memrealtime();
          for (;;) {
            okt = kt;
            ask(kt);
            WAIT_VM(0);
            look();
            if (all || okt != kt) break;
            if (__builtin_amdgcn_s_memrealtime() - t0 > kDeadline) { set_timeout(myws, call); failed = true; all = true; break; }
            __builtin_amdgcn_s_sleep(MSLEEP);
          }
        }
        if (!all) ask(okt);
      }
      sa = (sp < 36u ? abase + sp * kPanel : bbase + (sp - 36u) * kPanel) + kt * kBlk + so;
      da = (dp < 36u ? abase + dp * kPanel : bbase + (dp - 36u) * kPanel) + db * kBlk + dof;
      fa = kFlags + dp * 1024u + db * 8u + dh * 4u;
      fetch(v, sa);
    };
    unsigned m = 2u * (PU * NWS + k) + h;
    setup(m);
    STAMP(myws, 20 + wid);
    do {
      unsigned dv = 0;
      if (lane == 0) dv = draw(myws + queue);
      FENCE();
      put(v, da);
      raise();
      fetch(w, sa + s2);
      put(w, da + d2);
      pend = fa;
      if ((m & 1u) == 0u) { PSTAMP(myws, sid >> 1, (m >> 1) + 1u); if (m < 256u) PSTAMP(myws, 8u + (sid >> 1), ((m >> 1) + 1u) * 4u); }
      m = uni(dv) - qbase;
      if (m < nplain) setup(m);
      if (m >= ERAISE && m < nplain) { WAIT_VM(4); raise(); }
    } while (m < nplain);
    WAIT_VM(0);
    raise();
    if (cross) {
      unsigned hval = 0;
      asm volatile("" : "+v"(fw) :: "memory");
      auto stale = [&](unsigned o) {
        return o - wbase >= 63u || (int)((unsigned)__builtin_amdgcn_readlane((int)fw[0], (int)(o - wbase)) - call) < 0;
      };
      for (unsigned b = 0, pb = 0;; pb = b, b = b == 2u ? 0u : b + 1u) {
        const unsigned o = m - 1152u, s = o;
        unsigned cnt = 2048u, dv = 0;
        if (m >= total || stale(o)) {
          WAIT_VM(0);
          if (hval != 0u && lane == 0) hand[pb] = hval;
          hval = 0u;
          const unsigned long long t0 = __builtin_amdgcn_s_memrealtime();
          wbase = o;
          while (!failed && m < total) {
            fw = __builtin_bit_cast(u32x2, __hip_atomic_load((GAS unsigned long long*)(myws + fo + (lane < 63 ? o * 8u : 0u)), __ATOMIC_RELAXED, __HIP_MEMORY_SCOPE_SYSTEM));
            const unsigned long long bl = __ballot(lane < 63 ? (int)(fw[0] - call) >= 0 : fw[0] == call);
            if ((bl >> 63) != 0ull) { failed = true; break; }
            if (bl & 1ull) break;
            if (__builtin_amdgcn_s_memrealtime() - t0 > kDeadline) { set_timeout(myws, call); failed = true; }
            __builtin_amdgcn_s_sleep(MSLEEP);
          }
        }
        cnt = failed ? 0u : min((unsigned)__builtin_amdgcn_readlane((int)fw[1], (int)((o - wbase) & 63u)), 4096u);
        if (UNLIKELY(uni(hand[b]) != 0u)) {
          const unsigned long long t0 = __builtin_amdgcn_s_memrealtime();
          while (uni(hand[b]) != 0u)
            if (__builtin_amdgcn_s_memrealtime() - t0 > kDeadline) { set_timeout(myws, call); return; }
        }
        FENCE();
        if (m >= total) {
          if (lane == 0) hand[b] = ~0u;
          break;
        }
        const unsigned bytes = 512u + 4u * cnt, sb = obuf + (q * 224u + s) * kPiece, nbuf = buf0 + b * kPBuf;
        auto part = [&](unsigned f0, unsigned f1) {
          for (unsigned f = f0; f < f1; f += 1024u) {
            WAIT_VM(PWIN - 1u);
            asm volatile("s_mov_b32 m0, %0\n\ts_nop 0\n\tbuffer_load_dwordx4 %1, %2, %3 offen sc0 sc1 lds"
                         :: "s"(nbuf + f), "v"(f + l16 < bytes ? l16 : kOob), "s"(rs), "s"(sb + f) : "memory");
          }
        };
        asm volatile("s_mov_b64 exec, 1\n\tglobal_atomic_add %0, %1, %2, %3 sc0\n\ts_mov_b64 exec, -1"
                     : "=v"(dv) : "v"(queue), "v"(1u), "s"(myws) : "memory");
        part(0u, min(bytes, (PWIN - 1u) * 1024u));
        if (bytes <= (PWIN - 2u) * 1024u) WAIT_VM(0);
        WAIT_VM(PWIN - 1u);
        if (hval != 0u && lane == 0) hand[pb] = hval;
        hval = 4u * s + (failed ? 3u : 1u);
        if ((m & 1u) == 0u) PSTAMP(myws, sid >> 1, (m >> 1) + 1u);
        asm volatile("" : "+v"(dv));
        m = uni(dv) - qbase;
        if (m < total && stale(m - 1152u)) {
          wbase = m - 1152u;
          asm volatile("buffer_load_dwordx2 %0, %1, %2, 0 offen sc0 sc1" : "=v"(fw) : "v"(fo + (lane < 63 ? wbase * 8u : 0u)), "s"(rws) : "memory");
          part((PWIN - 1u) * 1024u, bytes);
          if (bytes <= (2u * PWIN - 3u) * 1024u) WAIT_VM(0);
          WAIT_VM(PWIN - 1u);
          asm volatile("" : "+v"(fw) :: "memory");
        } else
          part((PWIN - 1u) * 1024u, bytes);
      }
    }
    STAMP(myws, 1 + wid);
    return;
  }

  {
    constexpr unsigned kNConv = 8u * (26u - CVFROM);
    char* const pws = p.ws[lane & 7];
    const unsigned mine = 15u << (rank & 4u);
    STAMP(myws, wid == 0 ? 27 : 31);
    for (unsigned n = isconv ? (unsigned)(bid - kNMove - 8 * CVFROM) : 1024u; n < 1024u; n += kNConv) {
      unsigned ktg, sub;
      item(n, ktg, sub);
      if (sub < 4u) {
        const unsigned so = abase + (2u * sub + (wid >> 2)) * kPanel + ktg * kBlk + (wid & 3) * 2048u;
#pragma unroll
        for (int i = 0; i < 4; ++i) st16<CVAUX>(cvt8(cv[2 * i], cv[2 * i + 1]), rws, l16 + (i >> 1) * 1024u, so + (i & 1) * 8192u);
        if (sub == 2u && (wid >> 1) == 2) {
          const int from = (4 * fr + (fq ^ ((-(fr >> 2)) & 3))) * 4;
#pragma unroll
          for (int i = 0; i < 4; ++i) {
            u32x4 hv = __builtin_bit_cast(u32x4, cvt8(cv[2 * i], cv[2 * i + 1]));
#pragma unroll
            for (int e = 0; e < 4; ++e) hv[e] = (unsigned)__builtin_amdgcn_ds_bpermute(from, (int)hv[e]);
            __builtin_amdgcn_raw_buffer_store_b128(hv, rws, l16 + (i >> 1) * 1024u, abase + kLast + ktg * 8192u + (wid & 1) * 2048u + (i & 1) * 4096u, CVAUX);
          }
        }
      } else {
        const unsigned so = bbase + (2u * (sub - 4u) + (wid >> 2)) * kPanel + ktg * kBlk + ((wid >> 1) & 1) * 4096u + (wid & 1) * 8192u;
#pragma unroll
        for (int i = 0; i < 4; ++i) st16<CVAUX>(cvt_col(cv, i), rws, l16 + i * 1024u, so);
      }
      FENCE();
      const bool ahead = n >= kNConv && n + kNConv < 1024u;
      if (ahead) { loads(n + kNConv); WAIT_VM(8); } else { WAIT_VM(0); }
      if (n < kNConv) STAMP(myws, wid == 0 ? 30 : 31);
      BARRIER();
      if (n < 2u * kNConv) STAMP(myws, wid == 0 ? (n < kNConv ? 28 : 29) : 31);
      const unsigned to = ktg >= FEWF || (sub - 1u) < 3u ? 0u : sub == 0u ? mine : 1u << (og + sub - 4u);
      if (tid < 16) flag_store2(myws + kFlags + ((sub < 4u ? 2u * sub : 28u + 2u * sub) + ((unsigned)tid & 1u)) * 1024u + ktg * 8u + ((unsigned)tid >> 1) * kRep, call, call);
      else if (wid == 1 && lane < 8 && (((to & ~(1u << rank)) >> lane) & 1u)) flag_store(pws + kSrc + rank * 4096u + sub * 512u + ktg * 4u, call);
      FENCE();
      if (n < kNConv) loads(n + kNConv);
    }
    if (isconv && wid == 7) {
      unsigned dv = 0;
      if (lane == 0) dv = draw(myws + kCvt);
      if (uni(dv) == call * kNConv - 1u && lane < 8) flag_store(pws + kAll + rank * 4u, call);
    }
  }
  STAMP(myws, wid == 0 ? 1 : 31);

  const uint32_t* const tl = p.tiles + 3 * (bid - kNMove);
  const unsigned small = tl[(tl[0] >> 28) - 1u] & 16383u;
  const bool hass = ((small >> 10) & 3u) == 2u;
  const int ntiles = (int)(tl[0] >> 28) - (hass ? 1 : 0);
  const bool late = (small >> 13) != 0u;
  const unsigned tau = ((small >> 6) & 15u) * 6u + (small & 63u) - 23u + 3u * ((small >> 12) & 1u);
  if (wid == 7) return;

  if (wid >= 4) {
    const unsigned lw = (unsigned)wid - 4u;
    const unsigned total = 256u * (unsigned)ntiles;
    unsigned a = 0, aend = 0, so = lw * 8192u, src = 0, idle = 0;
    unsigned ok = 0, f0 = 0, f1 = 0, fb = 0, fv = 0;
    unsigned rdy = 0, askat = 0, lookat = ~0u;
    int ta = 0, tf = 0;
    auto next_flags = [&]() {
      while (tf < ntiles) {
        const unsigned code = tl[tf++];
        const unsigned ap = code & 63u, bp = (code >> 6) & 15u, shape = (code >> 10) & 3u;
        f0 = f1;
        f1 += 256u;
        if (shape == 3u && lw == 2u) { ok = f1; continue; }
        const unsigned grp = lw == 0u ? ap : shape == 0u ? (lw == 1u ? ap + 1u : 36u + bp) : shape == 1u ? 35u + bp + lw : 36u + bp;
        fb = kFlags + grp * 1024u + ((unsigned)bid & 7u) * kRep;
        break;
      }
    };
    auto known = [&]() {
      rdy = min(ok, aend);
      unsigned from = max(ok, LTHR);
      asm volatile("" : "+s"(from));
      askat = ok >= total ? ~0u : from - LTHR;
    };
    auto ask = [&](unsigned last) {
      const unsigned off = lane < ALANES ? fb + min(ok - f0 + (unsigned)lane, 255u) * 4u : lane == 63 ? last : kOob;
      asm volatile("buffer_load_dword %0, %1, %2, 0 offen sc0 sc1" : "=v"(fv) : "v"(off), "s"(rws) : "memory");
    };
    auto give_up = [&]() {
      ok = total;
      if (lane == 0) misc[3] = 1u;
      WAIT_LGKM0();
    };
    auto look = [&]() {
      asm volatile("" : "+v"(fv) :: "memory");
      const unsigned long long b = __ballot(lane < 63 ? (int)(fv - call) >= 0 : fv == call);
      if (UNLIKELY((b >> 63) != 0ull)) {
        give_up();
      } else {
        const unsigned run = (unsigned)__builtin_ctzll(~b);
        ok = min(ok + (run & ~1u), f1);
        if (UNLIKELY(ok == f1)) next_flags();
      }
      lookat = ~0u;
    };
    auto issue = [&](bool busy) {
      LAS char* const dl = lds + so;
      if (busy) {
        __builtin_amdgcn_raw_ptr_buffer_load_lds(rws, (LAS void*)dl, 16, l16, src, 0, kAuxDev);
        __builtin_amdgcn_raw_ptr_buffer_load_lds(rws, (LAS void*)dl, 16, l16, src, 1024, kAuxDev);
        __builtin_amdgcn_raw_ptr_buffer_load_lds(rws, (LAS void*)dl, 16, l16, src, 2048, kAuxDev);
        __builtin_amdgcn_raw_ptr_buffer_load_lds(rws, (LAS void*)dl, 16, l16, src, 3072, kAuxDev);
        __builtin_amdgcn_raw_ptr_buffer_load_lds(rws, (LAS void*)(dl + 4096), 16, l16, src + 4096u, 0, kAuxDev);
        __builtin_amdgcn_raw_ptr_buffer_load_lds(rws, (LAS void*)(dl + 4096), 16, l16, src + 4096u, 1024, kAuxDev);
        __builtin_amdgcn_raw_ptr_buffer_load_lds(rws, (LAS void*)(dl + 4096), 16, l16, src + 4096u, 2048, kAuxDev);
        __builtin_amdgcn_raw_ptr_buffer_load_lds(rws, (LAS void*)(dl + 4096), 16, l16, src + 4096u, 3072, kAuxDev);
      }
      FENCE();
      src += 8192u;
      so = so >= kRing - kSlot ? so - (kRing - kSlot) : so + kSlot;
      ++a;
    };
    next_flags();
    known();
    for (unsigned j = 0; j < total; ++j) {
      if (a == j + 4u) {
        unsigned e = min(rdy + 1u, askat);
        asm volatile("" : "+s"(e));
        e = min(e, lookat);
        if (idle != 0u) {
          WAIT_VM(0);
          while (a + 1u < e) { issue(false); BARRIER(); }
        }
        while (a + 1u < e) {
          issue(true);
          WAIT_VM(32);
          BARRIER();
        }
        j = a - 4u;
      }
      unsigned lim = min(j + 5u, rdy);
      if (UNLIKELY(a >= lim)) {
        if (a == aend) {
          const unsigned code = tl[ta++];
          const unsigned ap = code & 63u, bp = (code >> 6) & 15u, shape = (code >> 10) & 3u;
          if (lw == 0u || (lw == 1u && shape == 0u)) src = abase + (ap + lw) * kPanel;
          else src = bbase + (bp + (shape == 1u ? lw - 1u : 0u)) * kPanel;
          idle = shape == 3u && lw == 2u ? 1u : 0u;
          aend = ta == ntiles ? ~0u : aend + 256u;
          rdy = min(ok, aend);
        }
        if (a <= j && ok <= a) {
          WAIT_VM(0);
          if (lookat != ~0u) look();
          const unsigned long long t0 = __builtin_amdgcn_s_memrealtime();
          for (unsigned it = 0; ok <= a; ++it) {
            ask((it & TMOEV) == TMOEV ? kTmo : kOob);
            WAIT_VM(0);
            look();
            if (ok > a) break;
            if (__builtin_amdgcn_s_memrealtime() - t0 > kDeadline) { set_timeout(myws, call); give_up(); break; }
            __builtin_amdgcn_s_sleep(LSLEEP);
          }
          known();
        }
        lim = min(j + 5u, rdy);
      }
      if (a < lim) {
        issue(idle == 0u);
        if (UNLIKELY(a < lim)) issue(idle == 0u);
      }
      if (UNLIKELY(a >= askat)) { ask(PTMO); askat = ~0u; lookat = a + 4u; }
      const unsigned n = idle != 0u ? 0u : a - 1u - j;
      if (UNLIKELY(n < 4u)) {
        WAIT_VM(24);
        if (n < 3u) {
          WAIT_VM(16);
          if (n < 2u) {
            WAIT_VM(8);
            if (n < 1u) WAIT_VM(0);
          }
        }
      }
      WAIT_VM(32);
      if (UNLIKELY(j + 5u >= lookat)) { look(); known(); }
      BARRIER();
    }
    if (hass && lw < 2u) {
      const unsigned rep = kFlags + ((unsigned)bid & 7u) * kRep;
      const unsigned long long t0 = __builtin_amdgcn_s_memrealtime();
      unsigned long long mine = (small & 512u ? MASKL : MASKO) ^ (late ? 0ull : ~0ull);
      auto expired = [&](unsigned it) {
        bool bad = __builtin_amdgcn_s_memrealtime() - t0 > kDeadline;
        if (bad) set_timeout(myws, call);
        else if ((it & TMOEV) == TMOEV) bad = uni(__hip_atomic_load((GAS unsigned*)(myws + kTmo), __ATOMIC_RELAXED, __HIP_MEMORY_SCOPE_AGENT)) == call;
        if (UNLIKELY(bad)) {
          if (lane == 0) misc[3] = 1u;
          WAIT_LGKM0();
        }
        return bad;
      };
      BARRIER();
      if (lw == 1u) {
        const unsigned bp = (small >> 6) & 15u;
        for (unsigned it = 0;; ++it) {
          const u32x4 f = __builtin_amdgcn_raw_buffer_load_b128(rws, l16, rep + (36u + bp) * 1024u, kAuxSys);
          if (__ballot((int)((f[0] - call) | (f[1] - call) | (f[2] - call) | (f[3] - call)) < 0) == 0ull || expired(it)) break;
          __builtin_amdgcn_s_sleep(PSLEEP);
        }
        if (lane == 0) misc[5] = 1u;
        for (unsigned i = NPF; i-- > 0u;) {
          const unsigned pr = 63u - (unsigned)__builtin_clzll(mine);
          mine ^= 1ull << pr;
          const unsigned sb = bbase + bp * kPanel + ((small >> 12) & 1u) * 4096u + pr * 32768u;
          WAIT_VM(32);
#pragma unroll
          for (int w = 0; w < 4; ++w) {
            LAS char* const dl = lds + (w * (NPF * 4096) + i * 4096u);
            __builtin_amdgcn_raw_ptr_buffer_load_lds(rws, (LAS void*)dl, 16, l16, sb + w * 8192u, 0, kAuxDev);
            __builtin_amdgcn_raw_ptr_buffer_load_lds(rws, (LAS void*)dl, 16, l16, sb + w * 8192u, 1024, kAuxDev);
            __builtin_amdgcn_raw_ptr_buffer_load_lds(rws, (LAS void*)dl, 16, l16, sb + w * 8192u, 2048, kAuxDev);
            __builtin_amdgcn_raw_ptr_buffer_load_lds(rws, (LAS void*)dl, 16, l16, sb + w * 8192u, 3072, kAuxDev);
          }
          FENCE();
        }
        WAIT_VM(0);
        if (lane == 0) misc[6] = 1u;
        return;
      }
      const unsigned upto = 128u - 2u * (unsigned)__builtin_clzll(mine);
      unsigned have = 0;
      bool part = late;
      for (unsigned it = 0; have < upto || part; ++it) {
        {
          const unsigned k = min(have >> 1, 63u);
          const u32x2 f = __builtin_amdgcn_raw_buffer_load_b64(rws, have < upto && (unsigned)lane - (k & ~15u) < NARROW ? (unsigned)lane * 8u : kOob, rep + (small & 63u) * 1024u, kAuxSys);
          const unsigned g = __builtin_amdgcn_raw_buffer_load_b32(rws, part && have + PGATE >= upto ? (unsigned)(lane & 3) * 4u : kOob, kPFlags + tau * 128u, kAuxSys);
          const unsigned long long b0 = __ballot((int)(f[0] - call) < 0) >> k, b1 = __ballot((int)(f[1] - call) < 0) >> k;
          const unsigned n = max(have, min(b0 ? 2u * (k + (unsigned)__builtin_ctzll(b0)) : 128u, b1 ? 2u * (k + (unsigned)__builtin_ctzll(b1)) + 1u : 128u));
          if (n != have && lane == 0) misc[4] = n;
          if (part && __ballot((int)(g - call) < 0) == 0ull) {
#pragma unroll
            for (int i = 0; i < 16; ++i)
              __builtin_amdgcn_raw_ptr_buffer_load_lds(rws, (LAS void*)(lds + kPartLds + i * 1024), 16, l16, kPart + tau * 16384u + i * 1024u, 0, kAuxDev);
            part = false;
          }
          if (n != have) { have = n; continue; }
        }
        if (expired(it)) {
          if (lane == 0) misc[4] = 128u;
          break;
        }
        __builtin_amdgcn_s_sleep(PSLEEP);
      }
      WAIT_VM(0);
    }
    return;
  }

  {
    const unsigned slot = (unsigned)(4 * fr + (fq ^ ((-(fr >> 2)) & 3))) * 16u;
    unsigned so = 0;
    for (int ti = 0; ti < ntiles; ++ti) {
      const unsigned code = tl[ti];
      const unsigned shape = (code >> 10) & 3u;
      const unsigned wh = (unsigned)wid >> 1, wl = (unsigned)wid & 1u;
      const unsigned ap = (code & 63u) + (shape == 0u ? wh : 0u), bp = ((code >> 6) & 15u) + (shape == 1u ? wh : 0u);
      const unsigned jo = (bp - 8u) >> 1;
      const unsigned owner = (bp < 8u ? rank : og + jo) & 7u;
      const unsigned col0 = bp < 8u ? bp * 128u : 256u * q + (bp & 1u) * 128u;
      const __amdgpu_buffer_rsrc_t ro = mk_rsrc(p.out[owner], kInBytes);
      const unsigned vo = (unsigned)fq * 16384u + (unsigned)fr * 16u;

      auto run = [&](auto mconst, const unsigned aoff, const unsigned boff) {
        constexpr int M = decltype(mconst)::value;
        constexpr int NS = 256;
        const unsigned va = aoff + slot, vb = boff + l16;
        f32x4 acc[M][4];
#pragma unroll
        for (int m = 0; m < M; ++m)
#pragma unroll
          for (int n = 0; n < 4; ++n) acc[m][n] = f32x4{0.f, 0.f, 0.f, 0.f};
        f16x8 fa[2][M], fb[2][4];
        BARRIER();
        STAMP(myws, wid == 0 ? 2 + 3 * ti : 31);
        CSTAMP(myws, wid == 0 ? 12 + 2 * ti : 31);
#pragma unroll
        for (int i = 0; i < M; ++i) {
          if (i < 4) fb[0][i] = *(LAS f16x8*)(lds + (vb + so) + i * 1024);
          fa[0][i] = *(LAS f16x8*)(lds + (va + so) + i * 1024);
        }
        __builtin_amdgcn_s_waitcnt(0xC07F);
        BARRIER();
        so = so == kRing - kSlot ? 0u : so + kSlot;
        {
          unsigned ua = va + so, ub = vb + so;
          for (int t = 0; t < NS; t += 2) {
#pragma unroll
            for (int s = 0; s < 2; ++s) {
              __builtin_amdgcn_sched_barrier(0);
#pragma unroll
              for (int i = 0; i < 4 * M; ++i) {
                acc[i >> 2][i & 3] = __builtin_amdgcn_mfma_f32_16x16x32_f16(fa[s][i >> 2], fb[s][i & 3], acc[i >> 2][i & 3], 0, 0, 0);
                __builtin_amdgcn_sched_barrier(0);
                if (M == 8 && (i & 1) == 0 && i < 24) {
                  const int r = i >> 1;
                  if (r < 4) fb[s ^ 1][r] = *(LAS f16x8*)(lds + ub + r * 1024);
                  else fa[s ^ 1][r - 4] = *(LAS f16x8*)(lds + ua + (r - 4) * 1024);
                }
                if (M == 4 && i < 4) {
                  fb[s ^ 1][i] = *(LAS f16x8*)(lds + ub + i * 1024);
                  fa[s ^ 1][i] = *(LAS f16x8*)(lds + ua + i * 1024);
                }
                if (i == 4 * M - 10) { so = so == kRing - kSlot ? 0u : so + kSlot; asm volatile("" : "+s"(so)); }
                if (i == 4 * M - 6) { ua = va + so; asm volatile("" : "+v"(ua)); }
                if (i == 4 * M - 4) { ub = vb + so; asm volatile("" : "+v"(ub)); }
                if (i == 4 * M - 2) {
                  __builtin_amdgcn_s_waitcnt(0xC07F);
                  if (t < NS - 2) BARRIER();
                }
                __builtin_amdgcn_sched_barrier(0);
              }
            }
          }
          so = so == 0u ? kRing - kSlot : so - kSlot;
        }
        const bool hand_over = hass && ti == ntiles - 1;
        if (M == 4 && bp < 8u && hand_over) BARRIER();
        STAMP(myws, wid == 0 ? 3 + 3 * ti : 31);
        CSTAMP(myws, wid == 0 ? 13 + 2 * ti : 31);
        const int lo = misc[3] != 0u ? 0x7fc00000 : 0;
        if (bp >= 8u) {
          const __amdgpu_buffer_rsrc_t rw4 = __builtin_amdgcn_make_buffer_rsrc((void*)myws, 4, kWsNeed / 4u, 0x00020000);
          if (UNLIKELY(lo != 0)) {
#pragma unroll
            for (int m = 0; m < M; ++m)
#pragma unroll
              for (int n = 0; n < 4; ++n) acc[m][n] = __builtin_bit_cast(f32x4, u32x4{0x7fc00000u, 0x7fc00000u, 0x7fc00000u, 0x7fc00000u});
          }
          if constexpr (M == 4) {
            const unsigned hp = park_pos(ap, wh, jo);
            const unsigned s = hp * 4u + (bp & 1u) * 2u + wl;
            const unsigned pb0 = obuf + (jo * 224u + s) * kPiece;
            unsigned l0 = so + (unsigned)wid * kSlot;
            l0 = (unsigned)(size_t)lds + (l0 >= kRing ? l0 - kRing : l0);
            unsigned pos = l0 + 512u;
            const unsigned trash = l0 + kPBuf + (unsigned)lane * 4u;
            u32x2 mk = {0u, 0u};
#pragma unroll
            for (int m = 0; m < 4; ++m)
#pragma unroll
              for (int j = 0; j < 4; ++j)
#pragma unroll
                for (int n = 0; n < 4; ++n) {
                  const float f = acc[m][n][j];
                  const unsigned iv = __float_as_uint(f);
                  const bool keep = (int)iv >= 0;
                  const unsigned long long b = __ballot(keep);
                  unsigned a = lanes_below(b) * 4u + pos;
                  asm volatile("" : "+v"(a));
                  *(LAS unsigned*)(size_t)(keep ? a : trash) = iv;
                  advance(pos, b);
                  mk[m >> 1] = __builtin_amdgcn_alignbit(mk[m >> 1], iv, 31);
                }
            *(LAS u32x2*)(size_t)(l0 + (unsigned)lane * 8u) = mk;
            const unsigned bytes = hp < 56u ? pos - l0 : 0u;
            u32x4 v[17];
#pragma unroll
            for (int i = 0; i < 17; ++i) v[i] = *(LAS u32x4*)(size_t)(l0 + l16 + i * 1024);
            WAIT_LGKM0();
            if (hand_over) BARRIER();
#pragma unroll
            for (int i = 0; i < 17; ++i)
              if (i * 1024u < bytes) __builtin_amdgcn_raw_buffer_store_b128(v[i], rws, i * 1024u + l16 < bytes ? l16 : kOob, pb0 + i * 1024u, kAuxSys);
            WAIT_VM(0);
            if (lane == 0 && hp < 56u) flag_store2(p.ws[owner] + kOFlags + par * 8192u + q * 2048u + s * 8u, call, (bytes - 512u) >> 2);
          } else
#pragma unroll
          for (int rh = 0; rh < M / 4; ++rh) {
            const unsigned hp = park_pos(ap, M == 8 ? (unsigned)rh : wh, jo);
            const unsigned s = hp * 4u + (bp & 1u) * 2u + wl;
            const unsigned pb0 = obuf + (jo * 224u + s) * kPiece;
            unsigned pos = pb0 + 512u;
            if (hp < 56u) {
              u32x2 mk = {0u, 0u};
#pragma unroll
              for (int m = 0; m < 4; ++m)
#pragma unroll
                for (int j = 0; j < 4; ++j)
#pragma unroll
                  for (int n = 0; n < 4; ++n) {
                    const float f = acc[4 * rh + m][n][j];
                    const unsigned iv = __float_as_uint(f);
                    const unsigned long long b = __ballot((int)iv >= 0);
                    const unsigned idx = lanes_below(b) | (iv & kOob);
                    asm volatile("buffer_store_dword %0, %1, %2, %3 idxen sc0 sc1" :: "v"(iv), "v"(idx), "s"(rw4), "s"(pos) : "memory");
                    advance(pos, b);
                    mk[m >> 1] = __builtin_amdgcn_alignbit(mk[m >> 1], iv, 31);
                  }
              __builtin_amdgcn_raw_buffer_store_b64(mk, rws, (unsigned)lane * 8u, pb0, kAuxSys);
            }
            if (M == 4 && hand_over) BARRIER();
            if (hp < 56u) {
              WAIT_VM(0);
              if (lane == 0) flag_store2(p.ws[owner] + kOFlags + par * 8192u + q * 2048u + s * 8u, call, (pos - pb0 - 512u) >> 2);
            }
          }
        } else {
#pragma unroll
          for (int m = 0; m < M; ++m) {
            const unsigned ob = (panel_row(ap, M == 8 ? (unsigned)(m >> 2) : wh, rank) + 16u * (m & 3)) * 4096u + (col0 + 64u * wl) * 4u;
#pragma unroll
            for (int j = 0; j < 4; ++j) {
              u32x4 o;
#pragma unroll
              for (int n = 0; n < 4; ++n) { const float f = acc[m][n][j]; o[n] = (unsigned)max(__float_as_int(f), lo); }
              __builtin_amdgcn_raw_buffer_store_b128(o, ro, vo, ob + j * 4096u, kAuxSys);
            }
          }
        }
        if (M == 8 && hand_over) BARRIER();
      };
      if (shape == 3u) {
        run(std::integral_constant<int, 4>{}, wh * 4096u, 8192u + wl * 4096u);
      } else {
        run(std::integral_constant<int, 8>{}, shape == 0u ? wh * 8192u : 0u, (shape == 0u ? 16384u : 8192u + wh * 8192u) + wl * 4096u);
      }
      STAMP(myws, wid == 0 ? 4 + 3 * ti : 31);
    }
    if (!hass) return;
    const unsigned ap = small & 63u, bp = (small >> 6) & 15u, ch = (small >> 12) & 1u;
    const unsigned owner = (bp < 8u ? rank : og + ((bp - 8u) >> 1)) & 7u;
    const unsigned sa = abase + ap * kPanel + (unsigned)wid * 4096u, sb = bbase + bp * kPanel + ch * 4096u + (unsigned)wid * 8192u;
    unsigned long long mine = (small & 512u ? MASKL : MASKO) ^ (late ? 0ull : ~0ull);
    unsigned have = 0;
    constexpr int kNL = __builtin_popcountll(MASKL), kNE = 64 - kNL, kNO = __builtin_popcountll(MASKO);
    static_assert(kNO % SD == 0 && kNO >= NPF + SD && 64 - kNO >= NPF + SD, "");
    const int rounds = (__builtin_popcountll(mine) - NPF) / SD - 1;
    static_assert(kNE % SD == 0 && kNL % SD == 0 && NPF % SD == 0 && kNE >= NPF + SD && kNL >= NPF + SD && NPF >= 4, "");
    f32x4 acc[4][4];
#pragma unroll
    for (int m = 0; m < 4; ++m)
#pragma unroll
      for (int n = 0; n < 4; ++n) acc[m][n] = f32x4{0.f, 0.f, 0.f, 0.f};
    f32x4 ra[SD][4], rb[SD][4];
    auto await = [&](int word, unsigned need) {
      unsigned v;
      const unsigned long long t0 = __builtin_amdgcn_s_memrealtime();
      while ((v = uni(misc[word])) <= need)
        if (__builtin_amdgcn_s_memrealtime() - t0 > 2u * kDeadline) { misc[3] = 1u; v = 128u; break; }
      FENCE();
      return v;
    };
    auto fetch = [&](int u, bool cols) {
      const unsigned pr = (unsigned)__builtin_ctzll(mine), need = 2u * pr + ((unsigned)wid >> 1);
      mine &= mine - 1ull;
      if (cols) {
#pragma unroll
        for (int i = 0; i < 4; ++i) rb[u][i] = ld16<kAuxDev>(rws, l16 + i * 1024u, sb + pr * 32768u);
      }
      FENCE();
      if (have <= need) have = await(4, need);
#pragma unroll
      for (int i = 0; i < 4; ++i) ra[u][i] = ld16<kAuxDev>(rws, l16 + i * 1024u, sa + pr * 16384u);
      __builtin_amdgcn_sched_barrier(0);
    };
    auto mm = [&](int u) {
#pragma unroll
      for (int m = 0; m < 4; ++m)
#pragma unroll
        for (int n = 0; n < 4; ++n)
          acc[m][n] = __builtin_amdgcn_mfma_f32_16x16x32_f16(__builtin_bit_cast(f16x8, ra[u][m]), __builtin_bit_cast(f16x8, rb[u][n]), acc[m][n], 0, 0, 0);
    };
    LAS char* const mylds = lds + ((unsigned)wid * (NPF * 4096u) + l16);
    const __amdgpu_buffer_rsrc_t ro = mk_rsrc(p.out[owner], kInBytes);
    const unsigned ob = (panel_row(ap, 0u, rank) + 16u * (unsigned)wid) * 4096u + ((bp < 8u ? bp * 128u : 256u * q + (bp & 1u) * 128u) + 64u * ch) * 4u;
    STAMP(myws, wid == 0 ? 2 + 3 * ntiles : 31);
    await(5, 0u);
#pragma unroll
    for (int i = 0; i < SD - 1; ++i) fetch(i, true);
    for (int i = rounds; i > 0; --i) {
#pragma unroll
      for (int u = 0; u < SD; ++u) {
        fetch((u + SD - 1) % SD, true);
        mm(u);
      }
    }
#pragma unroll
    for (int u = 0; u < SD; ++u) {
      fetch((u + SD - 1) % SD, u == 0);
      mm(u);
    }
    await(6, 0u);
#pragma unroll
    for (int i = 0; i < NPF; ++i) {
#pragma unroll
      for (int n = 0; n < 4; ++n) rb[i % SD][n] = *(LAS f32x4*)(mylds + (i * 4096 + n * 1024));
      if (i + SD - 1 < NPF) fetch((i + SD - 1) % SD, false);
      mm(i % SD);
    }
    STAMP(myws, wid == 0 ? 3 + 3 * ntiles : 31);
    STAMP(myws, 18 + wid);
#pragma unroll
    for (int m = 0; m < 4; ++m)
#pragma unroll
      for (int n = 0; n < 4; ++n) *(LAS f32x4*)(mylds + (4 * m + n) * 1024) = acc[m][n];
    WAIT_LGKM0();
    BARRIER();
    STAMP(myws, wid == 0 ? 24 : 31);
    f32x4 sum[4];
#pragma unroll
    for (int n = 0; n < 4; ++n) {
      LAS char* const ps = lds + ((unsigned)wid * 4096u + n * 1024 + l16);
      sum[n] = (*(LAS f32x4*)ps + *(LAS f32x4*)(ps + NPF * 4096)) + (*(LAS f32x4*)(ps + 2 * NPF * 4096) + *(LAS f32x4*)(ps + 3 * NPF * 4096));
    }
    const int lo = misc[3] != 0u ? 0x7fc00000 : 0;
    if (!late) {
#pragma unroll
      for (int n = 0; n < 4; ++n) {
        if (UNLIKELY(lo != 0)) sum[n] = __builtin_bit_cast(f32x4, u32x4{0x7fc00000u, 0x7fc00000u, 0x7fc00000u, 0x7fc00000u});
        __builtin_amdgcn_raw_buffer_store_b128(__builtin_bit_cast(u32x4, sum[n]), rws, l16 + n * 1024u, kPart + tau * 16384u + (unsigned)wid * 4096u, kAuxDev);
      }
      WAIT_VM(0);
      if (lane == 0) flag_store(myws + kPFlags + tau * 128u + (unsigned)wid * 4u, call);
      STAMP(myws, wid == 0 ? 4 + 3 * ntiles : 31);
      return;
    }
#pragma unroll
    for (int n = 0; n < 4; ++n) sum[n] += *(LAS f32x4*)(lds + (kPartLds + (unsigned)wid * 4096u + n * 1024 + l16));
#pragma unroll
    for (int j = 0; j < 4; ++j) {
      u32x4 o;
#pragma unroll
      for (int n = 0; n < 4; ++n) { const float f = sum[n][j]; o[n] = (unsigned)max(__float_as_int(f), lo); }
      __builtin_amdgcn_raw_buffer_store_b128(o, ro, (unsigned)fq * 16384u + (unsigned)fr * 16u, ob + j * 4096u, kAuxSys);
    }
    STAMP(myws, wid == 0 ? 23 : 31);
    if (bp >= 8u) {
      WAIT_VM(0);
      if (lane == 0) __hip_atomic_fetch_add((GAS unsigned*)(p.ws[owner] + kDone), 1u, __ATOMIC_RELAXED, __HIP_MEMORY_SCOPE_SYSTEM);
    }
    STAMP(myws, wid == 0 ? 4 + 3 * ntiles : 31);
  }
}

extern "C" void kernel_launch(int rank, int nranks, ncclComm_t comm, hipStream_t stream,
                              void* const* d_in, const int* in_sizes, int n_in,
                              void* d_out, int out_size,
                              const int* in_dtypes, int out_dtype,
                              void* const* d_out_all, void* const* d_ws_all, size_t ws_bytes) {
  (void)comm; (void)in_dtypes; (void)out_dtype; (void)d_out;
  if (nranks != 8 || (unsigned)rank >= 8u || n_in < 2 || ws_bytes < (size_t)kWsNeed ||
      in_sizes[0] != 8 * 1024 * 1024 || in_sizes[1] != 8 * 1024 * 1024 || out_size != 8 * 1024 * 1024) {
    fprintf(stderr, "kernel_launch: unexpected problem geometry\n");
    return;
  }
  Params p;
  p.x = (const float*)d_in[0];
  p.w = (const float*)d_in[1];
  for (int i = 0; i < 8; ++i) { p.ws[i] = (char*)d_ws_all[i]; p.out[i] = (float*)d_out_all[i]; }
  p.rank = rank;
  p.pad = 0;
  static const int xcol[8] = {0, 4, 2, 3, 1, 5, 6, 7};
  auto tile = [](int a, int b, int shape) { return (unsigned)(a | (b << 6) | (shape << 10)); };
  for (int ci = 0; ci < kNComp; ++ci) {
    const int x = ci & 7, s = ci >> 3, bq = 2 * (x ^ 4), j = x >> 1;
    unsigned t[3] = {0u, 0u, 0u};
    int n = 0;
    auto farb = [&](int e) { const int idx = 3 * (x & 1) + e; return 8 + idx + (idx >= 2 * j ? 2 : 0); };
    if (s < 12) {
      t[n++] = tile(x < 4 ? 2 * (s >> 2) : 8 + 5 * (s >> 2), 4 * (x & 3) + (s & 3), 0);
      t[n++] = tile(10 + 5 * (s >> 2) + ((s >> 1) & 1), bq + (s & 1), 3);
    } else if (s == 12) {
      t[n++] = tile(6, xcol[x], 0);
      t[n++] = tile(27 + 2 * j, farb(1), 3);
    } else if (s < 16) {
      t[n++] = tile(12 + 5 * (s - 13), bq, 1);
    } else if (s < 18) {
      t[n++] = tile(34 + (s - 16), bq, 1);
    } else if (s < 23) {
      t[n++] = tile(26 + 2 * j, s < 22 ? 4 * (x & 1) + s - 18 : farb(2), 0);
    } else {
      t[n++] = tile(26 + 2 * j + ((s - 23) & 1), farb((s - 23) >> 1), 3);
    }
    const int i = s < 12 ? s : s == SKIPB || s == SKIPA ? -1 : s - 12 - (s > SKIPA ? 1 : 0) - (s > SKIPB ? 1 : 0);
    if (i >= 0) t[n++] = tile(23 + i % 3, i < 6 ? 8 + x : x, 2) | ((unsigned)((i / 3) & 1) << 12) | (s < 12 ? 8192u : 0u);
    p.tiles[3 * ci] = t[0] | ((unsigned)n << 28);
    p.tiles[3 * ci + 1] = t[1];
    p.tiles[3 * ci + 2] = t[2];
  }
  hipError_t e = hipFuncSetAttribute((const void*)ag_gemm_relu_kernel, hipFuncAttributeMaxDynamicSharedMemorySize, (int)kLdsBytes);
  if (e != hipSuccess) { fprintf(stderr, "hipFuncSetAttribute: %s\n", hipGetErrorString(e)); return; }
  hipLaunchKernelGGL(ag_gemm_relu_kernel, dim3(kNWG), dim3(512), kLdsBytes, stream, p);
  e = hipGetLastError();
  if (e != hipSuccess) fprintf(stderr, "launch: %s\n", hipGetErrorString(e));
}
```

```cpp
#include <hip/hip_runtime.h>
#include <rccl/rccl.h>
#include <cstdio>
#include <cstdint>
#include <type_traits>

typedef _Float16 f16x8 __attribute__((ext_vector_type(8)));
typedef float f32x4 __attribute__((ext_vector_type(4)));
typedef unsigned u32x4 __attribute__((ext_vector_type(4)));
typedef unsigned u32x2 __attribute__((ext_vector_type(2)));
#define LAS __attribute__((address_space(3)))
#define GAS __attribute__((address_space(1)))

#define NWS 9u
#define WPM 3u
#define PU 1u
#define PUW 1u
#define CVDELAY 32
#define CVFIRST 50
#define CVLEAD 40u
#define XSLEEP 4
#define CVFROM 13
#define PWIN 4u
#define CVAUX kAuxSys
#define MSLEEP 4
#define LSLEEP 2
#define WLIM 160u
#define SKIPA 22
#define ALANES 63
#define PTMO kOob
#define TMOEV 7u
#define SD 4
#define MASKL 0xffaaaaaaaaaa0000ull
#define MASKO MASKL
#define LDSPARK
#define FPOLL
#define URG
#define NARROW 32u
#define ERAISE 1320u
#define FEWF 80u
#define CVROT 10u
#define PSL 6
#define PSL0 16
#define PGATE 32u
#define CVSKIP ((PU + 1u) * NWS / 2u)
static_assert((PU + 1u) * NWS % 2u == 0u && 10u * NWS <= 8u * CVFROM, "");
#define USLEEP 2
#define FPS 8
#define NPF 8
#define SKIPB 12
#define PSLEEP 1
#define LTHR 24u

#define LDCOND(j) ((j) + 40u >= total)
namespace {

constexpr unsigned kBlk = 16384u;
constexpr unsigned kPanel = 2u << 20;
constexpr unsigned kAOff = 2u << 20;
constexpr unsigned kASize = 36u * kPanel;
constexpr unsigned kBOff = kAOff + 2u * kASize;
constexpr unsigned kBSize = 16u * kPanel;
constexpr unsigned kLast = 23u * kPanel + (1u << 20);
constexpr unsigned kPiece = 16896u;
constexpr unsigned kOOff = kBOff + 2u * kBSize;
constexpr unsigned kOSize = 4u * 224u * kPiece;
constexpr unsigned kPart = kOOff + 2u * kOSize;
constexpr unsigned kWsNeed = kPart + 96u * 16384u;
constexpr unsigned kPFlags = 131072;
constexpr unsigned kTmo = 128, kDone = 256, kCvt = 384, kAll = 512, kQueue = 1024;
constexpr unsigned kFirst = 2048;
constexpr unsigned kTicket = 4096;
constexpr unsigned kFlags = 1u << 20, kRep = 57600;
constexpr unsigned kOFlags = 81920;
constexpr unsigned kSrc = 98304;
constexpr unsigned kInBytes = 32u << 20;
constexpr int kNMove = 48, kNComp = 208, kNWG = 256;
constexpr unsigned kRowsI = 704u, kRowsC = 688u;
constexpr unsigned kDonePerCall = 192u;
constexpr unsigned long long kDeadline = 50000000ull;
constexpr unsigned kSlot = 24576u, kRing = 6u * kSlot;
constexpr unsigned kPBuf = 17408u;
constexpr unsigned kLdsMisc = 9u * kPBuf, kLdsBytes = kLdsMisc + 128u;
constexpr unsigned kPartLds = NPF * 16384u;
static_assert(kLdsMisc >= kRing && kPartLds + 16384u <= kRing, "");
constexpr unsigned kOob = 0x80000000u;
constexpr int kAuxSys = 17;
constexpr int kAuxDev = 16;

struct Params {
  const float* x;
  const float* w;
  char* ws[8];
  float* out[8];
  int rank;
  int pad;
  uint32_t tiles[3 * kNComp];
};

__device__ __forceinline__ __amdgpu_buffer_rsrc_t mk_rsrc(const void* p, unsigned bytes) {
  return __builtin_amdgcn_make_buffer_rsrc((void*)p, 0, bytes, 0x00020000);
}
template <int AUX>
__device__ __forceinline__ f32x4 ld16(__amdgpu_buffer_rsrc_t r, unsigned voff, unsigned soff) {
  return __builtin_bit_cast(f32x4, __builtin_amdgcn_raw_buffer_load_b128(r, voff, soff, AUX));
}
template <int AUX>
__device__ __forceinline__ void st16(f16x8 v, __amdgpu_buffer_rsrc_t r, unsigned voff, unsigned soff) {
  __builtin_amdgcn_raw_buffer_store_b128(__builtin_bit_cast(u32x4, v), r, voff, soff, AUX);
}
__device__ __forceinline__ f16x8 cvt8(f32x4 a, f32x4 b) {
  f16x8 h;
#pragma unroll
  for (int i = 0; i < 4; ++i) { h[i] = (_Float16)a[i]; h[4 + i] = (_Float16)b[i]; }
  return h;
}
__device__ __forceinline__ f16x8 cvt_col(const f32x4 (&v)[8], int i) {
  f16x8 h;
#pragma unroll
  for (int j = 0; j < 8; ++j) h[j] = (_Float16)v[j][i];
  return h;
}
#define WAIT_VM(n) asm volatile("s_waitcnt vmcnt(%0)" :: "n"(n) : "memory")
#define WAIT_LGKM0() asm volatile("s_waitcnt lgkmcnt(0)" ::: "memory")
#define FENCE() asm volatile("" ::: "memory")
#define UNLIKELY(c) __builtin_expect(!!(c), 0)
#define BARRIER() do { FENCE(); __builtin_amdgcn_s_barrier(); FENCE(); } while (0)

__device__ __forceinline__ void flag_store(char* p, unsigned v) {
  __hip_atomic_store((GAS unsigned*)p, v, __ATOMIC_RELAXED, __HIP_MEMORY_SCOPE_SYSTEM);
}
__device__ __forceinline__ void flag_store2(char* p, unsigned lo, unsigned hi) {
  __hip_atomic_store((GAS unsigned long long*)p, (unsigned long long)lo | ((unsigned long long)hi << 32), __ATOMIC_RELAXED, __HIP_MEMORY_SCOPE_SYSTEM);
}
__device__ __forceinline__ unsigned draw(char* p) {
  unsigned z = 0;
  asm volatile("" : "+v"(z));
  p += z;
  return __hip_atomic_fetch_add((GAS unsigned*)p, 1u, __ATOMIC_RELAXED, __HIP_MEMORY_SCOPE_AGENT);
}
__device__ __forceinline__ unsigned uni(unsigned v) { return (unsigned)__builtin_amdgcn_readfirstlane((int)v); }
__device__ __forceinline__ unsigned lanes_below(unsigned long long b) {
  return __builtin_amdgcn_mbcnt_hi((unsigned)(b >> 32), __builtin_amdgcn_mbcnt_lo((unsigned)b, 0u));
}
__device__ __forceinline__ void advance(unsigned& pos, unsigned long long b) {
  asm("s_lshl2_add_u32 %0, %1, %0" : "+s"(pos) : "s"((unsigned)__builtin_popcountll(b)) : "scc");
}

#define VSTAMP(ws, slot, val) do {} while (0)
#define STAMP(ws, slot) do {} while (0)
#define PSTAMP(ws, cls, n) do {} while (0)
#define CSTAMP(ws, slot) do {} while (0)

__device__ __forceinline__ void set_timeout(char* myws, unsigned call) {
  __hip_atomic_store((GAS unsigned*)(myws + kTmo), call, __ATOMIC_RELAXED, __HIP_MEMORY_SCOPE_AGENT);
}
__device__ __forceinline__ unsigned peer_of(unsigned r, unsigned i) { return (r & 4u) + ((r + i + 1u) & 3u); }
__device__ __forceinline__ unsigned panel_row(unsigned ap, unsigned rh, unsigned rank) {
  const unsigned og = (rank & 4u) ^ 4u;
  if (ap < 8u) return rank * 1024u + ap * 128u + rh * 64u;
  if (ap < 23u) { const unsigned i = (ap - 8u) / 5u, s = (ap - 8u) - 5u * i; return peer_of(rank, i) * 1024u + s * 128u + rh * 64u; }
  if (ap < 26u) return peer_of(rank, ap - 23u) * 1024u + 640u;
  if (ap < 34u) return (og + ((ap - 26u) >> 1)) * 1024u + 768u + (ap & 1u) * 128u + rh * 64u;
  return (og + 2u * (ap - 34u) + rh) * 1024u + 704u;
}
__device__ __forceinline__ unsigned park_pos(unsigned ap, unsigned rh, unsigned j) {
  if (ap < 8u) return 2u * ap + rh < 11u ? 2u * ap + rh : 99u;
  if (ap < 23u) {
    const unsigned i = (ap - 8u) / 5u, s = (ap - 8u) - 5u * i;
    return s < 2u ? 11u + 4u * i + 2u * s + rh : s < 4u ? 40u + 4u * i + 2u * s + rh : 26u + 2u * i + rh;
  }
  const unsigned src = ap < 34u ? (ap - 26u) >> 1 : 2u * (ap - 34u) + rh;
  if (src == j) return 99u;
  const unsigned e = src - (src > j ? 1u : 0u);
  return ap < 34u ? 32u + 4u * e + 2u * (ap & 1u) + rh : 23u + e;
}
__device__ __forceinline__ unsigned park_row(unsigned hp, unsigned src, unsigned rank) {
  const unsigned q = rank & 3u;
  if (hp < 11u) return src * 1024u + hp * 64u;
  if (hp < 23u) return peer_of(src, (hp - 11u) >> 2) * 1024u + ((hp - 11u) & 3u) * 64u;
  if (hp < 26u) { const unsigned e = hp - 23u; return ((rank & 4u) + e + (e >= q ? 1u : 0u)) * 1024u + 704u; }
  if (hp < 32u) return peer_of(src, (hp - 26u) >> 1) * 1024u + 512u + (hp & 1u) * 64u;
  if (hp < 44u) { const unsigned e = (hp - 32u) >> 2; return ((rank & 4u) + e + (e >= q ? 1u : 0u)) * 1024u + 768u + (hp & 3u) * 64u; }
  return peer_of(src, (hp - 44u) >> 2) * 1024u + 256u + (hp & 3u) * 64u;
}

}

__global__ void __launch_bounds__(512) __attribute__((amdgpu_waves_per_eu(2, 2)))
ag_gemm_relu_kernel(const Params p) {
  extern __shared__ __attribute__((aligned(16))) char smem_raw[];
  LAS char* const lds = (LAS char*)smem_raw;
  LAS volatile unsigned* const misc = (LAS volatile unsigned*)(lds + kLdsMisc);

  const int tid = threadIdx.x;
  const int lane = tid & 63;
  const int wid = __builtin_amdgcn_readfirstlane(tid >> 6);
  const int bid = blockIdx.x;
  const unsigned rank = (unsigned)p.rank & 7u;
  const unsigned q = rank & 3u, og = (rank & 4u) ^ 4u;
  char* const myws = p.ws[rank];
  const unsigned l16 = (unsigned)lane * 16u;
  const __amdgpu_buffer_rsrc_t rws = mk_rsrc(myws, kWsNeed);
  const int fr = lane & 15, fq = lane >> 4;

  STAMP(myws, wid == 0 ? 0 : 31);

  const unsigned lx = (((unsigned)lane >> 2) * 8192u + 4u * (unsigned)((lane & 3) ^ ((-(lane >> 4)) & 3))) * 4u;
  const unsigned lw = (4u * ((unsigned)lane >> 4) * 1024u + 4u * ((unsigned)lane & 15u)) * 4u;
  const __amdgpu_buffer_rsrc_t rx = mk_rsrc(p.x, kInBytes);
  const __amdgpu_buffer_rsrc_t rw = mk_rsrc(p.w, kInBytes);
  const unsigned pw = (unsigned)wid >= WPM ? (unsigned)wid - WPM : (unsigned)wid;
  const unsigned gw = (unsigned)bid * WPM + pw;
  const unsigned sid = gw % 14u, k = gw / 14u;
  const bool cross = sid >= 6u;
  const unsigned c = cross ? (sid - 6u) >> 1 : sid >> 1, h = sid & 1u;
  const unsigned sr = (cross ? og + c : peer_of(rank, c)) & 7u;
  char* const sws = p.ws[sr];
  const __amdgpu_buffer_rsrc_t rs = mk_rsrc(sws, kWsNeed);
  LAS volatile unsigned* const hand = misc + 4u + 4u * pw;
  const unsigned buf0 = pw * 3u * kPBuf;
  if (bid < kNMove && (unsigned)wid >= WPM && (unsigned)wid < 2u * WPM && gw < 14u * NWS) {
    unsigned call = 0;
    auto push = [&](auto wconst) {
      constexpr bool W = decltype(wconst)::value;
      f32x4 pv[PU][2][8];
#pragma unroll
      for (unsigned u = 0; u < PU; ++u)
#pragma unroll
        for (int i = 0; i < 16; ++i) {
          const unsigned m = u * NWS + k, kk = (m >> 1) * 64u + h * 32u;
          if (W) pv[u][i >> 3][i & 7] = ld16<0>(rw, lw + (i >> 3) * 256u, (kk * 1024u + 256u * c + (m & 1u) * 128u) * 4u + ((i & 3) + 16 * ((i >> 2) & 1)) * 4096u);
          else pv[u][i >> 3][i & 7] = ld16<0>(rx, lx + (i & 1) * 64u, ((m & 1u) * 128u * 8192u + kk) * 4u + (i >> 1) * 524288u);
        }
      __syncthreads();
      call = uni(misc[0]) + 1u;
      const unsigned base = W ? kBOff + (call & 1u) * kBSize + (8u + 2u * q) * kPanel : kAOff + (call & 1u) * kASize + (18u - 5u * c) * kPanel;
      char* const fl = sws + kFlags + (W ? 44u + 2u * q : 18u - 5u * c) * 1024u + h * 4u + (unsigned)lane * kRep;
#pragma unroll
      for (unsigned j = 0; j < 2u * PU; ++j) {
        const unsigned u = j >> 1, r = j & 1u, m = u * NWS + k;
        f16x8 hv[4];
#pragma unroll
        for (int i = 0; i < 4; ++i) hv[i] = W ? cvt_col(pv[u][r], i) : cvt8(pv[u][r][2 * i], pv[u][r][2 * i + 1]);
        asm volatile("" :: "v"(hv[0]), "v"(hv[1]), "v"(hv[2]), "v"(hv[3]));
        if (j >= PUW) WAIT_VM(4u * (PUW - 1u));
        if (j <= 2u) STAMP(myws, 8 + wid + 3 * j);
        if (u > 0u && r == PUW - 1u && lane < 8) flag_store(fl + ((m - NWS) & 1u) * 1024u + ((m - NWS) >> 1) * 8u, call);
#pragma unroll
        for (int i = 0; i < 4; ++i) st16<kAuxSys>(hv[i], rs, l16 + i * 1024u, base + (m & 1u) * kPanel + (m >> 1) * kBlk + h * 8192u + r * 4096u);
        FENCE();
      }
      WAIT_VM(0);
      const unsigned m = (PU - 1u) * NWS + k;
      if (lane < 8) flag_store(fl + (m & 1u) * 1024u + (m >> 1) * 8u, call);
    };
    if (cross) push(std::true_type{}); else push(std::false_type{});
    STAMP(myws, 4 + wid);
    if (!cross) return;
    const __amdgpu_buffer_rsrc_t ro = mk_rsrc(p.out[rank], kInBytes);
    const unsigned vo = (unsigned)fq * 16384u + (unsigned)fr * 16u;
    const unsigned long long t0 = __builtin_amdgcn_s_memrealtime();
    for (unsigned b = 0;; b = b == 2u ? 0u : b + 1u) {
      unsigned hv;
      while ((hv = uni(hand[b])) == 0u) {
        if (__builtin_amdgcn_s_memrealtime() - t0 > 2u * kDeadline) { set_timeout(myws, call); return; }
        __builtin_amdgcn_s_sleep(XSLEEP);
      }
      FENCE();
      if (hv == ~0u) break;
      const unsigned oa = park_row(hv >> 4, sr, rank) * 4096u + (256u * c + (hv & 12u) * 16u) * 4u;
      unsigned pos = (unsigned)(size_t)lds + buf0 + b * kPBuf, val[64];
      u32x2 mk = {~0u, ~0u};
      if ((hv & 2u) == 0u) mk = *(LAS u32x2*)(size_t)(pos + lane * 8);
      const unsigned fill = (unsigned)(size_t)lds + kLdsMisc + 64u + (hv & 2u) * 2u;
      pos += 512u;
#pragma unroll
      for (int i = 0; i < 64; ++i) {
        const bool keep = (int)(mk[i >> 5] << (i & 31)) >= 0;
        const unsigned long long bl = __ballot(keep);
        unsigned a = lanes_below(bl) * 4u + pos;
        asm volatile("" : "+v"(a));
        val[i] = *(LAS unsigned*)(size_t)(keep ? a : fill);
        advance(pos, bl);
      }
      WAIT_LGKM0();
      if (lane == 0) hand[b] = 0u;
      FENCE();
#pragma unroll
      for (int i = 0; i < 16; ++i)
        __builtin_amdgcn_raw_buffer_store_b128(u32x4{val[4 * i], val[4 * i + 1], val[4 * i + 2], val[4 * i + 3]}, ro, vo, oa + (i >> 2) * 65536u + (i & 3) * 4096u, kAuxSys);
    }
    STAMP(myws, 1 + wid);
    return;
  }
  if (wid == 7 && bid >= kNMove && (unsigned)(bid - kNMove) < 10u * NWS) {
    const unsigned t = (unsigned)(bid - kNMove) / (2u * NWS), e = (unsigned)(bid - kNMove) - t * 2u * NWS;
    const unsigned m = PU * NWS + (e >> 1), kk = (m >> 1) * 64u + (e & 1u) * 32u;
    auto stage = [&](auto wconst) {
      constexpr bool W = decltype(wconst)::value;
      f32x4 pv[2][8];
#pragma unroll
      for (int i = 0; i < 16; ++i) {
        if (W) pv[i >> 3][i & 7] = ld16<0>(rw, lw + (i >> 3) * 256u, (kk * 1024u + 256u * (t - 1u) + (m & 1u) * 128u) * 4u + ((i & 3) + 16 * ((i >> 2) & 1)) * 4096u);
        else pv[i >> 3][i & 7] = ld16<0>(rx, lx + (i & 1) * 64u, ((m & 1u) * 128u * 8192u + kk) * 4u + (i >> 1) * 524288u);
      }
      __syncthreads();
      const unsigned call = uni(misc[0]) + 1u;
      const unsigned base = (W ? kBOff + (call & 1u) * kBSize + 2u * (t - 1u) * kPanel : kAOff + (call & 1u) * kASize) + (m & 1u) * kPanel + (m >> 1) * kBlk + (e & 1u) * 8192u;
#pragma unroll
      for (int i = 0; i < 8; ++i) st16<CVAUX>(W ? cvt_col(pv[i >> 2], i & 3) : cvt8(pv[i >> 2][2 * (i & 3)], pv[i >> 2][2 * (i & 3) + 1]), rws, l16 + (i & 3) * 1024u, base + (i >> 2) * 4096u);
      STAMP(myws, 26);
      WAIT_VM(0);
      if (lane < (W ? 1 : 3)) flag_store(p.ws[(W ? og + t - 1u : peer_of(rank, (unsigned)lane)) & 7u] + kFirst + rank * 128u + e * 4u, call);
      STAMP(myws, 27);
    };
    if (t != 0u) stage(std::true_type{}); else stage(std::false_type{});
    return;
  }
  const bool isconv = bid >= kNMove + 8 * CVFROM;
  unsigned tk = 0;
  if (tid == 0) tk = draw(myws + kTicket + (unsigned)bid * 32u);
  FENCE();
  f32x4 cv[8];
  auto item = [&](unsigned n, unsigned& ktg, unsigned& sub) {
    if (n < 5u * CVLEAD) {
      ktg = n / 5u; sub = n - 5u * ktg; sub += sub ? 3u : 0u;
      if (ktg < CVROT + CVSKIP) ktg = ktg < CVROT ? ktg + CVSKIP : ktg - CVROT;
    }
    else if (n < 8u * CVLEAD) { ktg = (n - 5u * CVLEAD) / 3u; sub = n - 5u * CVLEAD - 3u * ktg + 1u; }
    else { ktg = n >> 3; sub = n & 7u; }
  };
  auto loads = [&](unsigned n) {
    unsigned ktg, sub;
    item(n, ktg, sub);
    if (sub < 4u) {
      const unsigned so = ((256u * sub + 32u * wid) * 8192u + ktg * 64u) * 4u;
#pragma unroll
      for (int s = 0; s < 2; ++s) {
        cv[4 * s + 0] = ld16<0>(rx, lx, so + s * 524288u); cv[4 * s + 1] = ld16<0>(rx, lx + 64u, so + s * 524288u);
        cv[4 * s + 2] = ld16<0>(rx, lx + 128u, so + s * 524288u); cv[4 * s + 3] = ld16<0>(rx, lx + 192u, so + s * 524288u);
      }
    } else {
      const unsigned so = ((ktg * 64u + 32u * (wid & 1)) * 1024u + 256u * (sub - 4u) + 64u * (wid >> 1)) * 4u;
#pragma unroll
      for (int i = 0; i < 8; ++i) cv[i] = ld16<0>(rw, lw, so + ((i & 3) + 16 * (i >> 2)) * 4096u);
    }
  };
  if (bid >= kNMove + 8 * CVFROM + CVFIRST) __builtin_amdgcn_s_sleep(CVDELAY);
  if (isconv) loads((unsigned)(bid - kNMove - 8 * CVFROM));
  FENCE();
  if (tid == 0) {
    misc[0] = tk;
    misc[3] = 0u;
  }
  if (tid >= 4 && tid < 18) misc[tid] = tid == 17 ? 0x7fc00000u : 0u;
  __syncthreads();
  const unsigned call = uni(misc[0]) + 1u;
  const unsigned par = call & 1u;
  const unsigned abase = kAOff + par * kASize, bbase = kBOff + par * kBSize, obuf = kOOff + par * kOSize;

  if (bid < kNMove) {
    if (wid == 7) {
      if (bid == 0) {
        const unsigned target = call * kDonePerCall, off = lane == 0 ? kDone : lane == 1 ? kTmo : kOob;
        const unsigned long long t0 = __builtin_amdgcn_s_memrealtime();
        unsigned res = 0;
        while (res == 0u) {
          const unsigned v = __builtin_amdgcn_raw_buffer_load_b32(rws, off, 0, kAuxSys);
          FENCE();
          res = (unsigned)__ballot(lane == 0 ? (int)(v - target) >= 0 : v == call) & 3u;
          if (res == 0u && __builtin_amdgcn_s_memrealtime() - t0 > kDeadline) res = 2u;
        }
        if ((res & 1u) == 0u && lane == 0) {
          set_timeout(myws, call);
          __hip_atomic_store((GAS unsigned*)(p.out[rank] + (size_t)rank * 1024u * 1024u), 0x7fc00000u, __ATOMIC_RELAXED, __HIP_MEMORY_SCOPE_SYSTEM);
        }
        STAMP(myws, 10);
      }
      return;
    }
    if ((unsigned)wid >= WPM || gw >= 14u * NWS) return;
    const unsigned total = cross ? 2u * kRowsC : 2u * kRowsI;
    const unsigned nplain = cross ? 1152u : 2u * kRowsI;
    const unsigned qbase = (call - 1u) * (total - PU * 2u * NWS) - (PU + 1u) * 2u * NWS;
    const unsigned queue = kQueue + (sid >> 1) * 128u;
    bool failed = false, all = false;
    unsigned cgrp = 99u, okf = 0, okt = 0, fv = 0, wbase = 0x10000u;
    u32x2 fw = {0u, 0u};
    const unsigned fo = lane < 63 ? kOFlags + par * 8192u + c * 2048u + (unsigned)lane * 8u : kTmo;
    unsigned sa = 0, da = 0, fa = 0, pend = 0;
    unsigned s2 = 4096u, d2 = 4096u;
    u32x4 v[4], w[4];
    auto raise = [&]() {
      if (pend != 0u && lane < 8) flag_store(myws + pend + (unsigned)lane * kRep, call);
      pend = 0u;
    };
    auto fetch = [&](u32x4 (&x)[4], unsigned so) {
#pragma unroll
      for (int i = 0; i < 4; ++i) x[i] = __builtin_amdgcn_raw_buffer_load_b128(rs, l16 + i * 1024u, so, kAuxSys);
      FENCE();
    };
    auto put = [&](const u32x4 (&x)[4], unsigned so) {
#pragma unroll
      for (int i = 0; i < 4; ++i) __builtin_amdgcn_raw_buffer_store_b128(x[i], rws, l16 + i * 1024u, so, kAuxDev);
      FENCE();
    };
    auto ask = [&](unsigned k0) {
      const unsigned off = lane < 62 ? kSrc + (sr * 8u + cgrp) * 512u + min(k0 + (unsigned)lane, 127u) * 4u : lane == 62 ? kAll + sr * 4u : kTmo;
      asm volatile("buffer_load_dword %0, %1, %2, 0 offen sc0 sc1" : "=v"(fv) : "v"(off), "s"(rws) : "memory");
    };
    auto look = [&]() {
      asm volatile("" : "+v"(fv) :: "memory");
      const unsigned long long b = __ballot(lane < 63 ? (int)(fv - call) >= 0 : fv == call);
      if ((b >> 62) != 0ull) { failed = (b >> 63) != 0ull; all = true; }
      okt += (unsigned)__builtin_ctzll(~b);
    };
    auto setup = [&](unsigned mh, bool first) {
      const unsigned m = mh >> 1, h = mh & 1u;
      unsigned sp, kt, dp, db, dh = h;
      unsigned so = h * 8192u, dof = h * 8192u;
      s2 = 4096u; d2 = 4096u;
      if (!cross) {
        if (m < 256u) { sp = m & 1u; kt = m >> 1; dp = 8u + 5u * c + sp; db = kt; }
        else if (m < 384u) { sp = 4u; kt = m - 256u; dp = 12u + 5u * c; db = kt; }
        else if (m < 640u) { sp = 2u + (m & 1u); kt = (m - 384u) >> 1; dp = 8u + 5u * c + sp; db = kt; }
        else { sp = 5u; db = m - 640u; kt = 2u * db + h; so = kLast - 5u * kPanel - kt * 8192u; dp = 23u + c; }
      } else {
        if (m < 256u) { sp = 36u + 2u * q + (m & 1u); kt = m >> 1; dp = 44u + 2u * c + (m & 1u); db = kt; }
        else {
          const unsigned g = ((m - 256u) * 52429u) >> 18, r = m - 256u - 5u * g;
          if (r == 2u) { sp = 5u; kt = 2u * g + h; so = 4096u; s2 = 8192u; dp = 34u + (c >> 1); db = kt; dh = c & 1u; dof = dh * 4096u; d2 = 8192u; }
          else { sp = 6u + (r % 3u); kt = 2u * g + (r > 2u ? 1u : 0u); dp = 20u + 2u * c + sp; db = kt; }
        }
      }
      const unsigned sub = sp < 36u ? sp >> 1 : (sp - 28u) >> 1;
      if (first) {
        const unsigned long long t0 = __builtin_amdgcn_s_memrealtime();
        for (;;) {
          const unsigned f = __builtin_amdgcn_raw_buffer_load_b32(rws, lane == 0 ? kFirst + sr * 128u + (mh - 2u * PU * NWS) * 4u : lane == 1 ? kTmo : kOob, 0, kAuxSys);
          FENCE();
          const unsigned b = (unsigned)__ballot(lane == 0 ? (int)(f - call) >= 0 : f == call) & 3u;
          if (b == 1u) break;
          if (b != 0u || __builtin_amdgcn_s_memrealtime() - t0 > kDeadline) { set_timeout(myws, call); failed = true; all = true; break; }
          __builtin_amdgcn_s_sleep(USLEEP);
        }
        cgrp = sub;
        okf = okt = CVSKIP;
      } else
      if (!all) {
        if (cgrp != 99u) look();
        if (!all && !(sub == cgrp && kt >= okf && kt < okt)) {
          WAIT_VM(0);
          raise();
          cgrp = sub;
          okf = kt;
          const unsigned long long t0 = __builtin_amdgcn_s_memrealtime();
          for (;;) {
            okt = kt;
            ask(kt);
            WAIT_VM(0);
            look();
            if (all || okt != kt) break;
            if (__builtin_amdgcn_s_memrealtime() - t0 > kDeadline) { set_timeout(myws, call); failed = true; all = true; break; }
            __builtin_amdgcn_s_sleep(MSLEEP);
          }
        }
        if (!all) ask(okt);
      }
      sa = (sp < 36u ? abase + sp * kPanel : bbase + (sp - 36u) * kPanel) + kt * kBlk + so;
      da = (dp < 36u ? abase + dp * kPanel : bbase + (dp - 36u) * kPanel) + db * kBlk + dof;
      fa = kFlags + dp * 1024u + db * 8u + dh * 4u;
      fetch(v, sa);
    };
    unsigned m = 2u * (PU * NWS + k) + h;
    setup(m, true);
    STAMP(myws, 20 + wid);
    bool first = true;
    do {
      unsigned dv = 0;
      if (lane == 0) dv = draw(myws + queue);
      FENCE();
      put(v, da);
      raise();
      if (first) ask(okt);
      first = false;
      fetch(w, sa + s2);
      put(w, da + d2);
      pend = fa;
      if ((m & 1u) == 0u) { PSTAMP(myws, sid >> 1, (m >> 1) + 1u); if (m < 256u) PSTAMP(myws, 8u + (sid >> 1), ((m >> 1) + 1u) * 4u); }
      m = uni(dv) - qbase;
      if (m < nplain) setup(m, false);
      if (m >= ERAISE && m < nplain) { WAIT_VM(4); raise(); }
    } while (m < nplain);
    WAIT_VM(0);
    raise();
    if (cross) {
      unsigned hval = 0;
      asm volatile("" : "+v"(fw) :: "memory");
      auto stale = [&](unsigned o) {
        return o - wbase >= 63u || (int)((unsigned)__builtin_amdgcn_readlane((int)fw[0], (int)(o - wbase)) - call) < 0;
      };
      for (unsigned b = 0, pb = 0;; pb = b, b = b == 2u ? 0u : b + 1u) {
        const unsigned o = m - 1152u, s = o;
        unsigned cnt = 2048u, dv = 0;
        if (m >= total || stale(o)) {
          WAIT_VM(0);
          if (hval != 0u && lane == 0) hand[pb] = hval;
          hval = 0u;
          const unsigned long long t0 = __builtin_amdgcn_s_memrealtime();
          wbase = o;
          while (!failed && m < total) {
            fw = __builtin_bit_cast(u32x2, __hip_atomic_load((GAS unsigned long long*)(myws + fo + (lane < 63 ? o * 8u : 0u)), __ATOMIC_RELAXED, __HIP_MEMORY_SCOPE_SYSTEM));
            const unsigned long long bl = __ballot(lane < 63 ? (int)(fw[0] - call) >= 0 : fw[0] == call);
            if ((bl >> 63) != 0ull) { failed = true; break; }
            if (bl & 1ull) break;
            if (__builtin_amdgcn_s_memrealtime() - t0 > kDeadline) { set_timeout(myws, call); failed = true; }
            __builtin_amdgcn_s_sleep(MSLEEP);
          }
        }
        cnt = failed ? 0u : min((unsigned)__builtin_amdgcn_readlane((int)fw[1], (int)((o - wbase) & 63u)), 4096u);
        if (UNLIKELY(uni(hand[b]) != 0u)) {
          const unsigned long long t0 = __builtin_amdgcn_s_memrealtime();
          while (uni(hand[b]) != 0u)
            if (__builtin_amdgcn_s_memrealtime() - t0 > kDeadline) { set_timeout(myws, call); return; }
        }
        FENCE();
        if (m >= total) {
          if (lane == 0) hand[b] = ~0u;
          break;
        }
        const unsigned bytes = 512u + 4u * cnt, sb = obuf + (q * 224u + s) * kPiece, nbuf = buf0 + b * kPBuf;
        auto part = [&](unsigned f0, unsigned f1) {
          for (unsigned f = f0; f < f1; f += 1024u) {
            WAIT_VM(PWIN - 1u);
            asm volatile("s_mov_b32 m0, %0\n\ts_nop 0\n\tbuffer_load_dwordx4 %1, %2, %3 offen sc0 sc1 lds"
                         :: "s"(nbuf + f), "v"(f + l16 < bytes ? l16 : kOob), "s"(rs), "s"(sb + f) : "memory");
          }
        };
        asm volatile("s_mov_b64 exec, 1\n\tglobal_atomic_add %0, %1, %2, %3 sc0\n\ts_mov_b64 exec, -1"
                     : "=v"(dv) : "v"(queue), "v"(1u), "s"(myws) : "memory");
        part(0u, min(bytes, (PWIN - 1u) * 1024u));
        if (bytes <= (PWIN - 2u) * 1024u) WAIT_VM(0);
        WAIT_VM(PWIN - 1u);
        if (hval != 0u && lane == 0) hand[pb] = hval;
        hval = 4u * s + (failed ? 3u : 1u);
        if ((m & 1u) == 0u) PSTAMP(myws, sid >> 1, (m >> 1) + 1u);
        asm volatile("" : "+v"(dv));
        m = uni(dv) - qbase;
        if (m < total && stale(m - 1152u)) {
          wbase = m - 1152u;
          asm volatile("buffer_load_dwordx2 %0, %1, %2, 0 offen sc0 sc1" : "=v"(fw) : "v"(fo + (lane < 63 ? wbase * 8u : 0u)), "s"(rws) : "memory");
          part((PWIN - 1u) * 1024u, bytes);
          if (bytes <= (2u * PWIN - 3u) * 1024u) WAIT_VM(0);
          WAIT_VM(PWIN - 1u);
          asm volatile("" : "+v"(fw) :: "memory");
        } else
          part((PWIN - 1u) * 1024u, bytes);
      }
    }
    STAMP(myws, 1 + wid);
    return;
  }

  {
    constexpr unsigned kNConv = 8u * (26u - CVFROM);
    char* const pws = p.ws[lane & 7];
    const unsigned mine = 15u << (rank & 4u);
    STAMP(myws, wid == 0 ? 27 : 31);
    for (unsigned n = isconv ? (unsigned)(bid - kNMove - 8 * CVFROM) : 1024u; n < 1024u; n += kNConv) {
      unsigned ktg, sub;
      item(n, ktg, sub);
      if (sub < 4u) {
        const unsigned so = abase + (2u * sub + (wid >> 2)) * kPanel + ktg * kBlk + (wid & 3) * 2048u;
#pragma unroll
        for (int i = 0; i < 4; ++i) st16<CVAUX>(cvt8(cv[2 * i], cv[2 * i + 1]), rws, l16 + (i >> 1) * 1024u, so + (i & 1) * 8192u);
        if (sub == 2u && (wid >> 1) == 2) {
          const int from = (4 * fr + (fq ^ ((-(fr >> 2)) & 3))) * 4;
#pragma unroll
          for (int i = 0; i < 4; ++i) {
            u32x4 hv = __builtin_bit_cast(u32x4, cvt8(cv[2 * i], cv[2 * i + 1]));
#pragma unroll
            for (int e = 0; e < 4; ++e) hv[e] = (unsigned)__builtin_amdgcn_ds_bpermute(from, (int)hv[e]);
            __builtin_amdgcn_raw_buffer_store_b128(hv, rws, l16 + (i >> 1) * 1024u, abase + kLast + ktg * 8192u + (wid & 1) * 2048u + (i & 1) * 4096u, CVAUX);
          }
        }
      } else {
        const unsigned so = bbase + (2u * (sub - 4u) + (wid >> 2)) * kPanel + ktg * kBlk + ((wid >> 1) & 1) * 4096u + (wid & 1) * 8192u;
#pragma unroll
        for (int i = 0; i < 4; ++i) st16<CVAUX>(cvt_col(cv, i), rws, l16 + i * 1024u, so);
      }
      FENCE();
      const bool ahead = n >= kNConv && n + kNConv < 1024u;
      if (ahead) { loads(n + kNConv); WAIT_VM(8); } else { WAIT_VM(0); }
      if (n < kNConv) STAMP(myws, wid == 0 ? 30 : 31);
      BARRIER();
      if (n < 2u * kNConv) STAMP(myws, wid == 0 ? (n < kNConv ? 28 : 29) : 31);
      const unsigned to = ktg >= FEWF || ktg < CVSKIP || (sub - 1u) < 3u ? 0u : sub == 0u ? mine : 1u << (og + sub - 4u);
      if (tid < 16) flag_store2(myws + kFlags + ((sub < 4u ? 2u * sub : 28u + 2u * sub) + ((unsigned)tid & 1u)) * 1024u + ktg * 8u + ((unsigned)tid >> 1) * kRep, call, call);
      else if (wid == 1 && lane < 8 && (((to & ~(1u << rank)) >> lane) & 1u)) flag_store(pws + kSrc + rank * 4096u + sub * 512u + ktg * 4u, call);
      FENCE();
      if (n < kNConv) loads(n + kNConv);
    }
    if (isconv && wid == 7) {
      unsigned dv = 0;
      if (lane == 0) dv = draw(myws + kCvt);
      if (uni(dv) == call * kNConv - 1u && lane < 8) flag_store(pws + kAll + rank * 4u, call);
    }
  }
  STAMP(myws, wid == 0 ? 1 : 31);

  const uint32_t* const tl = p.tiles + 3 * (bid - kNMove);
  const unsigned small = tl[(tl[0] >> 28) - 1u] & 16383u;
  const bool hass = ((small >> 10) & 3u) == 2u;
  const int ntiles = (int)(tl[0] >> 28) - (hass ? 1 : 0);
  const bool late = (small >> 13) != 0u;
  const unsigned tau = ((small >> 6) & 15u) * 6u + (small & 63u) - 23u + 3u * ((small >> 12) & 1u);
  if (wid == 7) return;

  if (wid >= 4) {
    const unsigned lw = (unsigned)wid - 4u;
    const unsigned total = 256u * (unsigned)ntiles;
    unsigned a = 0, aend = 0, so = lw * 8192u, src = 0, idle = 0;
    unsigned ok = 0, f0 = 0, f1 = 0, fb = 0, fv = 0;
    unsigned rdy = 0, askat = 0, lookat = ~0u;
    int ta = 0, tf = 0;
    auto next_flags = [&]() {
      while (tf < ntiles) {
        const unsigned code = tl[tf++];
        const unsigned ap = code & 63u, bp = (code >> 6) & 15u, shape = (code >> 10) & 3u;
        f0 = f1;
        f1 += 256u;
        if (shape == 3u && lw == 2u) { ok = f1; continue; }
        const unsigned grp = lw == 0u ? ap : shape == 0u ? (lw == 1u ? ap + 1u : 36u + bp) : shape == 1u ? 35u + bp + lw : 36u + bp;
        fb = kFlags + grp * 1024u + ((unsigned)bid & 7u) * kRep;
        break;
      }
    };
    auto known = [&]() {
      rdy = min(ok, aend);
      unsigned from = max(ok, LTHR);
      asm volatile("" : "+s"(from));
      askat = ok >= total ? ~0u : from - LTHR;
    };
    auto ask = [&](unsigned last) {
      const unsigned off = lane < ALANES ? fb + min(ok - f0 + (unsigned)lane, 255u) * 4u : lane == 63 ? last : kOob;
      asm volatile("buffer_load_dword %0, %1, %2, 0 offen sc0 sc1" : "=v"(fv) : "v"(off), "s"(rws) : "memory");
    };
    auto give_up = [&]() {
      ok = total;
      if (lane == 0) misc[3] = 1u;
      WAIT_LGKM0();
    };
    auto look = [&]() {
      asm volatile("" : "+v"(fv) :: "memory");
      const unsigned long long b = __ballot(lane < 63 ? (int)(fv - call) >= 0 : fv == call);
      if (UNLIKELY((b >> 63) != 0ull)) {
        give_up();
      } else {
        const unsigned run = (unsigned)__builtin_ctzll(~b);
        ok = min(ok + (run & ~1u), f1);
        if (UNLIKELY(ok == f1)) next_flags();
      }
      lookat = ~0u;
    };
    auto issue = [&](bool busy) {
      LAS char* const dl = lds + so;
      if (busy) {
        __builtin_amdgcn_raw_ptr_buffer_load_lds(rws, (LAS void*)dl, 16, l16, src, 0, kAuxDev);
        __builtin_amdgcn_raw_ptr_buffer_load_lds(rws, (LAS void*)dl, 16, l16, src, 1024, kAuxDev);
        __builtin_amdgcn_raw_ptr_buffer_load_lds(rws, (LAS void*)dl, 16, l16, src, 2048, kAuxDev);
        __builtin_amdgcn_raw_ptr_buffer_load_lds(rws, (LAS void*)dl, 16, l16, src, 3072, kAuxDev);
        __builtin_amdgcn_raw_ptr_buffer_load_lds(rws, (LAS void*)(dl + 4096), 16, l16, src + 4096u, 0, kAuxDev);
        __builtin_amdgcn_raw_ptr_buffer_load_lds(rws, (LAS void*)(dl + 4096), 16, l16, src + 4096u, 1024, kAuxDev);
        __builtin_amdgcn_raw_ptr_buffer_load_lds(rws, (LAS void*)(dl + 4096), 16, l16, src + 4096u, 2048, kAuxDev);
        __builtin_amdgcn_raw_ptr_buffer_load_lds(rws, (LAS void*)(dl + 4096), 16, l16, src + 4096u, 3072, kAuxDev);
      }
      FENCE();
      src += 8192u;
      so = so >= kRing - kSlot ? so - (kRing - kSlot) : so + kSlot;
      ++a;
    };
    next_flags();
    known();
    for (unsigned j = 0; j < total; ++j) {
      if (a == j + 4u) {
        unsigned e = min(rdy + 1u, askat);
        asm volatile("" : "+s"(e));
        e = min(e, lookat);
        if (idle != 0u) {
          WAIT_VM(0);
          while (a + 1u < e) { issue(false); BARRIER(); }
        }
        while (a + 1u < e) {
          issue(true);
          WAIT_VM(32);
          BARRIER();
        }
        j = a - 4u;
      }
      unsigned lim = min(j + 5u, rdy);
      if (UNLIKELY(a >= lim)) {
        if (a == aend) {
          const unsigned code = tl[ta++];
          const unsigned ap = code & 63u, bp = (code >> 6) & 15u, shape = (code >> 10) & 3u;
          if (lw == 0u || (lw == 1u && shape == 0u)) src = abase + (ap + lw) * kPanel;
          else src = bbase + (bp + (shape == 1u ? lw - 1u : 0u)) * kPanel;
          idle = shape == 3u && lw == 2u ? 1u : 0u;
          aend = ta == ntiles ? ~0u : aend + 256u;
          rdy = min(ok, aend);
        }
        if (a <= j && ok <= a) {
          WAIT_VM(0);
          if (lookat != ~0u) look();
          const unsigned long long t0 = __builtin_amdgcn_s_memrealtime();
          for (unsigned it = 0; ok <= a; ++it) {
            ask((it & TMOEV) == TMOEV ? kTmo : kOob);
            WAIT_VM(0);
            look();
            if (ok > a) break;
            if (__builtin_amdgcn_s_memrealtime() - t0 > kDeadline) { set_timeout(myws, call); give_up(); break; }
            __builtin_amdgcn_s_sleep(LSLEEP);
          }
          known();
        }
        lim = min(j + 5u, rdy);
      }
      if (a < lim) {
        issue(idle == 0u);
        if (UNLIKELY(a < lim)) issue(idle == 0u);
      }
      if (UNLIKELY(a >= askat)) { ask(PTMO); askat = ~0u; lookat = a + 4u; }
      const unsigned n = idle != 0u ? 0u : a - 1u - j;
      if (UNLIKELY(n < 4u)) {
        WAIT_VM(24);
        if (n < 3u) {
          WAIT_VM(16);
          if (n < 2u) {
            WAIT_VM(8);
            if (n < 1u) WAIT_VM(0);
          }
        }
      }
      WAIT_VM(32);
      if (UNLIKELY(j + 5u >= lookat)) { look(); known(); }
      BARRIER();
    }
    if (hass && lw < 2u) {
      const unsigned rep = kFlags + ((unsigned)bid & 7u) * kRep;
      const unsigned long long t0 = __builtin_amdgcn_s_memrealtime();
      unsigned long long mine = (small & 512u ? MASKL : MASKO) ^ (late ? 0ull : ~0ull);
      auto expired = [&](unsigned it) {
        bool bad = __builtin_amdgcn_s_memrealtime() - t0 > kDeadline;
        if (bad) set_timeout(myws, call);
        else if ((it & TMOEV) == TMOEV) bad = uni(__hip_atomic_load((GAS unsigned*)(myws + kTmo), __ATOMIC_RELAXED, __HIP_MEMORY_SCOPE_AGENT)) == call;
        if (UNLIKELY(bad)) {
          if (lane == 0) misc[3] = 1u;
          WAIT_LGKM0();
        }
        return bad;
      };
      BARRIER();
      if (lw == 1u) {
        const unsigned bp = (small >> 6) & 15u;
        for (unsigned it = 0;; ++it) {
          const u32x4 f = __builtin_amdgcn_raw_buffer_load_b128(rws, l16, rep + (36u + bp) * 1024u, kAuxSys);
          if (__ballot((int)((f[0] - call) | (f[1] - call) | (f[2] - call) | (f[3] - call)) < 0) == 0ull || expired(it)) break;
          __builtin_amdgcn_s_sleep(PSLEEP);
        }
        if (lane == 0) misc[5] = 1u;
        for (unsigned i = NPF; i-- > 0u;) {
          const unsigned pr = 63u - (unsigned)__builtin_clzll(mine);
          mine ^= 1ull << pr;
          const unsigned sb = bbase + bp * kPanel + ((small >> 12) & 1u) * 4096u + pr * 32768u;
          WAIT_VM(32);
#pragma unroll
          for (int w = 0; w < 4; ++w) {
            LAS char* const dl = lds + (w * (NPF * 4096) + i * 4096u);
            __builtin_amdgcn_raw_ptr_buffer_load_lds(rws, (LAS void*)dl, 16, l16, sb + w * 8192u, 0, kAuxDev);
            __builtin_amdgcn_raw_ptr_buffer_load_lds(rws, (LAS void*)dl, 16, l16, sb + w * 8192u, 1024, kAuxDev);
            __builtin_amdgcn_raw_ptr_buffer_load_lds(rws, (LAS void*)dl, 16, l16, sb + w * 8192u, 2048, kAuxDev);
            __builtin_amdgcn_raw_ptr_buffer_load_lds(rws, (LAS void*)dl, 16, l16, sb + w * 8192u, 3072, kAuxDev);
          }
          FENCE();
        }
        WAIT_VM(0);
        if (lane == 0) misc[6] = 1u;
        return;
      }
      const unsigned upto = 128u - 2u * (unsigned)__builtin_clzll(mine);
      unsigned have = 0;
      bool part = late;
      for (unsigned it = 0; have < upto || part; ++it) {
        {
          const unsigned k = min(have >> 1, 63u);
          const u32x2 f = __builtin_amdgcn_raw_buffer_load_b64(rws, have < upto && (unsigned)lane - (k & ~15u) < NARROW ? (unsigned)lane * 8u : kOob, rep + (small & 63u) * 1024u, kAuxSys);
          const unsigned g = __builtin_amdgcn_raw_buffer_load_b32(rws, part && have + PGATE >= upto ? (unsigned)(lane & 3) * 4u : kOob, kPFlags + tau * 128u, kAuxSys);
          const unsigned long long b0 = __ballot((int)(f[0] - call) < 0) >> k, b1 = __ballot((int)(f[1] - call) < 0) >> k;
          const unsigned n = max(have, min(b0 ? 2u * (k + (unsigned)__builtin_ctzll(b0)) : 128u, b1 ? 2u * (k + (unsigned)__builtin_ctzll(b1)) + 1u : 128u));
          if (n != have && lane == 0) misc[4] = n;
          if (part && __ballot((int)(g - call) < 0) == 0ull) {
#pragma unroll
            for (int i = 0; i < 16; ++i)
              __builtin_amdgcn_raw_ptr_buffer_load_lds(rws, (LAS void*)(lds + kPartLds + i * 1024), 16, l16, kPart + tau * 16384u + i * 1024u, 0, kAuxDev);
            part = false;
          }
          if (n != have) { have = n; continue; }
        }
        if (expired(it)) {
          if (lane == 0) misc[4] = 128u;
          break;
        }
        __builtin_amdgcn_s_sleep(PSLEEP);
      }
      WAIT_VM(0);
    }
    return;
  }

  {
    const unsigned slot = (unsigned)(4 * fr + (fq ^ ((-(fr >> 2)) & 3))) * 16u;
    unsigned so = 0;
    for (int ti = 0; ti < ntiles; ++ti) {
      const unsigned code = tl[ti];
      const unsigned shape = (code >> 10) & 3u;
      const unsigned wh = (unsigned)wid >> 1, wl = (unsigned)wid & 1u;
      const unsigned ap = (code & 63u) + (shape == 0u ? wh : 0u), bp = ((code >> 6) & 15u) + (shape == 1u ? wh : 0u);
      const unsigned jo = (bp - 8u) >> 1;
      const unsigned owner = (bp < 8u ? rank : og + jo) & 7u;
      const unsigned col0 = bp < 8u ? bp * 128u : 256u * q + (bp & 1u) * 128u;
      const __amdgpu_buffer_rsrc_t ro = mk_rsrc(p.out[owner], kInBytes);
      const unsigned vo = (unsigned)fq * 16384u + (unsigned)fr * 16u;

      auto run = [&](auto mconst, const unsigned aoff, const unsigned boff) {
        constexpr int M = decltype(mconst)::value;
        constexpr int NS = 256;
        const unsigned va = aoff + slot, vb = boff + l16;
        f32x4 acc[M][4];
#pragma unroll
        for (int m = 0; m < M; ++m)
#pragma unroll
          for (int n = 0; n < 4; ++n) acc[m][n] = f32x4{0.f, 0.f, 0.f, 0.f};
        f16x8 fa[2][M], fb[2][4];
        BARRIER();
        STAMP(myws, wid == 0 ? 2 + 3 * ti : 31);
        CSTAMP(myws, wid == 0 ? 12 + 2 * ti : 31);
#pragma unroll
        for (int i = 0; i < M; ++i) {
          if (i < 4) fb[0][i] = *(LAS f16x8*)(lds + (vb + so) + i * 1024);
          fa[0][i] = *(LAS f16x8*)(lds + (va + so) + i * 1024);
        }
        __builtin_amdgcn_s_waitcnt(0xC07F);
        BARRIER();
        so = so == kRing - kSlot ? 0u : so + kSlot;
        {
          unsigned ua = va + so, ub = vb + so;
          for (int t = 0; t < NS; t += 2) {
#pragma unroll
            for (int s = 0; s < 2; ++s) {
              __builtin_amdgcn_sched_barrier(0);
#pragma unroll
              for (int i = 0; i < 4 * M; ++i) {
                acc[i >> 2][i & 3] = __builtin_amdgcn_mfma_f32_16x16x32_f16(fa[s][i >> 2], fb[s][i & 3], acc[i >> 2][i & 3], 0, 0, 0);
                __builtin_amdgcn_sched_barrier(0);
                if (M == 8 && (i & 1) == 0 && i < 24) {
                  const int r = i >> 1;
                  if (r < 4) fb[s ^ 1][r] = *(LAS f16x8*)(lds + ub + r * 1024);
                  else fa[s ^ 1][r - 4] = *(LAS f16x8*)(lds + ua + (r - 4) * 1024);
                }
                if (M == 4 && i < 4) {
                  fb[s ^ 1][i] = *(LAS f16x8*)(lds + ub + i * 1024);
                  fa[s ^ 1][i] = *(LAS f16x8*)(lds + ua + i * 1024);
                }
                if (i == 4 * M - 10) { so = so == kRing - kSlot ? 0u : so + kSlot; asm volatile("" : "+s"(so)); }
                if (i == 4 * M - 6) { ua = va + so; asm volatile("" : "+v"(ua)); }
                if (i == 4 * M - 4) { ub = vb + so; asm volatile("" : "+v"(ub)); }
                if (i == 4 * M - 2) {
                  __builtin_amdgcn_s_waitcnt(0xC07F);
                  if (t < NS - 2) BARRIER();
                }
                __builtin_amdgcn_sched_barrier(0);
              }
            }
          }
          so = so == 0u ? kRing - kSlot : so - kSlot;
        }
        const bool hand_over = hass && ti == ntiles - 1;
        if (M == 4 && bp < 8u && hand_over) BARRIER();
        STAMP(myws, wid == 0 ? 3 + 3 * ti : 31);
        CSTAMP(myws, wid == 0 ? 13 + 2 * ti : 31);
        const int lo = misc[3] != 0u ? 0x7fc00000 : 0;
        if (bp >= 8u) {
          const __amdgpu_buffer_rsrc_t rw4 = __builtin_amdgcn_make_buffer_rsrc((void*)myws, 4, kWsNeed / 4u, 0x00020000);
          if (UNLIKELY(lo != 0)) {
#pragma unroll
            for (int m = 0; m < M; ++m)
#pragma unroll
              for (int n = 0; n < 4; ++n) acc[m][n] = __builtin_bit_cast(f32x4, u32x4{0x7fc00000u, 0x7fc00000u, 0x7fc00000u, 0x7fc00000u});
          }
          if constexpr (M == 4) {
            const unsigned hp = park_pos(ap, wh, jo);
            const unsigned s = hp * 4u + (bp & 1u) * 2u + wl;
            const unsigned pb0 = obuf + (jo * 224u + s) * kPiece;
            unsigned l0 = so + (unsigned)wid * kSlot;
            l0 = (unsigned)(size_t)lds + (l0 >= kRing ? l0 - kRing : l0);
            unsigned pos = l0 + 512u;
            const unsigned trash = l0 + kPBuf + (unsigned)lane * 4u;
            u32x2 mk = {0u, 0u};
#pragma unroll
            for (int m = 0; m < 4; ++m)
#pragma unroll
              for (int j = 0; j < 4; ++j)
#pragma unroll
                for (int n = 0; n < 4; ++n) {
                  const float f = acc[m][n][j];
                  const unsigned iv = __float_as_uint(f);
                  const bool keep = (int)iv >= 0;
                  const unsigned long long b = __ballot(keep);
                  unsigned a = lanes_below(b) * 4u + pos;
                  asm volatile("" : "+v"(a));
                  *(LAS unsigned*)(size_t)(keep ? a : trash) = iv;
                  advance(pos, b);
                  mk[m >> 1] = __builtin_amdgcn_alignbit(mk[m >> 1], iv, 31);
                }
            *(LAS u32x2*)(size_t)(l0 + (unsigned)lane * 8u) = mk;
            const unsigned bytes = hp < 56u ? pos - l0 : 0u;
            u32x4 v[17];
#pragma unroll
            for (int i = 0; i < 17; ++i) v[i] = *(LAS u32x4*)(size_t)(l0 + l16 + i * 1024);
            WAIT_LGKM0();
            if (hand_over) BARRIER();
#pragma unroll
            for (int i = 0; i < 17; ++i)
              if (i * 1024u < bytes) __builtin_amdgcn_raw_buffer_store_b128(v[i], rws, i * 1024u + l16 < bytes ? l16 : kOob, pb0 + i * 1024u, kAuxSys);
            WAIT_VM(0);
            if (lane == 0 && hp < 56u) flag_store2(p.ws[owner] + kOFlags + par * 8192u + q * 2048u + s * 8u, call, (bytes - 512u) >> 2);
          } else
#pragma unroll
          for (int rh = 0; rh < M / 4; ++rh) {
            const unsigned hp = park_pos(ap, M == 8 ? (unsigned)rh : wh, jo);
            const unsigned s = hp * 4u + (bp & 1u) * 2u + wl;
            const unsigned pb0 = obuf + (jo * 224u + s) * kPiece;
            unsigned pos = pb0 + 512u;
            if (hp < 56u) {
              u32x2 mk = {0u, 0u};
#pragma unroll
              for (int m = 0; m < 4; ++m)
#pragma unroll
                for (int j = 0; j < 4; ++j)
#pragma unroll
                  for (int n = 0; n < 4; ++n) {
                    const float f = acc[4 * rh + m][n][j];
                    const unsigned iv = __float_as_uint(f);
                    const unsigned long long b = __ballot((int)iv >= 0);
                    const unsigned idx = lanes_below(b) | (iv & kOob);
                    asm volatile("buffer_store_dword %0, %1, %2, %3 idxen sc0 sc1" :: "v"(iv), "v"(idx), "s"(rw4), "s"(pos) : "memory");
                    advance(pos, b);
                    mk[m >> 1] = __builtin_amdgcn_alignbit(mk[m >> 1], iv, 31);
                  }
              __builtin_amdgcn_raw_buffer_store_b64(mk, rws, (unsigned)lane * 8u, pb0, kAuxSys);
            }
            if (M == 4 && hand_over) BARRIER();
            if (hp < 56u) {
              WAIT_VM(0);
              if (lane == 0) flag_store2(p.ws[owner] + kOFlags + par * 8192u + q * 2048u + s * 8u, call, (pos - pb0 - 512u) >> 2);
            }
          }
        } else {
#pragma unroll
          for (int m = 0; m < M; ++m) {
            const unsigned ob = (panel_row(ap, M == 8 ? (unsigned)(m >> 2) : wh, rank) + 16u * (m & 3)) * 4096u + (col0 + 64u * wl) * 4u;
#pragma unroll
            for (int j = 0; j < 4; ++j) {
              u32x4 o;
#pragma unroll
              for (int n = 0; n < 4; ++n) { const float f = acc[m][n][j]; o[n] = (unsigned)max(__float_as_int(f), lo); }
              __builtin_amdgcn_raw_buffer_store_b128(o, ro, vo, ob + j * 4096u, kAuxSys);
            }
          }
        }
        if (M == 8 && hand_over) BARRIER();
      };
      if (shape == 3u) {
        run(std::integral_constant<int, 4>{}, wh * 4096u, 8192u + wl * 4096u);
      } else {
        run(std::integral_constant<int, 8>{}, shape == 0u ? wh * 8192u : 0u, (shape == 0u ? 16384u : 8192u + wh * 8192u) + wl * 4096u);
      }
      STAMP(myws, wid == 0 ? 4 + 3 * ti : 31);
    }
    if (!hass) return;
    const unsigned ap = small & 63u, bp = (small >> 6) & 15u, ch = (small >> 12) & 1u;
    const unsigned owner = (bp < 8u ? rank : og + ((bp - 8u) >> 1)) & 7u;
    const unsigned sa = abase + ap * kPanel + (unsigned)wid * 4096u, sb = bbase + bp * kPanel + ch * 4096u + (unsigned)wid * 8192u;
    unsigned long long mine = (small & 512u ? MASKL : MASKO) ^ (late ? 0ull : ~0ull);
    unsigned have = 0;
    constexpr int kNL = __builtin_popcountll(MASKL), kNE = 64 - kNL, kNO = __builtin_popcountll(MASKO);
    static_assert(kNO % SD == 0 && kNO >= NPF + SD && 64 - kNO >= NPF + SD, "");
    const int rounds = (__builtin_popcountll(mine) - NPF) / SD - 1;
    static_assert(kNE % SD == 0 && kNL % SD == 0 && NPF % SD == 0 && kNE >= NPF + SD && kNL >= NPF + SD && NPF >= 4, "");
    f32x4 acc[4][4];
#pragma unroll
    for (int m = 0; m < 4; ++m)
#pragma unroll
      for (int n = 0; n < 4; ++n) acc[m][n] = f32x4{0.f, 0.f, 0.f, 0.f};
    f32x4 ra[SD][4], rb[SD][4];
    auto await = [&](int word, unsigned need) {
      unsigned v;
      const unsigned long long t0 = __builtin_amdgcn_s_memrealtime();
      while ((v = uni(misc[word])) <= need)
        if (__builtin_amdgcn_s_memrealtime() - t0 > 2u * kDeadline) { misc[3] = 1u; v = 128u; break; }
      FENCE();
      return v;
    };
    auto fetch = [&](int u, bool cols) {
      const unsigned pr = (unsigned)__builtin_ctzll(mine), need = 2u * pr + ((unsigned)wid >> 1);
      mine &= mine - 1ull;
      if (cols) {
#pragma unroll
        for (int i = 0; i < 4; ++i) rb[u][i] = ld16<kAuxDev>(rws, l16 + i * 1024u, sb + pr * 32768u);
      }
      FENCE();
      if (have <= need) have = await(4, need);
#pragma unroll
      for (int i = 0; i < 4; ++i) ra[u][i] = ld16<kAuxDev>(rws, l16 + i * 1024u, sa + pr * 16384u);
      __builtin_amdgcn_sched_barrier(0);
    };
    auto mm = [&](int u) {
#pragma unroll
      for (int m = 0; m < 4; ++m)
#pragma unroll
        for (int n = 0; n < 4; ++n)
          acc[m][n] = __builtin_amdgcn_mfma_f32_16x16x32_f16(__builtin_bit_cast(f16x8, ra[u][m]), __builtin_bit_cast(f16x8, rb[u][n]), acc[m][n], 0, 0, 0);
    };
    LAS char* const mylds = lds + ((unsigned)wid * (NPF * 4096u) + l16);
    const __amdgpu_buffer_rsrc_t ro = mk_rsrc(p.out[owner], kInBytes);
    const unsigned ob = (panel_row(ap, 0u, rank) + 16u * (unsigned)wid) * 4096u + ((bp < 8u ? bp * 128u : 256u * q + (bp & 1u) * 128u) + 64u * ch) * 4u;
    STAMP(myws, wid == 0 ? 2 + 3 * ntiles : 31);
    await(5, 0u);
#pragma unroll
    for (int i = 0; i < SD - 1; ++i) fetch(i, true);
    for (int i = rounds; i > 0; --i) {
#pragma unroll
      for (int u = 0; u < SD; ++u) {
        fetch((u + SD - 1) % SD, true);
        mm(u);
      }
    }
#pragma unroll
    for (int u = 0; u < SD; ++u) {
      fetch((u + SD - 1) % SD, u == 0);
      mm(u);
    }
    await(6, 0u);
#pragma unroll
    for (int i = 0; i < NPF; ++i) {
#pragma unroll
      for (int n = 0; n < 4; ++n) rb[i % SD][n] = *(LAS f32x4*)(mylds + (i * 4096 + n * 1024));
      if (i + SD - 1 < NPF) fetch((i + SD - 1) % SD, false);
      mm(i % SD);
    }
    STAMP(myws, wid == 0 ? 3 + 3 * ntiles : 31);
    STAMP(myws, 18 + wid);
#pragma unroll
    for (int m = 0; m < 4; ++m)
#pragma unroll
      for (int n = 0; n < 4; ++n) *(LAS f32x4*)(mylds + (4 * m + n) * 1024) = acc[m][n];
    WAIT_LGKM0();
    BARRIER();
    STAMP(myws, wid == 0 ? 24 : 31);
    f32x4 sum[4];
#pragma unroll
    for (int n = 0; n < 4; ++n) {
      LAS char* const ps = lds + ((unsigned)wid * 4096u + n * 1024 + l16);
      sum[n] = (*(LAS f32x4*)ps + *(LAS f32x4*)(ps + NPF * 4096)) + (*(LAS f32x4*)(ps + 2 * NPF * 4096) + *(LAS f32x4*)(ps + 3 * NPF * 4096));
    }
    const int lo = misc[3] != 0u ? 0x7fc00000 : 0;
    if (!late) {
#pragma unroll
      for (int n = 0; n < 4; ++n) {
        if (UNLIKELY(lo != 0)) sum[n] = __builtin_bit_cast(f32x4, u32x4{0x7fc00000u, 0x7fc00000u, 0x7fc00000u, 0x7fc00000u});
        __builtin_amdgcn_raw_buffer_store_b128(__builtin_bit_cast(u32x4, sum[n]), rws, l16 + n * 1024u, kPart + tau * 16384u + (unsigned)wid * 4096u, kAuxDev);
      }
      WAIT_VM(0);
      if (lane == 0) flag_store(myws + kPFlags + tau * 128u + (unsigned)wid * 4u, call);
      STAMP(myws, wid == 0 ? 4 + 3 * ntiles : 31);
      return;
    }
#pragma unroll
    for (int n = 0; n < 4; ++n) sum[n] += *(LAS f32x4*)(lds + (kPartLds + (unsigned)wid * 4096u + n * 1024 + l16));
#pragma unroll
    for (int j = 0; j < 4; ++j) {
      u32x4 o;
#pragma unroll
      for (int n = 0; n < 4; ++n) { const float f = sum[n][j]; o[n] = (unsigned)max(__float_as_int(f), lo); }
      __builtin_amdgcn_raw_buffer_store_b128(o, ro, (unsigned)fq * 16384u + (unsigned)fr * 16u, ob + j * 4096u, kAuxSys);
    }
    STAMP(myws, wid == 0 ? 23 : 31);
    if (bp >= 8u) {
      WAIT_VM(0);
      if (lane == 0) __hip_atomic_fetch_add((GAS unsigned*)(p.ws[owner] + kDone), 1u, __ATOMIC_RELAXED, __HIP_MEMORY_SCOPE_SYSTEM);
    }
    STAMP(myws, wid == 0 ? 4 + 3 * ntiles : 31);
  }
}

extern "C" void kernel_launch(int rank, int nranks, ncclComm_t comm, hipStream_t stream,
                              void* const* d_in, const int* in_sizes, int n_in,
                              void* d_out, int out_size,
                              const int* in_dtypes, int out_dtype,
                              void* const* d_out_all, void* const* d_ws_all, size_t ws_bytes) {
  (void)comm; (void)in_dtypes; (void)out_dtype; (void)d_out;
  if (nranks != 8 || (unsigned)rank >= 8u || n_in < 2 || ws_bytes < (size_t)kWsNeed ||
      in_sizes[0] != 8 * 1024 * 1024 || in_sizes[1] != 8 * 1024 * 1024 || out_size != 8 * 1024 * 1024) {
    fprintf(stderr, "kernel_launch: unexpected problem geometry\n");
    return;
  }
  Params p;
  p.x = (const float*)d_in[0];
  p.w = (const float*)d_in[1];
  for (int i = 0; i < 8; ++i) { p.ws[i] = (char*)d_ws_all[i]; p.out[i] = (float*)d_out_all[i]; }
  p.rank = rank;
  p.pad = 0;
  static const int xcol[8] = {0, 4, 2, 3, 1, 5, 6, 7};
  auto tile = [](int a, int b, int shape) { return (unsigned)(a | (b << 6) | (shape << 10)); };
  for (int ci = 0; ci < kNComp; ++ci) {
    const int x = ci & 7, s = ci >> 3, bq = 2 * (x ^ 4), j = x >> 1;
    unsigned t[3] = {0u, 0u, 0u};
    int n = 0;
    auto farb = [&](int e) { const int idx = 3 * (x & 1) + e; return 8 + idx + (idx >= 2 * j ? 2 : 0); };
    if (s < 12) {
      t[n++] = tile(x < 4 ? 2 * (s >> 2) : 8 + 5 * (s >> 2), 4 * (x & 3) + (s & 3), 0);
      t[n++] = tile(10 + 5 * (s >> 2) + ((s >> 1) & 1), bq + (s & 1), 3);
    } else if (s == 12) {
      t[n++] = tile(6, xcol[x], 0);
      t[n++] = tile(27 + 2 * j, farb(1), 3);
    } else if (s < 16) {
      t[n++] = tile(12 + 5 * (s - 13), bq, 1);
    } else if (s < 18) {
      t[n++] = tile(34 + (s - 16), bq, 1);
    } else if (s < 23) {
      t[n++] = tile(26 + 2 * j, s < 22 ? 4 * (x & 1) + s - 18 : farb(2), 0);
    } else {
      t[n++] = tile(26 + 2 * j + ((s - 23) & 1), farb((s - 23) >> 1), 3);
    }
    const int i = s < 12 ? s : s == SKIPB || s == SKIPA ? -1 : s - 12 - (s > SKIPA ? 1 : 0) - (s > SKIPB ? 1 : 0);
    if (i >= 0) t[n++] = tile(23 + i % 3, i < 6 ? 8 + x : x, 2) | ((unsigned)((i / 3) & 1) << 12) | (s < 12 ? 8192u : 0u);
    p.tiles[3 * ci] = t[0] | ((unsigned)n << 28);
    p.tiles[3 * ci + 1] = t[1];
    p.tiles[3 * ci + 2] = t[2];
  }
  hipError_t e = hipFuncSetAttribute((const void*)ag_gemm_relu_kernel, hipFuncAttributeMaxDynamicSharedMemorySize, (int)kLdsBytes);
  if (e != hipSuccess) { fprintf(stderr, "hipFuncSetAttribute: %s\n", hipGetErrorString(e)); return; }
  hipLaunchKernelGGL(ag_gemm_relu_kernel, dim3(kNWG), dim3(512), kLdsBytes, stream, p);
  e = hipGetLastError();
  if (e != hipSuccess) fprintf(stderr, "launch: %s\n", hipGetErrorString(e));
}
```
